```python
import numpy as np

kernel_name = "gpu_ag_gemm_m8192_k8192_n8192_bf16_relu"


ABSMAX_RTOL = 0.008

COMPUTE_DTYPE = "bfloat16"
EPILOGUE = "relu"
IN_QUANT = None
OUT_QUANT = None
IN_DTYPES = {"x": "bfloat16", "w_mat": "bfloat16"}
WORLD_SIZE = 8

def _to_bf16(x):
    u = np.ascontiguousarray(x, dtype=np.float32).view(np.uint32)
    return ((u + 0x7FFF + ((u >> 16) & 1)) & 0xFFFF0000).view(np.float32)

def setup_inputs():

    x = _to_bf16(np.random.randn(8192, 8192).astype(np.float32))
    w_mat = _to_bf16((np.random.randn(8192, 8192) / np.sqrt(8192)).astype(np.float32))
    return {"x": x, "w_mat": w_mat}

def _epilogue(y):
    return np.maximum(y, 0.0).astype(np.float32)

def reference(x, w_mat):
    y = (x @ w_mat).astype(np.float32)
    return _epilogue(y)
```

```cpp
#include <hip/hip_runtime.h>
#include <rccl/rccl.h>
#include <cstdio>

#define HIP_CHECK(expr) do { hipError_t _e = (expr); if (_e != hipSuccess) { \
    fprintf(stderr, "HIP error %s at %s:%d\n", hipGetErrorString(_e), __FILE__, __LINE__); \
    return; } } while (0)

namespace {
#define LAS __attribute__((address_space(3)))
#define GAS __attribute__((address_space(1)))
typedef short bf16x8 __attribute__((ext_vector_type(8)));
typedef short s16x4 __attribute__((ext_vector_type(4)));
typedef unsigned short u16x2 __attribute__((ext_vector_type(2)));
typedef float f32x4 __attribute__((ext_vector_type(4)));
typedef unsigned u32x4 __attribute__((ext_vector_type(4)));
typedef unsigned u32x2 __attribute__((ext_vector_type(2)));

constexpr int NR = 8;
constexpr int KDIM = 8192, BK = 64, NT = KDIM / BK;
constexpr int APITCH = KDIM * 2, BPITCH = 2048, OPITCH = 2048;
constexpr int MP = 3;
constexpr int ECELL = 10, OCELL = 16 - ECELL;
constexpr int NSLOT = 3 * ECELL + 4 * OCELL;
constexpr int NCH = 64;
constexpr int RA_ITEMS = NSLOT * 2, RITEMS = RA_ITEMS + 32;
constexpr int NU = NCH * RITEMS, NSEND = NCH * 64;
constexpr int ITEM_PK = 6144;
constexpr size_t CELL_BYTES = (size_t)64 * APITCH;
constexpr size_t XA_BYTES = NSLOT * CELL_BYTES, W2_BYTES = (size_t)KDIM * BPITCH, PK_BYTES = (size_t)NU * ITEM_PK;
constexpr size_t OFF_XA = 0, OFF_W2 = XA_BYTES, OFF_PK = XA_BYTES + W2_BYTES, PAR_BYTES = OFF_PK + PK_BYTES;
constexpr size_t OFF_FLAG = 2 * PAR_BYTES, FLAG_BYTES = (size_t)NU * 128;
constexpr size_t OFF_CNT = OFF_FLAG + 2 * FLAG_BYTES, CNT_BYTES = (size_t)(NSLOT + 4) * NCH * 128;
constexpr size_t OFF_TICK = OFF_CNT + CNT_BYTES, TICK_BYTES = 2048 * 64;
constexpr size_t OFF_DONE = OFF_TICK + TICK_BYTES, OFF_TMO = OFF_DONE + 128, OFF_DBG = OFF_TMO + 128, DBG_BYTES = 2048 * 64;
constexpr size_t WS_NEED = OFF_DBG + DBG_BYTES;
constexpr int GRID = 256, NTHR = 512;
#define SEND_PER_XCD 4
constexpr int NSW = SEND_PER_XCD * 8 * 8;
constexpr int NUWG = 4 * (8 - SEND_PER_XCD) + 4 * (12 - SEND_PER_XCD), NUW = NUWG * 8;
constexpr unsigned DONE_PER_CALL = 80;
constexpr unsigned long long DEADLINE = 50000000ull;
constexpr int HTB = 16384, LDS_BYTES = 8 * HTB;
static_assert(ECELL == 10 && NSW % 64 == 0, "tile tables below are written for 10 + 6 cells");

struct Params {
    const char* x; const char* w;
    char* out; char* outx[4];
    char* wsx[NR];
    int rank; int pad;
};

#define WAIT_V(n) asm volatile("s_waitcnt vmcnt(" #n ")" ::: "memory")
#define WAIT_L(n) asm volatile("s_waitcnt lgkmcnt(" #n ")" ::: "memory")
#define BAR __builtin_amdgcn_s_barrier()
#define SCHED __builtin_amdgcn_sched_barrier(0)
#define STAMP(i) do {} while (0)

__device__ __forceinline__ unsigned ld_agent(const char* p) { return __hip_atomic_load((const GAS unsigned*)p, __ATOMIC_RELAXED, __HIP_MEMORY_SCOPE_AGENT); }
__device__ __forceinline__ unsigned ld_sys(const char* p) { return __hip_atomic_load((const GAS unsigned*)p, __ATOMIC_RELAXED, __HIP_MEMORY_SCOPE_SYSTEM); }
__device__ __forceinline__ bool give_up(char* ws, unsigned call, unsigned long long t0) {
    if ((unsigned)__builtin_amdgcn_readfirstlane(ld_agent(ws + OFF_TMO)) == call || __builtin_amdgcn_s_memrealtime() - t0 > DEADLINE) {
        __hip_atomic_store((GAS unsigned*)(ws + OFF_TMO), call, __ATOMIC_RELAXED, __HIP_MEMORY_SCOPE_AGENT);
        return true;
    }
    __builtin_amdgcn_s_sleep(2);
    return false;
}
__device__ __forceinline__ void store16_sys(char* base, unsigned voff, u32x4 v) {
    asm volatile("global_store_dwordx4 %0, %1, %2 sc0 sc1\n\ts_nop 1" :: "v"(voff), "v"(v), "s"(base) : "memory");
}

__device__ __forceinline__ void sender(const Params& P, int sw, unsigned call) {
    const int lane = threadIdx.x & 63;
    const size_t par = (call & 1) * PAR_BYTES, fpar = (call & 1) * FLAG_BYTES;
    const unsigned voffA = (lane >> 4) * APITCH + (lane & 15) * 16, voffB = (lane >> 5) * BPITCH + (lane & 31) * 16;
    for (int n = 0; n * NSW + sw < NSEND; ++n) {
        const int i = n * NSW + sw, chunk = i >> 6, j = (i + n) & 63, idx = j >> 1;
        const bool isb = j & 1;
        const char* src; size_t fin; unsigned voff, istep; int d0, nd, jr, jstep; size_t finstep;
        if (isb) { const int q = idx & 3, sub = idx >> 2; const size_t o = (size_t)(chunk * 128 + sub * 16) * BPITCH + q * 512;
            src = P.w + o; fin = OFF_W2 + o; voff = voffB; istep = 2 * BPITCH; d0 = 2 * q + 1; nd = 1; jr = RA_ITEMS + q * 8 + sub; jstep = 0; finstep = 0; }
        else { const int cell = idx >> 1, half = idx & 1; const size_t o = (size_t)half * 32 * APITCH + chunk * 256; const bool e = cell < ECELL;
            const int slot = e ? cell : 3 * ECELL + cell - ECELL, sstep = e ? ECELL : OCELL;
            src = P.x + cell * CELL_BYTES + o; fin = OFF_XA + slot * CELL_BYTES + o; voff = voffA; istep = 4 * APITCH; d0 = e ? 2 : 1; nd = e ? 3 : 4;
            jr = slot * 2 + half; jstep = sstep * 2; finstep = sstep * CELL_BYTES; }
        u32x4 raw[8];
#pragma unroll
        for (int v = 0; v < 8; ++v) raw[v] = *(const GAS u32x4*)(src + voff + (size_t)v * istep);
        u16x2 mx = {0, 0};
#pragma unroll
        for (int v = 0; v < 8; ++v)
#pragma unroll
            for (int c = 0; c < 4; ++c) mx = __builtin_elementwise_max(mx, __builtin_bit_cast(u16x2, raw[v][c] & 0x7fff7fffu));
        unsigned m = mx.x > mx.y ? mx.x : mx.y;
#pragma unroll
        for (int o = 32; o >= 1; o >>= 1) { const unsigned t = __shfl_xor(m, o); m = t > m ? t : m; }
        const unsigned base = (unsigned)__builtin_amdgcn_readfirstlane(m) >> 7, base4 = base * 0x01010101u;
        u32x4 pk[6]; unsigned esc = 0;
#pragma unroll
        for (int v = 0; v < 8; ++v) {
            unsigned r[4];
#pragma unroll
            for (int c = 0; c < 4; ++c) r[c] = ((raw[v][c] << 1) & 0xfffefffeu) | ((raw[v][c] >> 15) & 0x00010001u);
            pk[v >> 1][(v & 1) * 2] = __builtin_amdgcn_perm(r[1], r[0], 0x06040200u);
            pk[v >> 1][(v & 1) * 2 + 1] = __builtin_amdgcn_perm(r[3], r[2], 0x06040200u);
            const unsigned d0e = base4 - __builtin_amdgcn_perm(r[1], r[0], 0x07050301u), d1e = base4 - __builtin_amdgcn_perm(r[3], r[2], 0x07050301u);
            unsigned nib = d0e | (d1e << 4);
            if (((d0e | d1e) & 0xf0f0f0f0u) != 0u) nib = 0xffffffffu;
            if (nib == 0xffffffffu) esc |= 1u << v;
            pk[4 + (v >> 2)][v & 3] = nib;
        }
        const bool any_esc = __builtin_amdgcn_ballot_w64(esc != 0u) != 0ull;
        for (int k = 0; k < nd; ++k) {
            char* ws = P.wsx[(d0 + 2 * k) & 7];
            char* dst = ws + par + OFF_PK + (size_t)(chunk * RITEMS + jr + k * jstep) * ITEM_PK + 2048;
            asm volatile("global_store_dwordx4 %0, %1, %7 offset:-2048 sc0 sc1\n\tglobal_store_dwordx4 %0, %2, %7 offset:-1024 sc0 sc1\n\t"
                         "global_store_dwordx4 %0, %3, %7 sc0 sc1\n\tglobal_store_dwordx4 %0, %4, %7 offset:1024 sc0 sc1\n\t"
                         "global_store_dwordx4 %0, %5, %7 offset:2048 sc0 sc1\n\tglobal_store_dwordx4 %0, %6, %7 offset:3072 sc0 sc1\n\ts_nop 1"
                         :: "v"(lane * 16), "v"(pk[0]), "v"(pk[1]), "v"(pk[2]), "v"(pk[3]), "v"(pk[4]), "v"(pk[5]), "s"(dst) : "memory");
            if (any_esc) {
                char* f = ws + par + fin + k * finstep;
#pragma unroll
                for (int v = 0; v < 8; ++v) if (esc & (1u << v)) store16_sys(f, voff + v * istep, raw[v]);
            }
        }
        WAIT_V(0);
        if (lane < nd) {
            char* ws = P.wsx[(d0 + 2 * lane) & 7];
            __hip_atomic_store((GAS unsigned*)(ws + OFF_FLAG + fpar + (size_t)(chunk * RITEMS + jr + lane * jstep) * 128), (call << 8) | base, __ATOMIC_RELAXED, __HIP_MEMORY_SCOPE_SYSTEM);
        }
        if (n == 0) STAMP(1);
    }
}

__device__ __forceinline__ void unpacker(const Params& P, int uw, unsigned call) {
    const int lane = threadIdx.x & 63;
    char* ws = P.wsx[0];
    char* set = ws + (call & 1) * PAR_BYTES;
    const char* flags = ws + OFF_FLAG + (call & 1) * FLAG_BYTES;
    const unsigned voffA = (lane >> 4) * APITCH + (lane & 15) * 16, voffB = (lane >> 5) * BPITCH + (lane & 31) * 16;
    GAS unsigned* pending = nullptr;
    for (int u = uw; u < NU; u += NUW) {
        const int chunk = u / RITEMS, j = u - chunk * RITEMS;
        unsigned f = (unsigned)__builtin_amdgcn_readfirstlane(ld_sys(flags + (size_t)u * 128));
        WAIT_V(0);
        if (pending && lane == 0) __hip_atomic_fetch_add(pending, 1u, __ATOMIC_RELAXED, __HIP_MEMORY_SCOPE_AGENT);
        pending = nullptr;
        if ((f >> 8) < call) {
            const unsigned long long t0 = __builtin_amdgcn_s_memrealtime();
            bool dead = false;
            while (((f = (unsigned)__builtin_amdgcn_readfirstlane(ld_sys(flags + (size_t)u * 128))) >> 8) < call)
                if (give_up(ws, call, t0)) { dead = true; break; }
            if (dead) return;
        }
        u32x4 pk[6];
        asm volatile("global_load_dwordx4 %0, %6, %7 offset:-2048 sc0 sc1\n\tglobal_load_dwordx4 %1, %6, %7 offset:-1024 sc0 sc1\n\t"
                     "global_load_dwordx4 %2, %6, %7 sc0 sc1\n\tglobal_load_dwordx4 %3, %6, %7 offset:1024 sc0 sc1\n\t"
                     "global_load_dwordx4 %4, %6, %7 offset:2048 sc0 sc1\n\tglobal_load_dwordx4 %5, %6, %7 offset:3072 sc0 sc1\n\ts_waitcnt vmcnt(0)"
                     : "=&v"(pk[0]), "=&v"(pk[1]), "=&v"(pk[2]), "=&v"(pk[3]), "=&v"(pk[4]), "=&v"(pk[5])
                     : "v"(lane * 16), "s"(set + OFF_PK + (size_t)u * ITEM_PK + 2048) : "memory");
        char* fin; unsigned voff, istep; int stream;
        if (j < RA_ITEMS) { stream = j >> 1; fin = set + OFF_XA + stream * CELL_BYTES + (size_t)(j & 1) * 32 * APITCH + chunk * 256; voff = voffA; istep = 4 * APITCH; }
        else { const int q = (j - RA_ITEMS) >> 3, sub = (j - RA_ITEMS) & 7; stream = NSLOT + q;
            fin = set + OFF_W2 + (size_t)(chunk * 128 + sub * 16) * BPITCH + q * 512; voff = voffB; istep = 2 * BPITCH; }
        const unsigned base4 = (f & 255u) * 0x01010101u;
#pragma unroll
        for (int v = 0; v < 8; ++v) {
            const unsigned nib = pk[4 + (v >> 2)][v & 3], m0 = pk[v >> 1][(v & 1) * 2], m1 = pk[v >> 1][(v & 1) * 2 + 1];
            const unsigned e0 = base4 - (nib & 0x0f0f0f0fu), e1 = base4 - ((nib >> 4) & 0x0f0f0f0fu);
            u32x4 r;
            r[0] = __builtin_amdgcn_perm(e0, m0, 0x05010400u); r[1] = __builtin_amdgcn_perm(e0, m0, 0x07030602u);
            r[2] = __builtin_amdgcn_perm(e1, m1, 0x05010400u); r[3] = __builtin_amdgcn_perm(e1, m1, 0x07030602u);
#pragma unroll
            for (int c = 0; c < 4; ++c) r[c] = ((r[c] >> 1) & 0x7fff7fffu) | ((r[c] << 15) & 0x80008000u);
            if (nib != 0xffffffffu) store16_sys(fin, voff + v * istep, r);
        }
        pending = (GAS unsigned*)(ws + OFF_CNT + (size_t)(stream * NCH + chunk) * 128);
        if (u == uw) STAMP(1);
    }
    WAIT_V(0);
    if (pending && lane == 0) __hip_atomic_fetch_add(pending, 1u, __ATOMIC_RELAXED, __HIP_MEMORY_SCOPE_AGENT);
}

__device__ __forceinline__ int lds_byte(int r, int c) { const int st = (r >> 4) * 2 + (c >> 5), rr = r & 15, cc = c & 31, ob = rr * 64 + cc * 2; return st * 1024 + (ob ^ (((ob >> 9) & 1) << 5)); }
__device__ __forceinline__ void stage_rc(int b, int& R, int& C) { const int st = b / 1024, sb = b % 1024, swz = sb ^ (((sb >> 9) & 1) << 5); R = (st >> 1) * 16 + swz / 64; C = (st & 1) * 32 + (swz % 64) / 2; }
__device__ __forceinline__ unsigned cvt_pk_bf16(float lo, float hi) { unsigned r; asm("v_cvt_pk_bf16_f32 %0, %1, %2" : "=v"(r) : "v"(lo), "v"(hi)); return r; }
template <int OFF> __device__ __forceinline__ s16x4 tr_read(unsigned addr) { s16x4 r; asm volatile("ds_read_b64_tr_b16 %0, %1 offset:%2" : "=v"(r) : "v"(addr), "i"(OFF) : "memory"); return r; }

__device__ __forceinline__ int cell_of(int rt, int p, int skipq) {
    if (rt < 14) { const int c = rt * 3 + p; return c >= 40 ? -1 : (c < ECELL ? c : 16 + c - ECELL); }
    if (rt >= 22) return ECELL + (rt - 22) * 3 + p;
    const int k = (rt - 14) >> 1, q = k + (k >= skipq ? 1 : 0);
    return 16 + 3 * ECELL + q * OCELL + ((rt - 14) & 1) * 3 + p;
}
__device__ __forceinline__ int out_row_of(int id, int rank) {
    if (id < 16) return rank * 1024 + id * 64;
    const int slot = id - 16;
    if (slot < 3 * ECELL) return (rank ^ (2 * (slot / ECELL + 1))) * 1024 + (slot % ECELL) * 64;
    const int o = slot - 3 * ECELL;
    return (rank ^ (2 * (o / OCELL) + 1)) * 1024 + (ECELL + o % OCELL) * 64;
}

__device__ __forceinline__ void gemm_tile(const Params& P, LAS unsigned char* lds, int rt, int ct, bool remote, unsigned call) {
    int tid = threadIdx.x;
    int wid = __builtin_amdgcn_readfirstlane(tid >> 6), lane = tid & 63, wr = wid >> 2, wc = wid & 3, fr = lane & 15, fq = lane >> 4;
    char* ws = P.wsx[0];
    const char* set = ws + (call & 1) * PAR_BYTES;
    const int skipq = remote ? ct : 4;
    const int id0 = cell_of(rt, 0, skipq), id1r = cell_of(rt, 1, skipq), id2r = cell_of(rt, 2, skipq);
    const int id1 = id1r < 0 ? id0 : id1r, id2 = id2r < 0 ? id0 : id2r;
    const char* A0 = id0 < 16 ? P.x + id0 * CELL_BYTES : set + OFF_XA + (id0 - 16) * CELL_BYTES;
    const char* A1 = id1 < 16 ? P.x + id1 * CELL_BYTES : set + OFF_XA + (id1 - 16) * CELL_BYTES;
    const char* A2 = id2 < 16 ? P.x + id2 * CELL_BYTES : set + OFF_XA + (id2 - 16) * CELL_BYTES;
    const char* Bp = (remote ? set + OFF_W2 : P.w) + ct * 512;
    unsigned coff, need;
    { const int id = lane == 0 ? id0 : lane == 1 ? id1 : id2;
      const bool watch = wid == 0 && (lane < 3 ? id >= 16 : (lane == 3 && remote));
      coff = watch ? (unsigned)((lane < 3 ? id - 16 : NSLOT + ct) * NCH * 128) : 0u;
      need = watch ? call * (lane < 3 ? 2u : 8u) : 0u; }
    const unsigned long long wmask = wid == 0 ? 0xfull : 0ull;
    const char* cnt = ws + OFF_CNT;
    unsigned pre = 0;
#define SLOW_WAIT(chunk) do { const unsigned long long t0 = __builtin_amdgcn_s_memrealtime(); \
        while (__builtin_amdgcn_ballot_w64(ld_agent(cnt + coff + (chunk) * 128) < need) != 0ull) if (give_up(ws, call, t0)) break; \
        asm volatile("" ::: "memory"); } while (0)
#define PRE_LOAD(chunk) asm volatile("s_mov_b64 exec, %3\n\tglobal_load_dword %0, %1, %2 sc1\n\ts_mov_b64 exec, -1" : "+v"(pre) : "v"(coff), "s"(cnt + (chunk) * 128), "s"(wmask) : "memory")

    unsigned voffA, voffB;
    { int R, C; stage_rc(tid * 16, R, C); voffA = (unsigned)(R * APITCH + C * 2); }
    { const int b = tid * 16, k = b >> 8, ch = (b >> 4) & 15, f = (k & 3) | (((k >> 3) & 1) << 2), c = (ch >> 1) ^ f;
      voffB = (unsigned)(k * BPITCH + c * 32 + (ch & 1) * 16); }
    const unsigned ldsw = (unsigned)wid * 1024u;
    const int aoff = lds_byte(wr * 16 * MP + fr, fq * 8);
    unsigned boff0, boff1;
    { const int q = (lane & 15) >> 2, p = lane & 3, f = q | ((fq & 1) << 2);
      const unsigned base = (unsigned)(size_t)lds + 4 * HTB + (8 * fq + q) * 256 + p * 8;
      boff0 = base + (((wc * 2) ^ f) * 32); boff1 = base + (((wc * 2 + 1) ^ f) * 32); }
#define SBO(b, h) (((b) * 2 + (h)) * HTB)
#define GLDS(gptr, ldsoff) __builtin_amdgcn_global_load_lds((const unsigned*)(gptr), (LAS unsigned*)(lds + (ldsoff) + ldsw), 16, 0, 17)
#define STAGE_A_LO(b, koff) GLDS(A0 + (koff) + voffA, (b) * 2 * HTB)
#define STAGE_A_HI(b, koff) do { GLDS(A1 + (koff) + voffA, (b) * 2 * HTB + 8192); GLDS(A2 + (koff) + voffA, (b) * 2 * HTB + 16384); } while (0)
#define STAGE_B(b, h, gbase) do { GLDS((gbase) + (h) * 256 + voffB, 4 * HTB + SBO(b, h)); GLDS((gbase) + (h) * 256 + 32 * BPITCH + voffB, 4 * HTB + SBO(b, h) + 8192); } while (0)
#define LDA(dst, b, h) do { _Pragma("unroll") for (int m = 0; m < MP; ++m) _Pragma("unroll") for (int k = 0; k < 2; ++k) dst[m][k] = *(const LAS bf16x8*)(lds + (b) * 2 * HTB + (h) * MP * 4096 + aoff + m * 2048 + k * 1024); } while (0)
#define LDB1(dst, addr, off) do { const s16x4 _lo = tr_read<(off)>(addr), _hi = tr_read<(off) + 1024>(addr); dst = __builtin_shufflevector(_lo, _hi, 0, 1, 2, 3, 4, 5, 6, 7); } while (0)
#define LDB(dst, b, h) do { LDB1(dst[0][0], boff0, SBO(b, h)); LDB1(dst[0][1], boff0, SBO(b, h) + 8192); LDB1(dst[1][0], boff1, SBO(b, h)); LDB1(dst[1][1], boff1, SBO(b, h) + 8192); } while (0)
#define WAIT_LB(B) asm volatile("s_waitcnt lgkmcnt(0)" : "+v"(B[0][0]), "+v"(B[0][1]), "+v"(B[1][0]), "+v"(B[1][1]) :: "memory")
#define WAIT_PRE() asm volatile("s_waitcnt vmcnt(5)" : "+v"(pre) :: "memory")
#define MMA(ai, bj, At, Bt) do { __builtin_amdgcn_s_setprio(1); _Pragma("unroll") for (int k = 0; k < 2; ++k) _Pragma("unroll") for (int m = 0; m < MP; ++m) _Pragma("unroll") for (int n = 0; n < 2; ++n) \
        acc[ai][bj][m][n] = __builtin_amdgcn_mfma_f32_16x16x32_bf16(Bt[n][k], At[m][k], acc[ai][bj][m][n], 0, 0, 0); __builtin_amdgcn_s_setprio(0); } while (0)
    f32x4 acc[2][2][MP][2];
#pragma unroll
    for (int a = 0; a < 2; ++a)
#pragma unroll
        for (int b = 0; b < 2; ++b)
#pragma unroll
            for (int m = 0; m < MP; ++m)
#pragma unroll
                for (int n = 0; n < 2; ++n) acc[a][b][m][n] = (f32x4){0.f, 0.f, 0.f, 0.f};
    bf16x8 At[MP][2], B0[2][2], B1[2][2];
    constexpr size_t ka = BK * 2, kb = (size_t)BK * BPITCH;
    if (wid == 0) SLOW_WAIT(0);
    BAR;
    STAMP(1);
    PRE_LOAD(1);
    STAGE_B(0, 0, Bp); STAGE_A_LO(0, 0); STAGE_A_HI(0, 0); STAGE_B(0, 1, Bp);
    if (wr == 1) BAR;
    WAIT_V(2); BAR;
    STAGE_B(1, 0, Bp + kb); STAGE_A_LO(1, ka); STAGE_B(1, 1, Bp + kb);
    WAIT_PRE(); BAR;
    for (int t = 0; t < NT; t += 2) {
        const int t2 = (t == NT - 2) ? 0 : t + 2;
        const size_t k1 = (size_t)(t + 1) * ka, k2 = (size_t)t2 * ka; const char* b2 = Bp + (size_t)t2 * kb;
        const int c1 = (t >> 1) + 1 < NCH ? (t >> 1) + 1 : NCH - 1, c2 = (t >> 1) + 2 < NCH ? (t >> 1) + 2 : NCH - 1;
        if (__builtin_amdgcn_ballot_w64(pre < need) != 0ull) SLOW_WAIT(c1);
        LDB(B0, 0, 0); SCHED; LDA(At, 0, 0); STAGE_A_HI(1, k1);
        WAIT_L(6); BAR; WAIT_LB(B0); MMA(0, 0, At, B0); BAR; SCHED;
        LDB(B1, 0, 1); STAGE_B(0, 0, b2);
        BAR; WAIT_LB(B1); MMA(0, 1, At, B1); BAR;
        LDA(At, 0, 1); STAGE_A_LO(0, k2);
        BAR; WAIT_L(0); MMA(1, 0, At, B0); BAR; SCHED;
        STAGE_B(0, 1, b2);
        WAIT_V(5); BAR; MMA(1, 1, At, B1); BAR;
        LDB(B0, 1, 0); SCHED; LDA(At, 1, 0); STAGE_A_HI(0, k2); PRE_LOAD(c2);
        WAIT_L(6); BAR; WAIT_LB(B0); MMA(0, 0, At, B0); BAR; SCHED;
        LDB(B1, 1, 1); STAGE_B(1, 0, b2 + kb);
        BAR; WAIT_LB(B1); MMA(0, 1, At, B1); BAR;
        LDA(At, 1, 1); STAGE_A_LO(1, k2 + ka);
        BAR; WAIT_L(0); MMA(1, 0, At, B0); BAR; SCHED;
        STAGE_B(1, 1, b2 + kb);
        WAIT_PRE(); BAR; MMA(1, 1, At, B1); BAR;
    }
    const bool bad = (unsigned)__builtin_amdgcn_readfirstlane(ld_agent(ws + OFF_TMO)) == call;
    WAIT_V(0);
    if (wr == 0) BAR;
    BAR;
    STAMP(2);
    int tz = threadIdx.x; asm volatile("" : "+v"(tz));
    lane = tz & 63; fr = lane & 15; fq = lane >> 4;
#pragma unroll
    for (int ai = 0; ai < 2; ++ai)
#pragma unroll
        for (int m = 0; m < MP; ++m) {
            const int row = ai * MP * 32 + wr * MP * 16 + m * 16 + fr;
#pragma unroll
            for (int bj = 0; bj < 2; ++bj)
#pragma unroll
                for (int n = 0; n < 2; ++n) {
                    f32x4 v = acc[ai][bj][m][n];
#pragma unroll
                    for (int j = 0; j < 4; ++j) v[j] = v[j] < 0.f ? 0.f : v[j];
                    u32x2 w; w.x = cvt_pk_bf16(v[0], v[1]); w.y = cvt_pk_bf16(v[2], v[3]);
                    if (bad) { w.x = 0x7fc07fc0u; w.y = 0x7fc07fc0u; }
                    const int ch = bj * 16 + wc * 4 + n * 2 + (fq >> 1);
                    *(LAS u32x2*)(lds + row * 512 + ((ch ^ (fr & 7)) * 16) + (fq & 1) * 8) = w;
                }
        }
    WAIT_L(0); BAR; asm volatile("" ::: "memory");
    char* out = (remote ? P.outx[ct & 3] : P.out) + ct * 512;
#pragma unroll
    for (int p = 0; p < MP; ++p) {
        const int id = p == 0 ? id0 : p == 1 ? id1r : id2r;
        if (id >= 0) {
            char* O = out + (size_t)out_row_of(id, P.rank) * OPITCH;
#pragma unroll
            for (int i = 0; i < 4; ++i) {
                const int rl = wid * 8 + i * 2 + (lane >> 5), row = p * 64 + rl, ch = lane & 31;
                const u32x4 w = *(const LAS u32x4*)(lds + row * 512 + ((ch ^ (row & 7)) * 16));
                if (remote) store16_sys(O, rl * OPITCH + ch * 16, w);
                else *(GAS u32x4*)(O + rl * OPITCH + ch * 16) = w;
            }
        }
    }
    if (remote) {
        WAIT_V(0); BAR;
        if (threadIdx.x == 0) __hip_atomic_fetch_add((GAS unsigned*)(P.wsx[(2 * ct + 1) & 7] + OFF_DONE), 1u, __ATOMIC_RELAXED, __HIP_MEMORY_SCOPE_SYSTEM);
    } else if (rt == 22 && ct == 0 && wid == 0) {
        const unsigned long long t0 = __builtin_amdgcn_s_memrealtime();
        while ((unsigned)__builtin_amdgcn_readfirstlane(ld_sys(ws + OFF_DONE)) < call * DONE_PER_CALL)
            if (give_up(ws, call, t0)) { WAIT_V(0); if (lane == 0) *(GAS unsigned*)(out + (size_t)out_row_of(id0, P.rank) * OPITCH) = 0x7fc07fc0u; break; }
    }
    STAMP(3);
}

__global__ __launch_bounds__(NTHR, 2) void ag_gemm_kernel(const Params P) {
    extern __shared__ __attribute__((aligned(16))) unsigned char shm[];
    const int xcd = blockIdx.x & 7, slot = blockIdx.x >> 3, wid = __builtin_amdgcn_readfirstlane(threadIdx.x >> 6);
    STAMP(0);
    unsigned tick = 0;
    if ((threadIdx.x & 63) == 0) tick = __hip_atomic_fetch_add((GAS unsigned*)(P.wsx[0] + OFF_TICK + (size_t)(blockIdx.x * 8 + wid) * 64), 1u, __ATOMIC_RELAXED, __HIP_MEMORY_SCOPE_AGENT);
    const unsigned call = (unsigned)__builtin_amdgcn_readfirstlane(tick) + 1u;
    const int ncomm = xcd < 4 ? 8 : 12;
    if (slot < SEND_PER_XCD) {
        sender(P, (slot * 8 + xcd) * 8 + wid, call);
        STAMP(3);
    } else if (slot < ncomm) {
        const int uwg = xcd < 4 ? xcd * (8 - SEND_PER_XCD) + slot - SEND_PER_XCD : 4 * (8 - SEND_PER_XCD) + (xcd - 4) * (12 - SEND_PER_XCD) + slot - SEND_PER_XCD;
        unpacker(P, wid * NUWG + uwg, call);
        STAMP(3);
    } else {
        const int g = slot - ncomm, nrt = xcd < 4 ? 6 : 5;
        gemm_tile(P, (LAS unsigned char*)shm, (xcd & 3) * nrt + g % nrt, g / nrt, xcd >= 4, call);
    }
}
}

extern "C" void kernel_launch(int rank, int nranks, ncclComm_t comm, hipStream_t stream,
                              void* const* d_in, const int* in_sizes, int n_in,
                              void* d_out, int out_size, const int* in_dtypes, int out_dtype,
                              void* const* d_out_all, void* const* d_ws_all, size_t ws_bytes) {
    (void)comm; (void)in_dtypes; (void)out_dtype;
    if (nranks != NR || (unsigned)rank >= (unsigned)NR || n_in != 2 || in_sizes[0] != 1024 * KDIM || in_sizes[1] != KDIM * 1024 || out_size != KDIM * 1024 || ws_bytes < WS_NEED) {
        fprintf(stderr, "kernel_launch: unexpected problem geometry\n");
        return;
    }
    static bool attr_set = false;
    if (!attr_set) { HIP_CHECK(hipFuncSetAttribute((const void*)ag_gemm_kernel, hipFuncAttributeMaxDynamicSharedMemorySize, LDS_BYTES)); attr_set = true; }
    Params P;
    P.x = (const char*)d_in[0]; P.w = (const char*)d_in[1]; P.out = (char*)d_out;
    for (int q = 0; q < 4; ++q) P.outx[q] = (char*)d_out_all[rank ^ (2 * q + 1)];
    for (int d = 0; d < NR; ++d) P.wsx[d] = (char*)d_ws_all[rank ^ d];
    P.rank = rank; P.pad = 0;
    hipLaunchKernelGGL(ag_gemm_kernel, dim3(GRID), dim3(NTHR), LDS_BYTES, stream, P);
    HIP_CHECK(hipGetLastError());
}
```

```cpp
#include <hip/hip_runtime.h>
#include <rccl/rccl.h>
#include <cstdio>

#define HIP_CHECK(expr) do { hipError_t _e = (expr); if (_e != hipSuccess) { \
    fprintf(stderr, "HIP error %s at %s:%d\n", hipGetErrorString(_e), __FILE__, __LINE__); \
    return; } } while (0)

namespace {
#define LAS __attribute__((address_space(3)))
#define GAS __attribute__((address_space(1)))
typedef short bf16x8 __attribute__((ext_vector_type(8)));
typedef short s16x4 __attribute__((ext_vector_type(4)));
typedef unsigned short u16x2 __attribute__((ext_vector_type(2)));
typedef float f32x4 __attribute__((ext_vector_type(4)));
typedef unsigned u32x4 __attribute__((ext_vector_type(4)));
typedef unsigned u32x2 __attribute__((ext_vector_type(2)));

constexpr int NR = 8;
constexpr int KDIM = 8192, BK = 64, NT = KDIM / BK;
constexpr int APITCH = KDIM * 2, BPITCH = 2048, OPITCH = 2048;
constexpr int MP = 3;
constexpr int ECELL = 10, OCELL = 16 - ECELL;
constexpr int NSLOT = 3 * ECELL + 4 * OCELL;
constexpr int NCH = 64;
constexpr int RA_ITEMS = NSLOT * 2, RITEMS = RA_ITEMS + 32;
constexpr int NU = NCH * RITEMS;
constexpr int ITEM_PK = 6144;
constexpr size_t CELL_BYTES = (size_t)64 * APITCH;
constexpr size_t XA_BYTES = NSLOT * CELL_BYTES, W2_BYTES = (size_t)KDIM * BPITCH, PK_BYTES = (size_t)NU * ITEM_PK;
constexpr size_t OFF_XA = 0, OFF_W2 = XA_BYTES, OFF_PK = XA_BYTES + W2_BYTES, PAR_BYTES = OFF_PK + PK_BYTES;
constexpr size_t OFF_FLAG = 2 * PAR_BYTES, FLAG_BYTES = (size_t)NU * 128;
constexpr size_t OFF_CNT = OFF_FLAG + 2 * FLAG_BYTES, CNT_BYTES = (size_t)(NSLOT + 4) * NCH * 128;
constexpr size_t OFF_TICK = OFF_CNT + CNT_BYTES, TICK_BYTES = 2048 * 64;
constexpr size_t OFF_DONE = OFF_TICK + TICK_BYTES, OFF_TMO = OFF_DONE + 128, OFF_QUEUE = OFF_TMO + 128, OFF_DBG = OFF_QUEUE + 8 * 128, DBG_BYTES = 2048 * 64 + NU * 16 + 8 * 1280 * 8 + 256 * 64 * 8;
constexpr size_t WS_NEED = OFF_DBG + DBG_BYTES;
constexpr int GRID = 256, NTHR = 512;
#define SEND_WAVES 4
#define SEND_WGS 6
#define SEND_MAP 1
#define SEND_PIPE 0
#define OUT_THR 0
constexpr int NCOMM = 80;
constexpr int LINK_WGS = SEND_WGS + (SEND_MAP == 1 ? 1 : 0);
__host__ __device__ constexpr int nsend(int x) { return SEND_MAP == 2 ? ((x == 0 || x == 7) ? SEND_WGS / 2 : SEND_WGS) : x < 7 ? SEND_WGS : SEND_MAP == 1 ? 7 : 0; }
constexpr int NSWG = 7 * LINK_WGS, NUWG = NCOMM - NSWG, NUW = NUWG * 8;
constexpr int LITEMS = 20, NLINK = NCH * LITEMS;
constexpr unsigned DONE_PER_CALL = 80;
constexpr unsigned long long DEADLINE = 50000000ull;
constexpr int HTB = 16384, LDS_STAGE = 8 * HTB, LDS_BYTES = LDS_STAGE + 16;
static_assert(ECELL == 10 && 2 * ECELL == LITEMS && 2 * OCELL + 8 == LITEMS, "tile tables below are written for 10 + 6 cells");

struct Params {
    const char* x; const char* w;
    char* out; char* outx[4];
    char* wsx[NR];
    int rank; int pad;
};

#define WAIT_V(n) asm volatile("s_waitcnt vmcnt(" #n ")" ::: "memory")
#define WAIT_L(n) asm volatile("s_waitcnt lgkmcnt(" #n ")" ::: "memory")
#define BAR __builtin_amdgcn_s_barrier()
#define SCHED __builtin_amdgcn_sched_barrier(0)
#define STAMP(i) do {} while (0)
#define DBG_AT(off) do {} while (0)
#define TS_NOW(dst, k) do {} while (0)

__device__ __forceinline__ unsigned ld_agent(const char* p) { return __hip_atomic_load((const GAS unsigned*)p, __ATOMIC_RELAXED, __HIP_MEMORY_SCOPE_AGENT); }
__device__ __forceinline__ unsigned ld_sys(const char* p) { return __hip_atomic_load((const GAS unsigned*)p, __ATOMIC_RELAXED, __HIP_MEMORY_SCOPE_SYSTEM); }
__device__ __forceinline__ bool give_up(char* ws, unsigned call, unsigned long long t0) {
    if ((unsigned)__builtin_amdgcn_readfirstlane(ld_agent(ws + OFF_TMO)) == call || __builtin_amdgcn_s_memrealtime() - t0 > DEADLINE) {
        __hip_atomic_store((GAS unsigned*)(ws + OFF_TMO), call, __ATOMIC_RELAXED, __HIP_MEMORY_SCOPE_AGENT);
        return true;
    }
    __builtin_amdgcn_s_sleep(2);
    return false;
}
__device__ __forceinline__ void store16_sys(char* base, unsigned voff, u32x4 v) {
    asm volatile("global_store_dwordx4 %0, %1, %2 sc0 sc1\n\ts_nop 1" :: "v"(voff), "v"(v), "s"(base) : "memory");
}

struct Item { const char* src; size_t fin; unsigned voff, istep; int u; };
__device__ __forceinline__ Item item_of(const Params& P, int d, int i, int lane) {
    const int chunk = i / LITEMS, j = i - chunk * LITEMS;
    Item it;
    if ((d & 1) && j >= 2 * OCELL) { const int q = d >> 1, sub = j - 2 * OCELL; const size_t o = (size_t)(chunk * 128 + sub * 16) * BPITCH + q * 512;
        it.src = P.w + o; it.fin = OFF_W2 + o; it.voff = (lane >> 5) * BPITCH + (lane & 31) * 16; it.istep = 2 * BPITCH;
        it.u = chunk * RITEMS + RA_ITEMS + q * 8 + sub; }
    else { const int c = j >> 1, half = j & 1; const size_t o = (size_t)half * 32 * APITCH + chunk * 256;
        const int cell = (d & 1) ? ECELL + c : c, slot = (d & 1) ? 3 * ECELL + (d >> 1) * OCELL + c : ((d >> 1) - 1) * ECELL + c;
        it.src = P.x + cell * CELL_BYTES + o; it.fin = OFF_XA + slot * CELL_BYTES + o; it.voff = (lane >> 4) * APITCH + (lane & 15) * 16; it.istep = 4 * APITCH;
        it.u = chunk * RITEMS + slot * 2 + half; }
    return it;
}
__device__ __forceinline__ void sender(const Params& P, int d, int first, int nwaves, unsigned call) {
    const int lane = threadIdx.x & 63;
    char* const peer = P.wsx[d & 7];
    char* const set = peer + (call & 1) * PAR_BYTES;
    char* const flags = peer + OFF_FLAG + (call & 1) * FLAG_BYTES;
    char* const queue = P.wsx[0] + OFF_QUEUE + d * 128;
    const unsigned qbase = (call - 1u) * (NLINK + nwaves) - nwaves;
    u32x4 nxt[8];
    unsigned tick = 0;
#define FETCH(i) do { const Item _x = item_of(P, d, (i) < NLINK ? (i) : NLINK - 1, lane); \
        asm volatile("global_load_dwordx4 %0, %9, %17\n\tglobal_load_dwordx4 %1, %10, %17\n\tglobal_load_dwordx4 %2, %11, %17\n\tglobal_load_dwordx4 %3, %12, %17\n\t" \
                     "global_load_dwordx4 %4, %13, %17\n\tglobal_load_dwordx4 %5, %14, %17\n\tglobal_load_dwordx4 %6, %15, %17\n\tglobal_load_dwordx4 %7, %16, %17\n\t" \
                     "s_mov_b64 exec, 1\n\tglobal_atomic_add %8, %18, %19, %20 sc0\n\ts_mov_b64 exec, -1" \
                     : "=&v"(nxt[0]), "=&v"(nxt[1]), "=&v"(nxt[2]), "=&v"(nxt[3]), "=&v"(nxt[4]), "=&v"(nxt[5]), "=&v"(nxt[6]), "=&v"(nxt[7]), "=&v"(tick) \
                     : "v"(_x.voff), "v"(_x.voff + _x.istep), "v"(_x.voff + 2 * _x.istep), "v"(_x.voff + 3 * _x.istep), "v"(_x.voff + 4 * _x.istep), "v"(_x.voff + 5 * _x.istep), \
                       "v"(_x.voff + 6 * _x.istep), "v"(_x.voff + 7 * _x.istep), "s"(_x.src), "v"(0u), "v"(1u), "s"(queue) : "memory"); } while (0)
#define WAIT_FETCH(n) asm volatile("s_waitcnt vmcnt(" #n ")" : "+v"(tick), "+v"(nxt[0]), "+v"(nxt[1]), "+v"(nxt[2]), "+v"(nxt[3]), "+v"(nxt[4]), "+v"(nxt[5]), "+v"(nxt[6]), "+v"(nxt[7]) :: "memory")
#define PUBLISH() do { if (lane == 0) __hip_atomic_store((GAS unsigned*)(flags + (size_t)prev_u * 128), prev_flag, __ATOMIC_RELAXED, __HIP_MEMORY_SCOPE_SYSTEM); } while (0)
    int cur = first, prev_u = -1;
    unsigned prev_flag = 0;
    FETCH(cur); WAIT_FETCH(0);
    while (cur < NLINK) {
        const Item it = item_of(P, d, cur, lane);
        u32x4 raw[8];
#pragma unroll
        for (int v = 0; v < 8; ++v) raw[v] = nxt[v];
        const int next = (int)((unsigned)__builtin_amdgcn_readfirstlane(tick) - qbase);
        FETCH(next); SCHED;
        u16x2 mx = {0, 0};
#pragma unroll
        for (int v = 0; v < 8; ++v)
#pragma unroll
            for (int c = 0; c < 4; ++c) mx = __builtin_elementwise_max(mx, __builtin_bit_cast(u16x2, raw[v][c] & 0x7fff7fffu));
        unsigned m = mx.x > mx.y ? mx.x : mx.y;
#pragma unroll
        for (int o = 32; o >= 1; o >>= 1) { const unsigned t = __shfl_xor(m, o); m = t > m ? t : m; }
        const unsigned base = (unsigned)__builtin_amdgcn_readfirstlane(m) >> 7, base4 = base * 0x01010101u;
        u32x4 pk[6]; unsigned esc = 0;
#pragma unroll
        for (int v = 0; v < 8; ++v) {
            unsigned r[4];
#pragma unroll
            for (int c = 0; c < 4; ++c) r[c] = ((raw[v][c] << 1) & 0xfffefffeu) | ((raw[v][c] >> 15) & 0x00010001u);
            pk[v >> 1][(v & 1) * 2] = __builtin_amdgcn_perm(r[1], r[0], 0x06040200u);
            pk[v >> 1][(v & 1) * 2 + 1] = __builtin_amdgcn_perm(r[3], r[2], 0x06040200u);
            const unsigned d0e = base4 - __builtin_amdgcn_perm(r[1], r[0], 0x07050301u), d1e = base4 - __builtin_amdgcn_perm(r[3], r[2], 0x07050301u);
            unsigned nib = d0e | (d1e << 4);
            if (((d0e | d1e) & 0xf0f0f0f0u) != 0u) nib = 0xffffffffu;
            if (nib == 0xffffffffu) esc |= 1u << v;
            pk[4 + (v >> 2)][v & 3] = nib;
        }
        if (__builtin_amdgcn_ballot_w64(esc != 0u) != 0ull) {
#pragma unroll
            for (int v = 0; v < 8; ++v) if (esc & (1u << v)) store16_sys(set + it.fin, it.voff + v * it.istep, raw[v]);
        }
        asm volatile("global_store_dwordx4 %0, %1, %7 offset:-2048 sc0 sc1\n\tglobal_store_dwordx4 %0, %2, %7 offset:-1024 sc0 sc1\n\t"
                     "global_store_dwordx4 %0, %3, %7 sc0 sc1\n\tglobal_store_dwordx4 %0, %4, %7 offset:1024 sc0 sc1\n\t"
                     "global_store_dwordx4 %0, %5, %7 offset:2048 sc0 sc1\n\tglobal_store_dwordx4 %0, %6, %7 offset:3072 sc0 sc1\n\ts_nop 1"
                     :: "v"(lane * 16), "v"(pk[0]), "v"(pk[1]), "v"(pk[2]), "v"(pk[3]), "v"(pk[4]), "v"(pk[5]), "s"(set + OFF_PK + (size_t)it.u * ITEM_PK + 2048) : "memory");
        WAIT_FETCH(0); prev_u = it.u; prev_flag = (call << 8) | base;
        PUBLISH(); DBG_AT(NU * 16 + (d * 1280 + prev_i) * 8); prev_u = -1; cur = next; continue;
        WAIT_FETCH(6);
        if (prev_u >= 0) { PUBLISH(); DBG_AT(NU * 16 + (d * 1280 + prev_i) * 8); }
        prev_u = it.u; prev_flag = (call << 8) | base;
        cur = next;
    }
    WAIT_V(0);
    if (prev_u >= 0) { PUBLISH(); DBG_AT(NU * 16 + (d * 1280 + prev_i) * 8); }
}

__device__ __forceinline__ void unpacker(const Params& P, int uw, unsigned call) {
    const int lane = threadIdx.x & 63;
    char* ws = P.wsx[0];
    char* set = ws + (call & 1) * PAR_BYTES;
    const char* flags = ws + OFF_FLAG + (call & 1) * FLAG_BYTES;
    const unsigned voffA = (lane >> 4) * APITCH + (lane & 15) * 16, voffB = (lane >> 5) * BPITCH + (lane & 31) * 16;
    GAS unsigned* pending = nullptr;
    for (int u = uw; u < NU; u += NUW) {
        const int chunk = u / RITEMS, j = u - chunk * RITEMS;
        unsigned f = (unsigned)__builtin_amdgcn_readfirstlane(ld_sys(flags + (size_t)u * 128));
        WAIT_V(0);
        if (pending && lane == 0) __hip_atomic_fetch_add(pending, 1u, __ATOMIC_RELAXED, __HIP_MEMORY_SCOPE_AGENT);
        if (pending) DBG_AT((u - NUW) * 16 + 8);
        pending = nullptr;
        if ((f >> 8) < call) {
            const unsigned long long t0 = __builtin_amdgcn_s_memrealtime();
            bool dead = false;
            while (((f = (unsigned)__builtin_amdgcn_readfirstlane(ld_sys(flags + (size_t)u * 128))) >> 8) < call)
                if (give_up(ws, call, t0)) { dead = true; break; }
            if (dead) return;
        }
        DBG_AT(u * 16);
        u32x4 pk[6];
        asm volatile("global_load_dwordx4 %0, %6, %7 offset:-2048 sc0 sc1\n\tglobal_load_dwordx4 %1, %6, %7 offset:-1024 sc0 sc1\n\t"
                     "global_load_dwordx4 %2, %6, %7 sc0 sc1\n\tglobal_load_dwordx4 %3, %6, %7 offset:1024 sc0 sc1\n\t"
                     "global_load_dwordx4 %4, %6, %7 offset:2048 sc0 sc1\n\tglobal_load_dwordx4 %5, %6, %7 offset:3072 sc0 sc1\n\ts_waitcnt vmcnt(0)"
                     : "=&v"(pk[0]), "=&v"(pk[1]), "=&v"(pk[2]), "=&v"(pk[3]), "=&v"(pk[4]), "=&v"(pk[5])
                     : "v"(lane * 16), "s"(set + OFF_PK + (size_t)u * ITEM_PK + 2048) : "memory");
        char* fin; unsigned voff, istep; int stream;
        if (j < RA_ITEMS) { stream = j >> 1; fin = set + OFF_XA + stream * CELL_BYTES + (size_t)(j & 1) * 32 * APITCH + chunk * 256; voff = voffA; istep = 4 * APITCH; }
        else { const int q = (j - RA_ITEMS) >> 3, sub = (j - RA_ITEMS) & 7; stream = NSLOT + q;
            fin = set + OFF_W2 + (size_t)(chunk * 128 + sub * 16) * BPITCH + q * 512; voff = voffB; istep = 2 * BPITCH; }
        const unsigned base4 = (f & 255u) * 0x01010101u;
#pragma unroll
        for (int v = 0; v < 8; ++v) {
            const unsigned nib = pk[4 + (v >> 2)][v & 3], m0 = pk[v >> 1][(v & 1) * 2], m1 = pk[v >> 1][(v & 1) * 2 + 1];
            const unsigned e0 = base4 - (nib & 0x0f0f0f0fu), e1 = base4 - ((nib >> 4) & 0x0f0f0f0fu);
            u32x4 r;
            r[0] = __builtin_amdgcn_perm(e0, m0, 0x05010400u); r[1] = __builtin_amdgcn_perm(e0, m0, 0x07030602u);
            r[2] = __builtin_amdgcn_perm(e1, m1, 0x05010400u); r[3] = __builtin_amdgcn_perm(e1, m1, 0x07030602u);
#pragma unroll
            for (int c = 0; c < 4; ++c) r[c] = ((r[c] >> 1) & 0x7fff7fffu) | ((r[c] << 15) & 0x80008000u);
            if (nib != 0xffffffffu) store16_sys(fin, voff + v * istep, r);
        }
        pending = (GAS unsigned*)(ws + OFF_CNT + (size_t)(stream * NCH + chunk) * 128);
        if (u == uw) STAMP(1);
    }
    WAIT_V(0);
    if (pending && lane == 0) __hip_atomic_fetch_add(pending, 1u, __ATOMIC_RELAXED, __HIP_MEMORY_SCOPE_AGENT);
    if (pending) DBG_AT((uw + (NU - 1 - uw) / NUW * NUW) * 16 + 8);
}

__device__ __forceinline__ int lds_byte(int r, int c) { const int st = (r >> 4) * 2 + (c >> 5), rr = r & 15, cc = c & 31, ob = rr * 64 + cc * 2; return st * 1024 + (ob ^ (((ob >> 9) & 1) << 5)); }
__device__ __forceinline__ void stage_rc(int b, int& R, int& C) { const int st = b / 1024, sb = b % 1024, swz = sb ^ (((sb >> 9) & 1) << 5); R = (st >> 1) * 16 + swz / 64; C = (st & 1) * 32 + (swz % 64) / 2; }
__device__ __forceinline__ unsigned cvt_pk_bf16(float lo, float hi) { unsigned r; asm("v_cvt_pk_bf16_f32 %0, %1, %2" : "=v"(r) : "v"(lo), "v"(hi)); return r; }
template <int OFF> __device__ __forceinline__ s16x4 tr_read(unsigned addr) { s16x4 r; asm volatile("ds_read_b64_tr_b16 %0, %1 offset:%2" : "=v"(r) : "v"(addr), "i"(OFF) : "memory"); return r; }

__device__ __forceinline__ int cell_of(int rt, int p, int skipq) {
    if (rt < 14) { const int c = rt * 3 + p; return c >= 40 ? -1 : (c < ECELL ? c : 16 + c - ECELL); }
    if (rt >= 22) return ECELL + (rt - 22) * 3 + p;
    const int k = (rt - 14) >> 1, q = k + (k >= skipq ? 1 : 0);
    return 16 + 3 * ECELL + q * OCELL + ((rt - 14) & 1) * 3 + p;
}
__device__ __forceinline__ int out_row_of(int id, int rank) {
    if (id < 16) return rank * 1024 + id * 64;
    const int slot = id - 16;
    if (slot < 3 * ECELL) return (rank ^ (2 * (slot / ECELL + 1))) * 1024 + (slot % ECELL) * 64;
    const int o = slot - 3 * ECELL;
    return (rank ^ (2 * (o / OCELL) + 1)) * 1024 + (ECELL + o % OCELL) * 64;
}

__device__ __forceinline__ void gemm_tile(const Params& P, LAS unsigned char* lds, int rt, int ct, bool remote, unsigned call) {
    int tid = threadIdx.x;
    int wid = __builtin_amdgcn_readfirstlane(tid >> 6), lane = tid & 63, wr = wid >> 2, wc = wid & 3, fr = lane & 15, fq = lane >> 4;
    char* ws = P.wsx[0];
    const char* set = ws + (call & 1) * PAR_BYTES;
    const int skipq = remote ? ct : 4;
    const int id0 = cell_of(rt, 0, skipq), id1r = cell_of(rt, 1, skipq), id2r = cell_of(rt, 2, skipq);
    const int id1 = id1r < 0 ? id0 : id1r, id2 = id2r < 0 ? id0 : id2r;
    const char* A0 = id0 < 16 ? P.x + id0 * CELL_BYTES : set + OFF_XA + (id0 - 16) * CELL_BYTES;
    const char* A1 = id1 < 16 ? P.x + id1 * CELL_BYTES : set + OFF_XA + (id1 - 16) * CELL_BYTES;
    const char* A2 = id2 < 16 ? P.x + id2 * CELL_BYTES : set + OFF_XA + (id2 - 16) * CELL_BYTES;
    const char* Bp = (remote ? set + OFF_W2 : P.w) + ct * 512;
    unsigned coff, cstep, need;
    { const int id = lane == 0 ? id0 : lane == 1 ? id1 : id2;
      const bool watch = wid == 0 && (lane < 3 ? id >= 16 : (lane == 3 && remote));
      coff = watch ? (unsigned)(OFF_CNT + (lane < 3 ? id - 16 : NSLOT + ct) * NCH * 128) : (unsigned)OFF_TMO;
      cstep = watch ? 128u : 0u;
      need = watch ? call * (lane < 3 ? 2u : 8u) : 0u; }
    unsigned pre = 0;
#define SLOW_WAIT() do { const unsigned long long t0 = __builtin_amdgcn_s_memrealtime(); \
        while (__builtin_amdgcn_ballot_w64(ld_agent(ws + coff) < need) != 0ull) if (give_up(ws, call, t0)) break; \
        asm volatile("" ::: "memory"); } while (0)
#define PRE_LOAD(mask) do { coff += cstep; const unsigned long long _m = wid == 0 ? (mask) : 0ull; \
        asm volatile("s_mov_b64 exec, %3\n\tglobal_load_dword %0, %1, %2 sc1\n\ts_mov_b64 exec, -1" : "+v"(pre) : "v"(coff), "s"(ws), "s"(_m) : "memory"); } while (0)

    unsigned voffA, voffB;
    { int R, C; stage_rc(tid * 16, R, C); voffA = (unsigned)(R * APITCH + C * 2); }
    { const int b = tid * 16, k = b >> 8, ch = (b >> 4) & 15, f = (k & 3) | (((k >> 3) & 1) << 2), c = (ch >> 1) ^ f;
      voffB = (unsigned)(k * BPITCH + c * 32 + (ch & 1) * 16); }
    const unsigned ldsw = (unsigned)wid * 1024u;
    const int aoff = lds_byte(wr * 16 * MP + fr, fq * 8);
    unsigned boff0, boff1;
    { const int q = (lane & 15) >> 2, p = lane & 3, f = q | ((fq & 1) << 2);
      const unsigned base = (unsigned)(size_t)lds + 4 * HTB + (8 * fq + q) * 256 + p * 8;
      boff0 = base + (((wc * 2) ^ f) * 32); boff1 = base + (((wc * 2 + 1) ^ f) * 32); }
#define SBO(b, h) (((b) * 2 + (h)) * HTB)
#define GLDS(gptr, ldsoff) __builtin_amdgcn_global_load_lds((const unsigned*)(gptr), (LAS unsigned*)(lds + (ldsoff) + ldsw), 16, 0, 17)
#define STAGE_A_LO(b, koff) GLDS(A0 + (koff) + voffA, (b) * 2 * HTB)
#define STAGE_A_HI(b, koff) do { GLDS(A1 + (koff) + voffA, (b) * 2 * HTB + 8192); GLDS(A2 + (koff) + voffA, (b) * 2 * HTB + 16384); } while (0)
#define STAGE_B(b, h, gbase) do { GLDS((gbase) + (h) * 256 + voffB, 4 * HTB + SBO(b, h)); GLDS((gbase) + (h) * 256 + 32 * BPITCH + voffB, 4 * HTB + SBO(b, h) + 8192); } while (0)
#define LDA(dst, b, h) do { _Pragma("unroll") for (int m = 0; m < MP; ++m) _Pragma("unroll") for (int k = 0; k < 2; ++k) dst[m][k] = *(const LAS bf16x8*)(lds + (b) * 2 * HTB + (h) * MP * 4096 + aoff + m * 2048 + k * 1024); } while (0)
#define LDB1(dst, addr, off) do { const s16x4 _lo = tr_read<(off)>(addr), _hi = tr_read<(off) + 1024>(addr); dst = __builtin_shufflevector(_lo, _hi, 0, 1, 2, 3, 4, 5, 6, 7); } while (0)
#define LDB(dst, b, h) do { LDB1(dst[0][0], boff0, SBO(b, h)); LDB1(dst[0][1], boff0, SBO(b, h) + 8192); LDB1(dst[1][0], boff1, SBO(b, h)); LDB1(dst[1][1], boff1, SBO(b, h) + 8192); } while (0)
#define WAIT_LB(B) asm volatile("s_waitcnt lgkmcnt(0)" : "+v"(B[0][0]), "+v"(B[0][1]), "+v"(B[1][0]), "+v"(B[1][1]) :: "memory")
#define WAIT_PRE() asm volatile("s_waitcnt vmcnt(5)" : "+v"(pre) :: "memory")
#define MMA(ai, bj, At, Bt) do { __builtin_amdgcn_s_setprio(1); _Pragma("unroll") for (int k = 0; k < 2; ++k) _Pragma("unroll") for (int m = 0; m < MP; ++m) _Pragma("unroll") for (int n = 0; n < 2; ++n) \
        acc[ai][bj][m][n] = __builtin_amdgcn_mfma_f32_16x16x32_bf16(Bt[n][k], At[m][k], acc[ai][bj][m][n], 0, 0, 0); __builtin_amdgcn_s_setprio(0); } while (0)
    f32x4 acc[2][2][MP][2];
#pragma unroll
    for (int a = 0; a < 2; ++a)
#pragma unroll
        for (int b = 0; b < 2; ++b)
#pragma unroll
            for (int m = 0; m < MP; ++m)
#pragma unroll
                for (int n = 0; n < 2; ++n) acc[a][b][m][n] = (f32x4){0.f, 0.f, 0.f, 0.f};
    bf16x8 At[MP][2], B0[2][2], B1[2][2];
    constexpr size_t ka = BK * 2, kb = (size_t)BK * BPITCH;
    if (wid == 0) SLOW_WAIT();
    BAR;
    STAMP(1);
    PRE_LOAD(0x1full);
    STAGE_B(0, 0, Bp); STAGE_A_LO(0, 0); STAGE_A_HI(0, 0); STAGE_B(0, 1, Bp);
    if (wr == 1) BAR;
    WAIT_V(2); BAR;
    STAGE_B(1, 0, Bp + kb); STAGE_A_LO(1, ka); STAGE_B(1, 1, Bp + kb);
    WAIT_PRE(); BAR;
    for (int t = 0; t < NT; t += 2) {
        const int t2 = (t == NT - 2) ? t : t + 2;
        const size_t k1 = (size_t)(t + 1) * ka, k2 = (size_t)t2 * ka; const char* b2 = Bp + (size_t)t2 * kb;
        TS_NOW(ts0, t >> 1);
        if (t < NT - 2 && __builtin_amdgcn_ballot_w64(pre < need) != 0ull) SLOW_WAIT();
        TS_NOW(ts1, t >> 1);
        LDB(B0, 0, 0); SCHED; LDA(At, 0, 0); STAGE_A_HI(1, k1);
        WAIT_L(6); BAR; WAIT_LB(B0); MMA(0, 0, At, B0); BAR; SCHED;
        LDB(B1, 0, 1); STAGE_B(0, 0, b2);
        BAR; WAIT_LB(B1); MMA(0, 1, At, B1); BAR;
        LDA(At, 0, 1); STAGE_A_LO(0, k2);
        BAR; WAIT_L(0); MMA(1, 0, At, B0); BAR; SCHED;
        STAGE_B(0, 1, b2);
        WAIT_V(5); BAR; MMA(1, 1, At, B1); BAR;
        LDB(B0, 1, 0); SCHED; LDA(At, 1, 0); STAGE_A_HI(0, k2); PRE_LOAD(t < NT - 4 ? 0x1full : t < NT - 2 ? 0x10ull : 0ull);
        WAIT_L(6); BAR; WAIT_LB(B0); MMA(0, 0, At, B0); BAR; SCHED;
        LDB(B1, 1, 1); STAGE_B(1, 0, b2 + kb);
        BAR; WAIT_LB(B1); MMA(0, 1, At, B1); BAR;
        LDA(At, 1, 1); STAGE_A_LO(1, k2 + ka);
        BAR; WAIT_L(0); MMA(1, 0, At, B0); BAR; SCHED;
        STAGE_B(1, 1, b2 + kb);
        WAIT_PRE(); BAR; MMA(1, 1, At, B1); BAR;
    }
    if (threadIdx.x == 4) *(LAS unsigned*)(lds + LDS_STAGE) = pre;
    asm volatile("s_waitcnt vmcnt(0) lgkmcnt(0)" ::: "memory");
    if (wr == 0) BAR;
    BAR;
    asm volatile("" ::: "memory");
    const bool bad = *(const LAS unsigned*)(lds + LDS_STAGE) == call;
    STAMP(2);
    int tz = threadIdx.x; asm volatile("" : "+v"(tz));
    lane = tz & 63; fr = lane & 15; fq = lane >> 4;
#pragma unroll
    for (int ai = 0; ai < 2; ++ai)
#pragma unroll
        for (int m = 0; m < MP; ++m) {
            const int row = ai * MP * 32 + wr * MP * 16 + m * 16 + fr;
#pragma unroll
            for (int bj = 0; bj < 2; ++bj)
#pragma unroll
                for (int n = 0; n < 2; ++n) {
                    f32x4 v = acc[ai][bj][m][n];
#pragma unroll
                    for (int j = 0; j < 4; ++j) v[j] = v[j] < 0.f ? 0.f : v[j];
                    u32x2 w; w.x = cvt_pk_bf16(v[0], v[1]); w.y = cvt_pk_bf16(v[2], v[3]);
                    if (bad) { w.x = 0x7fc07fc0u; w.y = 0x7fc07fc0u; }
                    const int ch = bj * 16 + wc * 4 + n * 2 + (fq >> 1);
                    *(LAS u32x2*)(lds + row * 512 + ((ch ^ (fr & 7)) * 16) + (fq & 1) * 8) = w;
                }
        }
    WAIT_L(0); BAR; asm volatile("" ::: "memory");
    char* out = (remote ? P.outx[ct & 3] : P.out) + ct * 512;
    u32x4 ov[MP][4];
#pragma unroll
    for (int p = 0; p < MP; ++p)
#pragma unroll
        for (int i = 0; i < 4; ++i) { const int row = p * 64 + wid * 8 + i * 2 + (lane >> 5); ov[p][i] = *(const LAS u32x4*)(lds + row * 512 + (((lane & 31) ^ (row & 7)) * 16)); }
    const unsigned ooff = (wid * 8 + (lane >> 5)) * OPITCH + (lane & 31) * 16;
#pragma unroll
    for (int p = 0; p < MP; ++p) {
        const int id = p == 0 ? id0 : p == 1 ? id1r : id2r;
        if (id >= 0) {
            char* O = out + (size_t)out_row_of(id, P.rank) * OPITCH;
#pragma unroll
            for (int i = 0; i < 4; ++i) {
                store16_sys(O, ooff + i * 2 * OPITCH, ov[p][i]);
                if (remote) asm volatile("s_waitcnt vmcnt(%0)" :: "i"(OUT_THR) : "memory");
            }
        }
    }
    if (remote) {
        WAIT_V(0); BAR;
        if (threadIdx.x == 0) __hip_atomic_fetch_add((GAS unsigned*)(P.wsx[(2 * ct + 1) & 7] + OFF_DONE), 1u, __ATOMIC_RELAXED, __HIP_MEMORY_SCOPE_SYSTEM);
    } else if (rt == 22 && ct == 0 && wid == 0) {
        const unsigned long long t0 = __builtin_amdgcn_s_memrealtime();
        while ((unsigned)__builtin_amdgcn_readfirstlane(ld_sys(ws + OFF_DONE)) < call * DONE_PER_CALL)
            if (give_up(ws, call, t0)) { WAIT_V(0); if (lane == 0) *(GAS unsigned*)(out + (size_t)out_row_of(id0, P.rank) * OPITCH) = 0x7fc07fc0u; break; }
    }
    STAMP(3);
}

__global__ __launch_bounds__(NTHR, 2) void ag_gemm_kernel(const Params P) {
    extern __shared__ __attribute__((aligned(16))) unsigned char shm[];
    const int xcd = blockIdx.x & 7, slot = blockIdx.x >> 3, wid = __builtin_amdgcn_readfirstlane(threadIdx.x >> 6);
    STAMP(0);
    unsigned tick = 0;
    if ((threadIdx.x & 63) == 0) tick = __hip_atomic_fetch_add((GAS unsigned*)(P.wsx[0] + OFF_TICK + (size_t)(blockIdx.x * 8 + wid) * 64), 1u, __ATOMIC_RELAXED, __HIP_MEMORY_SCOPE_AGENT);
    const unsigned call = (unsigned)__builtin_amdgcn_readfirstlane(tick) + 1u;
    const int ncomm = xcd < 4 ? 8 : 12;
    if (slot < ncomm) {
        const bool send = slot < nsend(xcd);
        int link, swg, uwg = slot - nsend(xcd);
        for (int y = 0; y < xcd; ++y) uwg += (y < 4 ? 8 : 12) - nsend(y);
        if (SEND_MAP == 2) { const bool second = xcd == 7 || (xcd > 0 && slot >= SEND_WGS / 2); link = second ? xcd : xcd + 1; swg = (second ? SEND_WGS / 2 : 0) + slot % (SEND_WGS / 2); }
        else if (xcd == 7) { link = slot + 1; swg = SEND_WGS; }
        else { link = xcd + 1; swg = slot; }
        if (send) {
            if (wid < SEND_WAVES) sender(P, link, wid * LINK_WGS + swg, LINK_WGS * SEND_WAVES, call);
        } else {
            unpacker(P, wid * NUWG + uwg, call);
        }
        STAMP(3);
    } else {
        const int g = slot - ncomm, nrt = xcd < 4 ? 6 : 5;
        gemm_tile(P, (LAS unsigned char*)shm, (xcd & 3) * nrt + g % nrt, g / nrt, xcd >= 4, call);
    }
}
}

extern "C" void kernel_launch(int rank, int nranks, ncclComm_t comm, hipStream_t stream,
                              void* const* d_in, const int* in_sizes, int n_in,
                              void* d_out, int out_size, const int* in_dtypes, int out_dtype,
                              void* const* d_out_all, void* const* d_ws_all, size_t ws_bytes) {
    (void)comm; (void)in_dtypes; (void)out_dtype;
    if (nranks != NR || (unsigned)rank >= (unsigned)NR || n_in != 2 || in_sizes[0] != 1024 * KDIM || in_sizes[1] != KDIM * 1024 || out_size != KDIM * 1024 || ws_bytes < WS_NEED) {
        fprintf(stderr, "kernel_launch: unexpected problem geometry\n");
        return;
    }
    static bool attr_set = false;
    if (!attr_set) { HIP_CHECK(hipFuncSetAttribute((const void*)ag_gemm_kernel, hipFuncAttributeMaxDynamicSharedMemorySize, LDS_BYTES)); attr_set = true; }
    Params P;
    P.x = (const char*)d_in[0]; P.w = (const char*)d_in[1]; P.out = (char*)d_out;
    for (int q = 0; q < 4; ++q) P.outx[q] = (char*)d_out_all[rank ^ (2 * q + 1)];
    for (int d = 0; d < NR; ++d) P.wsx[d] = (char*)d_ws_all[rank ^ d];
    P.rank = rank; P.pad = 0;
    hipLaunchKernelGGL(ag_gemm_kernel, dim3(GRID), dim3(NTHR), LDS_BYTES, stream, P);
    HIP_CHECK(hipGetLastError());
}
```

```cpp
#include <hip/hip_runtime.h>
#include <rccl/rccl.h>
#include <cstdio>

#define HIP_CHECK(expr) do { hipError_t _e = (expr); if (_e != hipSuccess) { \
    fprintf(stderr, "HIP error %s at %s:%d\n", hipGetErrorString(_e), __FILE__, __LINE__); \
    return; } } while (0)

namespace {
#define LAS __attribute__((address_space(3)))
#define GAS __attribute__((address_space(1)))
typedef short bf16x8 __attribute__((ext_vector_type(8)));
typedef short s16x4 __attribute__((ext_vector_type(4)));
typedef unsigned short u16x2 __attribute__((ext_vector_type(2)));
typedef float f32x4 __attribute__((ext_vector_type(4)));
typedef unsigned u32x4 __attribute__((ext_vector_type(4)));
typedef unsigned u32x2 __attribute__((ext_vector_type(2)));

constexpr int NR = 8;
constexpr int KDIM = 8192, BK = 64, NT = KDIM / BK;
constexpr int APITCH = KDIM * 2, BPITCH = 2048, OPITCH = 2048;
constexpr int MP = 3;
constexpr int EH = 21, OH = 32 - EH;
constexpr int NSLOT = 3 * EH + 4 * OH;
constexpr int NCH = 64;
__host__ __device__ constexpr int litems(int d) { return (d & 1) ? OH + 8 : EH; }
constexpr int LSTRIDE = NCH * EH, NU = 7 * LSTRIDE;
constexpr int ITEM_PK = 6144;
constexpr size_t STRIP_BYTES = (size_t)32 * APITCH;
constexpr size_t XA_BYTES = NSLOT * STRIP_BYTES, W2_BYTES = (size_t)KDIM * BPITCH, PK_BYTES = (size_t)NU * ITEM_PK;
constexpr size_t OFF_XA = 0, OFF_W2 = XA_BYTES, OFF_PK = XA_BYTES + W2_BYTES, PAR_BYTES = OFF_PK + PK_BYTES;
constexpr size_t OFF_FLAG = 2 * PAR_BYTES, FLAG_BYTES = (size_t)NU * 128;
constexpr size_t OFF_CNT = OFF_FLAG + 2 * FLAG_BYTES, CNT_BYTES = (size_t)(NSLOT + 4) * NCH * 128;
constexpr size_t OFF_TICK = OFF_CNT + CNT_BYTES, TICK_BYTES = 2048 * 64;
constexpr size_t OFF_DONE = OFF_TICK + TICK_BYTES, OFF_TMO = OFF_DONE + 128, OFF_QUEUE = OFF_TMO + 128, OFF_DBG = OFF_QUEUE + 8 * 128, DBG_BYTES = 2048 * 64 + NU * 24 + 256 * 64 * 8;
constexpr size_t WS_NEED = OFF_DBG + DBG_BYTES;
constexpr int GRID = 256, NTHR = 512;
#define SEND_WAVES 4
#define SEND_WGS 6
#define SEND_HELP 0
#define OUT_THR 0
__host__ __device__ constexpr int ncomm(int x) { return x < 4 ? 8 : 12; }
__host__ __device__ constexpr int nsend(int x) { return x != 3 ? SEND_WGS : SEND_HELP ? 7 : 0; }
constexpr int LINK_WGS = SEND_WGS + SEND_HELP, NUWG = 80 - 7 * LINK_WGS, NUW = NUWG * 8;
constexpr unsigned DONE_PER_CALL = 80;
constexpr unsigned long long DEADLINE = 50000000ull;
constexpr int HTB = 16384, LDS_STAGE = 8 * HTB, LDS_BYTES = LDS_STAGE + 16;
static_assert(EH + 3 * OH == 54 && EH + 3 * OH + 3 * EH <= 120 && 32 + NSLOT <= 144, "tile lists: 9 + 11 remote tiles per quarter, 24 local tiles per column block");

struct Params {
    const char* x; const char* w;
    char* out; char* outx[4];
    char* wsx[NR];
    int rank; int pad;
};

#define WAIT_V(n) asm volatile("s_waitcnt vmcnt(" #n ")" ::: "memory")
#define WAIT_L(n) asm volatile("s_waitcnt lgkmcnt(" #n ")" ::: "memory")
#define BAR __builtin_amdgcn_s_barrier()
#define SCHED __builtin_amdgcn_sched_barrier(0)
#define STAMP(i) do {} while (0)
#define DBG_AT(off) do {} while (0)
#define TS_NOW(dst, k) do {} while (0)

__device__ __forceinline__ unsigned ld_agent(const char* p) { return __hip_atomic_load((const GAS unsigned*)p, __ATOMIC_RELAXED, __HIP_MEMORY_SCOPE_AGENT); }
__device__ __forceinline__ unsigned ld_sys(const char* p) { return __hip_atomic_load((const GAS unsigned*)p, __ATOMIC_RELAXED, __HIP_MEMORY_SCOPE_SYSTEM); }
__device__ __forceinline__ bool give_up(char* ws, unsigned call, unsigned long long t0) {
    if ((unsigned)__builtin_amdgcn_readfirstlane(ld_agent(ws + OFF_TMO)) == call || __builtin_amdgcn_s_memrealtime() - t0 > DEADLINE) {
        __hip_atomic_store((GAS unsigned*)(ws + OFF_TMO), call, __ATOMIC_RELAXED, __HIP_MEMORY_SCOPE_AGENT);
        return true;
    }
    __builtin_amdgcn_s_sleep(2);
    return false;
}
__device__ __forceinline__ void store16_sys(char* base, unsigned voff, u32x4 v) {
    asm volatile("global_store_dwordx4 %0, %1, %2 sc0 sc1\n\ts_nop 1" :: "v"(voff), "v"(v), "s"(base) : "memory");
}

struct Item { bool isw; unsigned src, fin, cnt, voff, istep; };
__device__ __forceinline__ Item item_of(int d, int i, int lane) {
    const int chunk = (d & 1) ? i / litems(1) : i / litems(0), j = i - chunk * litems(d);
    Item it;
    it.isw = (d & 1) && j >= OH;
    if (it.isw) { it.src = (unsigned)((chunk * 128 + (j - OH) * 16) * BPITCH + (d >> 1) * 512); it.fin = (unsigned)OFF_W2 + it.src;
        it.cnt = (unsigned)(OFF_CNT + ((NSLOT + (d >> 1)) * NCH + chunk) * 128); it.voff = (lane >> 5) * BPITCH + (lane & 31) * 16; it.istep = 2 * BPITCH; }
    else { const int strip = (d & 1) ? EH + j : j, slot = (d & 1) ? 3 * EH + (d >> 1) * OH + j : ((d >> 1) - 1) * EH + j;
        it.src = (unsigned)(strip * STRIP_BYTES + chunk * 256); it.fin = (unsigned)(OFF_XA + slot * STRIP_BYTES + chunk * 256);
        it.cnt = (unsigned)(OFF_CNT + (slot * NCH + chunk) * 128); it.voff = (lane >> 4) * APITCH + (lane & 15) * 16; it.istep = 4 * APITCH; }
    return it;
}
__device__ __forceinline__ void sender(const Params& P, int d, int first, char* ticket) {
    const int lane = threadIdx.x & 63, nlink = NCH * litems(d), nwaves = LINK_WGS * SEND_WAVES;
    char* const queue = P.wsx[0] + OFF_QUEUE + d * 128;
    const char* const src_x = P.x; const char* const src_w = P.w;
    u32x4 nxt[8];
    unsigned tick = 0, ctick = 0;
#define FETCH(i) do { const Item _x = item_of(d, (i) < nlink ? (i) : nlink - 1, lane); \
        asm volatile("global_load_dwordx4 %0, %9, %17\n\tglobal_load_dwordx4 %1, %10, %17\n\tglobal_load_dwordx4 %2, %11, %17\n\tglobal_load_dwordx4 %3, %12, %17\n\t" \
                     "global_load_dwordx4 %4, %13, %17\n\tglobal_load_dwordx4 %5, %14, %17\n\tglobal_load_dwordx4 %6, %15, %17\n\tglobal_load_dwordx4 %7, %16, %17\n\t" \
                     "s_mov_b64 exec, 1\n\tglobal_atomic_add %8, %18, %19, %20 sc0\n\ts_mov_b64 exec, -1" \
                     : "=&v"(nxt[0]), "=&v"(nxt[1]), "=&v"(nxt[2]), "=&v"(nxt[3]), "=&v"(nxt[4]), "=&v"(nxt[5]), "=&v"(nxt[6]), "=&v"(nxt[7]), "=&v"(tick) \
                     : "v"(_x.voff), "v"(_x.voff + _x.istep), "v"(_x.voff + 2 * _x.istep), "v"(_x.voff + 3 * _x.istep), "v"(_x.voff + 4 * _x.istep), "v"(_x.voff + 5 * _x.istep), \
                       "v"(_x.voff + 6 * _x.istep), "v"(_x.voff + 7 * _x.istep), "s"((_x.isw ? src_w : src_x) + _x.src), "v"(0u), "v"(1u), "s"(queue) : "memory"); } while (0)
#define WAIT_FETCH() asm volatile("s_waitcnt vmcnt(0)" : "+v"(ctick), "+v"(tick), "+v"(nxt[0]), "+v"(nxt[1]), "+v"(nxt[2]), "+v"(nxt[3]), "+v"(nxt[4]), "+v"(nxt[5]), "+v"(nxt[6]), "+v"(nxt[7]) :: "memory")
    int cur = first;
    asm volatile("s_mov_b64 exec, 1\n\tglobal_atomic_add %0, %1, %2, %3 sc0\n\ts_mov_b64 exec, -1" : "=&v"(ctick) : "v"(0u), "v"(1u), "s"(ticket) : "memory");
    FETCH(cur); WAIT_FETCH();
    const unsigned call = (unsigned)__builtin_amdgcn_readfirstlane(ctick) + 1u;
    char* const peer = P.wsx[d & 7];
    char* const set = peer + (call & 1) * PAR_BYTES;
    char* const flags = peer + OFF_FLAG + (call & 1) * FLAG_BYTES;
    const unsigned qbase = (call - 1u) * (nlink + nwaves) - nwaves;
    while (cur < nlink) {
        const Item it = item_of(d, cur, lane);
        const int u = (d - 1) * LSTRIDE + cur;
        u32x4 raw[8];
#pragma unroll
        for (int v = 0; v < 8; ++v) raw[v] = nxt[v];
        const int next = (int)((unsigned)__builtin_amdgcn_readfirstlane(tick) - qbase);
        FETCH(next); SCHED;
        u16x2 mx = {0, 0};
#pragma unroll
        for (int v = 0; v < 8; ++v)
#pragma unroll
            for (int c = 0; c < 4; ++c) mx = __builtin_elementwise_max(mx, __builtin_bit_cast(u16x2, raw[v][c] & 0x7fff7fffu));
        unsigned m = mx.x > mx.y ? mx.x : mx.y;
#pragma unroll
        for (int o = 32; o >= 1; o >>= 1) { const unsigned t = __shfl_xor(m, o); m = t > m ? t : m; }
        const unsigned base = (unsigned)__builtin_amdgcn_readfirstlane(m) >> 7, base4 = base * 0x01010101u;
        u32x4 pk[6]; unsigned esc = 0;
#pragma unroll
        for (int v = 0; v < 8; ++v) {
            unsigned r[4];
#pragma unroll
            for (int c = 0; c < 4; ++c) r[c] = ((raw[v][c] << 1) & 0xfffefffeu) | ((raw[v][c] >> 15) & 0x00010001u);
            pk[v >> 1][(v & 1) * 2] = __builtin_amdgcn_perm(r[1], r[0], 0x06040200u);
            pk[v >> 1][(v & 1) * 2 + 1] = __builtin_amdgcn_perm(r[3], r[2], 0x06040200u);
            const unsigned d0e = base4 - __builtin_amdgcn_perm(r[1], r[0], 0x07050301u), d1e = base4 - __builtin_amdgcn_perm(r[3], r[2], 0x07050301u);
            unsigned nib = d0e | (d1e << 4);
            if (((d0e | d1e) & 0xf0f0f0f0u) != 0u) nib = 0xffffffffu;
            if (nib == 0xffffffffu) esc |= 1u << v;
            pk[4 + (v >> 2)][v & 3] = nib;
        }
        if (__builtin_amdgcn_ballot_w64(esc != 0u) != 0ull) {
#pragma unroll
            for (int v = 0; v < 8; ++v) if (esc & (1u << v)) store16_sys(set + it.fin, it.voff + v * it.istep, raw[v]);
        }
        asm volatile("global_store_dwordx4 %0, %1, %7 offset:-2048 sc0 sc1\n\tglobal_store_dwordx4 %0, %2, %7 offset:-1024 sc0 sc1\n\t"
                     "global_store_dwordx4 %0, %3, %7 sc0 sc1\n\tglobal_store_dwordx4 %0, %4, %7 offset:1024 sc0 sc1\n\t"
                     "global_store_dwordx4 %0, %5, %7 offset:2048 sc0 sc1\n\tglobal_store_dwordx4 %0, %6, %7 offset:3072 sc0 sc1\n\ts_nop 1"
                     :: "v"(lane * 16), "v"(pk[0]), "v"(pk[1]), "v"(pk[2]), "v"(pk[3]), "v"(pk[4]), "v"(pk[5]), "s"(set + OFF_PK + (size_t)u * ITEM_PK + 2048) : "memory");
        WAIT_FETCH();
        if (lane == 0) __hip_atomic_store((GAS unsigned*)(flags + (size_t)u * 128), (call << 8) | base, __ATOMIC_RELAXED, __HIP_MEMORY_SCOPE_SYSTEM);
        DBG_AT(u * 24 + 16);
        cur = next;
    }
}

__device__ __forceinline__ void unpacker(const Params& P, int uw, unsigned call) {
    const int lane = threadIdx.x & 63;
    const int d = uw % 7 + 1, step = (NUW - uw % 7 + 6) / 7, nlink = NCH * litems(d);
    char* ws = P.wsx[0];
    char* set = ws + (call & 1) * PAR_BYTES;
    const char* flags = ws + OFF_FLAG + (call & 1) * FLAG_BYTES;
    GAS unsigned* pending = nullptr;
    for (int i = uw / 7; i < nlink; i += step) {
        const int u = (d - 1) * LSTRIDE + i;
        unsigned f = (unsigned)__builtin_amdgcn_readfirstlane(ld_sys(flags + (size_t)u * 128));
        WAIT_V(0);
        if (pending && lane == 0) __hip_atomic_fetch_add(pending, 1u, __ATOMIC_RELAXED, __HIP_MEMORY_SCOPE_AGENT);
        if (pending) DBG_AT((u - step) * 24 + 8);
        pending = nullptr;
        if ((f >> 8) < call) {
            const unsigned long long t0 = __builtin_amdgcn_s_memrealtime();
            bool dead = false;
            while (((f = (unsigned)__builtin_amdgcn_readfirstlane(ld_sys(flags + (size_t)u * 128))) >> 8) < call)
                if (give_up(ws, call, t0)) { dead = true; break; }
            if (dead) return;
        }
        DBG_AT(u * 24);
        u32x4 pk[6];
        asm volatile("global_load_dwordx4 %0, %6, %7 offset:-2048 sc0 sc1\n\tglobal_load_dwordx4 %1, %6, %7 offset:-1024 sc0 sc1\n\t"
                     "global_load_dwordx4 %2, %6, %7 sc0 sc1\n\tglobal_load_dwordx4 %3, %6, %7 offset:1024 sc0 sc1\n\t"
                     "global_load_dwordx4 %4, %6, %7 offset:2048 sc0 sc1\n\tglobal_load_dwordx4 %5, %6, %7 offset:3072 sc0 sc1\n\ts_waitcnt vmcnt(0)"
                     : "=&v"(pk[0]), "=&v"(pk[1]), "=&v"(pk[2]), "=&v"(pk[3]), "=&v"(pk[4]), "=&v"(pk[5])
                     : "v"(lane * 16), "s"(set + OFF_PK + (size_t)u * ITEM_PK + 2048) : "memory");
        const Item it = item_of(d, i, lane);
        const unsigned base4 = (f & 255u) * 0x01010101u;
#pragma unroll
        for (int v = 0; v < 8; ++v) {
            const unsigned nib = pk[4 + (v >> 2)][v & 3], m0 = pk[v >> 1][(v & 1) * 2], m1 = pk[v >> 1][(v & 1) * 2 + 1];
            const unsigned e0 = base4 - (nib & 0x0f0f0f0fu), e1 = base4 - ((nib >> 4) & 0x0f0f0f0fu);
            u32x4 r;
            r[0] = __builtin_amdgcn_perm(e0, m0, 0x05010400u); r[1] = __builtin_amdgcn_perm(e0, m0, 0x07030602u);
            r[2] = __builtin_amdgcn_perm(e1, m1, 0x05010400u); r[3] = __builtin_amdgcn_perm(e1, m1, 0x07030602u);
#pragma unroll
            for (int c = 0; c < 4; ++c) r[c] = ((r[c] >> 1) & 0x7fff7fffu) | ((r[c] << 15) & 0x80008000u);
            if (nib != 0xffffffffu) store16_sys(set + it.fin, it.voff + v * it.istep, r);
        }
        pending = (GAS unsigned*)(ws + it.cnt);
        if (i == uw / 7) STAMP(1);
    }
    WAIT_V(0);
    if (pending && lane == 0) __hip_atomic_fetch_add(pending, 1u, __ATOMIC_RELAXED, __HIP_MEMORY_SCOPE_AGENT);
    if (pending) DBG_AT(((d - 1) * LSTRIDE + uw / 7 + (nlink - 1 - uw / 7) / step * step) * 24 + 8);
}

__device__ __forceinline__ int lds_byte(int r, int c) { const int st = (r >> 4) * 2 + (c >> 5), rr = r & 15, cc = c & 31, ob = rr * 64 + cc * 2; return st * 1024 + (ob ^ (((ob >> 9) & 1) << 5)); }
__device__ __forceinline__ void stage_rc(int b, int& R, int& C) { const int st = b / 1024, sb = b % 1024, swz = sb ^ (((sb >> 9) & 1) << 5); R = (st >> 1) * 16 + swz / 64; C = (st & 1) * 32 + (swz % 64) / 2; }
__device__ __forceinline__ unsigned cvt_pk_bf16(float lo, float hi) { unsigned r; asm("v_cvt_pk_bf16_f32 %0, %1, %2" : "=v"(r) : "v"(lo), "v"(hi)); return r; }
template <int OFF> __device__ __forceinline__ s16x4 tr_read(unsigned addr) { s16x4 r; asm volatile("ds_read_b64_tr_b16 %0, %1 offset:%2" : "=v"(r) : "v"(addr), "i"(OFF) : "memory"); return r; }

__device__ __forceinline__ int strip_of(int pos, bool remote, int q) {
    if (!remote) return pos < 32 + NSLOT ? pos : -1;
    if (pos < EH) return pos;
    if (pos < EH + 3 * OH) { const int k = (pos - EH) / OH; return 32 + 3 * EH + (k + (k >= q ? 1 : 0)) * OH + (pos - EH) % OH; }
    return pos < 4 * EH + 3 * OH ? 32 + pos - (EH + 3 * OH) : -1;
}
__device__ __forceinline__ int out_row_of(int id, int rank) {
    if (id < 32) return rank * 1024 + id * 32;
    const int slot = id - 32;
    if (slot < 3 * EH) return (rank ^ (2 * (slot / EH + 1))) * 1024 + (slot % EH) * 32;
    const int o = slot - 3 * EH;
    return (rank ^ (2 * (o / OH) + 1)) * 1024 + (EH + o % OH) * 32;
}

__device__ __forceinline__ void gemm_tile(const Params& P, LAS unsigned char* lds, int rt, int ct, bool remote, unsigned call) {
    int tid = threadIdx.x;
    int wid = __builtin_amdgcn_readfirstlane(tid >> 6), lane = tid & 63, wr = wid >> 2, wc = wid & 3, fr = lane & 15, fq = lane >> 4;
    char* ws = P.wsx[0];
    const char* set = ws + (call & 1) * PAR_BYTES;
    const int idf = strip_of(rt * 6, remote, ct);
    const int id0 = strip_of(rt * 6 + wr, remote, ct), id1 = strip_of(rt * 6 + 2 + wr, remote, ct), id2 = strip_of(rt * 6 + 4 + wr, remote, ct);
#define STRIP_PTR(i_) ((((i_) < 0 ? idf : (i_)) < 32 ? P.x + ((i_) < 0 ? idf : (i_)) * STRIP_BYTES : set + OFF_XA + (((i_) < 0 ? idf : (i_)) - 32) * STRIP_BYTES) - wr * STRIP_BYTES)
    const char* A0 = STRIP_PTR(id0); const char* A1 = STRIP_PTR(id1); const char* A2 = STRIP_PTR(id2);
    const char* Bp = (remote ? set + OFF_W2 : P.w) + ct * 512;
    unsigned coff, cstep, need;
    { const int id = strip_of(rt * 6 + (lane < 6 ? lane : 0), remote, ct);
      const bool watch = wid == 0 && (lane < 6 ? id >= 32 : (lane == 6 && remote));
      coff = watch ? (unsigned)(OFF_CNT + (lane < 6 ? id - 32 : NSLOT + ct) * NCH * 128) : (unsigned)OFF_TMO;
      cstep = watch ? 128u : 0u;
      need = watch ? call * (lane < 6 ? 1u : 8u) : 0u; }
    unsigned pre = 0;
#define SLOW_WAIT() do { const unsigned long long t0 = __builtin_amdgcn_s_memrealtime(); \
        while (__builtin_amdgcn_ballot_w64(ld_agent(ws + coff) < need) != 0ull) if (give_up(ws, call, t0)) break; \
        asm volatile("" ::: "memory"); } while (0)
#define PRE_LOAD(mask) do { coff += cstep; const unsigned long long _m = wid == 0 ? (mask) : 0ull; \
        asm volatile("s_mov_b64 exec, %3\n\tglobal_load_dword %0, %1, %2 sc1\n\ts_mov_b64 exec, -1" : "+v"(pre) : "v"(coff), "s"(ws), "s"(_m) : "memory"); } while (0)

    unsigned voffA, voffB;
    { int R, C; stage_rc(tid * 16, R, C); voffA = (unsigned)(R * APITCH + C * 2); }
    { const int b = tid * 16, k = b >> 8, ch = (b >> 4) & 15, f = (k & 3) | (((k >> 3) & 1) << 2), c = (ch >> 1) ^ f;
      voffB = (unsigned)(k * BPITCH + c * 32 + (ch & 1) * 16); }
    const unsigned ldsw = (unsigned)wid * 1024u;
    const int aoff = lds_byte(wr * 16 * MP + fr, fq * 8);
    unsigned boff0, boff1;
    { const int q = (lane & 15) >> 2, p = lane & 3, f = q | ((fq & 1) << 2);
      const unsigned base = (unsigned)(size_t)lds + 4 * HTB + (8 * fq + q) * 256 + p * 8;
      boff0 = base + (((wc * 2) ^ f) * 32); boff1 = base + (((wc * 2 + 1) ^ f) * 32); }
#define SBO(b, h) (((b) * 2 + (h)) * HTB)
#define GLDS(gptr, ldsoff) __builtin_amdgcn_global_load_lds((const unsigned*)(gptr), (LAS unsigned*)(lds + (ldsoff) + ldsw), 16, 0, 17)
#define STAGE_A_LO(b, koff) GLDS(A0 + (koff) + voffA, (b) * 2 * HTB)
#define STAGE_A_HI(b, koff) do { GLDS(A1 + (koff) + voffA, (b) * 2 * HTB + 8192); GLDS(A2 + (koff) + voffA, (b) * 2 * HTB + 16384); } while (0)
#define STAGE_B(b, h, gbase) do { GLDS((gbase) + (h) * 256 + voffB, 4 * HTB + SBO(b, h)); GLDS((gbase) + (h) * 256 + 32 * BPITCH + voffB, 4 * HTB + SBO(b, h) + 8192); } while (0)
#define LDA(dst, b, h) do { _Pragma("unroll") for (int m = 0; m < MP; ++m) _Pragma("unroll") for (int k = 0; k < 2; ++k) dst[m][k] = *(const LAS bf16x8*)(lds + (b) * 2 * HTB + (h) * MP * 4096 + aoff + m * 2048 + k * 1024); } while (0)
#define LDB1(dst, addr, off) do { const s16x4 _lo = tr_read<(off)>(addr), _hi = tr_read<(off) + 1024>(addr); dst = __builtin_shufflevector(_lo, _hi, 0, 1, 2, 3, 4, 5, 6, 7); } while (0)
#define LDB(dst, b, h) do { LDB1(dst[0][0], boff0, SBO(b, h)); LDB1(dst[0][1], boff0, SBO(b, h) + 8192); LDB1(dst[1][0], boff1, SBO(b, h)); LDB1(dst[1][1], boff1, SBO(b, h) + 8192); } while (0)
#define WAIT_LB(B) asm volatile("s_waitcnt lgkmcnt(0)" : "+v"(B[0][0]), "+v"(B[0][1]), "+v"(B[1][0]), "+v"(B[1][1]) :: "memory")
#define WAIT_PRE() asm volatile("s_waitcnt vmcnt(5)" : "+v"(pre) :: "memory")
#define MMA(ai, bj, At, Bt) do { __builtin_amdgcn_s_setprio(1); _Pragma("unroll") for (int k = 0; k < 2; ++k) _Pragma("unroll") for (int m = 0; m < MP; ++m) _Pragma("unroll") for (int n = 0; n < 2; ++n) \
        acc[ai][bj][m][n] = __builtin_amdgcn_mfma_f32_16x16x32_bf16(Bt[n][k], At[m][k], acc[ai][bj][m][n], 0, 0, 0); __builtin_amdgcn_s_setprio(0); } while (0)
    f32x4 acc[2][2][MP][2];
#pragma unroll
    for (int a = 0; a < 2; ++a)
#pragma unroll
        for (int b = 0; b < 2; ++b)
#pragma unroll
            for (int m = 0; m < MP; ++m)
#pragma unroll
                for (int n = 0; n < 2; ++n) acc[a][b][m][n] = (f32x4){0.f, 0.f, 0.f, 0.f};
    bf16x8 At[MP][2], B0[2][2], B1[2][2];
    constexpr size_t ka = BK * 2, kb = (size_t)BK * BPITCH;
    if (wid == 0) SLOW_WAIT();
    BAR;
    STAMP(1);
    PRE_LOAD(0xffull);
    STAGE_B(0, 0, Bp); STAGE_A_LO(0, 0); STAGE_A_HI(0, 0); STAGE_B(0, 1, Bp);
    if (wr == 1) BAR;
    WAIT_V(2); BAR;
    STAGE_B(1, 0, Bp + kb); STAGE_A_LO(1, ka); STAGE_B(1, 1, Bp + kb);
    WAIT_PRE(); BAR;
    for (int t = 0; t < NT; t += 2) {
        const int t2 = (t == NT - 2) ? t : t + 2;
        const size_t k1 = (size_t)(t + 1) * ka, k2 = (size_t)t2 * ka; const char* b2 = Bp + (size_t)t2 * kb;
        TS_NOW(ts0, t >> 1);
        if (t < NT - 2 && __builtin_amdgcn_ballot_w64(pre < need) != 0ull) SLOW_WAIT();
        TS_NOW(ts1, t >> 1);
        LDB(B0, 0, 0); SCHED; LDA(At, 0, 0); STAGE_A_HI(1, k1);
        WAIT_L(6); BAR; WAIT_LB(B0); MMA(0, 0, At, B0); BAR; SCHED;
        LDB(B1, 0, 1); STAGE_B(0, 0, b2);
        BAR; WAIT_LB(B1); MMA(0, 1, At, B1); BAR;
        LDA(At, 0, 1); STAGE_A_LO(0, k2);
        BAR; WAIT_L(0); MMA(1, 0, At, B0); BAR; SCHED;
        STAGE_B(0, 1, b2);
        WAIT_V(5); BAR; MMA(1, 1, At, B1); BAR;
        LDB(B0, 1, 0); SCHED; LDA(At, 1, 0); STAGE_A_HI(0, k2); PRE_LOAD(t < NT - 4 ? 0xffull : t < NT - 2 ? 0x80ull : 0ull);
        WAIT_L(6); BAR; WAIT_LB(B0); MMA(0, 0, At, B0); BAR; SCHED;
        LDB(B1, 1, 1); STAGE_B(1, 0, b2 + kb);
        BAR; WAIT_LB(B1); MMA(0, 1, At, B1); BAR;
        LDA(At, 1, 1); STAGE_A_LO(1, k2 + ka);
        BAR; WAIT_L(0); MMA(1, 0, At, B0); BAR; SCHED;
        STAGE_B(1, 1, b2 + kb);
        WAIT_PRE(); BAR; MMA(1, 1, At, B1); BAR;
    }
    if (threadIdx.x == 7) *(LAS unsigned*)(lds + LDS_STAGE) = pre;
    asm volatile("s_waitcnt vmcnt(0) lgkmcnt(0)" ::: "memory");
    if (wr == 0) BAR;
    BAR;
    asm volatile("" ::: "memory");
    const bool bad = *(const LAS unsigned*)(lds + LDS_STAGE) == call;
    STAMP(2);
    int tz = threadIdx.x; asm volatile("" : "+v"(tz));
    lane = tz & 63; fr = lane & 15; fq = lane >> 4;
#pragma unroll
    for (int ai = 0; ai < 2; ++ai)
#pragma unroll
        for (int m = 0; m < MP; ++m) {
            const int row = ai * MP * 32 + wr * MP * 16 + m * 16 + fr;
#pragma unroll
            for (int bj = 0; bj < 2; ++bj)
#pragma unroll
                for (int n = 0; n < 2; ++n) {
                    f32x4 v = acc[ai][bj][m][n];
#pragma unroll
                    for (int j = 0; j < 4; ++j) v[j] = v[j] < 0.f ? 0.f : v[j];
                    u32x2 w; w.x = cvt_pk_bf16(v[0], v[1]); w.y = cvt_pk_bf16(v[2], v[3]);
                    if (bad) { w.x = 0x7fc07fc0u; w.y = 0x7fc07fc0u; }
                    const int ch = bj * 16 + wc * 4 + n * 2 + (fq >> 1);
                    *(LAS u32x2*)(lds + row * 512 + ((ch ^ (fr & 7)) * 16) + (fq & 1) * 8) = w;
                }
        }
    WAIT_L(0); BAR; asm volatile("" ::: "memory");
    char* out = (remote ? P.outx[ct & 3] : P.out) + ct * 512;
    u32x4 ov[MP][4];
#pragma unroll
    for (int p = 0; p < MP; ++p)
#pragma unroll
        for (int i = 0; i < 4; ++i) { const int row = p * 64 + wid * 8 + i * 2 + (lane >> 5); ov[p][i] = *(const LAS u32x4*)(lds + row * 512 + (((lane & 31) ^ (row & 7)) * 16)); }
    const unsigned ooff = (wc * 8 + (lane >> 5)) * OPITCH + (lane & 31) * 16;
#pragma unroll
    for (int p = 0; p < MP; ++p) {
        const int id = p == 0 ? id0 : p == 1 ? id1 : id2;
        if (id >= 0) {
            char* O = out + (size_t)out_row_of(id, P.rank) * OPITCH;
#pragma unroll
            for (int i = 0; i < 4; ++i) {
                store16_sys(O, ooff + i * 2 * OPITCH, ov[p][i]);
                if (remote) asm volatile("s_waitcnt vmcnt(%0)" :: "i"(OUT_THR) : "memory");
            }
        }
    }
    if (remote) {
        WAIT_V(0); BAR;
        if (threadIdx.x == 0) __hip_atomic_fetch_add((GAS unsigned*)(P.wsx[(2 * ct + 1) & 7] + OFF_DONE), 1u, __ATOMIC_RELAXED, __HIP_MEMORY_SCOPE_SYSTEM);
    } else if (rt == 0 && ct == 0 && wid == 0) {
        const unsigned long long t0 = __builtin_amdgcn_s_memrealtime();
        while ((unsigned)__builtin_amdgcn_readfirstlane(ld_sys(ws + OFF_DONE)) < call * DONE_PER_CALL)
            if (give_up(ws, call, t0)) { WAIT_V(0); if (lane == 0) *(GAS unsigned*)(out + (size_t)out_row_of(id0, P.rank) * OPITCH) = 0x7fc07fc0u; break; }
    }
    STAMP(3);
}

__global__ __launch_bounds__(NTHR, 2) void ag_gemm_kernel(const Params P) {
    extern __shared__ __attribute__((aligned(16))) unsigned char shm[];
    const int xcd = blockIdx.x & 7, slot = blockIdx.x >> 3, wid = __builtin_amdgcn_readfirstlane(threadIdx.x >> 6);
    char* const ticket = P.wsx[0] + OFF_TICK + (size_t)(blockIdx.x * 8 + wid) * 64;
    STAMP(0);
    if (slot < nsend(xcd)) {
        if (wid < SEND_WAVES) sender(P, xcd == 3 ? slot + 1 : xcd < 3 ? 2 * xcd + 2 : 2 * xcd - 7, wid * LINK_WGS + (xcd == 3 ? SEND_WGS : slot), ticket);
        STAMP(3);
        return;
    }
    unsigned tick = 0;
    if ((threadIdx.x & 63) == 0) tick = __hip_atomic_fetch_add((GAS unsigned*)ticket, 1u, __ATOMIC_RELAXED, __HIP_MEMORY_SCOPE_AGENT);
    const unsigned call = (unsigned)__builtin_amdgcn_readfirstlane(tick) + 1u;
    if (slot < ncomm(xcd)) {
        int uwg = slot - nsend(xcd);
        for (int y = 0; y < xcd; ++y) uwg += ncomm(y) - nsend(y);
        unpacker(P, wid * NUWG + uwg, call);
        STAMP(3);
    } else {
        const int g = slot - ncomm(xcd);
        gemm_tile(P, (LAS unsigned char*)shm, xcd < 4 ? xcd * 6 + g % 6 : g, xcd < 4 ? g / 6 : xcd - 4, xcd >= 4, call);
    }
}
}

extern "C" void kernel_launch(int rank, int nranks, ncclComm_t comm, hipStream_t stream,
                              void* const* d_in, const int* in_sizes, int n_in,
                              void* d_out, int out_size, const int* in_dtypes, int out_dtype,
                              void* const* d_out_all, void* const* d_ws_all, size_t ws_bytes) {
    (void)comm; (void)in_dtypes; (void)out_dtype;
    if (nranks != NR || (unsigned)rank >= (unsigned)NR || n_in != 2 || in_sizes[0] != 1024 * KDIM || in_sizes[1] != KDIM * 1024 || out_size != KDIM * 1024 || ws_bytes < WS_NEED) {
        fprintf(stderr, "kernel_launch: unexpected problem geometry\n");
        return;
    }
    static bool attr_set = false;
    if (!attr_set) { HIP_CHECK(hipFuncSetAttribute((const void*)ag_gemm_kernel, hipFuncAttributeMaxDynamicSharedMemorySize, LDS_BYTES)); attr_set = true; }
    Params P;
    P.x = (const char*)d_in[0]; P.w = (const char*)d_in[1]; P.out = (char*)d_out;
    for (int q = 0; q < 4; ++q) P.outx[q] = (char*)d_out_all[rank ^ (2 * q + 1)];
    for (int d = 0; d < NR; ++d) P.wsx[d] = (char*)d_ws_all[rank ^ d];
    P.rank = rank; P.pad = 0;
    hipLaunchKernelGGL(ag_gemm_kernel, dim3(GRID), dim3(NTHR), LDS_BYTES, stream, P);
    HIP_CHECK(hipGetLastError());
}
```

```cpp
#include <hip/hip_runtime.h>
#include <rccl/rccl.h>
#include <cstdio>

#define HIP_CHECK(expr) do { hipError_t _e = (expr); if (_e != hipSuccess) { \
    fprintf(stderr, "HIP error %s at %s:%d\n", hipGetErrorString(_e), __FILE__, __LINE__); \
    return; } } while (0)

namespace {
#define LAS __attribute__((address_space(3)))
#define GAS __attribute__((address_space(1)))
typedef short bf16x8 __attribute__((ext_vector_type(8)));
typedef short s16x4 __attribute__((ext_vector_type(4)));
typedef unsigned short u16x2 __attribute__((ext_vector_type(2)));
typedef float f32x4 __attribute__((ext_vector_type(4)));
typedef unsigned u32x4 __attribute__((ext_vector_type(4)));
typedef unsigned u32x2 __attribute__((ext_vector_type(2)));

constexpr int NR = 8;
constexpr int KDIM = 8192;
constexpr int APITCH = KDIM * 2, BPITCH = 2048, OPITCH = 2048;
constexpr int EH = 21, OH = 32 - EH;
constexpr int NSLOT = 3 * EH + 4 * OH;
#define SL 7
#define SR 5
constexpr int NTL = (32 + NSLOT + SL - 1) / SL, NTE = (EH + 3 * OH + SR - 1) / SR, NTR = NTE + (3 * EH + SR - 1) / SR, HPT = (SR + 1) / 2;
constexpr int NCH = 64;
__host__ __device__ constexpr int litems(int d) { return (d & 1) ? OH + 8 : EH; }
__host__ __device__ constexpr int lbase(int d) { return NCH * ((d >> 1) * litems(1) + ((d - 1) >> 1) * litems(0)); }
constexpr int NU = lbase(8);
constexpr int ITEM_PK = 6144;
constexpr size_t STRIP_BYTES = (size_t)32 * APITCH;
constexpr size_t XA_BYTES = NSLOT * STRIP_BYTES, W2_BYTES = (size_t)KDIM * BPITCH, PK_BYTES = (size_t)NU * ITEM_PK;
constexpr size_t OFF_XA = 0, OFF_W2 = XA_BYTES, OFF_PK = XA_BYTES + W2_BYTES, PAR_BYTES = OFF_PK + PK_BYTES;
constexpr size_t OFF_FLAG = 2 * PAR_BYTES, FLAG_BYTES = (size_t)NU * 64;
constexpr size_t OFF_CNT = OFF_FLAG + 2 * FLAG_BYTES, CNT_BYTES = (size_t)(NSLOT + 4) * NCH * 128;
constexpr size_t OFF_TICK = OFF_CNT + CNT_BYTES, TICK_BYTES = 2048 * 64;
constexpr int OITEM = 256 + 4096, NOI = 4 * NTR * 8 * HPT;
constexpr size_t OFF_OPK = OFF_TICK + TICK_BYTES, OFF_OFLAG = OFF_OPK + (size_t)NOI * OITEM;
constexpr size_t OFF_TMO = OFF_OFLAG + (size_t)NOI * 64, OFF_QUEUE = OFF_TMO + 128, OFF_DBG = OFF_QUEUE + 8 * 128, DBG_BYTES = 2048 * 64 + NU * 24 + 256 * 64 * 8;
constexpr size_t WS_NEED = OFF_DBG + DBG_BYTES;
constexpr int GRID = 256, NTHR = 512;
#define SEND_WAVES 4
#define SEND_WGS 6
#define SEND_HELP 0
__host__ __device__ constexpr int ncomm(int x) { return 32 - (x < 4 ? NTL : NTR); }
__host__ __device__ constexpr int nsend(int x) { return x != 3 ? SEND_WGS : SEND_HELP ? 7 : 0; }
constexpr int LINK_WGS = SEND_WGS + SEND_HELP, NUWG = 4 * (ncomm(0) + ncomm(4)) - 7 * LINK_WGS, NUW = NUWG * 8;
constexpr int NXW = NUW + 7 * LINK_WGS * SEND_WAVES;
constexpr unsigned long long DEADLINE = 50000000ull;
constexpr int LDS_PACK = SR * 16384, LDS_STAGE = 135168, LDS_BYTES = LDS_STAGE + 16;
static_assert(NTL % 4 == 0 && LDS_PACK + 8 * OITEM <= LDS_STAGE && ncomm(0) >= SEND_WGS && ncomm(4) >= SEND_WGS && ncomm(3) >= nsend(3), "roles");

struct Params {
    const char* x; const char* w;
    char* out;
    char* wsx[NR];
    int rank; int pad;
};

#define WAIT_V(n) asm volatile("s_waitcnt vmcnt(" #n ")" ::: "memory")
#define WAIT_L(n) asm volatile("s_waitcnt lgkmcnt(" #n ")" ::: "memory")
#define BAR __builtin_amdgcn_s_barrier()
#define SCHED __builtin_amdgcn_sched_barrier(0)
#define STAMP(i) do {} while (0)
#define DBG_AT(off) do {} while (0)
#define TS_NOW(dst, k) do {} while (0)

__device__ __forceinline__ unsigned ld_agent(const char* p) { return __hip_atomic_load((const GAS unsigned*)p, __ATOMIC_RELAXED, __HIP_MEMORY_SCOPE_AGENT); }
__device__ __forceinline__ unsigned ld_sys(const char* p) { return __hip_atomic_load((const GAS unsigned*)p, __ATOMIC_RELAXED, __HIP_MEMORY_SCOPE_SYSTEM); }
__device__ __forceinline__ bool give_up(char* ws, unsigned call, unsigned long long t0) {
    if ((unsigned)__builtin_amdgcn_readfirstlane(ld_agent(ws + OFF_TMO)) == call || __builtin_amdgcn_s_memrealtime() - t0 > DEADLINE) {
        __hip_atomic_store((GAS unsigned*)(ws + OFF_TMO), call, __ATOMIC_RELAXED, __HIP_MEMORY_SCOPE_AGENT);
        return true;
    }
    __builtin_amdgcn_s_sleep(2);
    return false;
}
__device__ __forceinline__ void store16_sys(char* base, unsigned voff, u32x4 v) {
    asm volatile("global_store_dwordx4 %0, %1, %2 sc0 sc1\n\ts_nop 1" :: "v"(voff), "v"(v), "s"(base) : "memory");
}

struct Item { bool isw; unsigned src, fin, cnt, voff, istep; };
__device__ __forceinline__ Item item_of(int d, int i, int lane) {
    const int chunk = (d & 1) ? i / litems(1) : i / litems(0), j = i - chunk * litems(d);
    Item it;
    it.isw = (d & 1) && j >= OH;
    if (it.isw) { it.src = (unsigned)((chunk * 128 + (j - OH) * 16) * BPITCH + (d >> 1) * 512); it.fin = (unsigned)OFF_W2 + it.src;
        it.cnt = (unsigned)(OFF_CNT + ((NSLOT + (d >> 1)) * NCH + chunk) * 128); it.voff = (lane >> 5) * BPITCH + (lane & 31) * 16; it.istep = 2 * BPITCH; }
    else { const int strip = (d & 1) ? EH + j : j, slot = (d & 1) ? 3 * EH + (d >> 1) * OH + j : ((d >> 1) - 1) * EH + j;
        it.src = (unsigned)(strip * STRIP_BYTES + chunk * 256); it.fin = (unsigned)(OFF_XA + slot * STRIP_BYTES + chunk * 256);
        it.cnt = (unsigned)(OFF_CNT + (slot * NCH + chunk) * 128); it.voff = (lane >> 4) * APITCH + (lane & 15) * 16; it.istep = 4 * APITCH; }
    return it;
}
__device__ __forceinline__ unsigned sender(const Params& P, int d, int first, char* ticket) {
    const int lane = threadIdx.x & 63, nlink = NCH * litems(d), nwaves = LINK_WGS * SEND_WAVES;
    char* const queue = P.wsx[0] + OFF_QUEUE + d * 128;
    const char* const src_x = P.x; const char* const src_w = P.w;
    u32x4 nxt[8];
    unsigned tick = 0, ctick = 0;
#define FETCH(i) do { const Item _x = item_of(d, (i) < nlink ? (i) : nlink - 1, lane); \
        asm volatile("global_load_dwordx4 %0, %9, %17\n\tglobal_load_dwordx4 %1, %10, %17\n\tglobal_load_dwordx4 %2, %11, %17\n\tglobal_load_dwordx4 %3, %12, %17\n\t" \
                     "global_load_dwordx4 %4, %13, %17\n\tglobal_load_dwordx4 %5, %14, %17\n\tglobal_load_dwordx4 %6, %15, %17\n\tglobal_load_dwordx4 %7, %16, %17\n\t" \
                     "s_mov_b64 exec, 1\n\tglobal_atomic_add %8, %18, %19, %20 sc0\n\ts_mov_b64 exec, -1" \
                     : "=&v"(nxt[0]), "=&v"(nxt[1]), "=&v"(nxt[2]), "=&v"(nxt[3]), "=&v"(nxt[4]), "=&v"(nxt[5]), "=&v"(nxt[6]), "=&v"(nxt[7]), "=&v"(tick) \
                     : "v"(_x.voff), "v"(_x.voff + _x.istep), "v"(_x.voff + 2 * _x.istep), "v"(_x.voff + 3 * _x.istep), "v"(_x.voff + 4 * _x.istep), "v"(_x.voff + 5 * _x.istep), \
                       "v"(_x.voff + 6 * _x.istep), "v"(_x.voff + 7 * _x.istep), "s"((_x.isw ? src_w : src_x) + _x.src), "v"(0u), "v"(1u), "s"(queue) : "memory"); } while (0)
#define WAIT_FETCH() asm volatile("s_waitcnt vmcnt(0)" : "+v"(ctick), "+v"(tick), "+v"(nxt[0]), "+v"(nxt[1]), "+v"(nxt[2]), "+v"(nxt[3]), "+v"(nxt[4]), "+v"(nxt[5]), "+v"(nxt[6]), "+v"(nxt[7]) :: "memory")
    int cur = first;
    asm volatile("s_mov_b64 exec, 1\n\tglobal_atomic_add %0, %1, %2, %3 sc0\n\ts_mov_b64 exec, -1" : "=&v"(ctick) : "v"(0u), "v"(1u), "s"(ticket) : "memory");
    FETCH(cur); WAIT_FETCH();
    const unsigned call = (unsigned)__builtin_amdgcn_readfirstlane(ctick) + 1u;
    char* const peer = P.wsx[d & 7];
    char* const set = peer + (call & 1) * PAR_BYTES;
    char* const flags = peer + OFF_FLAG + (call & 1) * FLAG_BYTES;
    const unsigned qbase = (call - 1u) * (nlink + nwaves) - nwaves;
    while (cur < nlink) {
        const Item it = item_of(d, cur, lane);
        const int u = lbase(d) + cur;
        u32x4 raw[8];
#pragma unroll
        for (int v = 0; v < 8; ++v) raw[v] = nxt[v];
        const int next = (int)((unsigned)__builtin_amdgcn_readfirstlane(tick) - qbase);
        FETCH(next); SCHED;
        u16x2 mx = {0, 0};
#pragma unroll
        for (int v = 0; v < 8; ++v)
#pragma unroll
            for (int c = 0; c < 4; ++c) mx = __builtin_elementwise_max(mx, __builtin_bit_cast(u16x2, raw[v][c] & 0x7fff7fffu));
        unsigned m = mx.x > mx.y ? mx.x : mx.y;
#pragma unroll
        for (int o = 32; o >= 1; o >>= 1) { const unsigned t = __shfl_xor(m, o); m = t > m ? t : m; }
        const unsigned base = (unsigned)__builtin_amdgcn_readfirstlane(m) >> 7, base4 = base * 0x01010101u;
        u32x4 pk[6]; unsigned esc = 0;
#pragma unroll
        for (int v = 0; v < 8; ++v) {
            unsigned r[4];
#pragma unroll
            for (int c = 0; c < 4; ++c) r[c] = ((raw[v][c] << 1) & 0xfffefffeu) | ((raw[v][c] >> 15) & 0x00010001u);
            pk[v >> 1][(v & 1) * 2] = __builtin_amdgcn_perm(r[1], r[0], 0x06040200u);
            pk[v >> 1][(v & 1) * 2 + 1] = __builtin_amdgcn_perm(r[3], r[2], 0x06040200u);
            const unsigned d0e = base4 - __builtin_amdgcn_perm(r[1], r[0], 0x07050301u), d1e = base4 - __builtin_amdgcn_perm(r[3], r[2], 0x07050301u);
            unsigned nib = d0e | (d1e << 4);
            if (((d0e | d1e) & 0xf0f0f0f0u) != 0u) nib = 0xffffffffu;
            if (nib == 0xffffffffu) esc |= 1u << v;
            pk[4 + (v >> 2)][v & 3] = nib;
        }
        if (__builtin_amdgcn_ballot_w64(esc != 0u) != 0ull) {
#pragma unroll
            for (int v = 0; v < 8; ++v) if (esc & (1u << v)) store16_sys(set + it.fin, it.voff + v * it.istep, raw[v]);
        }
        asm volatile("global_store_dwordx4 %0, %1, %7 offset:-2048 sc0 sc1\n\tglobal_store_dwordx4 %0, %2, %7 offset:-1024 sc0 sc1\n\t"
                     "global_store_dwordx4 %0, %3, %7 sc0 sc1\n\tglobal_store_dwordx4 %0, %4, %7 offset:1024 sc0 sc1\n\t"
                     "global_store_dwordx4 %0, %5, %7 offset:2048 sc0 sc1\n\tglobal_store_dwordx4 %0, %6, %7 offset:3072 sc0 sc1\n\ts_nop 1"
                     :: "v"(lane * 16), "v"(pk[0]), "v"(pk[1]), "v"(pk[2]), "v"(pk[3]), "v"(pk[4]), "v"(pk[5]), "s"(set + OFF_PK + (size_t)u * ITEM_PK + 2048) : "memory");
        WAIT_FETCH();
        if (lane == 0) __hip_atomic_store((GAS unsigned*)(flags + (size_t)u * 64), (call << 8) | base, __ATOMIC_RELAXED, __HIP_MEMORY_SCOPE_SYSTEM);
        DBG_AT(u * 24 + 16);
        cur = next;
    }
    return call;
}

__device__ __forceinline__ void unpacker(const Params& P, int uw, unsigned call) {
    const int lane = threadIdx.x & 63;
    const int d = uw % 7 + 1, step = (NUW - uw % 7 + 6) / 7, nlink = NCH * litems(d);
    char* ws = P.wsx[0];
    char* set = ws + (call & 1) * PAR_BYTES;
    const char* flags = ws + OFF_FLAG + (call & 1) * FLAG_BYTES;
    GAS unsigned* pending = nullptr;
    for (int i = uw / 7; i < nlink; i += step) {
        const int u = lbase(d) + i;
        unsigned f = (unsigned)__builtin_amdgcn_readfirstlane(ld_sys(flags + (size_t)u * 64));
        WAIT_V(0);
        if (pending && lane == 0) __hip_atomic_fetch_add(pending, 1u, __ATOMIC_RELAXED, __HIP_MEMORY_SCOPE_AGENT);
        if (pending) DBG_AT((u - step) * 24 + 8);
        pending = nullptr;
        if ((f >> 8) < call) {
            const unsigned long long t0 = __builtin_amdgcn_s_memrealtime();
            bool dead = false;
            while (((f = (unsigned)__builtin_amdgcn_readfirstlane(ld_sys(flags + (size_t)u * 64))) >> 8) < call)
                if (give_up(ws, call, t0)) { dead = true; break; }
            if (dead) break;
        }
        DBG_AT(u * 24);
        u32x4 pk[6];
        asm volatile("global_load_dwordx4 %0, %6, %7 offset:-2048 sc0 sc1\n\tglobal_load_dwordx4 %1, %6, %7 offset:-1024 sc0 sc1\n\t"
                     "global_load_dwordx4 %2, %6, %7 sc0 sc1\n\tglobal_load_dwordx4 %3, %6, %7 offset:1024 sc0 sc1\n\t"
                     "global_load_dwordx4 %4, %6, %7 offset:2048 sc0 sc1\n\tglobal_load_dwordx4 %5, %6, %7 offset:3072 sc0 sc1\n\ts_waitcnt vmcnt(0)"
                     : "=&v"(pk[0]), "=&v"(pk[1]), "=&v"(pk[2]), "=&v"(pk[3]), "=&v"(pk[4]), "=&v"(pk[5])
                     : "v"(lane * 16), "s"(set + OFF_PK + (size_t)u * ITEM_PK + 2048) : "memory");
        const Item it = item_of(d, i, lane);
        const unsigned base4 = (f & 255u) * 0x01010101u;
#pragma unroll
        for (int v = 0; v < 8; ++v) {
            const unsigned nib = pk[4 + (v >> 2)][v & 3], m0 = pk[v >> 1][(v & 1) * 2], m1 = pk[v >> 1][(v & 1) * 2 + 1];
            const unsigned e0 = base4 - (nib & 0x0f0f0f0fu), e1 = base4 - ((nib >> 4) & 0x0f0f0f0fu);
            u32x4 r;
            r[0] = __builtin_amdgcn_perm(e0, m0, 0x05010400u); r[1] = __builtin_amdgcn_perm(e0, m0, 0x07030602u);
            r[2] = __builtin_amdgcn_perm(e1, m1, 0x05010400u); r[3] = __builtin_amdgcn_perm(e1, m1, 0x07030602u);
#pragma unroll
            for (int c = 0; c < 4; ++c) r[c] = ((r[c] >> 1) & 0x7fff7fffu) | ((r[c] << 15) & 0x80008000u);
            if (nib != 0xffffffffu) store16_sys(set + it.fin, it.voff + v * it.istep, r);
        }
        pending = (GAS unsigned*)(ws + it.cnt);
        if (i == uw / 7) STAMP(1);
    }
    WAIT_V(0);
    if (pending && lane == 0) __hip_atomic_fetch_add(pending, 1u, __ATOMIC_RELAXED, __HIP_MEMORY_SCOPE_AGENT);
    if (pending) DBG_AT((lbase(d) + uw / 7 + (nlink - 1 - uw / 7) / step * step) * 24 + 8);
}

__device__ __forceinline__ int strip_of(int pos, bool remote, int q) {
    if (!remote) return pos < 32 + NSLOT ? pos : -1;
    if (pos < EH) return pos;
    if (pos < EH + 3 * OH) { const int k = (pos - EH) / OH; return 32 + 3 * EH + (k + (k >= q ? 1 : 0)) * OH + (pos - EH) % OH; }
    return (unsigned)(pos - NTE * SR) < 3u * EH ? 32 + pos - NTE * SR : -1;
}
__device__ __forceinline__ int out_row_of(int id, int rank) {
    if (id < 32) return rank * 1024 + id * 32;
    const int slot = id - 32;
    if (slot < 3 * EH) return (rank ^ (2 * (slot / EH + 1))) * 1024 + (slot % EH) * 32;
    const int o = slot - 3 * EH;
    return (rank ^ (2 * (o / OH) + 1)) * 1024 + (EH + o % OH) * 32;
}

__device__ __forceinline__ void expander(const Params& P, LAS unsigned char* buf, int xw, unsigned call) {
    const int lane = threadIdx.x & 63;
    char* ws = P.wsx[0];
    for (int n = xw; n < NOI; n += NXW) {
        const bool late = n >= NTE * HPT * 32;
        const int n1 = late ? n - NTE * HPT * 32 : n, per = (late ? NTR - NTE : NTE) * 32, h = n1 / per, rt = (late ? NTE : 0) + (n1 - h * per) / 32, w = (n1 >> 2) & 7, q = n1 & 3;
        const int ida = strip_of(rt * SR + 2 * h, true, q), idb = 2 * h + 1 < SR ? strip_of(rt * SR + 2 * h + 1, true, q) : -1;
        if (ida < 0) continue;
        const int o = ((q * NTR + rt) * 8 + w) * HPT + h;
        unsigned f; bool dead = false;
        { const unsigned long long t0 = __builtin_amdgcn_s_memrealtime();
          while (((f = (unsigned)__builtin_amdgcn_readfirstlane(ld_sys(ws + OFF_OFLAG + o * 64))) >> 12) < call)
              if (give_up(ws, call, t0)) { dead = true; break; } }
        u32x4 r[4];
        unsigned tmo = call;
        if (!dead) {
            const unsigned top = 256u + 2u * (f & 0xfffu) - 1u;
            const unsigned lim = (top < (unsigned)OITEM ? top : OITEM - 1u) & ~15u;
            u32x4 t[5]; unsigned a[5];
#pragma unroll
            for (int k = 0; k < 5; ++k) { a[k] = lane * 16 + k * 1024; a[k] = a[k] < lim ? a[k] : lim; }
            asm volatile("global_load_dwordx4 %0, %6, %11 sc0 sc1\n\tglobal_load_dwordx4 %1, %7, %11 sc0 sc1\n\tglobal_load_dwordx4 %2, %8, %11 sc0 sc1\n\t"
                         "global_load_dwordx4 %3, %9, %11 sc0 sc1\n\tglobal_load_dwordx4 %4, %10, %11 sc0 sc1\n\tglobal_load_dword %5, %12, %13 sc1\n\ts_waitcnt vmcnt(0)"
                         : "=&v"(t[0]), "=&v"(t[1]), "=&v"(t[2]), "=&v"(t[3]), "=&v"(t[4]), "=&v"(tmo)
                         : "v"(a[0]), "v"(a[1]), "v"(a[2]), "v"(a[3]), "v"(a[4]), "s"(ws + OFF_OPK + (size_t)o * OITEM), "v"(0u), "s"(ws + OFF_TMO) : "memory");
#pragma unroll
            for (int k = 0; k < 5; ++k) *(LAS u32x4*)(buf + a[k]) = t[k];
            asm volatile("" ::: "memory");
            const unsigned masks = *(const LAS unsigned*)(buf + lane * 4);
            unsigned pos = 0;
#pragma unroll
            for (int j = 0; j < 32; ++j) {
                const unsigned lo = __builtin_amdgcn_readlane(masks, 2 * j), hi = __builtin_amdgcn_readlane(masks, 2 * j + 1);
                unsigned idx = __builtin_amdgcn_mbcnt_hi(hi, __builtin_amdgcn_mbcnt_lo(lo, pos));
                idx = idx < 2047u ? idx : 2047u;
                unsigned val = *(const LAS unsigned short*)(buf + 256 + idx * 2);
                val = __builtin_amdgcn_inverse_ballot_w64(((unsigned long long)hi << 32) | lo) ? val : 0u;
                if (j & 1) r[j >> 3][(j >> 1) & 3] |= val << 16; else r[j >> 3][(j >> 1) & 3] = val;
                pos += __builtin_popcount(lo) + __builtin_popcount(hi);
            }
        }
        if (tmo == call) {
#pragma unroll
            for (int v = 0; v < 4; ++v) r[v] = (u32x4){0x7fc07fc0u, 0x7fc07fc0u, 0x7fc07fc0u, 0x7fc07fc0u};
        }
        const int src = P.rank ^ (2 * q + 1);
        const unsigned ooff = (w * 4 + (lane >> 5)) * OPITCH + q * 512 + (lane & 31) * 16;
        { char* O = P.out + (size_t)out_row_of(ida, src) * OPITCH; store16_sys(O, ooff, r[0]); store16_sys(O, ooff + 2 * OPITCH, r[1]); }
        if (idb >= 0) { char* O = P.out + (size_t)out_row_of(idb, src) * OPITCH; store16_sys(O, ooff, r[2]); store16_sys(O, ooff + 2 * OPITCH, r[3]); }
    }
}

__device__ __forceinline__ unsigned cvt_pk_bf16(float lo, float hi) { unsigned r; asm("v_cvt_pk_bf16_f32 %0, %1, %2" : "=v"(r) : "v"(lo), "v"(hi)); return r; }
template <int OFF> __device__ __forceinline__ s16x4 tr_read(unsigned addr) { s16x4 r; asm volatile("ds_read_b64_tr_b16 %0, %1 offset:%2" : "=v"(r) : "v"(addr), "i"(OFF) : "memory"); return r; }
template <int OFF> __device__ __forceinline__ bf16x8 row_read(unsigned addr) { bf16x8 r; asm volatile("ds_read_b128 %0, %1 offset:%2" : "=v"(r) : "v"(addr), "i"(OFF) : "memory"); return r; }
__device__ __forceinline__ unsigned below(int a, int b) { unsigned r; asm("s_ashr_i32 %0, %1, 31" : "=s"(r) : "s"(a - b) : "scc"); return r; }
__device__ __forceinline__ void glds(unsigned ldsaddr, unsigned voff, const char* g) {
    asm volatile("s_mov_b32 m0, %0\n\ts_nop 0\n\tglobal_load_lds_dwordx4 %1, %2 sc0 sc1" :: "s"(ldsaddr), "v"(voff), "s"(g) : "memory");
}
__device__ __forceinline__ void glds_masked(unsigned ldsaddr, unsigned voff, const char* g, unsigned mask) {
    asm volatile("s_mov_b32 m0, %0\n\ts_mov_b32 exec_lo, %3\n\ts_mov_b32 exec_hi, %3\n\tglobal_load_lds_dwordx4 %1, %2 sc0 sc1\n\ts_mov_b64 exec, -1" :: "s"(ldsaddr), "v"(voff), "s"(g), "s"(mask) : "memory");
}

template <int S> struct Frag { bf16x8 a[S], b[4]; };
template <int S> __device__ __forceinline__ void wait_step(Frag<S>& F, unsigned& pre) {
    asm volatile("s_waitcnt vmcnt(6) lgkmcnt(0)" : "+v"(F.a[0]), "+v"(F.a[1]), "+v"(F.a[2]), "+v"(F.a[3]), "+v"(F.a[S - 1]), "+v"(F.b[0]), "+v"(F.b[1]), "+v"(F.b[2]), "+v"(F.b[3]), "+v"(pre) :: "memory");
    if constexpr (S > 5) asm volatile("" : "+v"(F.a[4]));
    if constexpr (S > 6) asm volatile("" : "+v"(F.a[5]));
    if constexpr (S > 7) asm volatile("" : "+v"(F.a[6]));
}
template <int S, bool remote> __device__ __forceinline__ void gemm_tile(const Params& P, LAS unsigned char* lds, int rt, int ct, unsigned call) {
    constexpr int SLOT_A = S * 2048, SLOT = SLOT_A + 16384;
    static_assert(S >= 5 && S <= 8 && 4 * SLOT <= LDS_STAGE && S * 16384 <= LDS_STAGE, "tile");
    int wid = __builtin_amdgcn_readfirstlane(threadIdx.x >> 6), lane = threadIdx.x & 63, wr = wid >> 2, wc = wid & 3, fr = lane & 15, fq = lane >> 4;
    char* ws = P.wsx[0];
    const char* set = ws + (call & 1) * PAR_BYTES;
    const char* const own_x = P.x;
    const unsigned ldst = (unsigned)(size_t)lds + wid * 1024;
    unsigned voffA, voffB;
    { const int k0 = lane >> 4, ch = lane & 15; voffB = (unsigned)((wid * 4 + k0) * BPITCH + (((ch >> 1) ^ (k0 | ((wid & 2) << 1))) * 32) + (ch & 1) * 16);
      const int sb = lane * 16, swz = sb ^ (((sb >> 9) & 1) << 5); voffA = (unsigned)(((wid & 1) * 16 + (swz >> 6)) * APITCH + (swz & 63)); }
    auto strip_ptr = [&](int k) { int id = k < S ? strip_of(rt * S + k, remote, ct) : -1; if (id < 0) id = strip_of(rt * S, remote, ct);
                                  return id < 32 ? own_x + id * STRIP_BYTES : set + OFF_XA + (id - 32) * STRIP_BYTES; };
    const char* const gb = (remote ? set + OFF_W2 : P.w) + ct * 512;
    const char* const ga0 = strip_ptr(wid >> 1);
    const char* const ga1 = strip_ptr(4 + (wid >> 1));
    const unsigned amask = below(wid, 2 * S - 8);
    unsigned aoff[2], boff[2][2];
    { const unsigned lbase = (unsigned)(size_t)lds; const int ob = fr * 64 + fq * 16; aoff[0] = lbase + wr * S * 1024 + (ob ^ (((ob >> 9) & 1) << 5)); aoff[1] = aoff[0] + 2 * SLOT;
      const int q = (lane & 15) >> 2, p = lane & 3, f = q | ((fq & 1) << 2);
      for (int n = 0; n < 2; ++n) { boff[0][n] = lbase + SLOT_A + (8 * fq + q) * 256 + p * 8 + (((wc * 2 + n) ^ f) * 32); boff[1][n] = boff[0][n] + 2 * SLOT; } }
    unsigned coff, cstep, need;
    { const int id = strip_of(rt * S + (lane < S ? lane : 0), remote, ct);
      const bool watch = wid == 0 && (lane < S ? id >= 32 : (lane == 8 && remote));
      coff = watch ? (unsigned)(OFF_CNT + (lane < S ? id - 32 : NSLOT + ct) * NCH * 128) : (unsigned)OFF_TMO;
      cstep = watch ? 128u : 0u;
      need = watch ? call * (lane < S ? 1u : 8u) : 0u; }
    const unsigned w0 = below(wid, 1);
    unsigned pre = 0;
#define SLOW_WAIT() do { const unsigned long long t0 = __builtin_amdgcn_s_memrealtime(); \
        while (__builtin_amdgcn_ballot_w64(ld_agent(ws + coff) < need) != 0ull) if (give_up(ws, call, t0)) break; \
        asm volatile("" ::: "memory"); } while (0)
#define PRE_LOAD(mask) do { coff += cstep; \
        asm volatile("s_mov_b32 exec_lo, %3\n\ts_mov_b32 exec_hi, 0\n\tglobal_load_dword %0, %1, %2 sc1\n\ts_mov_b64 exec, -1" : "+v"(pre) : "v"(coff), "s"(ws), "s"((mask) & w0) : "memory"); } while (0)
#define STAGE_B(h, sl, st) glds(ldst + (sl) * SLOT + SLOT_A + (h) * 8192, voffB, gb + (h) * 256 + (size_t)(st) * (32 * BPITCH))
#define STAGE_A0(sl, st) glds(ldst + (sl) * SLOT, voffA, ga0 + (st) * 64)
#define STAGE_A1(sl, st) glds_masked(ldst + (sl) * SLOT + 8192, voffA, ga1 + (st) * 64, amask)
#define STAGE_ALL(sl, st) do { STAGE_B(0, sl, st); STAGE_B(1, sl, st); STAGE_A0(sl, st); STAGE_A1(sl, st); } while (0)
#define RDA(F, m, sl) F.a[m] = row_read<((sl) & 1) * SLOT + (m) * 1024>(aoff[(sl) >> 1])
#define RDB(F, j, sl) do { const s16x4 _lo = tr_read<((sl) & 1) * SLOT + ((j) >> 1) * 8192>(boff[(sl) >> 1][(j) & 1]), _hi = tr_read<((sl) & 1) * SLOT + ((j) >> 1) * 8192 + 1024>(boff[(sl) >> 1][(j) & 1]); \
        F.b[j] = __builtin_shufflevector(_lo, _hi, 0, 1, 2, 3, 4, 5, 6, 7); } while (0)
#define MM(F, i) acc[(i) >> 2][(i) & 3] = __builtin_amdgcn_mfma_f32_16x16x32_bf16(F.b[(i) & 3], F.a[(i) >> 2], acc[(i) >> 2][(i) & 3], 0, 0, 0)
#define PAIR(C, i) do { MM(C, i); MM(C, (i) + 1); SCHED; } while (0)
#define STEP(C, N, sl, st, GATE) do { constexpr int nx = ((sl) + 1) & 3; \
        RDB(N, 0, nx); PAIR(C, 0); RDB(N, 1, nx); PAIR(C, 2); RDA(N, 0, nx); RDA(N, 1, nx); PAIR(C, 4); RDB(N, 2, nx); PAIR(C, 6); RDB(N, 3, nx); \
        STAGE_B(0, sl, st); PAIR(C, 8); RDA(N, 2, nx); RDA(N, 3, nx); STAGE_B(1, sl, st); PAIR(C, 10); \
        RDA(N, 4, nx); if constexpr (S > 5) RDA(N, 5, nx); STAGE_A0(sl, st); PAIR(C, 12); \
        if constexpr (S > 6) RDA(N, 6, nx); if constexpr (S > 7) RDA(N, 7, nx); STAGE_A1(sl, st); \
        _Pragma("unroll") for (int i = 14; i < 4 * S - 4; i += 2) PAIR(C, i); \
        wait_step(N, pre); GATE; BAR; SCHED; PAIR(C, 4 * S - 4); PAIR(C, 4 * S - 2); } while (0)
    f32x4 acc[S][4];
#pragma unroll
    for (int m = 0; m < S; ++m)
#pragma unroll
        for (int j = 0; j < 4; ++j) acc[m][j] = (f32x4){0.f, 0.f, 0.f, 0.f};
    Frag<S> F0, F1;
    if (wid == 0) SLOW_WAIT();
    BAR;
    STAMP(1);
    PRE_LOAD(0x3ffu);
    STAGE_ALL(0, 0); STAGE_ALL(1, 1); STAGE_ALL(2, 2); STAGE_ALL(3, 3);
    WAIT_V(6); BAR;
    RDB(F0, 0, 0); RDB(F0, 1, 0); RDB(F0, 2, 0); RDB(F0, 3, 0);
    RDA(F0, 0, 0); RDA(F0, 1, 0); RDA(F0, 2, 0); RDA(F0, 3, 0); RDA(F0, 4, 0);
    if constexpr (S > 5) RDA(F0, 5, 0);
    if constexpr (S > 6) RDA(F0, 6, 0);
    if constexpr (S > 7) RDA(F0, 7, 0);
    wait_step(F0, pre);
    if (__builtin_amdgcn_ballot_w64(pre < need) != 0ull) SLOW_WAIT();
    BAR;
    for (int c = 0; c < NCH; ++c) {
        const int st = (c < NCH - 1 ? c + 1 : c) * 4;
        STEP(F0, F1, 0, st, (void)0);
        STEP(F1, F0, 1, st + 1, (void)0);
        PRE_LOAD(c < NCH - 2 ? 0x3ffu : c < NCH - 1 ? 0x200u : 0u);
        STEP(F0, F1, 2, st + 2, (void)0);
        STEP(F1, F0, 3, st + 3, { TS_NOW(ts0, c); if (c < NCH - 2 && __builtin_amdgcn_ballot_w64(pre < need) != 0ull) SLOW_WAIT(); TS_NOW(ts1, c); });
    }
    if (threadIdx.x == 9) *(LAS unsigned*)(lds + LDS_STAGE) = pre;
    asm volatile("s_waitcnt vmcnt(0) lgkmcnt(0)" ::: "memory");
    BAR;
    asm volatile("" ::: "memory");
    const bool bad = *(const LAS unsigned*)(lds + LDS_STAGE) == call;
    STAMP(2);
    int tz = threadIdx.x; asm volatile("" : "+v"(tz));
    lane = tz & 63; fr = lane & 15; fq = lane >> 4;
#pragma unroll
    for (int m = 0; m < S; ++m) {
        const int row = wr * S * 16 + m * 16 + fr;
#pragma unroll
        for (int j = 0; j < 4; ++j) {
            f32x4 v = acc[m][j];
#pragma unroll
            for (int e = 0; e < 4; ++e) v[e] = v[e] < 0.f ? 0.f : v[e];
            u32x2 w; w.x = cvt_pk_bf16(v[0], v[1]); w.y = cvt_pk_bf16(v[2], v[3]);
            if (bad) { w.x = 0x7fc07fc0u; w.y = 0x7fc07fc0u; }
            const int ch = (j >> 1) * 16 + wc * 4 + (j & 1) * 2 + (fq >> 1);
            *(LAS u32x2*)(lds + row * 512 + ((ch ^ (fr & 7)) * 16) + (fq & 1) * 8) = w;
        }
    }
    WAIT_L(0); BAR; asm volatile("" ::: "memory");
    u32x4 ov[S][2];
#pragma unroll
    for (int k = 0; k < S; ++k)
#pragma unroll
        for (int i = 0; i < 2; ++i) { const int row = k * 32 + wid * 4 + i * 2 + (lane >> 5); ov[k][i] = *(const LAS u32x4*)(lds + row * 512 + (((lane & 31) ^ (row & 7)) * 16)); }
    if constexpr (!remote) {
        const unsigned ooff = (wid * 4 + (lane >> 5)) * OPITCH + ct * 512 + (lane & 31) * 16;
#pragma unroll
        for (int k = 0; k < S; ++k) {
            const int id = strip_of(rt * S + k, false, 0);
            if (id >= 0) { char* O = P.out + (size_t)out_row_of(id, P.rank) * OPITCH; store16_sys(O, ooff, ov[k][0]); store16_sys(O, ooff + 2 * OPITCH, ov[k][1]); }
        }
    } else {
        char* const peer = P.wsx[(2 * ct + 1) & 7];
        LAS unsigned char* buf = lds + LDS_PACK + wid * OITEM;
        const int o0 = ((ct * NTR + rt) * 8 + wid) * HPT;
#pragma unroll
        for (int h = 0; h < HPT; ++h) {
            if (strip_of(rt * S + 2 * h, true, ct) < 0) break;
            const bool second = 2 * h + 1 < S && strip_of(rt * S + 2 * h + 1, true, ct) >= 0;
            unsigned masks = 0, pos = 0;
#pragma unroll
            for (int j = 0; j < 32; ++j) {
                const unsigned dw = ov[2 * h + (j >> 4) < S ? 2 * h + (j >> 4) : 0][(j >> 3) & 1][(j >> 1) & 3];
                const unsigned val = (j & 1) ? dw >> 16 : dw & 0xffffu;
                const bool nz = val != 0u && (j < 16 || second);
                const unsigned long long m = __builtin_amdgcn_ballot_w64(nz);
                const unsigned lo = (unsigned)m, hi = (unsigned)(m >> 32);
                const unsigned idx = __builtin_amdgcn_mbcnt_hi(hi, __builtin_amdgcn_mbcnt_lo(lo, pos));
                if (nz) *(LAS unsigned short*)(buf + 256 + idx * 2) = (unsigned short)val;
                asm("s_nop 1\n\tv_writelane_b32 %0, %1, %3\n\tv_writelane_b32 %0, %2, %4" : "+v"(masks) : "s"(lo), "s"(hi), "i"(2 * j), "i"(2 * j + 1));
                pos += __builtin_popcount(lo) + __builtin_popcount(hi);
            }
            *(LAS unsigned*)(buf + lane * 4) = masks;
            asm volatile("" ::: "memory");
            const unsigned total = 256u + 2u * pos;
            u32x4 t[5];
#pragma unroll
            for (int k = 0; k < 5; ++k) { const unsigned a = lane * 16 + k * 1024; t[k] = *(const LAS u32x4*)(buf + (a < (unsigned)OITEM ? a : 0u)); }
            char* dst = peer + OFF_OPK + (size_t)(o0 + h) * OITEM;
#pragma unroll
            for (int k = 0; k < 5; ++k) if (lane * 16 + k * 1024 < total) store16_sys(dst, lane * 16 + k * 1024, t[k]);
            WAIT_V(0);
            if (lane == 0) __hip_atomic_store((GAS unsigned*)(peer + OFF_OFLAG + (o0 + h) * 64), (call << 12) | pos, __ATOMIC_RELAXED, __HIP_MEMORY_SCOPE_SYSTEM);
        }
    }
    STAMP(3);
}

__global__ __launch_bounds__(NTHR, 2) void ag_gemm_kernel(const Params P) {
    extern __shared__ __attribute__((aligned(16))) unsigned char shm[];
    const int xcd = blockIdx.x & 7, slot = blockIdx.x >> 3, wid = __builtin_amdgcn_readfirstlane(threadIdx.x >> 6);
    char* const ticket = P.wsx[0] + OFF_TICK + (size_t)(blockIdx.x * 8 + wid) * 64;
    STAMP(0);
    if (slot < nsend(xcd)) {
        if (wid >= SEND_WAVES) return;
        const int link = xcd == 3 ? slot + 1 : xcd < 3 ? 2 * xcd + 2 : 2 * xcd - 7, first = wid * LINK_WGS + (xcd == 3 ? SEND_WGS : slot);
        const unsigned call = sender(P, link, first, ticket);
        STAMP(2);
        expander(P, (LAS unsigned char*)shm + wid * OITEM, NUW + first * 7 + link - 1, call);
        STAMP(3);
        return;
    }
    unsigned tick = 0;
    if ((threadIdx.x & 63) == 0) tick = __hip_atomic_fetch_add((GAS unsigned*)ticket, 1u, __ATOMIC_RELAXED, __HIP_MEMORY_SCOPE_AGENT);
    const unsigned call = (unsigned)__builtin_amdgcn_readfirstlane(tick) + 1u;
    if (slot < ncomm(xcd)) {
        int uwg = slot - nsend(xcd);
        for (int y = 0; y < xcd; ++y) uwg += ncomm(y) - nsend(y);
        unpacker(P, wid * NUWG + uwg, call);
        STAMP(2);
        expander(P, (LAS unsigned char*)shm + wid * OITEM, wid * NUWG + uwg, call);
        STAMP(3);
    } else {
        const int g = slot - ncomm(xcd);
                if (xcd < 4) gemm_tile<SL, false>(P, (LAS unsigned char*)shm, xcd * (NTL / 4) + g % (NTL / 4), g / (NTL / 4), call);
        else gemm_tile<SR, true>(P, (LAS unsigned char*)shm, g, xcd - 4, call);
    }
}
}

extern "C" void kernel_launch(int rank, int nranks, ncclComm_t comm, hipStream_t stream,
                              void* const* d_in, const int* in_sizes, int n_in,
                              void* d_out, int out_size, const int* in_dtypes, int out_dtype,
                              void* const* d_out_all, void* const* d_ws_all, size_t ws_bytes) {
    (void)comm; (void)in_dtypes; (void)out_dtype; (void)d_out_all;
    if (nranks != NR || (unsigned)rank >= (unsigned)NR || n_in != 2 || in_sizes[0] != 1024 * KDIM || in_sizes[1] != KDIM * 1024 || out_size != KDIM * 1024 || ws_bytes < WS_NEED) {
        fprintf(stderr, "kernel_launch: unexpected problem geometry\n");
        return;
    }
    static bool attr_set = false;
    if (!attr_set) { HIP_CHECK(hipFuncSetAttribute((const void*)ag_gemm_kernel, hipFuncAttributeMaxDynamicSharedMemorySize, LDS_BYTES)); attr_set = true; }
    Params P;
    P.x = (const char*)d_in[0]; P.w = (const char*)d_in[1]; P.out = (char*)d_out;
    for (int d = 0; d < NR; ++d) P.wsx[d] = (char*)d_ws_all[rank ^ d];
    P.rank = rank; P.pad = 0;
    hipLaunchKernelGGL(ag_gemm_kernel, dim3(GRID), dim3(NTHR), LDS_BYTES, stream, P);
    HIP_CHECK(hipGetLastError());
}
```

```cpp
#include <hip/hip_runtime.h>
#include <rccl/rccl.h>
#include <cstdio>

#define HIP_CHECK(expr) do { hipError_t _e = (expr); if (_e != hipSuccess) { \
    fprintf(stderr, "HIP error %s at %s:%d\n", hipGetErrorString(_e), __FILE__, __LINE__); \
    return; } } while (0)

namespace {
#define LAS __attribute__((address_space(3)))
#define GAS __attribute__((address_space(1)))
typedef short bf16x8 __attribute__((ext_vector_type(8)));
typedef short s16x4 __attribute__((ext_vector_type(4)));
typedef float f32x4 __attribute__((ext_vector_type(4)));
typedef unsigned u32x4 __attribute__((ext_vector_type(4)));
typedef unsigned u32x2 __attribute__((ext_vector_type(2)));

constexpr int NR = 8;
constexpr int KDIM = 8192;
constexpr int APITCH = KDIM * 2, BPITCH = 2048, OPITCH = 2048;
constexpr int EH = 21, OH = 32 - EH;
constexpr int NSLOT = 3 * EH + 4 * OH;
#define SL 7
#define SR 5
constexpr int NTL = (32 + NSLOT + SL - 1) / SL, NTE = (EH + 3 * OH + SR - 1) / SR, NTR = NTE + (3 * EH + SR - 1) / SR, HPT = (SR + 1) / 2;
constexpr int NCH = 64;
__host__ __device__ constexpr int litems(int d) { return (d & 1) ? OH + 8 : EH; }
__host__ __device__ constexpr int lbase(int d) { return NCH * ((d >> 1) * litems(1) + ((d - 1) >> 1) * litems(0)); }
constexpr int NU = lbase(8);
constexpr int ITEM_PK = 6144;
constexpr size_t STRIP_BYTES = (size_t)32 * APITCH;
constexpr size_t XA_BYTES = NSLOT * STRIP_BYTES, W2_BYTES = (size_t)KDIM * BPITCH, PK_BYTES = (size_t)NU * ITEM_PK;
constexpr size_t OFF_XA = 0, OFF_W2 = XA_BYTES, OFF_PK = XA_BYTES + W2_BYTES, PAR_BYTES = OFF_PK + PK_BYTES;
constexpr size_t OFF_FLAG = 2 * PAR_BYTES, FLAG_BYTES = (size_t)NU * 64;
constexpr size_t OFF_CNT = OFF_FLAG + 2 * FLAG_BYTES, CNT_BYTES = (size_t)(NSLOT + 4) * NCH * 128;
constexpr size_t OFF_TICK = OFF_CNT + CNT_BYTES, TICK_BYTES = 2048 * 64;
constexpr int OITEM = 256 + 4096, NOI = 4 * NTR * 8 * HPT;
constexpr size_t OFF_OPK = OFF_TICK + TICK_BYTES, OFF_OFLAG = OFF_OPK + (size_t)NOI * OITEM;
constexpr size_t OFF_TMO = OFF_OFLAG + (size_t)NOI * 64, OFF_QUEUE = OFF_TMO + 128, OFF_DBG = OFF_QUEUE + 8 * 128, DBG_BYTES = 2048 * 64 + NU * 24 + 256 * 64 * 8 + 1024 * 32;
constexpr size_t WS_NEED = OFF_DBG + DBG_BYTES;
constexpr int GRID = 256, NTHR = 512;
#define SEND_WAVES 6
#define SEND_WGS 6
#define BQ 4
#define MIX 0
#define SEND_PIPE 0
#define POLL_SLEEP 2
__host__ __device__ constexpr int ncomm(int x) { return 32 - (x < 4 ? NTL : NTR); }
__host__ __device__ constexpr int nsend(int x) { return x == 3 ? 0 : MIX ? ncomm(x) : SEND_WGS; }
__host__ __device__ constexpr int link_of(int x) { return x < 3 ? 2 * x + 2 : 2 * x - 7; }
__host__ __device__ constexpr int nunp(int x) { return (ncomm(x) - nsend(x)) * 8 + (MIX ? nsend(x) * (8 - SEND_WAVES) : 0); }
__host__ __device__ constexpr int unp_before(int x) { return x == 0 ? 0 : unp_before(x - 1) + nunp(x - 1); }
__host__ __device__ constexpr int snd_before(int x) { return x == 0 ? 0 : snd_before(x - 1) + nsend(x - 1) * SEND_WAVES; }
constexpr int NUW = unp_before(8), NXW = NUW + snd_before(8);
constexpr unsigned long long DEADLINE = 50000000ull;
constexpr int LDS_PACK = SR * 16384, LDS_STAGE = 135168, LDS_BYTES = LDS_STAGE + 16;
static_assert(NTL % 4 == 0 && NTR % BQ == 0 && LDS_PACK + 8 * OITEM <= LDS_STAGE && ncomm(0) >= nsend(0) && ncomm(4) >= nsend(4), "roles");

struct Params {
    const char* x; const char* w;
    char* out;
    char* wsx[NR];
    int rank; int pad;
};

#define WAIT_V(n) asm volatile("s_waitcnt vmcnt(" #n ")" ::: "memory")
#define WAIT_L(n) asm volatile("s_waitcnt lgkmcnt(" #n ")" ::: "memory")
#define BAR __builtin_amdgcn_s_barrier()
#define SCHED __builtin_amdgcn_sched_barrier(0)
#define STAMP(i) do {} while (0)
#define DBG_AT(off) do {} while (0)
#define TS_NOW(dst, k) do {} while (0)

__device__ __forceinline__ unsigned ld_agent(const char* p) { return __hip_atomic_load((const GAS unsigned*)p, __ATOMIC_RELAXED, __HIP_MEMORY_SCOPE_AGENT); }
__device__ __forceinline__ unsigned ld_sys(const char* p) { return __hip_atomic_load((const GAS unsigned*)p, __ATOMIC_RELAXED, __HIP_MEMORY_SCOPE_SYSTEM); }
__device__ __forceinline__ bool give_up(char* ws, unsigned call, unsigned long long t0) {
    if ((unsigned)__builtin_amdgcn_readfirstlane(ld_agent(ws + OFF_TMO)) == call || __builtin_amdgcn_s_memrealtime() - t0 > DEADLINE) {
        __hip_atomic_store((GAS unsigned*)(ws + OFF_TMO), call, __ATOMIC_RELAXED, __HIP_MEMORY_SCOPE_AGENT);
        return true;
    }
    __builtin_amdgcn_s_sleep(POLL_SLEEP);
    return false;
}
__device__ __forceinline__ void store16_sys(char* base, unsigned voff, u32x4 v) {
    asm volatile("global_store_dwordx4 %0, %1, %2 sc0 sc1\n\ts_nop 1" :: "v"(voff), "v"(v), "s"(base) : "memory");
}

__device__ __forceinline__ unsigned below(int a, int b) { unsigned r; asm("s_ashr_i32 %0, %1, 31" : "=s"(r) : "s"(a - b) : "scc"); return r; }
__device__ __forceinline__ unsigned wave_scan(unsigned x) {
    x += __builtin_amdgcn_update_dpp(0u, x, 0x111, 0xf, 0xf, false); x += __builtin_amdgcn_update_dpp(0u, x, 0x112, 0xf, 0xf, false);
    x += __builtin_amdgcn_update_dpp(0u, x, 0x114, 0xf, 0xf, false); x += __builtin_amdgcn_update_dpp(0u, x, 0x118, 0xf, 0xf, false);
    x += __builtin_amdgcn_update_dpp(0u, x, 0x142, 0xa, 0xf, false); x += __builtin_amdgcn_update_dpp(0u, x, 0x143, 0xc, 0xf, false);
    return x;
}

constexpr unsigned RAWG = 0x0f0f0f0fu;
constexpr int CODEC_LUT = 0, CODEC_STREAM = 2048, CODEC_LDS = CODEC_STREAM + 1088;
__device__ __forceinline__ unsigned esc_mask(unsigned c) {
    unsigned e = c & (c >> 1) & 0x05050505u; e |= e >> 7; e |= e >> 12;
    return c == RAWG ? 0u : e & 0xffu;
}
template <bool PACK> __device__ __forceinline__ void build_lut(LAS unsigned char* buf, int lane) {
    for (int m = lane; m < 256; m += 64) {
        unsigned long long sel = 0x0c0c0c0c0c0c0c0cull; int k = 0;
#pragma unroll
        for (int b = 0; b < 8; ++b) if (m >> b & 1) { const int pos = (b & 1) | ((b & 2) << 1) | ((b & 4) >> 1);
            sel ^= PACK ? (unsigned long long)(0x0c ^ pos) << (8 * k) : (unsigned long long)(0x0c ^ k) << (8 * pos); ++k; }
        *(LAS unsigned long long*)(buf + CODEC_LUT + m * 8) = sel;
    }
    asm volatile("" ::: "memory");
}

struct Item { bool isw; unsigned src, fin, cnt, voff, istep; };
__device__ __forceinline__ Item item_of(int d, int i, int lane) {
    const int chunk = (d & 1) ? i / litems(1) : i / litems(0), j = i - chunk * litems(d);
    Item it;
    it.isw = (d & 1) && j >= OH;
    if (it.isw) { it.src = (unsigned)((chunk * 128 + (j - OH) * 16) * BPITCH + (d >> 1) * 512); it.fin = (unsigned)OFF_W2 + it.src;
        it.cnt = (unsigned)(OFF_CNT + ((NSLOT + (d >> 1)) * NCH + chunk) * 128); it.voff = (lane >> 5) * BPITCH + (lane & 31) * 16; it.istep = 2 * BPITCH; }
    else { const int strip = (d & 1) ? EH + j : j, slot = (d & 1) ? 3 * EH + (d >> 1) * OH + j : ((d >> 1) - 1) * EH + j;
        it.src = (unsigned)(strip * STRIP_BYTES + chunk * 256); it.fin = (unsigned)(OFF_XA + slot * STRIP_BYTES + chunk * 256);
        it.cnt = (unsigned)(OFF_CNT + (slot * NCH + chunk) * 128); it.voff = (lane >> 4) * APITCH + (lane & 15) * 16; it.istep = 4 * APITCH; }
    return it;
}
__device__ __forceinline__ unsigned sender(const Params& P, LAS unsigned char* buf, int d, int first, int nwaves, char* ticket) {
    const int lane = threadIdx.x & 63, nlink = NCH * litems(d);
    char* const queue = P.wsx[0] + OFF_QUEUE + d * 128;
    const char* const src_x = P.x; const char* const src_w = P.w;
    u32x4 nxt[8];
    unsigned tick = 0, ctick = 0;
#define FETCH(i) do { const Item _x = item_of(d, (i) < nlink ? (i) : nlink - 1, lane); \
        asm volatile("global_load_dwordx4 %0, %9, %17\n\tglobal_load_dwordx4 %1, %10, %17\n\tglobal_load_dwordx4 %2, %11, %17\n\tglobal_load_dwordx4 %3, %12, %17\n\t" \
                     "global_load_dwordx4 %4, %13, %17\n\tglobal_load_dwordx4 %5, %14, %17\n\tglobal_load_dwordx4 %6, %15, %17\n\tglobal_load_dwordx4 %7, %16, %17\n\t" \
                     "s_mov_b64 exec, 1\n\tglobal_atomic_add %8, %18, %19, %20 sc0\n\ts_mov_b64 exec, -1" \
                     : "=&v"(nxt[0]), "=&v"(nxt[1]), "=&v"(nxt[2]), "=&v"(nxt[3]), "=&v"(nxt[4]), "=&v"(nxt[5]), "=&v"(nxt[6]), "=&v"(nxt[7]), "=&v"(tick) \
                     : "v"(_x.voff), "v"(_x.voff + _x.istep), "v"(_x.voff + 2 * _x.istep), "v"(_x.voff + 3 * _x.istep), "v"(_x.voff + 4 * _x.istep), "v"(_x.voff + 5 * _x.istep), \
                       "v"(_x.voff + 6 * _x.istep), "v"(_x.voff + 7 * _x.istep), "s"((_x.isw ? src_w : src_x) + _x.src), "v"(0u), "v"(1u), "s"(queue) : "memory"); } while (0)
#define WAIT_FETCH(n) asm volatile("s_waitcnt vmcnt(" #n ")" : "+v"(ctick), "+v"(tick), "+v"(nxt[0]), "+v"(nxt[1]), "+v"(nxt[2]), "+v"(nxt[3]), "+v"(nxt[4]), "+v"(nxt[5]), "+v"(nxt[6]), "+v"(nxt[7]) :: "memory")
    int cur = first;
    asm volatile("s_mov_b64 exec, 1\n\tglobal_atomic_add %0, %1, %2, %3 sc0\n\ts_mov_b64 exec, -1" : "=&v"(ctick) : "v"(0u), "v"(1u), "s"(ticket) : "memory");
    FETCH(cur);
    build_lut<true>(buf, lane);
    WAIT_FETCH(0);
    const unsigned call = (unsigned)__builtin_amdgcn_readfirstlane(ctick) + 1u;
    char* const peer = P.wsx[d & 7];
    char* const set = peer + (call & 1) * PAR_BYTES;
    char* const flags = peer + OFF_FLAG + (call & 1) * FLAG_BYTES;
    const unsigned qbase = (call - 1u) * (nlink + nwaves) - nwaves;
    int prev_u = -1; unsigned prev_f = 0; (void)prev_u; (void)prev_f;
    while (cur < nlink) {
        const Item it = item_of(d, cur, lane);
        const int u = lbase(d) + cur;
        u32x4 raw[8];
#pragma unroll
        for (int v = 0; v < 8; ++v) raw[v] = nxt[v];
        const int next = (int)((unsigned)__builtin_amdgcn_readfirstlane(tick) - qbase);
        FETCH(next); SCHED;
        u32x4 pk[6]; unsigned ea[8], eb[8], esum = 0;
#pragma unroll
        for (int v = 0; v < 8; ++v) {
            unsigned r[4];
#pragma unroll
            for (int c = 0; c < 4; ++c) r[c] = ((raw[v][c] << 1) & 0xfffefffeu) | ((raw[v][c] >> 15) & 0x00010001u);
            pk[v >> 1][(v & 1) * 2] = __builtin_amdgcn_perm(r[1], r[0], 0x06040200u);
            pk[v >> 1][(v & 1) * 2 + 1] = __builtin_amdgcn_perm(r[3], r[2], 0x06040200u);
            ea[v] = __builtin_amdgcn_perm(r[1], r[0], 0x07050301u); eb[v] = __builtin_amdgcn_perm(r[3], r[2], 0x07050301u);
            esum = __builtin_amdgcn_sad_u8(ea[v], 0u, esum); esum = __builtin_amdgcn_sad_u8(eb[v], 0u, esum);
        }
        *(LAS u32x4*)(buf + CODEC_STREAM + lane * 16) = (u32x4){0u, 0u, 0u, 0u};
        unsigned top = (((unsigned)__builtin_amdgcn_readlane(wave_scan(esum), 63) + 4095u) >> 12) + 3u;
        top = top < 18u ? 18u : top > 255u ? 255u : top;
        unsigned cg[8], xs[8], cnt[8], tot = 0;
#pragma unroll
        for (int v = 0; v < 8; ++v) {
            const unsigned ta = top * 0x01010101u - ea[v], tb = top * 0x01010101u - eb[v];
            const unsigned ca = __builtin_amdgcn_perm(0x03030302u, 0x01000303u, ta | ((ta & 0x08080808u) << 4)) & 0x03030303u;
            const unsigned cb = __builtin_amdgcn_perm(0x03030302u, 0x01000303u, tb | ((tb & 0x08080808u) << 4)) & 0x03030303u;
            cg[v] = ((ta | tb) & 0xf0f0f0f0u) != 0u ? RAWG : ca | (cb << 2);
            const unsigned m = esc_mask(cg[v]);
            const u32x2 sel = *(const LAS u32x2*)(buf + CODEC_LUT + m * 8);
            const unsigned lo = __builtin_amdgcn_perm(tb, ta, sel.x), hi = __builtin_amdgcn_perm(tb, ta, sel.y);
            xs[v] = __builtin_amdgcn_perm(hi | (hi >> 4), lo | (lo >> 4), 0x06040200u);
            cnt[v] = __builtin_popcount(m); tot += cnt[v];
        }
        const unsigned incl = wave_scan(tot);
        unsigned total = (unsigned)__builtin_amdgcn_readlane(incl, 63);
        if (total > 2048u) {
            total = 0;
#pragma unroll
            for (int v = 0; v < 8; ++v) cg[v] = RAWG;
        } else {
            unsigned pos = (incl - tot) * 4u;
            asm volatile("" ::: "memory");
#pragma unroll
            for (int v = 0; v < 8; ++v) {
                const unsigned long long w = (unsigned long long)xs[v] << (pos & 31u);
                LAS unsigned* q = (LAS unsigned*)(buf + CODEC_STREAM + ((pos >> 5) << 2));
                __hip_atomic_fetch_or(q, (unsigned)w, __ATOMIC_RELAXED, __HIP_MEMORY_SCOPE_WAVEFRONT);
                __hip_atomic_fetch_or(q + 1, (unsigned)(w >> 32), __ATOMIC_RELAXED, __HIP_MEMORY_SCOPE_WAVEFRONT);
                pos += cnt[v] * 4u;
            }
        }
        asm volatile("" ::: "memory");
        pk[5] = *(const LAS u32x4*)(buf + CODEC_STREAM + lane * 16);
        bool anyraw = false;
#pragma unroll
        for (int v = 0; v < 8; ++v) anyraw |= cg[v] == RAWG;
#pragma unroll
        for (int c = 0; c < 4; ++c) pk[4][c] = cg[2 * c] | (cg[2 * c + 1] << 4);
        if (__builtin_amdgcn_ballot_w64(anyraw) != 0ull) {
#pragma unroll
            for (int v = 0; v < 8; ++v) if (cg[v] == RAWG) store16_sys(set + it.fin, it.voff + v * it.istep, raw[v]);
        }
        const unsigned long long smask = __builtin_amdgcn_ballot_w64((unsigned)lane * 32u < total);
        asm volatile("s_mov_b64 exec, %8\n\tglobal_store_dwordx4 %0, %6, %7 offset:3072 sc0 sc1\n\ts_mov_b64 exec, -1\n\t"
                     "global_store_dwordx4 %0, %1, %7 offset:-2048 sc0 sc1\n\tglobal_store_dwordx4 %0, %2, %7 offset:-1024 sc0 sc1\n\t"
                     "global_store_dwordx4 %0, %3, %7 sc0 sc1\n\tglobal_store_dwordx4 %0, %4, %7 offset:1024 sc0 sc1\n\t"
                     "global_store_dwordx4 %0, %5, %7 offset:2048 sc0 sc1\n\ts_nop 1"
                     :: "v"(lane * 16), "v"(pk[0]), "v"(pk[1]), "v"(pk[2]), "v"(pk[3]), "v"(pk[4]), "v"(pk[5]), "s"(set + OFF_PK + (size_t)u * ITEM_PK + 2048), "s"(smask) : "memory");
        const unsigned base = top;
        WAIT_FETCH(0);
        if (lane == 0) __hip_atomic_store((GAS unsigned*)(flags + (size_t)u * 64), (call << 8) | base, __ATOMIC_RELAXED, __HIP_MEMORY_SCOPE_SYSTEM);
        DBG_AT(u * 24 + 16);
        cur = next;
    }
    return call;
}

__device__ __forceinline__ void unpacker(const Params& P, LAS unsigned char* buf, int uw, unsigned call) {
    const int lane = threadIdx.x & 63;
    const int d = uw % 7 + 1, step = (NUW - uw % 7 + 6) / 7, nlink = NCH * litems(d);
    char* ws = P.wsx[0];
    char* set = ws + (call & 1) * PAR_BYTES;
    const char* pkb = set + OFF_PK + (size_t)lbase(d) * ITEM_PK + 2048;
    const char* flags = ws + OFF_FLAG + (call & 1) * FLAG_BYTES + (size_t)lbase(d) * 64;
    build_lut<false>(buf, lane);
    u32x4 nx[6]; unsigned nf = 0;
#pragma unroll
    for (int k = 0; k < 6; ++k) nx[k] = (u32x4){0u, 0u, 0u, 0u};
#define LOAD_ITEM(i, mask) asm volatile("s_mov_b32 exec_lo, %11\n\ts_mov_b32 exec_hi, %11\n\t" \
        "global_load_dwordx4 %0, %7, %8 offset:-2048 sc0 sc1\n\tglobal_load_dwordx4 %1, %7, %8 offset:-1024 sc0 sc1\n\tglobal_load_dwordx4 %2, %7, %8 sc0 sc1\n\t" \
        "global_load_dwordx4 %3, %7, %8 offset:1024 sc0 sc1\n\tglobal_load_dwordx4 %4, %7, %8 offset:2048 sc0 sc1\n\tglobal_load_dwordx4 %5, %7, %8 offset:3072 sc0 sc1\n\t" \
        "global_load_dword %6, %9, %10 sc0 sc1\n\ts_mov_b64 exec, -1" \
        : "+v"(nx[0]), "+v"(nx[1]), "+v"(nx[2]), "+v"(nx[3]), "+v"(nx[4]), "+v"(nx[5]), "+v"(nf) \
        : "v"(lane * 16), "s"(pkb + (size_t)(i) * ITEM_PK), "v"(0u), "s"(flags + (size_t)((i) + step < nlink ? (i) + step : (i)) * 64), "s"(mask) : "memory")
#define WAIT_ITEM() asm volatile("s_waitcnt vmcnt(0)" : "+v"(nx[0]), "+v"(nx[1]), "+v"(nx[2]), "+v"(nx[3]), "+v"(nx[4]), "+v"(nx[5]), "+v"(nf) :: "memory")
    unsigned f = 0; bool have = false;
    for (int i = uw / 7; i < nlink; i += step) {
        if (!have) {
            const unsigned long long t0 = __builtin_amdgcn_s_memrealtime();
            bool dead = false;
            while (((f = (unsigned)__builtin_amdgcn_readfirstlane(ld_sys(flags + (size_t)i * 64))) >> 8) < call)
                if (give_up(ws, call, t0)) { dead = true; break; }
            if (dead) break;
            LOAD_ITEM(i, ~0u); WAIT_ITEM();
        }
        DBG_AT((lbase(d) + i) * 24);
        u32x4 pk[6];
#pragma unroll
        for (int k = 0; k < 6; ++k) pk[k] = nx[k];
        const unsigned top4 = (f & 255u) * 0x01010101u;
        f = (unsigned)__builtin_amdgcn_readfirstlane(nf);
        const unsigned ahead = below(i + step, nlink) & ~below((int)(f >> 8), (int)call);
        have = ahead != 0u;
        LOAD_ITEM(i + step < nlink ? i + step : i, ahead); SCHED;
        *(LAS u32x4*)(buf + CODEC_STREAM + lane * 16) = pk[5];
        unsigned cg[8], m[8], tot = 0;
#pragma unroll
        for (int v = 0; v < 8; ++v) { cg[v] = (pk[4][v >> 1] >> (4 * (v & 1))) & 0x0f0f0f0fu; m[v] = esc_mask(cg[v]); tot += __builtin_popcount(m[v]); }
        unsigned pos = (wave_scan(tot) - tot) * 4u;
        asm volatile("" ::: "memory");
        const Item it = item_of(d, i, lane);
        unsigned xs[8]; u32x2 sels[8];
#pragma unroll
        for (int v = 0; v < 8; ++v) {
            const unsigned p = pos & 8191u;
            const LAS unsigned* q = (const LAS unsigned*)(buf + CODEC_STREAM + ((p >> 5) << 2));
            xs[v] = __builtin_amdgcn_alignbit(q[1], q[0], p & 31u);
            sels[v] = *(const LAS u32x2*)(buf + CODEC_LUT + m[v] * 8);
            pos += __builtin_popcount(m[v]) * 4u;
        }
#pragma unroll
        for (int v = 0; v < 8; ++v) {
            const unsigned x = xs[v]; const u32x2 sel = sels[v];
            const unsigned pl = __builtin_amdgcn_perm(0u, x, 0x01010000u), ph = __builtin_amdgcn_perm(0u, x, 0x03030202u);
            const unsigned lo = ((pl & 0x00ff00ffu) | ((pl >> 4) & 0xff00ff00u)) & 0x0f0f0f0fu, hi = ((ph & 0x00ff00ffu) | ((ph >> 4) & 0xff00ff00u)) & 0x0f0f0f0fu;
            const unsigned e0 = top4 - (__builtin_amdgcn_perm(hi, lo, sel.x) | __builtin_amdgcn_perm(0u, 0x00040302u, cg[v] & 0x03030303u));
            const unsigned e1 = top4 - (__builtin_amdgcn_perm(hi, lo, sel.y) | __builtin_amdgcn_perm(0u, 0x00040302u, (cg[v] >> 2) & 0x03030303u));
            const unsigned m0 = pk[v >> 1][(v & 1) * 2], m1 = pk[v >> 1][(v & 1) * 2 + 1];
            u32x4 r;
            r[0] = __builtin_amdgcn_perm(e0, m0, 0x05010400u); r[1] = __builtin_amdgcn_perm(e0, m0, 0x07030602u);
            r[2] = __builtin_amdgcn_perm(e1, m1, 0x05010400u); r[3] = __builtin_amdgcn_perm(e1, m1, 0x07030602u);
#pragma unroll
            for (int c = 0; c < 4; ++c) r[c] = ((r[c] >> 1) & 0x7fff7fffu) | ((r[c] << 15) & 0x80008000u);
            if (cg[v] != RAWG) store16_sys(set + it.fin, it.voff + v * it.istep, r);
        }
        WAIT_ITEM();
        if (lane == 0) __hip_atomic_fetch_add((GAS unsigned*)(ws + it.cnt), 1u, __ATOMIC_RELAXED, __HIP_MEMORY_SCOPE_AGENT);
        DBG_AT((lbase(d) + i) * 24 + 8);
        if (i == uw / 7) STAMP(1);
    }
}

__device__ __forceinline__ int strip_of(int pos, bool remote, int q) {
    if (!remote) return pos < 32 + NSLOT ? pos : -1;
    if (pos < EH) return pos;
    if (pos < EH + 3 * OH) { const int k = (pos - EH) / OH; return 32 + 3 * EH + (k + (k >= q ? 1 : 0)) * OH + (pos - EH) % OH; }
    return (unsigned)(pos - NTE * SR) < 3u * EH ? 32 + pos - NTE * SR : -1;
}
__device__ __forceinline__ int out_row_of(int id, int rank) {
    if (id < 32) return rank * 1024 + id * 32;
    const int slot = id - 32;
    if (slot < 3 * EH) return (rank ^ (2 * (slot / EH + 1))) * 1024 + (slot % EH) * 32;
    const int o = slot - 3 * EH;
    return (rank ^ (2 * (o / OH) + 1)) * 1024 + (EH + o % OH) * 32;
}

__device__ __forceinline__ void expander(const Params& P, LAS unsigned char* buf, int xw, unsigned call) {
    const int lane = threadIdx.x & 63;
    char* ws = P.wsx[0];
    for (int n = xw; n < NOI; n += NXW) {
        const bool late = n >= NTE * HPT * 32;
        const int n1 = late ? n - NTE * HPT * 32 : n, per = (late ? NTR - NTE : NTE) * 32, h = n1 / per, rt = (late ? NTE : 0) + (n1 - h * per) / 32, w = (n1 >> 2) & 7, q = n1 & 3;
        const int ida = strip_of(rt * SR + 2 * h, true, q), idb = 2 * h + 1 < SR ? strip_of(rt * SR + 2 * h + 1, true, q) : -1;
        if (ida < 0) continue;
        const int o = ((q * NTR + rt) * 8 + w) * HPT + h;
        unsigned f; bool dead = false;
        { const unsigned long long t0 = __builtin_amdgcn_s_memrealtime();
          while (((f = (unsigned)__builtin_amdgcn_readfirstlane(ld_sys(ws + OFF_OFLAG + o * 64))) >> 12) < call)
              if (give_up(ws, call, t0)) { dead = true; break; } }
        u32x4 r[4];
        unsigned tmo = call;
        if (!dead) {
            const unsigned top = 256u + 2u * (f & 0xfffu) - 1u;
            const unsigned lim = (top < (unsigned)OITEM ? top : OITEM - 1u) & ~15u;
            u32x4 t[5]; unsigned a[5];
#pragma unroll
            for (int k = 0; k < 5; ++k) { a[k] = lane * 16 + k * 1024; a[k] = a[k] < lim ? a[k] : lim; }
            asm volatile("global_load_dwordx4 %0, %6, %11 sc0 sc1\n\tglobal_load_dwordx4 %1, %7, %11 sc0 sc1\n\tglobal_load_dwordx4 %2, %8, %11 sc0 sc1\n\t"
                         "global_load_dwordx4 %3, %9, %11 sc0 sc1\n\tglobal_load_dwordx4 %4, %10, %11 sc0 sc1\n\tglobal_load_dword %5, %12, %13 sc1\n\ts_waitcnt vmcnt(0)"
                         : "=&v"(t[0]), "=&v"(t[1]), "=&v"(t[2]), "=&v"(t[3]), "=&v"(t[4]), "=&v"(tmo)
                         : "v"(a[0]), "v"(a[1]), "v"(a[2]), "v"(a[3]), "v"(a[4]), "s"(ws + OFF_OPK + (size_t)o * OITEM), "v"(0u), "s"(ws + OFF_TMO) : "memory");
#pragma unroll
            for (int k = 0; k < 5; ++k) *(LAS u32x4*)(buf + a[k]) = t[k];
            asm volatile("" ::: "memory");
            const unsigned masks = *(const LAS unsigned*)(buf + lane * 4);
            unsigned pos = 0;
#pragma unroll
            for (int j = 0; j < 32; ++j) {
                const unsigned lo = __builtin_amdgcn_readlane(masks, 2 * j), hi = __builtin_amdgcn_readlane(masks, 2 * j + 1);
                unsigned idx = __builtin_amdgcn_mbcnt_hi(hi, __builtin_amdgcn_mbcnt_lo(lo, pos));
                idx = idx < 2047u ? idx : 2047u;
                unsigned val = *(const LAS unsigned short*)(buf + 256 + idx * 2);
                val = __builtin_amdgcn_inverse_ballot_w64(((unsigned long long)hi << 32) | lo) ? val : 0u;
                if (j & 1) r[j >> 3][(j >> 1) & 3] |= val << 16; else r[j >> 3][(j >> 1) & 3] = val;
                pos += __builtin_popcount(lo) + __builtin_popcount(hi);
            }
        }
        if (tmo == call) {
#pragma unroll
            for (int v = 0; v < 4; ++v) r[v] = (u32x4){0x7fc07fc0u, 0x7fc07fc0u, 0x7fc07fc0u, 0x7fc07fc0u};
        }
        const int src = P.rank ^ (2 * q + 1);
        const unsigned ooff = (w * 4 + (lane >> 5)) * OPITCH + q * 512 + (lane & 31) * 16;
        { char* O = P.out + (size_t)out_row_of(ida, src) * OPITCH; store16_sys(O, ooff, r[0]); store16_sys(O, ooff + 2 * OPITCH, r[1]); }
        if (idb >= 0) { char* O = P.out + (size_t)out_row_of(idb, src) * OPITCH; store16_sys(O, ooff, r[2]); store16_sys(O, ooff + 2 * OPITCH, r[3]); }
    }
}

__device__ __forceinline__ unsigned cvt_pk_bf16(float lo, float hi) { unsigned r; asm("v_cvt_pk_bf16_f32 %0, %1, %2" : "=v"(r) : "v"(lo), "v"(hi)); return r; }
template <int OFF> __device__ __forceinline__ s16x4 tr_read(unsigned addr) { s16x4 r; asm volatile("ds_read_b64_tr_b16 %0, %1 offset:%2" : "=v"(r) : "v"(addr), "i"(OFF) : "memory"); return r; }
template <int OFF> __device__ __forceinline__ bf16x8 row_read(unsigned addr) { bf16x8 r; asm volatile("ds_read_b128 %0, %1 offset:%2" : "=v"(r) : "v"(addr), "i"(OFF) : "memory"); return r; }
__device__ __forceinline__ void glds(unsigned ldsaddr, unsigned voff, const char* g) {
    asm volatile("s_mov_b32 m0, %0\n\ts_nop 0\n\tglobal_load_lds_dwordx4 %1, %2 sc0 sc1" :: "s"(ldsaddr), "v"(voff), "s"(g) : "memory");
}
__device__ __forceinline__ void glds_masked(unsigned ldsaddr, unsigned voff, const char* g, unsigned mask) {
    asm volatile("s_mov_b32 m0, %0\n\ts_mov_b32 exec_lo, %3\n\ts_mov_b32 exec_hi, %3\n\tglobal_load_lds_dwordx4 %1, %2 sc0 sc1\n\ts_mov_b64 exec, -1" :: "s"(ldsaddr), "v"(voff), "s"(g), "s"(mask) : "memory");
}

template <int S> struct Frag { bf16x8 a[S], b[4]; };
template <int S> __device__ __forceinline__ void wait_step(Frag<S>& F, unsigned& pre) {
    asm volatile("s_waitcnt vmcnt(6) lgkmcnt(0)" : "+v"(F.a[0]), "+v"(F.a[1]), "+v"(F.a[2]), "+v"(F.a[3]), "+v"(F.a[S - 1]), "+v"(F.b[0]), "+v"(F.b[1]), "+v"(F.b[2]), "+v"(F.b[3]), "+v"(pre) :: "memory");
    if constexpr (S > 5) asm volatile("" : "+v"(F.a[4]));
    if constexpr (S > 6) asm volatile("" : "+v"(F.a[5]));
    if constexpr (S > 7) asm volatile("" : "+v"(F.a[6]));
}
template <int S, bool remote> __device__ __forceinline__ void gemm_tile(const Params& P, LAS unsigned char* lds, int rt, int ct, unsigned call) {
    constexpr int SLOT_A = S * 2048, SLOT = SLOT_A + 16384;
    static_assert(S >= 5 && S <= 8 && 4 * SLOT <= LDS_STAGE && S * 16384 <= LDS_STAGE, "tile");
    int wid = __builtin_amdgcn_readfirstlane(threadIdx.x >> 6), lane = threadIdx.x & 63, wr = wid >> 2, wc = wid & 3, fr = lane & 15, fq = lane >> 4;
    char* ws = P.wsx[0];
    const char* set = ws + (call & 1) * PAR_BYTES;
    const char* const own_x = P.x;
    const unsigned ldst = (unsigned)(size_t)lds + wid * 1024;
    unsigned voffA, voffB;
    { const int k0 = lane >> 4, ch = lane & 15; voffB = (unsigned)((wid * 4 + k0) * BPITCH + (((ch >> 1) ^ (k0 | ((wid & 2) << 1))) * 32) + (ch & 1) * 16);
      const int sb = lane * 16, swz = sb ^ (((sb >> 9) & 1) << 5); voffA = (unsigned)(((wid & 1) * 16 + (swz >> 6)) * APITCH + (swz & 63)); }
    auto strip_ptr = [&](int k) { int id = k < S ? strip_of(rt * S + k, remote, ct) : -1; if (id < 0) id = strip_of(rt * S, remote, ct);
                                  return id < 32 ? own_x + id * STRIP_BYTES : set + OFF_XA + (id - 32) * STRIP_BYTES; };
    const char* const gb = (remote ? set + OFF_W2 : P.w) + ct * 512;
    const char* const ga0 = strip_ptr(wid >> 1);
    const char* const ga1 = strip_ptr(4 + (wid >> 1));
    const unsigned amask = below(wid, 2 * S - 8);
    unsigned aoff[2], boff[2][2];
    { const unsigned lbase = (unsigned)(size_t)lds; const int ob = fr * 64 + fq * 16; aoff[0] = lbase + wr * S * 1024 + (ob ^ (((ob >> 9) & 1) << 5)); aoff[1] = aoff[0] + 2 * SLOT;
      const int q = (lane & 15) >> 2, p = lane & 3, f = q | ((fq & 1) << 2);
      for (int n = 0; n < 2; ++n) { boff[0][n] = lbase + SLOT_A + (8 * fq + q) * 256 + p * 8 + (((wc * 2 + n) ^ f) * 32); boff[1][n] = boff[0][n] + 2 * SLOT; } }
    unsigned coff, cstep, need;
    { const int id = strip_of(rt * S + (lane < S ? lane : 0), remote, ct);
      const bool watch = wid == 0 && (lane < S ? id >= 32 : (lane == 8 && remote));
      coff = watch ? (unsigned)(OFF_CNT + (lane < S ? id - 32 : NSLOT + ct) * NCH * 128) : (unsigned)OFF_TMO;
      cstep = watch ? 128u : 0u;
      need = watch ? call * (lane < S ? 1u : 8u) : 0u; }
    const unsigned w0 = below(wid, 1);
    unsigned pre = 0;
#define SLOW_WAIT() do { const unsigned long long t0 = __builtin_amdgcn_s_memrealtime(); \
        while (__builtin_amdgcn_ballot_w64(ld_agent(ws + coff) < need) != 0ull) if (give_up(ws, call, t0)) break; \
        asm volatile("" ::: "memory"); } while (0)
#define PRE_LOAD(mask) do { coff += cstep; \
        asm volatile("s_mov_b32 exec_lo, %3\n\ts_mov_b32 exec_hi, 0\n\tglobal_load_dword %0, %1, %2 sc1\n\ts_mov_b64 exec, -1" : "+v"(pre) : "v"(coff), "s"(ws), "s"((mask) & w0) : "memory"); } while (0)
#define STAGE_B(h, sl, st) glds(ldst + (sl) * SLOT + SLOT_A + (h) * 8192, voffB, gb + (h) * 256 + (size_t)(st) * (32 * BPITCH))
#define STAGE_A0(sl, st) glds(ldst + (sl) * SLOT, voffA, ga0 + (st) * 64)
#define STAGE_A1(sl, st) glds_masked(ldst + (sl) * SLOT + 8192, voffA, ga1 + (st) * 64, amask)
#define STAGE_ALL(sl, st) do { STAGE_B(0, sl, st); STAGE_B(1, sl, st); STAGE_A0(sl, st); STAGE_A1(sl, st); } while (0)
#define RDA(F, m, sl) F.a[m] = row_read<((sl) & 1) * SLOT + (m) * 1024>(aoff[(sl) >> 1])
#define RDB(F, j, sl) do { const s16x4 _lo = tr_read<((sl) & 1) * SLOT + ((j) >> 1) * 8192>(boff[(sl) >> 1][(j) & 1]), _hi = tr_read<((sl) & 1) * SLOT + ((j) >> 1) * 8192 + 1024>(boff[(sl) >> 1][(j) & 1]); \
        F.b[j] = __builtin_shufflevector(_lo, _hi, 0, 1, 2, 3, 4, 5, 6, 7); } while (0)
#define MM(F, i) acc[(i) >> 2][(i) & 3] = __builtin_amdgcn_mfma_f32_16x16x32_bf16(F.b[(i) & 3], F.a[(i) >> 2], acc[(i) >> 2][(i) & 3], 0, 0, 0)
#define PAIR(C, i) do { MM(C, i); MM(C, (i) + 1); SCHED; } while (0)
#define STEP(C, N, sl, st, GATE) do { constexpr int nx = ((sl) + 1) & 3; \
        RDB(N, 0, nx); PAIR(C, 0); RDB(N, 1, nx); PAIR(C, 2); RDA(N, 0, nx); RDA(N, 1, nx); PAIR(C, 4); RDB(N, 2, nx); PAIR(C, 6); RDB(N, 3, nx); \
        STAGE_B(0, sl, st); PAIR(C, 8); RDA(N, 2, nx); RDA(N, 3, nx); STAGE_B(1, sl, st); PAIR(C, 10); \
        RDA(N, 4, nx); if constexpr (S > 5) RDA(N, 5, nx); STAGE_A0(sl, st); PAIR(C, 12); \
        if constexpr (S > 6) RDA(N, 6, nx); if constexpr (S > 7) RDA(N, 7, nx); STAGE_A1(sl, st); \
        _Pragma("unroll") for (int i = 14; i < 4 * S - 4; i += 2) PAIR(C, i); \
        wait_step(N, pre); GATE; BAR; SCHED; PAIR(C, 4 * S - 4); PAIR(C, 4 * S - 2); } while (0)
    f32x4 acc[S][4];
#pragma unroll
    for (int m = 0; m < S; ++m)
#pragma unroll
        for (int j = 0; j < 4; ++j) acc[m][j] = (f32x4){0.f, 0.f, 0.f, 0.f};
    Frag<S> F0, F1;
    if (wid == 0) SLOW_WAIT();
    BAR;
    STAMP(1);
    PRE_LOAD(0x3ffu);
    STAGE_ALL(0, 0); STAGE_ALL(1, 1); STAGE_ALL(2, 2); STAGE_ALL(3, 3);
    WAIT_V(6); BAR;
    RDB(F0, 0, 0); RDB(F0, 1, 0); RDB(F0, 2, 0); RDB(F0, 3, 0);
    RDA(F0, 0, 0); RDA(F0, 1, 0); RDA(F0, 2, 0); RDA(F0, 3, 0); RDA(F0, 4, 0);
    if constexpr (S > 5) RDA(F0, 5, 0);
    if constexpr (S > 6) RDA(F0, 6, 0);
    if constexpr (S > 7) RDA(F0, 7, 0);
    wait_step(F0, pre);
    if (__builtin_amdgcn_ballot_w64(pre < need) != 0ull) SLOW_WAIT();
    BAR;
    for (int c = 0; c < NCH; ++c) {
        const int st = (c < NCH - 1 ? c + 1 : c) * 4;
        STEP(F0, F1, 0, st, (void)0);
        STEP(F1, F0, 1, st + 1, (void)0);
        PRE_LOAD(c < NCH - 2 ? 0x3ffu : c < NCH - 1 ? 0x200u : 0u);
        STEP(F0, F1, 2, st + 2, (void)0);
        STEP(F1, F0, 3, st + 3, { TS_NOW(ts0, c); if (c < NCH - 2 && __builtin_amdgcn_ballot_w64(pre < need) != 0ull) SLOW_WAIT(); TS_NOW(ts1, c); });
    }
    if (threadIdx.x == 9) *(LAS unsigned*)(lds + LDS_STAGE) = pre;
    asm volatile("s_waitcnt vmcnt(0) lgkmcnt(0)" ::: "memory");
    BAR;
    asm volatile("" ::: "memory");
    const bool bad = *(const LAS unsigned*)(lds + LDS_STAGE) == call;
    STAMP(2);
    int tz = threadIdx.x; asm volatile("" : "+v"(tz));
    lane = tz & 63; fr = lane & 15; fq = lane >> 4;
#pragma unroll
    for (int m = 0; m < S; ++m) {
        const int row = wr * S * 16 + m * 16 + fr;
#pragma unroll
        for (int j = 0; j < 4; ++j) {
            f32x4 v = acc[m][j];
#pragma unroll
            for (int e = 0; e < 4; ++e) v[e] = v[e] < 0.f ? 0.f : v[e];
            u32x2 w; w.x = cvt_pk_bf16(v[0], v[1]); w.y = cvt_pk_bf16(v[2], v[3]);
            if (bad) { w.x = 0x7fc07fc0u; w.y = 0x7fc07fc0u; }
            const int ch = (j >> 1) * 16 + wc * 4 + (j & 1) * 2 + (fq >> 1);
            *(LAS u32x2*)(lds + row * 512 + ((ch ^ (fr & 7)) * 16) + (fq & 1) * 8) = w;
        }
    }
    WAIT_L(0); BAR; asm volatile("" ::: "memory");
    u32x4 ov[S][2];
#pragma unroll
    for (int k = 0; k < S; ++k)
#pragma unroll
        for (int i = 0; i < 2; ++i) { const int row = k * 32 + wid * 4 + i * 2 + (lane >> 5); ov[k][i] = *(const LAS u32x4*)(lds + row * 512 + (((lane & 31) ^ (row & 7)) * 16)); }
    if constexpr (!remote) {
        const unsigned ooff = (wid * 4 + (lane >> 5)) * OPITCH + ct * 512 + (lane & 31) * 16;
#pragma unroll
        for (int k = 0; k < S; ++k) {
            const int id = strip_of(rt * S + k, false, 0);
            if (id >= 0) { char* O = P.out + (size_t)out_row_of(id, P.rank) * OPITCH; store16_sys(O, ooff, ov[k][0]); store16_sys(O, ooff + 2 * OPITCH, ov[k][1]); }
        }
    } else {
        char* const peer = P.wsx[(2 * ct + 1) & 7];
        LAS unsigned char* buf = lds + LDS_PACK + wid * OITEM;
        const int o0 = ((ct * NTR + rt) * 8 + wid) * HPT;
#pragma unroll
        for (int h = 0; h < HPT; ++h) {
            if (strip_of(rt * S + 2 * h, true, ct) < 0) break;
            const bool second = 2 * h + 1 < S && strip_of(rt * S + 2 * h + 1, true, ct) >= 0;
            unsigned masks = 0, pos = 0;
#pragma unroll
            for (int j = 0; j < 32; ++j) {
                const unsigned dw = ov[2 * h + (j >> 4) < S ? 2 * h + (j >> 4) : 0][(j >> 3) & 1][(j >> 1) & 3];
                const unsigned val = (j & 1) ? dw >> 16 : dw & 0xffffu;
                const bool nz = val != 0u && (j < 16 || second);
                const unsigned long long m = __builtin_amdgcn_ballot_w64(nz);
                const unsigned lo = (unsigned)m, hi = (unsigned)(m >> 32);
                const unsigned idx = __builtin_amdgcn_mbcnt_hi(hi, __builtin_amdgcn_mbcnt_lo(lo, pos));
                if (nz) *(LAS unsigned short*)(buf + 256 + idx * 2) = (unsigned short)val;
                asm("s_nop 1\n\tv_writelane_b32 %0, %1, %3\n\tv_writelane_b32 %0, %2, %4" : "+v"(masks) : "s"(lo), "s"(hi), "i"(2 * j), "i"(2 * j + 1));
                pos += __builtin_popcount(lo) + __builtin_popcount(hi);
            }
            *(LAS unsigned*)(buf + lane * 4) = masks;
            asm volatile("" ::: "memory");
            const unsigned total = 256u + 2u * pos;
            u32x4 t[5];
#pragma unroll
            for (int k = 0; k < 5; ++k) { const unsigned a = lane * 16 + k * 1024; t[k] = *(const LAS u32x4*)(buf + (a < (unsigned)OITEM ? a : 0u)); }
            char* dst = peer + OFF_OPK + (size_t)(o0 + h) * OITEM;
#pragma unroll
            for (int k = 0; k < 5; ++k) if (lane * 16 + k * 1024 < total) store16_sys(dst, lane * 16 + k * 1024, t[k]);
            WAIT_V(0);
            if (lane == 0) __hip_atomic_store((GAS unsigned*)(peer + OFF_OFLAG + (o0 + h) * 64), (call << 12) | pos, __ATOMIC_RELAXED, __HIP_MEMORY_SCOPE_SYSTEM);
        }
    }
    STAMP(3);
}

__global__ __launch_bounds__(NTHR, 2) void ag_gemm_kernel(const Params P) {
    extern __shared__ __attribute__((aligned(16))) unsigned char shm[];
    const int xcd = blockIdx.x & 7, slot = blockIdx.x >> 3, wid = __builtin_amdgcn_readfirstlane(threadIdx.x >> 6);
    char* const ticket = P.wsx[0] + OFF_TICK + (size_t)(blockIdx.x * 8 + wid) * 64;
    STAMP(0);
    int ns = 0, nc = 0, ub = 0, sb = 0;
#pragma unroll
    for (int y = 0; y < 8; ++y) if (y == xcd) { ns = nsend(y); nc = ncomm(y); ub = unp_before(y); sb = snd_before(y); }
    if (slot < ns && wid < SEND_WAVES) {
        const unsigned call = sender(P, (LAS unsigned char*)shm + wid * OITEM, link_of(xcd), wid * ns + slot, ns * SEND_WAVES, ticket);
        STAMP(2);
        expander(P, (LAS unsigned char*)shm + wid * OITEM, NUW + sb + wid * ns + slot, call);
        STAMP(3);
        return;
    }
    if (!MIX && slot < ns) return;
    unsigned tick = 0;
    if ((threadIdx.x & 63) == 0) tick = __hip_atomic_fetch_add((GAS unsigned*)ticket, 1u, __ATOMIC_RELAXED, __HIP_MEMORY_SCOPE_AGENT);
    const unsigned call = (unsigned)__builtin_amdgcn_readfirstlane(tick) + 1u;
    if (slot < nc) {
        const int uw = ub + (slot < ns ? slot * (8 - SEND_WAVES) + wid - SEND_WAVES : (MIX ? ns * (8 - SEND_WAVES) : 0) + (slot - ns) * 8 + wid);
        unpacker(P, (LAS unsigned char*)shm + wid * OITEM, uw, call);
        STAMP(2);
        expander(P, (LAS unsigned char*)shm + wid * OITEM, uw, call);
        STAMP(3);
    } else {
        const int g = slot - nc;
                if (xcd < 4) gemm_tile<SL, false>(P, (LAS unsigned char*)shm, xcd * (NTL / 4) + g % (NTL / 4), g / (NTL / 4), call);
        else {
            const int y = xcd - 4;
            gemm_tile<SR, true>(P, (LAS unsigned char*)shm, (g / BQ) * BQ + y % BQ, (y / BQ) * BQ + g % BQ, call);
        }
    }
}
}

extern "C" void kernel_launch(int rank, int nranks, ncclComm_t comm, hipStream_t stream,
                              void* const* d_in, const int* in_sizes, int n_in,
                              void* d_out, int out_size, const int* in_dtypes, int out_dtype,
                              void* const* d_out_all, void* const* d_ws_all, size_t ws_bytes) {
    (void)comm; (void)in_dtypes; (void)out_dtype; (void)d_out_all;
    if (nranks != NR || (unsigned)rank >= (unsigned)NR || n_in != 2 || in_sizes[0] != 1024 * KDIM || in_sizes[1] != KDIM * 1024 || out_size != KDIM * 1024 || ws_bytes < WS_NEED) {
        fprintf(stderr, "kernel_launch: unexpected problem geometry\n");
        return;
    }
    static bool attr_set = false;
    if (!attr_set) { HIP_CHECK(hipFuncSetAttribute((const void*)ag_gemm_kernel, hipFuncAttributeMaxDynamicSharedMemorySize, LDS_BYTES)); attr_set = true; }
    Params P;
    P.x = (const char*)d_in[0]; P.w = (const char*)d_in[1]; P.out = (char*)d_out;
    for (int d = 0; d < NR; ++d) P.wsx[d] = (char*)d_ws_all[rank ^ d];
    P.rank = rank; P.pad = 0;
    hipLaunchKernelGGL(ag_gemm_kernel, dim3(GRID), dim3(NTHR), LDS_BYTES, stream, P);
    HIP_CHECK(hipGetLastError());
}
```

```cpp
#include <hip/hip_runtime.h>
#include <rccl/rccl.h>
#include <cstdio>

#define HIP_CHECK(expr) do { hipError_t _e = (expr); if (_e != hipSuccess) { \
    fprintf(stderr, "HIP error %s at %s:%d\n", hipGetErrorString(_e), __FILE__, __LINE__); \
    return; } } while (0)

namespace {
#define LAS __attribute__((address_space(3)))
#define GAS __attribute__((address_space(1)))
typedef short bf16x8 __attribute__((ext_vector_type(8)));
typedef short s16x4 __attribute__((ext_vector_type(4)));
typedef float f32x4 __attribute__((ext_vector_type(4)));
typedef unsigned u32x4 __attribute__((ext_vector_type(4)));
typedef unsigned u32x2 __attribute__((ext_vector_type(2)));

constexpr int NR = 8;
constexpr int KDIM = 8192;
constexpr int APITCH = KDIM * 2, BPITCH = 2048, OPITCH = 2048;
constexpr int EH = 21, OH = 32 - EH;
constexpr int NSLOT = 3 * EH + 4 * OH;
#define SL 7
#define SR 5
constexpr int NTL = (32 + NSLOT + SL - 1) / SL, NTE = (EH + 3 * OH + SR - 1) / SR, NTR = NTE + (3 * EH + SR - 1) / SR, HPT = (SR + 1) / 2;
constexpr int NCH = 64;
__host__ __device__ constexpr int litems(int d) { return (d & 1) ? OH + 8 : EH; }
__host__ __device__ constexpr int lbase(int d) { return NCH * ((d >> 1) * litems(1) + ((d - 1) >> 1) * litems(0)); }
constexpr int NU = lbase(8);
constexpr int ITEM_PK = 6144;
constexpr size_t STRIP_BYTES = (size_t)32 * APITCH;
constexpr size_t XA_BYTES = NSLOT * STRIP_BYTES, W2_BYTES = (size_t)KDIM * BPITCH, PK_BYTES = (size_t)NU * ITEM_PK;
constexpr size_t OFF_XA = 0, OFF_W2 = XA_BYTES, OFF_PK = XA_BYTES + W2_BYTES, PAR_BYTES = OFF_PK + PK_BYTES;
constexpr size_t OFF_FLAG = 2 * PAR_BYTES, FLAG_BYTES = (size_t)NU * 64;
constexpr size_t OFF_CNT = OFF_FLAG + 2 * FLAG_BYTES, CNT_BYTES = (size_t)(NSLOT + 4) * NCH * 128;
constexpr size_t OFF_TICK = OFF_CNT + CNT_BYTES, TICK_BYTES = 2048 * 64;
constexpr int OITEM = 256 + 4096, NOI = 4 * NTR * 8 * HPT;
constexpr size_t OFF_OPK = OFF_TICK + TICK_BYTES, OFF_OFLAG = OFF_OPK + (size_t)NOI * OITEM;
constexpr size_t OFF_TMO = OFF_OFLAG + (size_t)NOI * 64, OFF_QUEUE = OFF_TMO + 128, OFF_SINK = OFF_QUEUE + 8 * 128, OFF_DBG = OFF_SINK + 512 * 1024, DBG_OUT = NU * 24 + 256 * 64 * 8, DBG_BYTES = 2048 * 64 + DBG_OUT + NOI * 32 + 1024 * 32;
constexpr size_t WS_NEED = OFF_DBG + DBG_BYTES;
constexpr int GRID = 256, NTHR = 512;
#define SEND_WAVES 5
#define SEND_WGS 6
#define GLF " sc0 sc1"
#define BQ 4
#define MIX 0
#define SEND_PIPE 0
#define POLL_SLEEP 2
__host__ __device__ constexpr int ncomm(int x) { return 32 - (x < 4 ? NTL : NTR); }
__host__ __device__ constexpr int nsend(int x) { return x == 3 ? 0 : MIX ? ncomm(x) : SEND_WGS; }
__host__ __device__ constexpr int link_of(int x) { return x < 3 ? 2 * x + 2 : 2 * x - 7; }
__host__ __device__ constexpr int nunp(int x) { return (ncomm(x) - nsend(x)) * 8 + (MIX ? nsend(x) * (8 - SEND_WAVES) : 0); }
__host__ __device__ constexpr int unp_before(int x) { return x == 0 ? 0 : unp_before(x - 1) + nunp(x - 1); }
__host__ __device__ constexpr int snd_before(int x) { return x == 0 ? 0 : snd_before(x - 1) + nsend(x - 1) * (MIX ? SEND_WAVES : 8); }
constexpr int NUW = unp_before(8), NXW = NUW + snd_before(8);
constexpr unsigned long long DEADLINE = 50000000ull;
constexpr int LDS_STAGE = 135168, LDS_BYTES = LDS_STAGE + 16;
static_assert(NUW <= 512 && NTL % 4 == 0 && NTR % BQ == 0 && 8 * OITEM <= LDS_STAGE && ncomm(0) >= nsend(0) && ncomm(4) >= nsend(4), "roles");

struct Params {
    const char* x; const char* w;
    char* out;
    char* wsx[NR];
    int rank; int pad;
};

#define WAIT_V(n) asm volatile("s_waitcnt vmcnt(" #n ")" ::: "memory")
#define WAIT_L(n) asm volatile("s_waitcnt lgkmcnt(" #n ")" ::: "memory")
#define BAR __builtin_amdgcn_s_barrier()
#define SCHED __builtin_amdgcn_sched_barrier(0)
#define STAMP(i) do {} while (0)
#define CSTAMP(i) do {} while (0)
#define DBG_AT(off) do {} while (0)
#define TS_NOW(dst, k) do {} while (0)

__device__ __forceinline__ unsigned ld_agent(const char* p) { return __hip_atomic_load((const GAS unsigned*)p, __ATOMIC_RELAXED, __HIP_MEMORY_SCOPE_AGENT); }
__device__ __forceinline__ unsigned ld_sys(const char* p) { return __hip_atomic_load((const GAS unsigned*)p, __ATOMIC_RELAXED, __HIP_MEMORY_SCOPE_SYSTEM); }
__device__ __forceinline__ bool give_up(char* ws, unsigned call, unsigned long long t0) {
    if ((unsigned)__builtin_amdgcn_readfirstlane(ld_agent(ws + OFF_TMO)) == call || __builtin_amdgcn_s_memrealtime() - t0 > DEADLINE) {
        __hip_atomic_store((GAS unsigned*)(ws + OFF_TMO), call, __ATOMIC_RELAXED, __HIP_MEMORY_SCOPE_AGENT);
        return true;
    }
    __builtin_amdgcn_s_sleep(POLL_SLEEP);
    return false;
}
__device__ __forceinline__ void store16_sys(char* base, unsigned voff, u32x4 v) {
    asm volatile("global_store_dwordx4 %0, %1, %2 sc0 sc1\n\ts_nop 1" :: "v"(voff), "v"(v), "s"(base) : "memory");
}

__device__ __forceinline__ unsigned below(int a, int b) { unsigned r; asm("s_ashr_i32 %0, %1, 31" : "=s"(r) : "s"(a - b) : "scc"); return r; }
__device__ __forceinline__ unsigned wave_scan(unsigned x) {
    x += __builtin_amdgcn_update_dpp(0u, x, 0x111, 0xf, 0xf, false); x += __builtin_amdgcn_update_dpp(0u, x, 0x112, 0xf, 0xf, false);
    x += __builtin_amdgcn_update_dpp(0u, x, 0x114, 0xf, 0xf, false); x += __builtin_amdgcn_update_dpp(0u, x, 0x118, 0xf, 0xf, false);
    x += __builtin_amdgcn_update_dpp(0u, x, 0x142, 0xa, 0xf, false); x += __builtin_amdgcn_update_dpp(0u, x, 0x143, 0xc, 0xf, false);
    return x;
}

constexpr int CODEC_LUT = 0, CODEC_STREAM = 2048, CODEC_LDS = CODEC_STREAM + 1088;
static_assert(CODEC_LDS <= OITEM, "per-wave LDS");
__device__ __forceinline__ unsigned esc_bits(unsigned c) { return c == 0u ? 0u : ~(c | (c >> 1)) & 0x05050505u; }
__device__ __forceinline__ u32x2 lut_of(unsigned e, unsigned lut) {
    const LAS unsigned* q = (const LAS unsigned*)(size_t)__builtin_amdgcn_udot4(e, 0x80400804u, lut, false);
    return (u32x2){q[0], q[256]};
}
template <bool PACK> __device__ __forceinline__ void build_lut(LAS unsigned char* buf, int lane) {
    for (int m = lane; m < 256; m += 64) {
        unsigned long long sel = 0x0c0c0c0c0c0c0c0cull; int k = 0;
#pragma unroll
        for (int b = 0; b < 8; ++b) if (m >> b & 1) { const int pos = (b & 1) | ((b & 2) << 1) | ((b & 4) >> 1), slot = (k >> 1) | ((k & 1) << 2);
            sel ^= PACK ? (unsigned long long)(0x0c ^ pos) << (8 * slot) : (unsigned long long)(0x0c ^ slot) << (8 * pos); ++k; }
        *(LAS unsigned*)(buf + CODEC_LUT + m * 4) = (unsigned)sel; *(LAS unsigned*)(buf + CODEC_LUT + 1024 + m * 4) = (unsigned)(sel >> 32);
    }
    asm volatile("" ::: "memory");
}
__device__ __forceinline__ unsigned bfi(unsigned m, unsigned a, unsigned b) { return (a & m) | (b & ~m); }

struct Item { bool isw; unsigned src, fin, cnt, voff, istep; };
__device__ __forceinline__ Item item_of(int d, int i, int lane) {
    const int chunk = (d & 1) ? i / litems(1) : i / litems(0), j = i - chunk * litems(d);
    Item it;
    it.isw = (d & 1) && j >= OH;
    if (it.isw) { it.src = (unsigned)((chunk * 128 + (j - OH) * 16) * BPITCH + (d >> 1) * 512); it.fin = (unsigned)OFF_W2 + it.src;
        it.cnt = (unsigned)(OFF_CNT + ((NSLOT + (d >> 1)) * NCH + chunk) * 128); it.voff = (lane >> 5) * BPITCH + (lane & 31) * 16; it.istep = 2 * BPITCH; }
    else { const int strip = (d & 1) ? EH + j : j, slot = (d & 1) ? 3 * EH + (d >> 1) * OH + j : ((d >> 1) - 1) * EH + j;
        it.src = (unsigned)(strip * STRIP_BYTES + chunk * 256); it.fin = (unsigned)(OFF_XA + slot * STRIP_BYTES + chunk * 256);
        it.cnt = (unsigned)(OFF_CNT + (slot * NCH + chunk) * 128); it.voff = (lane >> 4) * APITCH + (lane & 15) * 16; it.istep = 4 * APITCH; }
    return it;
}
__device__ __forceinline__ unsigned sender(const Params& P, LAS unsigned char* buf, int d, int first, int nwaves, char* ticket) {
    const int lane = threadIdx.x & 63, nlink = NCH * litems(d);
    char* const queue = P.wsx[0] + OFF_QUEUE + d * 128;
    const char* const src_x = P.x; const char* const src_w = P.w;
    const unsigned lds0 = (unsigned)(size_t)buf;
    u32x4 nxt[8];
    unsigned tick = 0, ctick = 0;
#define FETCH(i) do { const Item _x = item_of(d, (i) < nlink ? (i) : nlink - 1, lane); \
        asm volatile("global_load_dwordx4 %0, %9, %17\n\tglobal_load_dwordx4 %1, %10, %17\n\tglobal_load_dwordx4 %2, %11, %17\n\tglobal_load_dwordx4 %3, %12, %17\n\t" \
                     "global_load_dwordx4 %4, %13, %17\n\tglobal_load_dwordx4 %5, %14, %17\n\tglobal_load_dwordx4 %6, %15, %17\n\tglobal_load_dwordx4 %7, %16, %17\n\t" \
                     "s_mov_b64 exec, 1\n\tglobal_atomic_add %8, %18, %19, %20 sc0\n\ts_mov_b64 exec, -1" \
                     : "=&v"(nxt[0]), "=&v"(nxt[1]), "=&v"(nxt[2]), "=&v"(nxt[3]), "=&v"(nxt[4]), "=&v"(nxt[5]), "=&v"(nxt[6]), "=&v"(nxt[7]), "=&v"(tick) \
                     : "v"(_x.voff), "v"(_x.voff + _x.istep), "v"(_x.voff + 2 * _x.istep), "v"(_x.voff + 3 * _x.istep), "v"(_x.voff + 4 * _x.istep), "v"(_x.voff + 5 * _x.istep), \
                       "v"(_x.voff + 6 * _x.istep), "v"(_x.voff + 7 * _x.istep), "s"((_x.isw ? src_w : src_x) + _x.src), "v"(0u), "v"(1u), "s"(queue) : "memory"); } while (0)
#define WAIT_FETCH(n) asm volatile("s_waitcnt vmcnt(" #n ")" : "+v"(ctick), "+v"(tick), "+v"(nxt[0]), "+v"(nxt[1]), "+v"(nxt[2]), "+v"(nxt[3]), "+v"(nxt[4]), "+v"(nxt[5]), "+v"(nxt[6]), "+v"(nxt[7]) :: "memory")
    int cur = first;
    asm volatile("s_mov_b64 exec, 1\n\tglobal_atomic_add %0, %1, %2, %3 sc0\n\ts_mov_b64 exec, -1" : "=&v"(ctick) : "v"(0u), "v"(1u), "s"(ticket) : "memory");
    FETCH(cur);
    build_lut<true>(buf, lane);
    WAIT_FETCH(0);
    const unsigned call = (unsigned)__builtin_amdgcn_readfirstlane(ctick) + 1u;
    char* const peer = P.wsx[d & 7];
    char* const set = peer + (call & 1) * PAR_BYTES;
    char* const flags = peer + OFF_FLAG + (call & 1) * FLAG_BYTES;
    const unsigned qbase = (call - 1u) * (nlink + nwaves) - nwaves;
    int prev_u = -1; unsigned prev_f = 0; (void)prev_u; (void)prev_f;
    while (cur < nlink) {
        const Item it = item_of(d, cur, lane);
        const int u = lbase(d) + cur;
        u32x4 raw[8];
#pragma unroll
        for (int v = 0; v < 8; ++v) raw[v] = nxt[v];
        const int next = (int)((unsigned)__builtin_amdgcn_readfirstlane(tick) - qbase);
        FETCH(next); SCHED;
        u32x4 pk[6]; unsigned ex[16], esum = 0;
#pragma unroll
        for (int g = 0; g < 16; ++g) {
            const unsigned r0 = raw[g >> 1][(g & 1) * 2], r1 = raw[g >> 1][(g & 1) * 2 + 1];
            const unsigned lo = __builtin_amdgcn_perm(r1, r0, 0x06040200u), hi = __builtin_amdgcn_perm(r1, r0, 0x07050301u);
            pk[g >> 2][g & 3] = bfi(0x80808080u, hi, lo);
            ex[g] = bfi(0x01010101u, lo >> 7, hi << 1);
            esum = __builtin_amdgcn_sad_u8(ex[g], 0u, esum);
        }
        *(LAS u32x4*)(buf + CODEC_STREAM + lane * 16) = (u32x4){0u, 0u, 0u, 0u};
        unsigned top = (((unsigned)__builtin_amdgcn_readlane(wave_scan(esum), 63) + 4095u) >> 12) + 2u;
        top = top < 18u ? 18u : top > 255u ? 255u : top;
        unsigned cg[8], xs[8], at[8], tot = 0;
#pragma unroll
        for (int v = 0; v < 8; ++v) {
            const unsigned ua = top * 0x01010101u - ex[2 * v], ub = top * 0x01010101u - ex[2 * v + 1];
            const unsigned ca = __builtin_amdgcn_perm(0u, 0x03020100u, (((ua >> 1) & 0x04040404u) | ua) & 0x07070707u);
            const unsigned cb = __builtin_amdgcn_perm(0u, 0x0c080400u, (((ub >> 1) & 0x04040404u) | ub) & 0x07070707u);
            cg[v] = ((ua | ub) & 0xf0f0f0f0u) != 0u ? 0u : ca | cb;
            const unsigned e = esc_bits(cg[v]);
            const u32x2 sel = lut_of(e, lds0 + CODEC_LUT);
            xs[v] = __builtin_amdgcn_perm(ub, ua, sel.x) | (__builtin_amdgcn_perm(ub, ua, sel.y) << 4);
            at[v] = tot; tot += __builtin_popcount(e);
        }
        const unsigned incl = wave_scan(tot);
        unsigned total = (unsigned)__builtin_amdgcn_readlane(incl, 63);
        if (total > 2048u) {
            total = 0;
#pragma unroll
            for (int v = 0; v < 8; ++v) cg[v] = 0u;
        } else {
            const unsigned n0 = incl - tot + (lds0 + CODEC_STREAM) * 2u;
            asm volatile("" ::: "memory");
#pragma unroll
            for (int v = 0; v < 8; ++v) {
                const unsigned n = n0 + at[v];
                const unsigned long long w = (unsigned long long)xs[v] << ((n * 4u) & 31u);
                LAS unsigned* q = (LAS unsigned*)(size_t)((n >> 1) & ~3u);
                __hip_atomic_fetch_or(q, (unsigned)w, __ATOMIC_RELAXED, __HIP_MEMORY_SCOPE_WAVEFRONT);
                __hip_atomic_fetch_or(q + 1, (unsigned)(w >> 32), __ATOMIC_RELAXED, __HIP_MEMORY_SCOPE_WAVEFRONT);
            }
        }
        asm volatile("" ::: "memory");
        pk[5] = *(const LAS u32x4*)(buf + CODEC_STREAM + lane * 16);
        unsigned least = cg[0];
#pragma unroll
        for (int v = 1; v < 8; ++v) least = cg[v] < least ? cg[v] : least;
#pragma unroll
        for (int c = 0; c < 4; ++c) pk[4][c] = cg[2 * c] | (cg[2 * c + 1] << 4);
        if (__builtin_amdgcn_ballot_w64(least == 0u) != 0ull) {
#pragma unroll
            for (int v = 0; v < 8; ++v) if (cg[v] == 0u) store16_sys(set + it.fin, it.voff + v * it.istep, raw[v]);
        }
        const unsigned long long smask = __builtin_amdgcn_ballot_w64((unsigned)lane * 32u < total);
        asm volatile("s_mov_b64 exec, %8\n\tglobal_store_dwordx4 %0, %6, %7 offset:3072 sc0 sc1\n\ts_mov_b64 exec, -1\n\t"
                     "global_store_dwordx4 %0, %1, %7 offset:-2048 sc0 sc1\n\tglobal_store_dwordx4 %0, %2, %7 offset:-1024 sc0 sc1\n\t"
                     "global_store_dwordx4 %0, %3, %7 sc0 sc1\n\tglobal_store_dwordx4 %0, %4, %7 offset:1024 sc0 sc1\n\t"
                     "global_store_dwordx4 %0, %5, %7 offset:2048 sc0 sc1\n\ts_nop 1"
                     :: "v"(lane * 16), "v"(pk[0]), "v"(pk[1]), "v"(pk[2]), "v"(pk[3]), "v"(pk[4]), "v"(pk[5]), "s"(set + OFF_PK + (size_t)u * ITEM_PK + 2048), "s"(smask) : "memory");
        const unsigned base = top;
        WAIT_FETCH(0);
        if (lane == 0) __hip_atomic_store((GAS unsigned*)(flags + (size_t)u * 64), (call << 8) | base, __ATOMIC_RELAXED, __HIP_MEMORY_SCOPE_SYSTEM);
        DBG_AT(u * 24 + 16);
        cur = next;
    }
    return call;
}

__device__ __forceinline__ void unpacker(const Params& P, LAS unsigned char* buf, int uw, unsigned call) {
    const int lane = threadIdx.x & 63;
    const int d = uw % 7 + 1, step = (NUW - uw % 7 + 6) / 7, nlink = NCH * litems(d);
    char* ws = P.wsx[0];
    char* set = ws + (call & 1) * PAR_BYTES;
    const char* pkb = set + OFF_PK + (size_t)lbase(d) * ITEM_PK + 2048;
    const char* flags = ws + OFF_FLAG + (call & 1) * FLAG_BYTES + (size_t)lbase(d) * 64;
    const unsigned lds0 = (unsigned)(size_t)buf;
    build_lut<false>(buf, lane);
    u32x4 nx[6]; unsigned nf = 0;
#pragma unroll
    for (int k = 0; k < 6; ++k) nx[k] = (u32x4){0u, 0u, 0u, 0u};
#define LOAD_ITEM(i, mask) asm volatile("s_mov_b32 exec_lo, %11\n\ts_mov_b32 exec_hi, %11\n\t" \
        "global_load_dwordx4 %0, %7, %8 offset:-2048 sc0 sc1\n\tglobal_load_dwordx4 %1, %7, %8 offset:-1024 sc0 sc1\n\tglobal_load_dwordx4 %2, %7, %8 sc0 sc1\n\t" \
        "global_load_dwordx4 %3, %7, %8 offset:1024 sc0 sc1\n\tglobal_load_dwordx4 %4, %7, %8 offset:2048 sc0 sc1\n\tglobal_load_dwordx4 %5, %7, %8 offset:3072 sc0 sc1\n\t" \
        "global_load_dword %6, %9, %10 sc0 sc1\n\ts_mov_b64 exec, -1" \
        : "+v"(nx[0]), "+v"(nx[1]), "+v"(nx[2]), "+v"(nx[3]), "+v"(nx[4]), "+v"(nx[5]), "+v"(nf) \
        : "v"(lane * 16), "s"(pkb + (size_t)(i) * ITEM_PK), "v"(0u), "s"(flags + (size_t)((i) + step < nlink ? (i) + step : (i)) * 64), "s"(mask) : "memory")
#define WAIT_ITEM(n) asm volatile("s_waitcnt vmcnt(" #n ")" : "+v"(nx[0]), "+v"(nx[1]), "+v"(nx[2]), "+v"(nx[3]), "+v"(nx[4]), "+v"(nx[5]), "+v"(nf) :: "memory")
#define SIGNAL() do { if (pending) { if (lane == 0) __hip_atomic_fetch_add((GAS unsigned*)(ws + pending), 1u, __ATOMIC_RELAXED, __HIP_MEMORY_SCOPE_AGENT); \
        DBG_AT((lbase(d) + pend_i) * 24 + 8); pending = 0; } } while (0)
    unsigned f = 0, pending = 0; bool have = false;
    int pend_i = 0; (void)pend_i;
    for (int i = uw / 7; i < nlink; i += step) {
        if (!have) {
            WAIT_V(0); SIGNAL();
            const unsigned long long t0 = __builtin_amdgcn_s_memrealtime();
            bool dead = false;
            while (((f = (unsigned)__builtin_amdgcn_readfirstlane(ld_sys(flags + (size_t)i * 64))) >> 8) < call)
                if (give_up(ws, call, t0)) { dead = true; break; }
            if (dead) break;
            LOAD_ITEM(i, ~0u); WAIT_ITEM(0);
        }
        DBG_AT((lbase(d) + i) * 24);
        u32x4 pk[6];
#pragma unroll
        for (int k = 0; k < 6; ++k) pk[k] = nx[k];
        const unsigned top4 = (f & 255u) * 0x01010101u;
        f = (unsigned)__builtin_amdgcn_readfirstlane(nf);
        const unsigned ahead = below(i + step, nlink) & ~below((int)(f >> 8), (int)call);
        have = ahead != 0u;
        LOAD_ITEM(i + step < nlink ? i + step : i, ahead); SCHED;
        *(LAS u32x4*)(buf + CODEC_STREAM + lane * 16) = pk[5];
        unsigned cg[8], at[8], tot = 0; u32x2 sel[8];
#pragma unroll
        for (int v = 0; v < 8; ++v) {
            cg[v] = (pk[4][v >> 1] >> (4 * (v & 1))) & 0x0f0f0f0fu;
            const unsigned e = esc_bits(cg[v]);
            sel[v] = lut_of(e, lds0 + CODEC_LUT);
            at[v] = tot; tot += __builtin_popcount(e);
        }
        const unsigned n0 = wave_scan(tot) - tot + (lds0 + CODEC_STREAM) * 2u;
        asm volatile("" ::: "memory");
        const Item it = item_of(d, i, lane);
        const unsigned sink = (unsigned)(OFF_SINK - (call & 1) * PAR_BYTES) - it.fin + uw * 1024 + lane * 16;
        unsigned xs[8];
#pragma unroll
        for (int v = 0; v < 8; ++v) {
            const unsigned n = n0 + at[v];
            const LAS unsigned* q = (const LAS unsigned*)(size_t)((n >> 1) & ~3u);
            xs[v] = __builtin_amdgcn_alignbit(q[1], q[0], n * 4u);
        }
#pragma unroll
        for (int v = 0; v < 8; ++v) {
            const unsigned xe = xs[v] & 0x0f0f0f0fu, xo = (xs[v] >> 4) & 0x0f0f0f0fu;
            const unsigned e0 = top4 - ((cg[v] & 0x03030303u) | __builtin_amdgcn_perm(xo, xe, sel[v].x));
            const unsigned e1 = top4 - (((cg[v] >> 2) & 0x03030303u) | __builtin_amdgcn_perm(xo, xe, sel[v].y));
            const unsigned m0 = pk[v >> 1][(v & 1) * 2], m1 = pk[v >> 1][(v & 1) * 2 + 1];
            const unsigned h0 = bfi(0x80808080u, m0, e0 >> 1), l0 = bfi(0x7f7f7f7fu, m0, e0 << 7), h1 = bfi(0x80808080u, m1, e1 >> 1), l1 = bfi(0x7f7f7f7fu, m1, e1 << 7);
            u32x4 r;
            r[0] = __builtin_amdgcn_perm(h0, l0, 0x05010400u); r[1] = __builtin_amdgcn_perm(h0, l0, 0x07030602u);
            r[2] = __builtin_amdgcn_perm(h1, l1, 0x05010400u); r[3] = __builtin_amdgcn_perm(h1, l1, 0x07030602u);
            store16_sys(set + it.fin, cg[v] != 0u ? it.voff + v * it.istep : sink, r);
        }
        WAIT_ITEM(8);
        SIGNAL();
        pending = it.cnt; pend_i = i;
        if (i == uw / 7) STAMP(1);
    }
    WAIT_V(0); SIGNAL();
}

__device__ __forceinline__ int strip_of(int pos, bool remote, int q) {
    if (!remote) return pos < 32 + NSLOT ? pos : -1;
    if (pos < EH) return pos;
    if (pos < EH + 3 * OH) { const int k = (pos - EH) / OH; return 32 + 3 * EH + (k + (k >= q ? 1 : 0)) * OH + (pos - EH) % OH; }
    return (unsigned)(pos - NTE * SR) < 3u * EH ? 32 + pos - NTE * SR : -1;
}
__device__ __forceinline__ int out_row_of(int id, int rank) {
    if (id < 32) return rank * 1024 + id * 32;
    const int slot = id - 32;
    if (slot < 3 * EH) return (rank ^ (2 * (slot / EH + 1))) * 1024 + (slot % EH) * 32;
    const int o = slot - 3 * EH;
    return (rank ^ (2 * (o / OH) + 1)) * 1024 + (EH + o % OH) * 32;
}

__device__ __forceinline__ void expander(const Params& P, LAS unsigned char* buf, int xw, unsigned call, bool idle) {
    const int lane = threadIdx.x & 63;
    char* ws = P.wsx[0];
    const unsigned lds0 = (unsigned)(size_t)buf;
    for (int n = xw; n < NOI; n += NXW) {
        const bool late = n >= NTE * HPT * 32;
        const int n1 = late ? n - NTE * HPT * 32 : n, per = (late ? NTR - NTE : NTE) * 32, h = n1 / per, rt = (late ? NTE : 0) + (n1 - h * per) / 32, w = (n1 >> 2) & 7, q = n1 & 3;
        const int f0 = (w >> 2) * SR + 2 * h;
        const int ida = strip_of(rt * SR + (f0 >> 1), true, q), idb = 2 * h + 1 < SR ? strip_of(rt * SR + ((f0 + 1) >> 1), true, q) : -1;
        if (ida < 0) continue;
        const int o = ((q * NTR + rt) * 8 + w) * HPT + h;
        unsigned f; bool dead = false;
        DBG_AT(DBG_OUT + o * 32);
        { const unsigned long long t0 = __builtin_amdgcn_s_memrealtime();
          while (((f = (unsigned)__builtin_amdgcn_readfirstlane(ld_sys(ws + OFF_OFLAG + o * 64))) >> 12) < call) {
              if (give_up(ws, call, t0)) { dead = true; break; }
              if (idle) __builtin_amdgcn_s_sleep(48);
          } }
        idle = false;
        DBG_AT(DBG_OUT + o * 32 + 8);
        u32x4 r[4];
        unsigned tmo = call;
        if (!dead) {
            const unsigned top = 256u + 2u * (f & 0xfffu) - 1u;
            const unsigned lim = (top < (unsigned)OITEM ? top : OITEM - 1u) & ~15u;
            u32x4 t[5]; unsigned a[5];
#pragma unroll
            for (int k = 0; k < 5; ++k) { a[k] = lane * 16 + k * 1024; a[k] = a[k] < lim ? a[k] : lim; }
            asm volatile("global_load_dwordx4 %0, %6, %11 sc0 sc1\n\tglobal_load_dwordx4 %1, %7, %11 sc0 sc1\n\tglobal_load_dwordx4 %2, %8, %11 sc0 sc1\n\t"
                         "global_load_dwordx4 %3, %9, %11 sc0 sc1\n\tglobal_load_dwordx4 %4, %10, %11 sc0 sc1\n\tglobal_load_dword %5, %12, %13 sc1\n\ts_waitcnt vmcnt(0)"
                         : "=&v"(t[0]), "=&v"(t[1]), "=&v"(t[2]), "=&v"(t[3]), "=&v"(t[4]), "=&v"(tmo)
                         : "v"(a[0]), "v"(a[1]), "v"(a[2]), "v"(a[3]), "v"(a[4]), "s"(ws + OFF_OPK + (size_t)o * OITEM), "v"(0u), "s"(ws + OFF_TMO) : "memory");
#pragma unroll
            for (int k = 0; k < 5; ++k) *(LAS u32x4*)(buf + a[k]) = t[k];
            asm volatile("" ::: "memory");
            const unsigned bits = *(const LAS unsigned*)(buf + lane * 4);
            unsigned pos = lds0 + 256, val[32];
#pragma unroll
            for (int j = 0; j < 32; ++j) {
                const bool nz = (int)(bits << j) < 0;
                const unsigned long long m = __builtin_amdgcn_ballot_w64(nz);
                const unsigned idx = __builtin_amdgcn_mbcnt_hi((unsigned)(m >> 32), __builtin_amdgcn_mbcnt_lo((unsigned)m, 0u));
                val[j] = *(const LAS unsigned short*)(size_t)(idx * 2u + pos);
                val[j] = nz ? val[j] : 0u;
                pos += __builtin_popcountll(m) * 2u;
            }
            asm volatile("" ::: "memory");
#pragma unroll
            for (int k = 0; k < 8; ++k) {
                const int row = (k >> 2) * 16 + (lane & 15), ch = (k & 3) * 2 + (lane >> 5);
                *(LAS u32x2*)(buf + row * 128 + ((ch ^ (row & 7)) * 16) + ((lane >> 4) & 1) * 8) = (u32x2){val[4 * k] | (val[4 * k + 1] << 16), val[4 * k + 2] | (val[4 * k + 3] << 16)};
            }
            asm volatile("" ::: "memory");
#pragma unroll
            for (int v = 0; v < 4; ++v) { const int row = v * 8 + (lane >> 3); r[v] = *(const LAS u32x4*)(buf + row * 128 + (((lane & 7) ^ (row & 7)) * 16)); }
        }
        if (tmo == call) {
#pragma unroll
            for (int v = 0; v < 4; ++v) r[v] = (u32x4){0x7fc07fc0u, 0x7fc07fc0u, 0x7fc07fc0u, 0x7fc07fc0u};
        }
        const int src = P.rank ^ (2 * q + 1);
        const unsigned ooff = (lane >> 3) * OPITCH + q * 512 + ((lane >> 2) & 1) * 256 + (w & 3) * 64 + (lane & 3) * 16;
        { char* O = P.out + (size_t)(out_row_of(ida, src) + (f0 & 1) * 16) * OPITCH; store16_sys(O, ooff, r[0]); store16_sys(O, ooff + 8 * OPITCH, r[1]); }
        if (idb >= 0) { char* O = P.out + (size_t)(out_row_of(idb, src) + ((f0 + 1) & 1) * 16) * OPITCH; store16_sys(O, ooff, r[2]); store16_sys(O, ooff + 8 * OPITCH, r[3]); }
        DBG_AT(DBG_OUT + o * 32 + 16);
    }
}

__device__ __forceinline__ unsigned cvt_pk_bf16(float lo, float hi) { unsigned r; asm("v_cvt_pk_bf16_f32 %0, %1, %2" : "=v"(r) : "v"(lo), "v"(hi)); return r; }
template <int OFF> __device__ __forceinline__ s16x4 tr_read(unsigned addr) { s16x4 r; asm volatile("ds_read_b64_tr_b16 %0, %1 offset:%2" : "=v"(r) : "v"(addr), "i"(OFF) : "memory"); return r; }
template <int OFF> __device__ __forceinline__ bf16x8 row_read(unsigned addr) { bf16x8 r; asm volatile("ds_read_b128 %0, %1 offset:%2" : "=v"(r) : "v"(addr), "i"(OFF) : "memory"); return r; }
__device__ __forceinline__ void glds(unsigned ldsaddr, unsigned voff, const char* g) {
    asm volatile("s_mov_b32 m0, %0\n\ts_nop 0\n\tglobal_load_lds_dwordx4 %1, %2" GLF :: "s"(ldsaddr), "v"(voff), "s"(g) : "memory");
}
__device__ __forceinline__ void glds_masked(unsigned ldsaddr, unsigned voff, const char* g, unsigned mask) {
    asm volatile("s_mov_b32 m0, %0\n\ts_mov_b32 exec_lo, %3\n\ts_mov_b32 exec_hi, %3\n\tglobal_load_lds_dwordx4 %1, %2" GLF "\n\ts_mov_b64 exec, -1" :: "s"(ldsaddr), "v"(voff), "s"(g), "s"(mask) : "memory");
}

template <int S> struct Frag { bf16x8 a[S], b[4]; };
template <int S> __device__ __forceinline__ void wait_step(Frag<S>& F, unsigned& pre) {
    asm volatile("s_waitcnt vmcnt(6) lgkmcnt(0)" : "+v"(F.a[0]), "+v"(F.a[1]), "+v"(F.a[2]), "+v"(F.a[3]), "+v"(F.a[S - 1]), "+v"(F.b[0]), "+v"(F.b[1]), "+v"(F.b[2]), "+v"(F.b[3]), "+v"(pre) :: "memory");
    if constexpr (S > 5) asm volatile("" : "+v"(F.a[4]));
    if constexpr (S > 6) asm volatile("" : "+v"(F.a[5]));
    if constexpr (S > 7) asm volatile("" : "+v"(F.a[6]));
}
template <int S, bool remote> __device__ __forceinline__ void gemm_tile(const Params& P, LAS unsigned char* lds, int rt, int ct, unsigned call) {
    constexpr int SLOT_A = S * 2048, SLOT = SLOT_A + 16384;
    static_assert(S >= 5 && S <= 8 && 4 * SLOT <= LDS_STAGE && S * 16384 <= LDS_STAGE, "tile");
    int wid = __builtin_amdgcn_readfirstlane(threadIdx.x >> 6), lane = threadIdx.x & 63, wr = wid >> 2, wc = wid & 3, fr = lane & 15, fq = lane >> 4;
    char* ws = P.wsx[0];
    const char* set = ws + (call & 1) * PAR_BYTES;
    const char* const own_x = P.x;
    const unsigned ldst = (unsigned)(size_t)lds + wid * 1024;
    unsigned voffA, voffB;
    { const int k0 = lane >> 4, ch = lane & 15; voffB = (unsigned)((wid * 4 + k0) * BPITCH + (((ch >> 1) ^ (k0 | ((wid & 2) << 1))) * 32) + (ch & 1) * 16);
      const int sb = lane * 16, swz = sb ^ (((sb >> 9) & 1) << 5); voffA = (unsigned)(((wid & 1) * 16 + (swz >> 6)) * APITCH + (swz & 63)); }
    auto strip_ptr = [&](int k) { int id = k < S ? strip_of(rt * S + k, remote, ct) : -1; if (id < 0) id = strip_of(rt * S, remote, ct);
                                  return id < 32 ? own_x + id * STRIP_BYTES : set + OFF_XA + (id - 32) * STRIP_BYTES; };
    const char* const gb = (remote ? set + OFF_W2 : P.w) + ct * 512;
    const char* const ga0 = strip_ptr(wid >> 1);
    const char* const ga1 = strip_ptr(4 + (wid >> 1));
    const unsigned amask = below(wid, 2 * S - 8);
    unsigned aoff[2], boff[2][2];
    { const unsigned lbase = (unsigned)(size_t)lds; const int ob = fr * 64 + fq * 16; aoff[0] = lbase + wr * S * 1024 + (ob ^ (((ob >> 9) & 1) << 5)); aoff[1] = aoff[0] + 2 * SLOT;
      const int q = (lane & 15) >> 2, p = lane & 3, f = q | ((fq & 1) << 2);
      for (int n = 0; n < 2; ++n) { boff[0][n] = lbase + SLOT_A + (8 * fq + q) * 256 + p * 8 + (((wc * 2 + n) ^ f) * 32); boff[1][n] = boff[0][n] + 2 * SLOT; } }
    unsigned coff, cstep, need;
    { const int id = strip_of(rt * S + (lane < S ? lane : 0), remote, ct);
      const bool watch = wid == 0 && (lane < S ? id >= 32 : (lane == 8 && remote));
      coff = watch ? (unsigned)(OFF_CNT + (lane < S ? id - 32 : NSLOT + ct) * NCH * 128) : (unsigned)OFF_TMO;
      cstep = watch ? 128u : 0u;
      need = watch ? call * (lane < S ? 1u : 8u) : 0u; }
    const unsigned w0 = below(wid, 1);
    unsigned pre = 0;
#define SLOW_WAIT() do { const unsigned long long t0 = __builtin_amdgcn_s_memrealtime(); \
        while (__builtin_amdgcn_ballot_w64(ld_agent(ws + coff) < need) != 0ull) if (give_up(ws, call, t0)) break; \
        asm volatile("" ::: "memory"); } while (0)
#define PRE_LOAD(mask) do { coff += cstep; \
        asm volatile("s_mov_b32 exec_lo, %3\n\ts_mov_b32 exec_hi, 0\n\tglobal_load_dword %0, %1, %2 sc1\n\ts_mov_b64 exec, -1" : "+v"(pre) : "v"(coff), "s"(ws), "s"((mask) & w0) : "memory"); } while (0)
#define STAGE_B(h, sl, st) glds(ldst + (sl) * SLOT + SLOT_A + (h) * 8192, voffB, gb + (h) * 256 + (size_t)(st) * (32 * BPITCH))
#define STAGE_A0(sl, st) glds(ldst + (sl) * SLOT, voffA, ga0 + (st) * 64)
#define STAGE_A1(sl, st) glds_masked(ldst + (sl) * SLOT + 8192, voffA, ga1 + (st) * 64, amask)
#define STAGE_ALL(sl, st) do { STAGE_B(0, sl, st); STAGE_B(1, sl, st); STAGE_A0(sl, st); STAGE_A1(sl, st); } while (0)
#define RDA(F, m, sl) F.a[m] = row_read<((sl) & 1) * SLOT + (m) * 1024>(aoff[(sl) >> 1])
#define RDB(F, j, sl) do { const s16x4 _lo = tr_read<((sl) & 1) * SLOT + ((j) >> 1) * 8192>(boff[(sl) >> 1][(j) & 1]), _hi = tr_read<((sl) & 1) * SLOT + ((j) >> 1) * 8192 + 1024>(boff[(sl) >> 1][(j) & 1]); \
        F.b[j] = __builtin_shufflevector(_lo, _hi, 0, 1, 2, 3, 4, 5, 6, 7); } while (0)
#define MM(F, i) acc[(i) >> 2][(i) & 3] = __builtin_amdgcn_mfma_f32_16x16x32_bf16(F.b[(i) & 3], F.a[(i) >> 2], acc[(i) >> 2][(i) & 3], 0, 0, 0)
#define PAIR(C, i) do { MM(C, i); MM(C, (i) + 1); SCHED; } while (0)
#define STEP(C, N, sl, st, GATE) do { constexpr int nx = ((sl) + 1) & 3; \
        RDB(N, 0, nx); PAIR(C, 0); RDB(N, 1, nx); PAIR(C, 2); RDA(N, 0, nx); RDA(N, 1, nx); PAIR(C, 4); RDB(N, 2, nx); PAIR(C, 6); RDB(N, 3, nx); \
        STAGE_B(0, sl, st); PAIR(C, 8); RDA(N, 2, nx); RDA(N, 3, nx); STAGE_B(1, sl, st); PAIR(C, 10); \
        RDA(N, 4, nx); if constexpr (S > 5) RDA(N, 5, nx); STAGE_A0(sl, st); PAIR(C, 12); \
        if constexpr (S > 6) RDA(N, 6, nx); if constexpr (S > 7) RDA(N, 7, nx); STAGE_A1(sl, st); \
        _Pragma("unroll") for (int i = 14; i < 4 * S - 4; i += 2) PAIR(C, i); \
        wait_step(N, pre); GATE; BAR; SCHED; PAIR(C, 4 * S - 4); PAIR(C, 4 * S - 2); } while (0)
    f32x4 acc[S][4];
#pragma unroll
    for (int m = 0; m < S; ++m)
#pragma unroll
        for (int j = 0; j < 4; ++j) acc[m][j] = (f32x4){0.f, 0.f, 0.f, 0.f};
    Frag<S> F0, F1;
    if (wid == 0) SLOW_WAIT();
    BAR;
    STAMP(1); CSTAMP(6);
    PRE_LOAD(0x3ffu);
    STAGE_ALL(0, 0); STAGE_ALL(1, 1); STAGE_ALL(2, 2); STAGE_ALL(3, 3);
    WAIT_V(6); BAR;
    RDB(F0, 0, 0); RDB(F0, 1, 0); RDB(F0, 2, 0); RDB(F0, 3, 0);
    RDA(F0, 0, 0); RDA(F0, 1, 0); RDA(F0, 2, 0); RDA(F0, 3, 0); RDA(F0, 4, 0);
    if constexpr (S > 5) RDA(F0, 5, 0);
    if constexpr (S > 6) RDA(F0, 6, 0);
    if constexpr (S > 7) RDA(F0, 7, 0);
    wait_step(F0, pre);
    if (__builtin_amdgcn_ballot_w64(pre < need) != 0ull) SLOW_WAIT();
    BAR;
    for (int c = 0; c < NCH; ++c) {
        const int st = (c < NCH - 1 ? c + 1 : c) * 4;
        STEP(F0, F1, 0, st, (void)0);
        STEP(F1, F0, 1, st + 1, (void)0);
        PRE_LOAD(c < NCH - 2 ? 0x3ffu : c < NCH - 1 ? 0x200u : 0u);
        STEP(F0, F1, 2, st + 2, (void)0);
        STEP(F1, F0, 3, st + 3, { TS_NOW(ts0, c); if (c < NCH - 2 && __builtin_amdgcn_ballot_w64(pre < need) != 0ull) SLOW_WAIT(); TS_NOW(ts1, c); });
    }
    if (threadIdx.x == 9) *(LAS unsigned*)(lds + LDS_STAGE) = pre;
    asm volatile("s_waitcnt vmcnt(0) lgkmcnt(0)" ::: "memory");
    BAR;
    asm volatile("" ::: "memory");
    const bool bad = *(const LAS unsigned*)(lds + LDS_STAGE) == call;
    STAMP(2); CSTAMP(7);
    int tz = threadIdx.x; asm volatile("" : "+v"(tz));
    lane = tz & 63; fr = lane & 15; fq = lane >> 4;
    auto relu_pack = [&](int m, int j) {
        f32x4 v = acc[m][j];
#pragma unroll
        for (int e = 0; e < 4; ++e) v[e] = v[e] < 0.f ? 0.f : v[e];
        u32x2 w; w.x = cvt_pk_bf16(v[0], v[1]); w.y = cvt_pk_bf16(v[2], v[3]);
        if (bad) { w.x = 0x7fc07fc0u; w.y = 0x7fc07fc0u; }
        return w;
    };
    if constexpr (!remote) {
#pragma unroll
        for (int m = 0; m < S; ++m) {
            const int row = wr * S * 16 + m * 16 + fr;
#pragma unroll
            for (int j = 0; j < 4; ++j) {
                const int ch = (j >> 1) * 16 + wc * 4 + (j & 1) * 2 + (fq >> 1);
                *(LAS u32x2*)(lds + row * 512 + ((ch ^ (fr & 7)) * 16) + (fq & 1) * 8) = relu_pack(m, j);
            }
        }
        WAIT_L(0); BAR; asm volatile("" ::: "memory");
        u32x4 ov[S][2];
#pragma unroll
        for (int k = 0; k < S; ++k)
#pragma unroll
            for (int i = 0; i < 2; ++i) { const int row = k * 32 + wid * 4 + i * 2 + (lane >> 5); ov[k][i] = *(const LAS u32x4*)(lds + row * 512 + (((lane & 31) ^ (row & 7)) * 16)); }
        const unsigned ooff = (wid * 4 + (lane >> 5)) * OPITCH + ct * 512 + (lane & 31) * 16;
#pragma unroll
        for (int k = 0; k < S; ++k) {
            const int id = strip_of(rt * S + k, false, 0);
            if (id >= 0) { char* O = P.out + (size_t)out_row_of(id, P.rank) * OPITCH; store16_sys(O, ooff, ov[k][0]); store16_sys(O, ooff + 2 * OPITCH, ov[k][1]); }
        }
    } else {
        char* const peer = P.wsx[(2 * ct + 1) & 7];
        LAS unsigned char* buf = lds + wid * OITEM;
        const unsigned lds0 = (unsigned)(size_t)buf;
        const int o0 = ((ct * NTR + rt) * 8 + wid) * HPT;
        int sent = -1; unsigned sent_flag = 0;
#define PUBLISH_OUT() do { WAIT_V(0); if (sent >= 0) { if (lane == 0) __hip_atomic_store((GAS unsigned*)(peer + OFF_OFLAG + sent * 64), sent_flag, __ATOMIC_RELAXED, __HIP_MEMORY_SCOPE_SYSTEM); \
            DBG_AT(DBG_OUT + sent * 32 + 24); } } while (0)
#pragma unroll
        for (int h = 0; h < HPT; ++h) {
            const int f0 = wr * S + 2 * h;
            if (strip_of(rt * S + (f0 >> 1), true, ct) < 0) break;
            const bool second = 2 * h + 1 < S && strip_of(rt * S + ((f0 + 1) >> 1), true, ct) >= 0;
            unsigned bits = 0, pos = lds0 + 256;
#pragma unroll
            for (int k = 0; k < 8; ++k) {
                u32x2 w = relu_pack(2 * h + (k >> 2) < S ? 2 * h + (k >> 2) : 0, k & 3);
                if (k >= 4 && !second) w = (u32x2){0u, 0u};
#pragma unroll
                for (int e = 0; e < 4; ++e) {
                    const unsigned val = (e & 1) ? w[e >> 1] >> 16 : w[e >> 1] & 0xffffu;
                    const bool nz = val != 0u;
                    const unsigned long long m = __builtin_amdgcn_ballot_w64(nz);
                    const unsigned idx = __builtin_amdgcn_mbcnt_hi((unsigned)(m >> 32), __builtin_amdgcn_mbcnt_lo((unsigned)m, 0u));
                    if (nz) *(LAS unsigned short*)(size_t)(idx * 2u + pos) = (unsigned short)val;
                    bits = bits + bits + (nz ? 1u : 0u);
                    pos += __builtin_popcountll(m) * 2u;
                }
            }
            *(LAS unsigned*)(buf + lane * 4) = bits;
            asm volatile("" ::: "memory");
            const unsigned total = pos - lds0;
            u32x4 t[5];
#pragma unroll
            for (int k = 0; k < 5; ++k) { const unsigned a = lane * 16 + k * 1024; t[k] = *(const LAS u32x4*)(buf + (a < (unsigned)OITEM ? a : 0u)); }
            asm volatile("" : "+v"(t[0]), "+v"(t[1]), "+v"(t[2]), "+v"(t[3]), "+v"(t[4]) :: "memory");
            PUBLISH_OUT();
            char* dst = peer + OFF_OPK + (size_t)(o0 + h) * OITEM;
#pragma unroll
            for (int k = 0; k < 5; ++k) if (lane * 16 + k * 1024 < total) store16_sys(dst, lane * 16 + k * 1024, t[k]);
            sent = o0 + h; sent_flag = (call << 12) | ((total - 256u) >> 1);
        }
        PUBLISH_OUT();
    }
    STAMP(3);
}

__global__ __launch_bounds__(NTHR, 2) void ag_gemm_kernel(const Params P) {
    extern __shared__ __attribute__((aligned(16))) unsigned char shm[];
    const int xcd = blockIdx.x & 7, slot = blockIdx.x >> 3, wid = __builtin_amdgcn_readfirstlane(threadIdx.x >> 6);
    char* const ticket = P.wsx[0] + OFF_TICK + (size_t)(blockIdx.x * 8 + wid) * 64;
    STAMP(0);
    int ns = 0, nc = 0, ub = 0, sb = 0;
#pragma unroll
    for (int y = 0; y < 8; ++y) if (y == xcd) { ns = nsend(y); nc = ncomm(y); ub = unp_before(y); sb = snd_before(y); }
    if (slot < ns && wid < SEND_WAVES) {
        const unsigned call = sender(P, (LAS unsigned char*)shm + wid * OITEM, link_of(xcd), wid * ns + slot, ns * SEND_WAVES, ticket);
        STAMP(2);
        expander(P, (LAS unsigned char*)shm + wid * OITEM, NUW + sb + wid * ns + slot, call, false);
        STAMP(3);
        return;
    }
    unsigned tick = 0;
    if ((threadIdx.x & 63) == 0) tick = __hip_atomic_fetch_add((GAS unsigned*)ticket, 1u, __ATOMIC_RELAXED, __HIP_MEMORY_SCOPE_AGENT);
    const unsigned call = (unsigned)__builtin_amdgcn_readfirstlane(tick) + 1u;
    if (!MIX && slot < ns) {
        expander(P, (LAS unsigned char*)shm + wid * OITEM, NUW + sb + wid * ns + slot, call, true);
    } else if (slot < nc) {
        const int uw = ub + (slot < ns ? slot * (8 - SEND_WAVES) + wid - SEND_WAVES : (MIX ? ns * (8 - SEND_WAVES) : 0) + (slot - ns) * 8 + wid);
        unpacker(P, (LAS unsigned char*)shm + wid * OITEM, uw, call);
        STAMP(2);
        expander(P, (LAS unsigned char*)shm + wid * OITEM, uw, call, false);
        STAMP(3);
    } else {
        const int g = slot - nc;
                if (xcd < 4) gemm_tile<SL, false>(P, (LAS unsigned char*)shm, xcd * (NTL / 4) + g % (NTL / 4), g / (NTL / 4), call);
        else {
            const int y = xcd - 4;
            gemm_tile<SR, true>(P, (LAS unsigned char*)shm, (g / BQ) * BQ + y % BQ, (y / BQ) * BQ + g % BQ, call);
        }
    }
}
}

extern "C" void kernel_launch(int rank, int nranks, ncclComm_t comm, hipStream_t stream,
                              void* const* d_in, const int* in_sizes, int n_in,
                              void* d_out, int out_size, const int* in_dtypes, int out_dtype,
                              void* const* d_out_all, void* const* d_ws_all, size_t ws_bytes) {
    (void)comm; (void)in_dtypes; (void)out_dtype; (void)d_out_all;
    if (nranks != NR || (unsigned)rank >= (unsigned)NR || n_in != 2 || in_sizes[0] != 1024 * KDIM || in_sizes[1] != KDIM * 1024 || out_size != KDIM * 1024 || ws_bytes < WS_NEED) {
        fprintf(stderr, "kernel_launch: unexpected problem geometry\n");
        return;
    }
    static bool attr_set = false;
    if (!attr_set) { HIP_CHECK(hipFuncSetAttribute((const void*)ag_gemm_kernel, hipFuncAttributeMaxDynamicSharedMemorySize, LDS_BYTES)); attr_set = true; }
    Params P;
    P.x = (const char*)d_in[0]; P.w = (const char*)d_in[1]; P.out = (char*)d_out;
    for (int d = 0; d < NR; ++d) P.wsx[d] = (char*)d_ws_all[rank ^ d];
    P.rank = rank; P.pad = 0;
    hipLaunchKernelGGL(ag_gemm_kernel, dim3(GRID), dim3(NTHR), LDS_BYTES, stream, P);
    HIP_CHECK(hipGetLastError());
}
```

```cpp
#include <hip/hip_runtime.h>
#include <rccl/rccl.h>
#include <cstdio>

#define HIP_CHECK(expr) do { hipError_t _e = (expr); if (_e != hipSuccess) { \
    fprintf(stderr, "HIP error %s at %s:%d\n", hipGetErrorString(_e), __FILE__, __LINE__); \
    return; } } while (0)

namespace {
#define LAS __attribute__((address_space(3)))
#define GAS __attribute__((address_space(1)))
typedef short bf16x8 __attribute__((ext_vector_type(8)));
typedef short s16x4 __attribute__((ext_vector_type(4)));
typedef float f32x4 __attribute__((ext_vector_type(4)));
typedef unsigned u32x4 __attribute__((ext_vector_type(4)));
typedef unsigned u32x2 __attribute__((ext_vector_type(2)));
typedef unsigned short u16x2 __attribute__((ext_vector_type(2)));

constexpr int NR = 8;
constexpr int KDIM = 8192;
constexpr int APITCH = KDIM * 2, BPITCH = 2048, OPITCH = 2048;
constexpr int EH = 21, OH = 32 - EH;
constexpr int NSLOT = 3 * EH + 4 * OH;
#define SL 7
#define SR 5
constexpr int NTL = (32 + NSLOT + SL - 1) / SL, NTE = (EH + 3 * OH + SR - 1) / SR, NTR = NTE + (3 * EH + SR - 1) / SR, HPT = (SR + 1) / 2;
constexpr int NCH = 64;
__host__ __device__ constexpr int litems(int d) { return (d & 1) ? OH + 8 : EH; }
__host__ __device__ constexpr int lbase(int d) { return NCH * ((d >> 1) * litems(1) + ((d - 1) >> 1) * litems(0)); }
constexpr int NU = lbase(8);
constexpr int ITEM_PK = 6144;
constexpr size_t STRIP_BYTES = (size_t)32 * APITCH;
constexpr size_t XA_BYTES = NSLOT * STRIP_BYTES, W2_BYTES = (size_t)KDIM * BPITCH, PK_BYTES = (size_t)NU * ITEM_PK;
constexpr size_t OFF_XA = 0, OFF_W2 = XA_BYTES, OFF_PK = XA_BYTES + W2_BYTES, PAR_BYTES = OFF_PK + PK_BYTES;
constexpr size_t OFF_FLAG = 2 * PAR_BYTES, FLAG_BYTES = (size_t)NU * 64;
constexpr size_t OFF_CNT = OFF_FLAG + 2 * FLAG_BYTES, CNT_BYTES = (size_t)(NSLOT + 4) * NCH * 128;
constexpr size_t OFF_TICK = OFF_CNT + CNT_BYTES, TICK_BYTES = 2048 * 64;
constexpr int OITEM = 4096, NOI = 4 * NTR * 8 * HPT;
constexpr size_t OFF_OPK = OFF_TICK + TICK_BYTES, OFF_OFLAG = OFF_OPK + (size_t)NOI * OITEM;
constexpr size_t OFF_TMO = OFF_OFLAG + (size_t)NOI * 64, OFF_QUEUE = OFF_TMO + 128, OFF_SINK = OFF_QUEUE + 8 * 128, OFF_DBG = OFF_SINK + 768 * 1024, DBG_OUT = NU * 24 + 256 * 64 * 8, DBG_BYTES = 2048 * 64 + DBG_OUT + NOI * 32 + 1024 * 32;
constexpr size_t WS_NEED = OFF_DBG + DBG_BYTES;
constexpr int GRID = 256, NTHR = 512;
#define SEND_WAVES 8
#define SEND_WGS 6
#define GLF " sc0 sc1"
#define OUT_SLOTS 6
#define SEND_SLOTS 3
#define TURN_SLEEP 8
#define POLL_SLEEP 2
__host__ __device__ constexpr int ncomm(int x) { return 32 - 4 * (((x < 4 ? NTL : NTR) + x % 4) / 4); }
__host__ __device__ constexpr int nsend(int x) { return x == 3 ? 0 : SEND_WGS; }
__host__ __device__ constexpr int link_of(int x) { return x < 3 ? 2 * x + 2 : 2 * x - 7; }
__host__ __device__ constexpr int unp_before(int x) { return x == 0 ? 0 : unp_before(x - 1) + (ncomm(x - 1) - nsend(x - 1)) * 8; }
__host__ __device__ constexpr int snd_before(int x) { return x == 0 ? 0 : snd_before(x - 1) + nsend(x - 1) * 8; }
constexpr int NUW = unp_before(8), NXW = NUW + snd_before(8);
constexpr unsigned long long DEADLINE = 50000000ull;
constexpr int LDS_STAGE = 135168, LDS_BYTES = LDS_STAGE + 16;
static_assert(NOI <= 8 * NXW && NUW <= 768 && ncomm(2) >= nsend(2) && ncomm(3) >= 0 && ncomm(7) >= nsend(7), "roles");

struct Params {
    const char* x; const char* w;
    char* out;
    char* wsx[NR];
    int rank; int pad;
};

#define WAIT_V(n) asm volatile("s_waitcnt vmcnt(" #n ")" ::: "memory")
#define WAIT_L(n) asm volatile("s_waitcnt lgkmcnt(" #n ")" ::: "memory")
#define BAR __builtin_amdgcn_s_barrier()
#define SCHED __builtin_amdgcn_sched_barrier(0)
#define STAMP(i) do {} while (0)
#define CSTAMP(i) do {} while (0)
#define DBG_AT(off) do {} while (0)
#define TS_NOW(dst, k) do {} while (0)

__device__ __forceinline__ unsigned ld_agent(const char* p) { return __hip_atomic_load((const GAS unsigned*)p, __ATOMIC_RELAXED, __HIP_MEMORY_SCOPE_AGENT); }
__device__ __forceinline__ unsigned ld_sys(const char* p) { return __hip_atomic_load((const GAS unsigned*)p, __ATOMIC_RELAXED, __HIP_MEMORY_SCOPE_SYSTEM); }
struct Wait {
    unsigned long long t0; unsigned polls;
    __device__ __forceinline__ Wait() : t0(__builtin_amdgcn_s_memrealtime()), polls(0) {}
    __device__ __forceinline__ bool give_up(char* ws, unsigned call) {
        if ((++polls & 15u) == 0u && ((unsigned)__builtin_amdgcn_readfirstlane(ld_agent(ws + OFF_TMO)) == call || __builtin_amdgcn_s_memrealtime() - t0 > DEADLINE)) {
            __hip_atomic_store((GAS unsigned*)(ws + OFF_TMO), call, __ATOMIC_RELAXED, __HIP_MEMORY_SCOPE_AGENT);
            return true;
        }
        __builtin_amdgcn_s_sleep(POLL_SLEEP);
        return false;
    }
};
__device__ __forceinline__ void store16_sys(char* base, unsigned voff, u32x4 v) {
    asm volatile("global_store_dwordx4 %0, %1, %2 sc0 sc1\n\ts_nop 1" :: "v"(voff), "v"(v), "s"(base) : "memory");
}

__device__ __forceinline__ void store4_sys(char* base, unsigned voff, unsigned v) { asm volatile("global_store_dword %0, %1, %2 sc0 sc1" :: "v"(voff), "v"(v), "s"(base) : "memory"); }
template <int CTRL, int ROWS> __device__ __forceinline__ unsigned max_dpp(unsigned x) { const unsigned y = (unsigned)__builtin_amdgcn_update_dpp(0u, x, CTRL, ROWS, 0xf, false); return x > y ? x : y; }
__device__ __forceinline__ unsigned wave_max(unsigned x) {
    return max_dpp<0x143, 0xc>(max_dpp<0x142, 0xa>(max_dpp<0x118, 0xf>(max_dpp<0x114, 0xf>(max_dpp<0x112, 0xf>(max_dpp<0x111, 0xf>(x))))));
}
__device__ __forceinline__ unsigned nonzero2(unsigned w) { unsigned r; asm("v_pk_min_u16 %0, %1, %2" : "=v"(r) : "v"(w), "s"(0x00010001u)); return r; }
__device__ __forceinline__ unsigned below(int a, int b) { unsigned r; asm("s_ashr_i32 %0, %1, 31" : "=s"(r) : "s"(a - b) : "scc"); return r; }
__device__ __forceinline__ unsigned wave_scan(unsigned x) {
    x += __builtin_amdgcn_update_dpp(0u, x, 0x111, 0xf, 0xf, false); x += __builtin_amdgcn_update_dpp(0u, x, 0x112, 0xf, 0xf, false);
    x += __builtin_amdgcn_update_dpp(0u, x, 0x114, 0xf, 0xf, false); x += __builtin_amdgcn_update_dpp(0u, x, 0x118, 0xf, 0xf, false);
    x += __builtin_amdgcn_update_dpp(0u, x, 0x142, 0xa, 0xf, false); x += __builtin_amdgcn_update_dpp(0u, x, 0x143, 0xc, 0xf, false);
    return x;
}

__device__ __forceinline__ unsigned draw_turn(volatile LAS unsigned* turn, int lane) {
    unsigned t = 0;
    if (lane == 0) t = __hip_atomic_fetch_add((LAS unsigned*)turn, 1u, __ATOMIC_RELAXED, __HIP_MEMORY_SCOPE_WORKGROUP);
    return (unsigned)__builtin_amdgcn_readfirstlane(t);
}
__device__ __forceinline__ void await_turn(volatile LAS unsigned* turn, unsigned mine, unsigned slots) {
    while (mine >= (unsigned)__builtin_amdgcn_readfirstlane(turn[1]) + slots) __builtin_amdgcn_s_sleep(TURN_SLEEP);
}
__device__ __forceinline__ void end_turn(volatile LAS unsigned* turn) { __hip_atomic_fetch_add((LAS unsigned*)turn + 1, 1u, __ATOMIC_RELAXED, __HIP_MEMORY_SCOPE_WORKGROUP); }

constexpr int CODEC_LUT = 0, CODEC_STREAM = 2048, CODEC_LDS = CODEC_STREAM + 1088;
constexpr int WAVE_LDS = 8448;
static_assert(CODEC_LDS <= WAVE_LDS && 8 * WAVE_LDS + 16 <= LDS_STAGE, "per-wave LDS");
__device__ __forceinline__ unsigned esc_bits(unsigned c) { return c == 0u ? 0u : ~(c | (c >> 1)) & 0x05050505u; }
__device__ __forceinline__ u32x2 lut_of(unsigned e, unsigned lut) {
    const LAS unsigned* q = (const LAS unsigned*)(size_t)__builtin_amdgcn_udot4(e, 0x80400804u, lut, false);
    return (u32x2){q[0], q[256]};
}
template <bool PACK> __device__ __forceinline__ void build_lut(LAS unsigned char* buf, int lane) {
    for (int m = lane; m < 256; m += 64) {
        unsigned long long sel = 0x0c0c0c0c0c0c0c0cull; int k = 0;
#pragma unroll
        for (int b = 0; b < 8; ++b) if (m >> b & 1) { const int pos = (b & 1) | ((b & 2) << 1) | ((b & 4) >> 1), slot = (k >> 1) | ((k & 1) << 2);
            sel ^= PACK ? (unsigned long long)(0x0c ^ pos) << (8 * slot) : (unsigned long long)(0x0c ^ slot) << (8 * pos); ++k; }
        *(LAS unsigned*)(buf + CODEC_LUT + m * 4) = (unsigned)sel; *(LAS unsigned*)(buf + CODEC_LUT + 1024 + m * 4) = (unsigned)(sel >> 32);
    }
    asm volatile("" ::: "memory");
}
__device__ __forceinline__ unsigned bfi(unsigned m, unsigned a, unsigned b) { return (a & m) | (b & ~m); }
__device__ __forceinline__ void split4(unsigned r0, unsigned r1, unsigned& m, unsigned& e) {
    const unsigned lo = __builtin_amdgcn_perm(r1, r0, 0x06040200u), hi = __builtin_amdgcn_perm(r1, r0, 0x07050301u);
    m = bfi(0x80808080u, hi, lo); e = bfi(0x01010101u, lo >> 7, hi << 1);
}
__device__ __forceinline__ u32x4 join8(unsigned m0, unsigned e0, unsigned m1, unsigned e1) {
    const unsigned h0 = bfi(0x80808080u, m0, e0 >> 1), l0 = bfi(0x7f7f7f7fu, m0, e0 << 7), h1 = bfi(0x80808080u, m1, e1 >> 1), l1 = bfi(0x7f7f7f7fu, m1, e1 << 7);
    return (u32x4){__builtin_amdgcn_perm(h0, l0, 0x05010400u), __builtin_amdgcn_perm(h0, l0, 0x07030602u), __builtin_amdgcn_perm(h1, l1, 0x05010400u), __builtin_amdgcn_perm(h1, l1, 0x07030602u)};
}

struct Item { bool isw; unsigned src, fin, cnt; };
__device__ __forceinline__ Item item_of(int d, int i) {
    const int chunk = (int)((d & 1) ? (unsigned)i / (unsigned)litems(1) : (unsigned)i / (unsigned)litems(0)), j = i - chunk * litems(d);
    Item it;
    it.isw = (d & 1) && j >= OH;
    if (it.isw) { it.src = (unsigned)((chunk * 128 + (j - OH) * 16) * BPITCH + (d >> 1) * 512); it.fin = (unsigned)(OFF_W2 + ((d >> 1) * NCH * 8 + chunk * 8 + j - OH) * 8192);
        it.cnt = (unsigned)(OFF_CNT + ((NSLOT + (d >> 1)) * NCH + chunk) * 128); }
    else { const int strip = (d & 1) ? EH + j : j, slot = (d & 1) ? 3 * EH + (d >> 1) * OH + j : ((d >> 1) - 1) * EH + j;
        it.src = (unsigned)(strip * STRIP_BYTES + chunk * 256); it.fin = (unsigned)(OFF_XA + slot * STRIP_BYTES + chunk * 8192);
        it.cnt = (unsigned)(OFF_CNT + (slot * NCH + chunk) * 128); }
    return it;
}
__device__ __forceinline__ unsigned sender(const Params& P, LAS unsigned char* buf, volatile LAS unsigned* turn, int d, int first, int nwaves, char* ticket) {
    const int lane = threadIdx.x & 63, nlink = NCH * litems(d);
    char* const queue = P.wsx[0] + OFF_QUEUE + d * 128;
    const char* const src_x = P.x; const char* const src_w = P.w;
    const unsigned lds0 = (unsigned)(size_t)buf, voff_x = (lane >> 4) * APITCH + (lane & 15) * 16, voff_w = (lane >> 5) * BPITCH + (lane & 31) * 16;
    u32x4 nxt[8];
    unsigned tick = 0, ctick = 0;
#define FETCH(i) do { const Item _x = item_of(d, (i) < nlink ? (i) : nlink - 1); const unsigned _v = _x.isw ? voff_w : voff_x, _s = _x.isw ? 2 * BPITCH : 4 * APITCH; \
        asm volatile("global_load_dwordx4 %0, %9, %17\n\tglobal_load_dwordx4 %1, %10, %17\n\tglobal_load_dwordx4 %2, %11, %17\n\tglobal_load_dwordx4 %3, %12, %17\n\t" \
                     "global_load_dwordx4 %4, %13, %17\n\tglobal_load_dwordx4 %5, %14, %17\n\tglobal_load_dwordx4 %6, %15, %17\n\tglobal_load_dwordx4 %7, %16, %17\n\t" \
                     "s_mov_b64 exec, 1\n\tglobal_atomic_add %8, %18, %19, %20 sc0\n\ts_mov_b64 exec, -1" \
                     : "=&v"(nxt[0]), "=&v"(nxt[1]), "=&v"(nxt[2]), "=&v"(nxt[3]), "=&v"(nxt[4]), "=&v"(nxt[5]), "=&v"(nxt[6]), "=&v"(nxt[7]), "=&v"(tick) \
                     : "v"(_v), "v"(_v + _s), "v"(_v + 2 * _s), "v"(_v + 3 * _s), "v"(_v + 4 * _s), "v"(_v + 5 * _s), "v"(_v + 6 * _s), "v"(_v + 7 * _s), \
                       "s"((_x.isw ? src_w : src_x) + _x.src), "v"(0u), "v"(1u), "s"(queue) : "memory"); } while (0)
#define WAIT_FETCH(n) asm volatile("s_waitcnt vmcnt(" #n ")" : "+v"(ctick), "+v"(tick), "+v"(nxt[0]), "+v"(nxt[1]), "+v"(nxt[2]), "+v"(nxt[3]), "+v"(nxt[4]), "+v"(nxt[5]), "+v"(nxt[6]), "+v"(nxt[7]) :: "memory")
    int cur = first;
    asm volatile("s_mov_b64 exec, 1\n\tglobal_atomic_add %0, %1, %2, %3 sc0\n\ts_mov_b64 exec, -1" : "=&v"(ctick) : "v"(0u), "v"(1u), "s"(ticket) : "memory");
    FETCH(cur);
    build_lut<true>(buf, lane);
    unsigned mine = (unsigned)__builtin_amdgcn_readfirstlane(threadIdx.x >> 6);
    if (threadIdx.x == 0) { turn[0] = SEND_WAVES; turn[1] = 0u; }
    WAIT_L(0); BAR;
    WAIT_FETCH(0);
    const unsigned call = (unsigned)__builtin_amdgcn_readfirstlane(ctick) + 1u;
    char* const peer = P.wsx[d & 7];
    char* const set = peer + (call & 1) * PAR_BYTES;
    char* const flags = peer + OFF_FLAG + (call & 1) * FLAG_BYTES;
    const unsigned qbase = (call - 1u) * (nlink + nwaves) - nwaves;
    while (cur < nlink) {
        const int u = lbase(d) + cur;
        u32x4 raw[8];
#pragma unroll
        for (int v = 0; v < 8; ++v) raw[v] = nxt[v];
        const int next = (int)((unsigned)__builtin_amdgcn_readfirstlane(tick) - qbase);
        FETCH(next); SCHED;
        u32x4 pk[6]; unsigned ex[16], esum = 0;
#pragma unroll
        for (int g = 0; g < 16; ++g) {
            unsigned m; split4(raw[g >> 1][(g & 1) * 2], raw[g >> 1][(g & 1) * 2 + 1], m, ex[g]);
            pk[g >> 2][g & 3] = m;
            if ((g & 3) == 0) esum = __builtin_amdgcn_sad_u8(ex[g], 0u, esum);
        }
        *(LAS u32x4*)(buf + CODEC_STREAM + lane * 16) = (u32x4){0u, 0u, 0u, 0u};
        unsigned top = (((unsigned)__builtin_amdgcn_readlane(wave_scan(esum), 63) + 1023u + 174u) >> 10) + 2u;
        top = top < 18u ? 18u : top > 255u ? 255u : top;
        unsigned cg[8], xs[8], at[8], tot = 0;
#pragma unroll
        for (int v = 0; v < 8; ++v) {
            const unsigned ua = top * 0x01010101u - ex[2 * v], ub = top * 0x01010101u - ex[2 * v + 1];
            const unsigned ca = __builtin_amdgcn_perm(0u, 0x03020100u, (((ua >> 1) & 0x04040404u) | ua) & 0x07070707u);
            const unsigned cb = __builtin_amdgcn_perm(0u, 0x0c080400u, (((ub >> 1) & 0x04040404u) | ub) & 0x07070707u);
            cg[v] = ((ua | ub) & 0xf0f0f0f0u) != 0u ? 0u : ca | cb;
            const unsigned e = esc_bits(cg[v]);
            const u32x2 sel = lut_of(e, lds0 + CODEC_LUT);
            xs[v] = __builtin_amdgcn_perm(ub, ua, sel.x) | (__builtin_amdgcn_perm(ub, ua, sel.y) << 4);
            at[v] = tot; tot += __builtin_popcount(e);
        }
        const unsigned incl = wave_scan(tot);
        unsigned total = (unsigned)__builtin_amdgcn_readlane(incl, 63);
        if (total > 2048u) {
            total = 0;
#pragma unroll
            for (int v = 0; v < 8; ++v) cg[v] = 0u;
        } else {
            const unsigned n0 = incl - tot + (lds0 + CODEC_STREAM) * 2u;
            asm volatile("" ::: "memory");
#pragma unroll
            for (int v = 0; v < 8; ++v) {
                const unsigned n = n0 + at[v];
                const unsigned long long w = (unsigned long long)xs[v] << ((n * 4u) & 31u);
                LAS unsigned* q = (LAS unsigned*)(size_t)((n >> 1) & ~3u);
                __hip_atomic_fetch_or(q, (unsigned)w, __ATOMIC_RELAXED, __HIP_MEMORY_SCOPE_WAVEFRONT);
                __hip_atomic_fetch_or(q + 1, (unsigned)(w >> 32), __ATOMIC_RELAXED, __HIP_MEMORY_SCOPE_WAVEFRONT);
            }
        }
        asm volatile("" ::: "memory");
        pk[5] = *(const LAS u32x4*)(buf + CODEC_STREAM + lane * 16);
        unsigned least = cg[0];
#pragma unroll
        for (int v = 1; v < 8; ++v) least = cg[v] < least ? cg[v] : least;
#pragma unroll
        for (int c = 0; c < 4; ++c) pk[4][c] = cg[2 * c] | (cg[2 * c + 1] << 4);
        if (cur != first) mine = draw_turn(turn, lane);
        await_turn(turn, mine, SEND_SLOTS);
        if (__builtin_amdgcn_ballot_w64(least == 0u) != 0ull) {
            char* fin = set + item_of(d, cur).fin;
#pragma unroll
            for (int v = 0; v < 8; ++v) if (cg[v] == 0u) store16_sys(fin, v * 1024 + lane * 16, raw[v]);
        }
        const unsigned long long smask = __builtin_amdgcn_ballot_w64((unsigned)lane * 32u < total);
        asm volatile("s_mov_b64 exec, %8\n\tglobal_store_dwordx4 %0, %6, %7 offset:3072 sc0 sc1\n\ts_mov_b64 exec, -1\n\t"
                     "global_store_dwordx4 %0, %1, %7 offset:-2048 sc0 sc1\n\tglobal_store_dwordx4 %0, %2, %7 offset:-1024 sc0 sc1\n\t"
                     "global_store_dwordx4 %0, %3, %7 sc0 sc1\n\tglobal_store_dwordx4 %0, %4, %7 offset:1024 sc0 sc1\n\t"
                     "global_store_dwordx4 %0, %5, %7 offset:2048 sc0 sc1\n\ts_nop 1"
                     :: "v"(lane * 16), "v"(pk[0]), "v"(pk[1]), "v"(pk[2]), "v"(pk[3]), "v"(pk[4]), "v"(pk[5]), "s"(set + OFF_PK + (size_t)u * ITEM_PK + 2048), "s"(smask) : "memory");
        WAIT_FETCH(0);
        if (lane == 0) {
            end_turn(turn);
            __hip_atomic_store((GAS unsigned*)(flags + (size_t)u * 64), (call << 8) | top, __ATOMIC_RELAXED, __HIP_MEMORY_SCOPE_SYSTEM);
        }
        DBG_AT(u * 24 + 16);
        cur = next;
    }
    return call;
}

__device__ __forceinline__ void unpacker(const Params& P, LAS unsigned char* buf, int uw, unsigned call) {
    const int lane = threadIdx.x & 63;
    const int d = __builtin_amdgcn_readfirstlane(uw % 7 + 1), i0 = __builtin_amdgcn_readfirstlane(uw / 7);
    const int step = __builtin_amdgcn_readfirstlane(NUW / 7 + (d <= NUW % 7 ? 1 : 0)), nlink = NCH * litems(d);
    char* ws = P.wsx[0];
    char* set = ws + (call & 1) * PAR_BYTES;
    const char* pkb = set + OFF_PK + (size_t)lbase(d) * ITEM_PK + 2048;
    const char* flags = ws + OFF_FLAG + (call & 1) * FLAG_BYTES + (size_t)lbase(d) * 64;
    const unsigned lds0 = (unsigned)(size_t)buf;
    build_lut<false>(buf, lane);
    u32x4 nx[6]; unsigned nf = 0;
#pragma unroll
    for (int k = 0; k < 6; ++k) nx[k] = (u32x4){0u, 0u, 0u, 0u};
#define LOAD_ITEM(i, mask) asm volatile("s_mov_b32 exec_lo, %11\n\ts_mov_b32 exec_hi, %11\n\t" \
        "global_load_dwordx4 %0, %7, %8 offset:-2048 sc0 sc1\n\tglobal_load_dwordx4 %1, %7, %8 offset:-1024 sc0 sc1\n\tglobal_load_dwordx4 %2, %7, %8 sc0 sc1\n\t" \
        "global_load_dwordx4 %3, %7, %8 offset:1024 sc0 sc1\n\tglobal_load_dwordx4 %4, %7, %8 offset:2048 sc0 sc1\n\tglobal_load_dwordx4 %5, %7, %8 offset:3072 sc0 sc1\n\t" \
        "global_load_dword %6, %9, %10 sc0 sc1\n\ts_mov_b64 exec, -1" \
        : "+v"(nx[0]), "+v"(nx[1]), "+v"(nx[2]), "+v"(nx[3]), "+v"(nx[4]), "+v"(nx[5]), "+v"(nf) \
        : "v"(lane * 16), "s"(pkb + (size_t)(i) * ITEM_PK), "v"(0u), "s"(flags + (size_t)((i) + step < nlink ? (i) + step : (i)) * 64), "s"(mask) : "memory")
#define WAIT_ITEM(n) asm volatile("s_waitcnt vmcnt(" #n ")" : "+v"(nx[0]), "+v"(nx[1]), "+v"(nx[2]), "+v"(nx[3]), "+v"(nx[4]), "+v"(nx[5]), "+v"(nf) :: "memory")
#define SIGNAL() do { if (pending) { if (lane == 0) __hip_atomic_fetch_add((GAS unsigned*)(ws + pending), 1u, __ATOMIC_RELAXED, __HIP_MEMORY_SCOPE_AGENT); \
        DBG_AT((lbase(d) + pend_i) * 24 + 8); pending = 0; } } while (0)
    unsigned f = 0, pending = 0; bool have = false;
    int pend_i = 0; (void)pend_i;
    for (int i = i0; i < nlink; i += step) {
        if (!have) {
            WAIT_V(0); SIGNAL();
            Wait wait;
            bool dead = false;
            while (((f = (unsigned)__builtin_amdgcn_readfirstlane(ld_sys(flags + (size_t)i * 64))) >> 8) < call)
                if (wait.give_up(ws, call)) { dead = true; break; }
            if (dead) break;
            LOAD_ITEM(i, ~0u); WAIT_ITEM(0);
        }
        DBG_AT((lbase(d) + i) * 24);
        u32x4 pk[6];
#pragma unroll
        for (int k = 0; k < 6; ++k) pk[k] = nx[k];
        const unsigned top4 = (f & 255u) * 0x01010101u;
        f = (unsigned)__builtin_amdgcn_readfirstlane(nf);
        const unsigned ahead = below(i + step, nlink) & ~below((int)(f >> 8), (int)call);
        have = ahead != 0u;
        LOAD_ITEM(i + step < nlink ? i + step : i, ahead); SCHED;
        *(LAS u32x4*)(buf + CODEC_STREAM + lane * 16) = pk[5];
        unsigned cg[8], at[8], tot = 0; u32x2 sel[8];
#pragma unroll
        for (int v = 0; v < 8; ++v) {
            cg[v] = (pk[4][v >> 1] >> (4 * (v & 1))) & 0x0f0f0f0fu;
            const unsigned e = esc_bits(cg[v]);
            sel[v] = lut_of(e, lds0 + CODEC_LUT);
            at[v] = tot; tot += __builtin_popcount(e);
        }
        const unsigned n0 = wave_scan(tot) - tot + (lds0 + CODEC_STREAM) * 2u;
        asm volatile("" ::: "memory");
        const Item it = item_of(d, i);
        const unsigned sink = (unsigned)(OFF_SINK - (call & 1) * PAR_BYTES) - it.fin + uw * 1024 + lane * 16;
        unsigned xs[8];
#pragma unroll
        for (int v = 0; v < 8; ++v) {
            const unsigned n = n0 + at[v];
            const LAS unsigned* q = (const LAS unsigned*)(size_t)((n >> 1) & ~3u);
            xs[v] = __builtin_amdgcn_alignbit(q[1], q[0], n * 4u);
        }
#pragma unroll
        for (int v = 0; v < 8; ++v) {
            const unsigned xe = xs[v] & 0x0f0f0f0fu, xo = (xs[v] >> 4) & 0x0f0f0f0fu;
            const unsigned e0 = top4 - ((cg[v] & 0x03030303u) | __builtin_amdgcn_perm(xo, xe, sel[v].x));
            const unsigned e1 = top4 - (((cg[v] >> 2) & 0x03030303u) | __builtin_amdgcn_perm(xo, xe, sel[v].y));
            const u32x4 r = join8(pk[v >> 1][(v & 1) * 2], e0, pk[v >> 1][(v & 1) * 2 + 1], e1);
            store16_sys(set + it.fin, cg[v] != 0u ? v * 1024 + lane * 16 : sink, r);
        }
        WAIT_ITEM(8);
        SIGNAL();
        pending = it.cnt; pend_i = i;
        if (i == i0) STAMP(1);
    }
    WAIT_V(0); SIGNAL();
}

__device__ __forceinline__ int strip_of(int pos, bool remote, int q) {
    if (!remote) return pos < 32 + NSLOT ? pos : -1;
    if (pos < EH) return pos;
    if (pos < EH + 3 * OH) { const int k = (pos - EH) / OH; return 32 + 3 * EH + (k + (k >= q ? 1 : 0)) * OH + (pos - EH) % OH; }
    return (unsigned)(pos - NTE * SR) < 3u * EH ? 32 + pos - NTE * SR : -1;
}
__device__ __forceinline__ int out_row_of(int id, int rank) {
    if (id < 32) return rank * 1024 + id * 32;
    const int slot = id - 32;
    if (slot < 3 * EH) return (rank ^ (2 * (slot / EH + 1))) * 1024 + (slot % EH) * 32;
    const int o = slot - 3 * EH;
    return (rank ^ (2 * (o / OH) + 1)) * 1024 + (EH + o % OH) * 32;
}

constexpr int OUT_M = 256, OUT_N = OUT_M + 2048, OUT_LDS = 4160;
template <bool PACK> __device__ __forceinline__ u32x4 out_lut(int m) {
    unsigned long long sm = 0x0c0c0c0c0c0c0c0cull, sn = sm; int k = 0;
#pragma unroll
    for (int b = 0; b < 8; ++b) if (m >> b & 1) { const int slot = (k >> 1) | ((k & 1) << 2);
        sm ^= PACK ? (unsigned long long)(0x0c ^ b) << (8 * k) : (unsigned long long)(0x0c ^ k) << (8 * b);
        sn ^= PACK ? (unsigned long long)(0x0c ^ b) << (8 * slot) : (unsigned long long)(0x0c ^ slot) << (8 * b); ++k; }
    return (u32x4){(unsigned)sm, (unsigned)(sm >> 32), (unsigned)sn, (unsigned)(sn >> 32)};
}
struct OutItem { int o, q, w, f0, ida, idb; };
__device__ __forceinline__ OutItem out_item(int n) {
    const bool late = n >= NTE * HPT * 32;
    const int n1 = late ? n - NTE * HPT * 32 : n, per = (late ? NTR - NTE : NTE) * 32, h = late ? n1 / ((NTR - NTE) * 32) : n1 / (NTE * 32), rt = (late ? NTE : 0) + ((n1 - h * per) >> 5);
    OutItem t; t.w = (n1 >> 2) & 7; t.q = n1 & 3; t.f0 = (t.w >> 2) * SR + 2 * h;
    t.ida = n < NOI ? strip_of(rt * SR + (t.f0 >> 1), true, t.q) : -1; t.idb = 2 * h + 1 < SR ? strip_of(rt * SR + ((t.f0 + 1) >> 1), true, t.q) : -1;
    t.o = ((t.q * NTR + rt) * 8 + t.w) * HPT + h;
    return t;
}
__device__ __forceinline__ void expander(const Params& P, LAS unsigned char* buf, int xw, unsigned call, bool idle) {
    const int lane = threadIdx.x & 63;
    char* ws = P.wsx[0];
    const unsigned lds0 = (unsigned)(size_t)buf;
    for (int m = lane; m < 256; m += 64) *(LAS u32x4*)(buf + OUT_LDS + m * 16) = out_lut<false>(m);
    const OutItem mine = out_item(xw + (lane & 7) * NXW);
    unsigned long long todo = __builtin_amdgcn_ballot_w64(lane < 8 && mine.ida >= 0);
    const char* watch = ws + OFF_OFLAG + (mine.ida >= 0 ? mine.o : 0) * 64;
    while (todo) {
        unsigned long long fl, ready; bool dead = false;
        { Wait wait;
          while ((ready = __builtin_amdgcn_ballot_w64((unsigned)((fl = __hip_atomic_load((const GAS unsigned long long*)watch, __ATOMIC_RELAXED, __HIP_MEMORY_SCOPE_SYSTEM)) >> 32) >= call) & todo) == 0ull) {
              if (wait.give_up(ws, call)) { dead = true; ready = todo; break; }
              if (idle) __builtin_amdgcn_s_sleep(48);
          } }
        idle = false;
        const int k = __builtin_ctzll(ready);
        todo &= ~(1ull << k);
        const unsigned f = (unsigned)__builtin_amdgcn_readlane((unsigned)fl, k);
        const OutItem t = out_item(xw + k * NXW);
#define UNI(x) __builtin_amdgcn_readfirstlane(x)
        const int o = UNI(t.o), q = UNI(t.q), w = UNI(t.w), f0 = UNI(t.f0), ida = UNI(t.ida), idb = UNI(t.idb);
        DBG_AT(DBG_OUT + o * 32 + 8);
        u32x4 r[4];
        unsigned tmo = call;
        if (!dead) {
            u32x4 t[4]; unsigned bits;
            const unsigned raw = below(0, (int)(f & 0x100000u)), a0 = lane * 16 + (OUT_M & ~raw);
            asm volatile("global_load_dwordx4 %0, %6, %8 sc0 sc1\n\tglobal_load_dwordx4 %1, %6, %8 offset:1024 sc0 sc1\n\tglobal_load_dwordx4 %2, %6, %8 offset:2048 sc0 sc1\n\t"
                         "global_load_dword %4, %7, %8 sc0 sc1\n\tglobal_load_dword %5, %9, %10 sc1\n\t"
                         "s_mov_b32 exec_lo, %11\n\ts_mov_b32 exec_hi, %11\n\tglobal_load_dwordx4 %3, %6, %8 offset:3072 sc0 sc1\n\ts_mov_b64 exec, -1\n\ts_waitcnt vmcnt(0)"
                         : "=&v"(t[0]), "=&v"(t[1]), "=&v"(t[2]), "=&v"(t[3]), "=&v"(bits), "=&v"(tmo)
                         : "v"(a0), "v"(lane * 4), "s"(ws + OFF_OPK + (size_t)o * OITEM), "v"(0u), "s"(ws + OFF_TMO), "s"(raw) : "memory");
            if (!raw) {
#pragma unroll
                for (int k = 0; k < 3; ++k) *(LAS u32x4*)(buf + OUT_M + lane * 16 + k * 1024) = t[k];
                const unsigned tot = __builtin_popcount(bits), base4 = ((f >> 12) & 255u) * 0x01010101u;
                unsigned n = (wave_scan(tot) - tot) & 2047u;
                asm volatile("" ::: "memory");
#pragma unroll
                for (int k = 0; k < 4; ++k) {
                    const unsigned m = (bits >> (8 * k)) & 255u;
                    const u32x4 sel = *(const LAS u32x4*)(buf + OUT_LDS + m * 16);
                    const LAS unsigned* qm = (const LAS unsigned*)(size_t)(lds0 + OUT_M + (n & ~3u));
                    const LAS unsigned* qn = (const LAS unsigned*)(size_t)(lds0 + OUT_N + ((n >> 1) & ~3u));
                    const unsigned m0 = __builtin_amdgcn_alignbyte(qm[1], qm[0], n), m1 = __builtin_amdgcn_alignbyte(qm[2], qm[1], n), x = __builtin_amdgcn_alignbit(qn[1], qn[0], n * 4u);
                    const unsigned xe = x & 0x0f0f0f0fu, xo = (x >> 4) & 0x0f0f0f0fu;
                    const unsigned e0 = (base4 - __builtin_amdgcn_perm(xo, xe, sel.z)) & __builtin_amdgcn_perm(~0u, ~0u, sel.x);
                    const unsigned e1 = (base4 - __builtin_amdgcn_perm(xo, xe, sel.w)) & __builtin_amdgcn_perm(~0u, ~0u, sel.y);
                    t[k] = join8(__builtin_amdgcn_perm(m1, m0, sel.x), e0, __builtin_amdgcn_perm(m1, m0, sel.y), e1);
                    n += __builtin_popcount(m);
                }
                asm volatile("" ::: "memory");
            }
#pragma unroll
            for (int k = 0; k < 8; ++k) {
                const int row = (k >> 2) * 16 + (lane & 15), ch = (k & 3) * 2 + (lane >> 5);
                *(LAS u32x2*)(buf + row * 128 + ((ch ^ (row & 7)) * 16) + ((lane >> 4) & 1) * 8) = (u32x2){t[k >> 1][(k & 1) * 2], t[k >> 1][(k & 1) * 2 + 1]};
            }
            asm volatile("" ::: "memory");
#pragma unroll
            for (int v = 0; v < 4; ++v) { const int row = v * 8 + (lane >> 3); r[v] = *(const LAS u32x4*)(buf + row * 128 + (((lane & 7) ^ (row & 7)) * 16)); }
        }
        if (tmo == call) {
#pragma unroll
            for (int v = 0; v < 4; ++v) r[v] = (u32x4){0x7fc07fc0u, 0x7fc07fc0u, 0x7fc07fc0u, 0x7fc07fc0u};
        }
        const int src = P.rank ^ (2 * q + 1);
        const unsigned ooff = (lane >> 3) * OPITCH + q * 512 + ((lane >> 2) & 1) * 256 + (w & 3) * 64 + (lane & 3) * 16;
        { char* O = P.out + (size_t)(out_row_of(ida, src) + (f0 & 1) * 16) * OPITCH; store16_sys(O, ooff, r[0]); store16_sys(O, ooff + 8 * OPITCH, r[1]); }
        if (idb >= 0) { char* O = P.out + (size_t)(out_row_of(idb, src) + ((f0 + 1) & 1) * 16) * OPITCH; store16_sys(O, ooff, r[2]); store16_sys(O, ooff + 8 * OPITCH, r[3]); }
        DBG_AT(DBG_OUT + o * 32 + 16);
    }
}

__device__ __forceinline__ unsigned cvt_pk_bf16(float lo, float hi) { unsigned r; asm("v_cvt_pk_bf16_f32 %0, %1, %2" : "=v"(r) : "v"(lo), "v"(hi)); return r; }
template <int OFF> __device__ __forceinline__ s16x4 tr_read(unsigned addr) { s16x4 r; asm volatile("ds_read_b64_tr_b16 %0, %1 offset:%2" : "=v"(r) : "v"(addr), "i"(OFF) : "memory"); return r; }
template <int OFF> __device__ __forceinline__ bf16x8 row_read(unsigned addr) { bf16x8 r; asm volatile("ds_read_b128 %0, %1 offset:%2" : "=v"(r) : "v"(addr), "i"(OFF) : "memory"); return r; }
__device__ __forceinline__ void glds(unsigned ldsaddr, unsigned voff, const char* g) {
    asm volatile("s_mov_b32 m0, %0\n\ts_nop 0\n\tglobal_load_lds_dwordx4 %1, %2" GLF :: "s"(ldsaddr), "v"(voff), "s"(g) : "memory");
}
__device__ __forceinline__ void glds_masked(unsigned ldsaddr, unsigned voff, const char* g, unsigned mask) {
    asm volatile("s_mov_b32 m0, %0\n\ts_mov_b32 exec_lo, %3\n\ts_mov_b32 exec_hi, %3\n\tglobal_load_lds_dwordx4 %1, %2" GLF "\n\ts_mov_b64 exec, -1" :: "s"(ldsaddr), "v"(voff), "s"(g), "s"(mask) : "memory");
}

template <int S> struct Frag { bf16x8 a[S], b[4]; };
template <int S> __device__ __forceinline__ void wait_step(Frag<S>& F, unsigned& pre) {
    asm volatile("s_waitcnt vmcnt(6) lgkmcnt(0)" : "+v"(F.a[0]), "+v"(F.a[1]), "+v"(F.a[2]), "+v"(F.a[3]), "+v"(F.a[S - 1]), "+v"(F.b[0]), "+v"(F.b[1]), "+v"(F.b[2]), "+v"(F.b[3]), "+v"(pre) :: "memory");
    if constexpr (S > 5) asm volatile("" : "+v"(F.a[4]));
    if constexpr (S > 6) asm volatile("" : "+v"(F.a[5]));
    if constexpr (S > 7) asm volatile("" : "+v"(F.a[6]));
}
template <int S, bool remote> __device__ __forceinline__ void gemm_tile(const Params& P, LAS unsigned char* lds, int rt, int ct, unsigned call) {
    constexpr int SLOT_A = S * 2048, SLOT = SLOT_A + 16384;
    static_assert(S >= 5 && S <= 8 && 4 * SLOT <= LDS_STAGE && S * 16384 <= LDS_STAGE, "tile");
    int wid = __builtin_amdgcn_readfirstlane(threadIdx.x >> 6), lane = threadIdx.x & 63, wr = wid >> 2, wc = wid & 3, fr = lane & 15, fq = lane >> 4;
    char* ws = P.wsx[0];
    const char* set = ws + (call & 1) * PAR_BYTES;
    const char* const own_x = P.x;
    const unsigned ldst = (unsigned)(size_t)lds + wid * 1024;
    constexpr int BP = remote ? 512 : BPITCH;
    auto strip_at = [&](int k) { const int id = k < S ? strip_of(rt * S + k, remote, ct) : -1; return id < 0 ? strip_of(rt * S, remote, ct) : id; };
    const int id0 = strip_at(wid >> 1), id1 = strip_at(4 + (wid >> 1));
    const char* const ga0 = id0 < 32 ? own_x + id0 * STRIP_BYTES : set + OFF_XA + (id0 - 32) * STRIP_BYTES;
    const char* const ga1 = id1 < 32 ? own_x + id1 * STRIP_BYTES : set + OFF_XA + (id1 - 32) * STRIP_BYTES;
    const unsigned cs0 = id0 < 32 ? 256 : 8192, cs1 = id1 < 32 ? 256 : 8192;
    const char* const gb = remote ? set + OFF_W2 + ct * (W2_BYTES / 4) : P.w + ct * 512;
    unsigned voffA0, voffA1, voffB;
    { const int k0 = lane >> 4, ch = lane & 15; voffB = (unsigned)((wid * 4 + k0) * BP + (((ch >> 1) ^ (k0 | ((wid & 2) << 1))) * 32) + (ch & 1) * 16);
      const int sb = lane * 16, swz = sb ^ (((sb >> 9) & 1) << 5), row = (wid & 1) * 16 + (swz >> 6);
      voffA0 = (unsigned)(row * (id0 < 32 ? APITCH : 256) + (swz & 63)); voffA1 = (unsigned)(row * (id1 < 32 ? APITCH : 256) + (swz & 63)); }
    const unsigned amask = below(wid, 2 * S - 8);
    unsigned aoff[2], boff[2][2];
    { const unsigned lbase = (unsigned)(size_t)lds; const int ob = fr * 64 + fq * 16; aoff[0] = lbase + wr * S * 1024 + (ob ^ (((ob >> 9) & 1) << 5)); aoff[1] = aoff[0] + 2 * SLOT;
      const int q = (lane & 15) >> 2, p = lane & 3, f = q | ((fq & 1) << 2);
      for (int n = 0; n < 2; ++n) { boff[0][n] = lbase + SLOT_A + (8 * fq + q) * 256 + p * 8 + (((wc * 2 + n) ^ f) * 32); boff[1][n] = boff[0][n] + 2 * SLOT; } }
    unsigned coff, cstep, need;
    { const int id = strip_of(rt * S + (lane < S ? lane : 0), remote, ct);
      const bool watch = wid == 0 && (lane < S ? id >= 32 : (lane == 8 && remote));
      coff = watch ? (unsigned)(OFF_CNT + (lane < S ? id - 32 : NSLOT + ct) * NCH * 128) : (unsigned)OFF_TMO;
      cstep = watch ? 128u : 0u;
      need = watch ? call * (lane < S ? 1u : 8u) : 0u; }
    const unsigned w0 = below(wid, 1);
    unsigned pre = 0;
#define SLOW_WAIT() do { Wait wait; \
        while (__builtin_amdgcn_ballot_w64(ld_agent(ws + coff) < need) != 0ull) if (wait.give_up(ws, call)) break; \
        asm volatile("" ::: "memory"); } while (0)
#define PRE_LOAD(mask) do { coff += cstep; \
        asm volatile("s_mov_b32 exec_lo, %3\n\ts_mov_b32 exec_hi, 0\n\tglobal_load_dword %0, %1, %2 sc1\n\ts_mov_b64 exec, -1" : "+v"(pre) : "v"(coff), "s"(ws), "s"((mask) & w0) : "memory"); } while (0)
#define STAGE_B(h, sl, ch) glds(ldst + (sl) * SLOT + SLOT_A + (h) * 8192, voffB, gb + (h) * 256 + (size_t)((ch) * 4 + (sl)) * (32 * BP))
#define STAGE_A0(sl, ch) glds(ldst + (sl) * SLOT, voffA0, ga0 + (ch) * cs0 + (sl) * 64)
#define STAGE_A1(sl, ch) glds_masked(ldst + (sl) * SLOT + 8192, voffA1, ga1 + (ch) * cs1 + (sl) * 64, amask)
#define STAGE_ALL(sl, ch) do { STAGE_B(0, sl, ch); STAGE_B(1, sl, ch); STAGE_A0(sl, ch); STAGE_A1(sl, ch); } while (0)
#define RDA(F, m, sl) F.a[m] = row_read<((sl) & 1) * SLOT + (m) * 1024>(aoff[(sl) >> 1])
#define RDB(F, j, sl) do { const s16x4 _lo = tr_read<((sl) & 1) * SLOT + ((j) >> 1) * 8192>(boff[(sl) >> 1][(j) & 1]), _hi = tr_read<((sl) & 1) * SLOT + ((j) >> 1) * 8192 + 1024>(boff[(sl) >> 1][(j) & 1]); \
        F.b[j] = __builtin_shufflevector(_lo, _hi, 0, 1, 2, 3, 4, 5, 6, 7); } while (0)
#define MM(F, i) acc[(i) >> 2][(i) & 3] = __builtin_amdgcn_mfma_f32_16x16x32_bf16(F.b[(i) & 3], F.a[(i) >> 2], acc[(i) >> 2][(i) & 3], 0, 0, 0)
#define PAIR(C, i) do { MM(C, i); MM(C, (i) + 1); SCHED; } while (0)
#define STEP(C, N, sl, st, GATE) do { constexpr int nx = ((sl) + 1) & 3; \
        RDB(N, 0, nx); PAIR(C, 0); RDB(N, 1, nx); PAIR(C, 2); RDA(N, 0, nx); RDA(N, 1, nx); PAIR(C, 4); RDB(N, 2, nx); PAIR(C, 6); RDB(N, 3, nx); \
        STAGE_B(0, sl, st); PAIR(C, 8); RDA(N, 2, nx); RDA(N, 3, nx); STAGE_B(1, sl, st); PAIR(C, 10); \
        RDA(N, 4, nx); if constexpr (S > 5) RDA(N, 5, nx); STAGE_A0(sl, st); PAIR(C, 12); \
        if constexpr (S > 6) RDA(N, 6, nx); if constexpr (S > 7) RDA(N, 7, nx); STAGE_A1(sl, st); \
        _Pragma("unroll") for (int i = 14; i < 4 * S - 4; i += 2) PAIR(C, i); \
        wait_step(N, pre); GATE; BAR; SCHED; PAIR(C, 4 * S - 4); PAIR(C, 4 * S - 2); } while (0)
    f32x4 acc[S][4];
#pragma unroll
    for (int m = 0; m < S; ++m)
#pragma unroll
        for (int j = 0; j < 4; ++j) acc[m][j] = (f32x4){0.f, 0.f, 0.f, 0.f};
    Frag<S> F0, F1;
    constexpr int OUT_LUT = LDS_STAGE - 4096;
    static_assert(!remote || (4 * SLOT <= OUT_LUT && 8 * OUT_LDS <= OUT_LUT), "pack tables");
    if (remote && threadIdx.x >= 256) *(LAS u32x4*)(lds + OUT_LUT + (threadIdx.x - 256) * 16) = out_lut<true>(threadIdx.x - 256);
    if (wid == 0) SLOW_WAIT();
    BAR;
    STAMP(1); CSTAMP(6);
    PRE_LOAD(0x3ffu);
    STAGE_ALL(0, 0u); STAGE_ALL(1, 0u); STAGE_ALL(2, 0u); STAGE_ALL(3, 0u);
    WAIT_V(6); BAR;
    RDB(F0, 0, 0); RDB(F0, 1, 0); RDB(F0, 2, 0); RDB(F0, 3, 0);
    RDA(F0, 0, 0); RDA(F0, 1, 0); RDA(F0, 2, 0); RDA(F0, 3, 0); RDA(F0, 4, 0);
    if constexpr (S > 5) RDA(F0, 5, 0);
    if constexpr (S > 6) RDA(F0, 6, 0);
    if constexpr (S > 7) RDA(F0, 7, 0);
    wait_step(F0, pre);
    if (__builtin_amdgcn_ballot_w64(pre < need) != 0ull) SLOW_WAIT();
    BAR;
    for (int c = 0; c < NCH; ++c) {
        const unsigned st = c < NCH - 1 ? c + 1 : c;
        STEP(F0, F1, 0, st, (void)0);
        STEP(F1, F0, 1, st, (void)0);
        PRE_LOAD(c < NCH - 2 ? 0x3ffu : c < NCH - 1 ? 0x200u : 0u);
        STEP(F0, F1, 2, st, (void)0);
        STEP(F1, F0, 3, st, { TS_NOW(ts0, c); if (c < NCH - 2 && __builtin_amdgcn_ballot_w64(pre < need) != 0ull) SLOW_WAIT(); TS_NOW(ts1, c); });
    }
    volatile LAS unsigned* turn = (volatile LAS unsigned*)(lds + LDS_STAGE + 4);
    if (threadIdx.x == 9) { *(LAS unsigned*)(lds + LDS_STAGE) = pre; turn[0] = 0u; turn[1] = 0u; }
    asm volatile("s_waitcnt vmcnt(0) lgkmcnt(0)" ::: "memory");
    BAR;
    asm volatile("" ::: "memory");
    const bool bad = *(const LAS unsigned*)(lds + LDS_STAGE) == call;
    STAMP(2); CSTAMP(7);
    int tz = threadIdx.x; asm volatile("" : "+v"(tz));
    lane = tz & 63; fr = lane & 15; fq = lane >> 4;
    auto relu_pack = [&](int m, int j) {
        f32x4 v = acc[m][j];
#pragma unroll
        for (int e = 0; e < 4; ++e) v[e] = v[e] < 0.f ? 0.f : v[e];
        u32x2 w; w.x = cvt_pk_bf16(v[0], v[1]); w.y = cvt_pk_bf16(v[2], v[3]);
        if (bad) { w.x = 0x7fc07fc0u; w.y = 0x7fc07fc0u; }
        return w;
    };
    if constexpr (!remote) {
#pragma unroll
        for (int m = 0; m < S; ++m) {
            const int row = wr * S * 16 + m * 16 + fr;
#pragma unroll
            for (int j = 0; j < 4; ++j) {
                const int ch = (j >> 1) * 16 + wc * 4 + (j & 1) * 2 + (fq >> 1);
                *(LAS u32x2*)(lds + row * 512 + ((ch ^ (fr & 7)) * 16) + (fq & 1) * 8) = relu_pack(m, j);
            }
        }
        WAIT_L(0); BAR; asm volatile("" ::: "memory");
        u32x4 ov[S][2];
#pragma unroll
        for (int k = 0; k < S; ++k)
#pragma unroll
            for (int i = 0; i < 2; ++i) { const int row = k * 32 + wid * 4 + i * 2 + (lane >> 5); ov[k][i] = *(const LAS u32x4*)(lds + row * 512 + (((lane & 31) ^ (row & 7)) * 16)); }
        const unsigned ooff = (wid * 4 + (lane >> 5)) * OPITCH + ct * 512 + (lane & 31) * 16;
#pragma unroll
        for (int k = 0; k < S; ++k) {
            const int id = strip_of(rt * S + k, false, 0);
            if (id >= 0) { char* O = P.out + (size_t)out_row_of(id, P.rank) * OPITCH; store16_sys(O, ooff, ov[k][0]); store16_sys(O, ooff + 2 * OPITCH, ov[k][1]); }
        }
    } else {
        char* const peer = P.wsx[(2 * ct + 1) & 7];
        LAS unsigned char* buf = lds + wid * OUT_LDS;
        const unsigned lds0 = (unsigned)(size_t)buf;
        const int o0 = ((ct * NTR + rt) * 8 + wid) * HPT;
        int sent = -1; unsigned sent_meta = 0;
#define PUBLISH_OUT() do { WAIT_V(0); if (sent >= 0) { if (lane == 0) { end_turn(turn); \
            __hip_atomic_store((GAS unsigned long long*)(peer + OFF_OFLAG + sent * 64), ((unsigned long long)call << 32) | sent_meta, __ATOMIC_RELAXED, __HIP_MEMORY_SCOPE_SYSTEM); } \
            DBG_AT(DBG_OUT + sent * 32 + 24); sent = -1; } } while (0)
#pragma unroll
        for (int h = 0; h < HPT; ++h) {
            const int f0 = wr * S + 2 * h;
            if (strip_of(rt * S + (f0 >> 1), true, ct) < 0) break;
            const bool second = 2 * h + 1 < S && strip_of(rt * S + ((f0 + 1) >> 1), true, ct) >= 0;
            u32x4 t[4]; u16x2 mx = {0, 0};
#pragma unroll
            for (int k = 0; k < 8; ++k) {
                u32x2 w = relu_pack(2 * h + (k >> 2) < S ? 2 * h + (k >> 2) : 0, k & 3);
                if (k >= 4 && !second) w = (u32x2){0u, 0u};
                t[k >> 1][(k & 1) * 2] = w.x; t[k >> 1][(k & 1) * 2 + 1] = w.y;
                mx = __builtin_elementwise_max(mx, __builtin_elementwise_max(__builtin_bit_cast(u16x2, w.x & 0x7fff7fffu), __builtin_bit_cast(u16x2, w.y & 0x7fff7fffu)));
            }
#pragma unroll
            for (int k = 0; k < 3; ++k) *(LAS u32x4*)(buf + OUT_M + lane * 16 + k * 1024) = (u32x4){0u, 0u, 0u, 0u};
            const unsigned emax = (unsigned)__builtin_amdgcn_readlane(wave_max(mx.x > mx.y ? mx.x : mx.y), 63) >> 7;
            unsigned bits = 0, wide = 0, tot = 0, at[4], cm0[4], cm1[4], cn[4];
#pragma unroll
            for (int k = 0; k < 4; ++k) {
                unsigned z[4], m0, e0, m1, e1;
#pragma unroll
                for (int c = 0; c < 4; ++c) z[c] = nonzero2(t[k][c]);
                const unsigned m = __builtin_amdgcn_udot4(__builtin_amdgcn_perm(z[3], z[2], 0x06040200u), 0x80402010u, __builtin_amdgcn_udot4(__builtin_amdgcn_perm(z[1], z[0], 0x06040200u), 0x08040201u, 0u, false), false);
                const u32x4 sel = *(const LAS u32x4*)(lds + OUT_LUT + m * 16);
                split4(t[k].x, t[k].y, m0, e0); split4(t[k].z, t[k].w, m1, e1);
                e0 = emax * 0x01010101u - e0; e1 = emax * 0x01010101u - e1;
                cm0[k] = __builtin_amdgcn_perm(m1, m0, sel.x); cm1[k] = __builtin_amdgcn_perm(m1, m0, sel.y);
                const unsigned de = __builtin_amdgcn_perm(e1, e0, sel.z), dd = __builtin_amdgcn_perm(e1, e0, sel.w);
                wide |= de | dd; cn[k] = de | (dd << 4);
                bits |= m << (8 * k); at[k] = tot; tot += __builtin_popcount(m);
            }
            const unsigned incl = wave_scan(tot), count = (unsigned)__builtin_amdgcn_readlane(incl, 63);
            const bool rawout = __builtin_amdgcn_ballot_w64((wide & 0xf0f0f0f0u) != 0u) != 0ull;
            char* dst = peer + OFF_OPK + (size_t)(o0 + h) * OITEM;
            if (!rawout) {
                asm volatile("" ::: "memory");
#pragma unroll
                for (int k = 0; k < 4; ++k) {
                    const unsigned n = incl - tot + at[k];
                    const unsigned long long a = (unsigned long long)cm0[k] << ((n & 3u) * 8u), b = (unsigned long long)cm1[k] << ((n & 3u) * 8u), x = (unsigned long long)cn[k] << ((n * 4u) & 31u);
                    LAS unsigned* qm = (LAS unsigned*)(size_t)(lds0 + OUT_M + (n & ~3u));
                    LAS unsigned* qn = (LAS unsigned*)(size_t)(lds0 + OUT_N + ((n >> 1) & ~3u));
#define LDS_OR(p, v) __hip_atomic_fetch_or(p, v, __ATOMIC_RELAXED, __HIP_MEMORY_SCOPE_WAVEFRONT)
                    LDS_OR(qm, (unsigned)a); LDS_OR(qm + 1, (unsigned)(a >> 32) | (unsigned)b); LDS_OR(qm + 2, (unsigned)(b >> 32));
                    LDS_OR(qn, (unsigned)x); LDS_OR(qn + 1, (unsigned)(x >> 32));
                }
                asm volatile("" ::: "memory");
#pragma unroll
                for (int k = 0; k < 3; ++k) t[k] = *(const LAS u32x4*)(buf + OUT_M + lane * 16 + k * 1024);
                asm volatile("" : "+v"(t[0]), "+v"(t[1]), "+v"(t[2]) :: "memory");
            }
            PUBLISH_OUT();
            await_turn(turn, draw_turn(turn, lane), OUT_SLOTS);
            if (rawout) {
#pragma unroll
                for (int k = 0; k < 4; ++k) store16_sys(dst, lane * 16 + k * 1024, t[k]);
            } else {
                store4_sys(dst, lane * 4, bits);
                if (lane * 16u < count) store16_sys(dst, OUT_M + lane * 16, t[0]);
                if (lane * 16u + 1024u < count) store16_sys(dst, OUT_M + 1024 + lane * 16, t[1]);
                if (lane * 32u < count) store16_sys(dst, OUT_N + lane * 16, t[2]);
            }
            sent = o0 + h; sent_meta = (rawout ? 0x100000u : 0u) | (emax << 12) | count;
        }
        PUBLISH_OUT();
    }
    STAMP(3);
}

__global__ __launch_bounds__(NTHR, 2) void ag_gemm_kernel(const Params P) {
    extern __shared__ __attribute__((aligned(16))) unsigned char shm[];
    const int xcd = blockIdx.x & 7, slot = blockIdx.x >> 3, wid = __builtin_amdgcn_readfirstlane(threadIdx.x >> 6);
    char* const ticket = P.wsx[0] + OFF_TICK + (size_t)(blockIdx.x * 8 + wid) * 64;
    STAMP(0);
    int ns = 0, nc = 0, ub = 0, sb = 0;
#pragma unroll
    for (int y = 0; y < 8; ++y) if (y == xcd) { ns = nsend(y); nc = ncomm(y); ub = unp_before(y); sb = snd_before(y); }
    ns = __builtin_amdgcn_readfirstlane(ns); nc = __builtin_amdgcn_readfirstlane(nc); ub = __builtin_amdgcn_readfirstlane(ub); sb = __builtin_amdgcn_readfirstlane(sb);
    if (slot < ns) {
      if (wid < SEND_WAVES) {
        const unsigned call = sender(P, (LAS unsigned char*)shm + wid * WAVE_LDS, (volatile LAS unsigned*)((LAS unsigned char*)shm + 8 * WAVE_LDS), link_of(xcd), wid * ns + slot, ns * SEND_WAVES, ticket);
        STAMP(2);
        expander(P, (LAS unsigned char*)shm + wid * WAVE_LDS, NUW + sb + wid * ns + slot, call, false);
        STAMP(3);
        return;
      }
      BAR;
    }
    unsigned tick = 0;
    if ((threadIdx.x & 63) == 0) tick = __hip_atomic_fetch_add((GAS unsigned*)ticket, 1u, __ATOMIC_RELAXED, __HIP_MEMORY_SCOPE_AGENT);
    const unsigned call = (unsigned)__builtin_amdgcn_readfirstlane(tick) + 1u;
    if (slot < ns) {
        expander(P, (LAS unsigned char*)shm + wid * WAVE_LDS, NUW + sb + wid * ns + slot, call, true);
    } else if (slot < nc) {
        const int uw = ub + (slot - ns) * 8 + wid;
        unpacker(P, (LAS unsigned char*)shm + wid * WAVE_LDS, uw, call);
        STAMP(2);
        expander(P, (LAS unsigned char*)shm + wid * WAVE_LDS, uw, call, false);
        STAMP(3);
    } else {
        const int g = slot - nc;
                const int rt = (g >> 2) * 4 + 3 - (xcd & 3);
        if (xcd < 4) gemm_tile<SL, false>(P, (LAS unsigned char*)shm, rt, g & 3, call);
        else gemm_tile<SR, true>(P, (LAS unsigned char*)shm, rt, g & 3, call);
    }
}
}

extern "C" void kernel_launch(int rank, int nranks, ncclComm_t comm, hipStream_t stream,
                              void* const* d_in, const int* in_sizes, int n_in,
                              void* d_out, int out_size, const int* in_dtypes, int out_dtype,
                              void* const* d_out_all, void* const* d_ws_all, size_t ws_bytes) {
    (void)comm; (void)in_dtypes; (void)out_dtype; (void)d_out_all;
    if (nranks != NR || (unsigned)rank >= (unsigned)NR || n_in != 2 || in_sizes[0] != 1024 * KDIM || in_sizes[1] != KDIM * 1024 || out_size != KDIM * 1024 || ws_bytes < WS_NEED) {
        fprintf(stderr, "kernel_launch: unexpected problem geometry\n");
        return;
    }
    static bool attr_set = false;
    if (!attr_set) { HIP_CHECK(hipFuncSetAttribute((const void*)ag_gemm_kernel, hipFuncAttributeMaxDynamicSharedMemorySize, LDS_BYTES)); attr_set = true; }
    Params P;
    P.x = (const char*)d_in[0]; P.w = (const char*)d_in[1]; P.out = (char*)d_out;
    for (int d = 0; d < NR; ++d) P.wsx[d] = (char*)d_ws_all[rank ^ d];
    P.rank = rank; P.pad = 0;
    hipLaunchKernelGGL(ag_gemm_kernel, dim3(GRID), dim3(NTHR), LDS_BYTES, stream, P);
    HIP_CHECK(hipGetLastError());
}
```

```cpp
#include <hip/hip_runtime.h>
#include <rccl/rccl.h>
#include <cstdio>

#define HIP_CHECK(expr) do { hipError_t _e = (expr); if (_e != hipSuccess) { \
    fprintf(stderr, "HIP error %s at %s:%d\n", hipGetErrorString(_e), __FILE__, __LINE__); \
    return; } } while (0)

namespace {
#define LAS __attribute__((address_space(3)))
#define GAS __attribute__((address_space(1)))
typedef short bf16x8 __attribute__((ext_vector_type(8)));
typedef short s16x4 __attribute__((ext_vector_type(4)));
typedef float f32x4 __attribute__((ext_vector_type(4)));
typedef unsigned u32x4 __attribute__((ext_vector_type(4)));
typedef unsigned u32x2 __attribute__((ext_vector_type(2)));
typedef unsigned short u16x2 __attribute__((ext_vector_type(2)));

constexpr int NR = 8;
constexpr int KDIM = 8192;
constexpr int APITCH = KDIM * 2, BPITCH = 2048, OPITCH = 2048;
#define E_STRIPS 21
constexpr int EH = E_STRIPS, OH = 32 - EH;
constexpr int NSLOT = 3 * EH + 4 * OH;
#define SL 7
#define SR 5
constexpr int NTL = (32 + NSLOT + SL - 1) / SL, NTE = (EH + 3 * OH + SR - 1) / SR, NTR = NTE + (3 * EH + SR - 1) / SR, HPT = (SR + 1) / 2;
constexpr int NCH = 64;
__host__ __device__ constexpr int litems(int d) { return (d & 1) ? OH + 8 : EH; }
__host__ __device__ constexpr int lbase(int d) { return NCH * ((d >> 1) * litems(1) + ((d - 1) >> 1) * litems(0)); }
constexpr int NU = lbase(8);
constexpr int ITEM_PK = 6144;
constexpr size_t STRIP_BYTES = (size_t)32 * APITCH;
constexpr size_t XA_BYTES = NSLOT * STRIP_BYTES, W2_BYTES = (size_t)KDIM * BPITCH, PK_BYTES = (size_t)NU * ITEM_PK;
constexpr size_t OFF_XA = 0, OFF_W2 = XA_BYTES, OFF_PK = XA_BYTES + W2_BYTES, PAR_BYTES = OFF_PK + PK_BYTES;
constexpr size_t OFF_FLAG = 2 * PAR_BYTES, FLAG_BYTES = (size_t)NU * 64;
constexpr size_t OFF_CNT = OFF_FLAG + 2 * FLAG_BYTES, CNT_BYTES = (size_t)(NSLOT + 4) * NCH * 64;
constexpr size_t OFF_TICK = OFF_CNT + CNT_BYTES, TICK_BYTES = 512 * 64;
#define OSTRIDE 4096
constexpr int OITEM = OSTRIDE, NOI = 4 * NTR * 8 * HPT;
constexpr size_t OFF_OPK = OFF_TICK + TICK_BYTES, OFF_OFLAG = OFF_OPK + (size_t)NOI * OITEM;
constexpr size_t OFF_TMO = OFF_OFLAG + (size_t)NOI * 64, OFF_QUEUE = OFF_TMO + 128, OFF_SINK = OFF_QUEUE + 8 * 128, OFF_DBG = OFF_SINK + 384 * 1024, DBG_OUT = NU * 24 + 256 * 64 * 8, DBG_BYTES = 2048 * 64 + DBG_OUT + NOI * 32 + 1024 * 32;
constexpr size_t WS_NEED = OFF_DBG;
static_assert(WS_NEED <= ((size_t)257 << 20), "workspace");
constexpr int GRID = 256, NTHR = 512;
#define SEND_WGS 6
#define GLF " sc0 sc1"
#define OUT_SLOTS 6
#define SEND_SLOTS 3
#define TURN_SLEEP 8
#define POLL_SLEEP 2
__host__ __device__ constexpr int ncomm(int x) { return 32 - 4 * (((x < 4 ? NTL : NTR) + x % 4) / 4); }
__host__ __device__ constexpr int nsend(int x) { return x == 3 ? 0 : SEND_WGS; }
__host__ __device__ constexpr int link_of(int x) { return x < 3 ? 2 * x + 2 : x < 4 ? 1 : 2 * x - 7; }
struct Before { int unp[9], snd[9]; };
constexpr Before before() { Before b = {}; for (int x = 0; x < 8; ++x) { b.unp[x + 1] = b.unp[x] + (ncomm(x) - nsend(x)) * 8; b.snd[x + 1] = b.snd[x] + nsend(x) * 8; } return b; }
template <int X> struct BeforeAt { static constexpr int unp = before().unp[X], snd = before().snd[X]; };
constexpr int NUW = BeforeAt<8>::unp, NXW = NUW + BeforeAt<8>::snd;
constexpr unsigned long long DEADLINE = 50000000ull;
constexpr int LDS_STAGE = 135168, LDS_BYTES = LDS_STAGE + 16;
static_assert(NOI <= 8 * NXW && NUW <= 384 && ncomm(2) >= nsend(2) && ncomm(3) >= 0 && ncomm(7) >= nsend(7), "roles");

__host__ __device__ constexpr int renum(int r) { return r == 3 ? 4 : r == 4 ? 3 : r; }
struct Params {
    const char* x; const char* w;
    char* out;
    char* wsx[NR];
    int rank; int pad;
};

#define WAIT_V(n) asm volatile("s_waitcnt vmcnt(" #n ")" ::: "memory")
#define WAIT_L(n) asm volatile("s_waitcnt lgkmcnt(" #n ")" ::: "memory")
#define BAR __builtin_amdgcn_s_barrier()
#define SCHED __builtin_amdgcn_sched_barrier(0)
#define STAMP(i) do {} while (0)
#define CSTAMP(i) do {} while (0)
#define DBG_AT(off) do {} while (0)
#define TS_NOW(dst, k) do {} while (0)

__device__ __forceinline__ unsigned ld_agent(const char* p) { return __hip_atomic_load((const GAS unsigned*)p, __ATOMIC_RELAXED, __HIP_MEMORY_SCOPE_AGENT); }
__device__ __forceinline__ unsigned ld_sys(const char* p) { return __hip_atomic_load((const GAS unsigned*)p, __ATOMIC_RELAXED, __HIP_MEMORY_SCOPE_SYSTEM); }
struct Wait {
    unsigned long long t0; unsigned polls;
    __device__ __forceinline__ Wait() : t0(__builtin_amdgcn_s_memrealtime()), polls(0) {}
    __device__ __forceinline__ bool give_up(char* ws, unsigned call) {
        if ((++polls & 15u) == 0u && ((unsigned)__builtin_amdgcn_readfirstlane(ld_agent(ws + OFF_TMO)) == call || __builtin_amdgcn_s_memrealtime() - t0 > DEADLINE)) {
            __hip_atomic_store((GAS unsigned*)(ws + OFF_TMO), call, __ATOMIC_RELAXED, __HIP_MEMORY_SCOPE_AGENT);
            return true;
        }
        __builtin_amdgcn_s_sleep(POLL_SLEEP);
        return false;
    }
};
__device__ __forceinline__ void store16_sys(char* base, unsigned voff, u32x4 v) {
    asm volatile("global_store_dwordx4 %0, %1, %2 sc0 sc1\n\ts_nop 1" :: "v"(voff), "v"(v), "s"(base) : "memory");
}

template <int CTRL, int ROWS> __device__ __forceinline__ unsigned max_dpp(unsigned x) { const unsigned y = (unsigned)__builtin_amdgcn_update_dpp(0u, x, CTRL, ROWS, 0xf, false); return x > y ? x : y; }
__device__ __forceinline__ unsigned wave_max(unsigned x) {
    return max_dpp<0x143, 0xc>(max_dpp<0x142, 0xa>(max_dpp<0x118, 0xf>(max_dpp<0x114, 0xf>(max_dpp<0x112, 0xf>(max_dpp<0x111, 0xf>(x))))));
}
__device__ __forceinline__ unsigned nonzero2(unsigned w) { unsigned r; asm("v_pk_min_u16 %0, %1, %2" : "=v"(r) : "v"(w), "s"(0x00010001u)); return r; }
__device__ __forceinline__ unsigned below(int a, int b) { unsigned r; asm("s_ashr_i32 %0, %1, 31" : "=s"(r) : "s"(a - b) : "scc"); return r; }
__device__ __forceinline__ unsigned wave_scan(unsigned x) {
    x += __builtin_amdgcn_update_dpp(0u, x, 0x111, 0xf, 0xf, false); x += __builtin_amdgcn_update_dpp(0u, x, 0x112, 0xf, 0xf, false);
    x += __builtin_amdgcn_update_dpp(0u, x, 0x114, 0xf, 0xf, false); x += __builtin_amdgcn_update_dpp(0u, x, 0x118, 0xf, 0xf, false);
    x += __builtin_amdgcn_update_dpp(0u, x, 0x142, 0xa, 0xf, false); x += __builtin_amdgcn_update_dpp(0u, x, 0x143, 0xc, 0xf, false);
    return x;
}

__device__ __forceinline__ unsigned draw_turn(volatile LAS unsigned* turn, int lane) {
    unsigned t = 0;
    if (lane == 0) t = __hip_atomic_fetch_add((LAS unsigned*)turn, 1u, __ATOMIC_RELAXED, __HIP_MEMORY_SCOPE_WORKGROUP);
    return (unsigned)__builtin_amdgcn_readfirstlane(t);
}
__device__ __forceinline__ void await_turn(volatile LAS unsigned* turn, unsigned mine, unsigned slots) {
    while (mine >= (unsigned)__builtin_amdgcn_readfirstlane(turn[1]) + slots) __builtin_amdgcn_s_sleep(TURN_SLEEP);
}
__device__ __forceinline__ void end_turn(volatile LAS unsigned* turn) { __hip_atomic_fetch_add((LAS unsigned*)turn + 1, 1u, __ATOMIC_RELAXED, __HIP_MEMORY_SCOPE_WORKGROUP); }

constexpr int CODEC_LUT = 0, CODEC_STREAM = 2048, CODEC_LDS = CODEC_STREAM + 1088;
constexpr int WAVE_LDS = 8448;
static_assert(CODEC_LDS <= WAVE_LDS && 8 * WAVE_LDS + 16 <= LDS_STAGE, "per-wave LDS");
__device__ __forceinline__ unsigned esc_bits(unsigned c) { return c == 0u ? 0u : ~(c | (c >> 1)) & 0x05050505u; }
__device__ __forceinline__ u32x2 lut_of(unsigned e, unsigned lut) {
    const LAS unsigned* q = (const LAS unsigned*)(size_t)__builtin_amdgcn_udot4(e, 0x80400804u, lut, false);
    return (u32x2){q[0], q[256]};
}
template <bool PACK> __device__ __forceinline__ void build_lut(LAS unsigned char* buf, int lane) {
    for (int m = lane; m < 256; m += 64) {
        unsigned long long sel = 0x0c0c0c0c0c0c0c0cull; int k = 0;
#pragma unroll
        for (int b = 0; b < 8; ++b) if (m >> b & 1) { const int pos = (b & 1) | ((b & 2) << 1) | ((b & 4) >> 1), slot = (k >> 1) | ((k & 1) << 2);
            sel ^= PACK ? (unsigned long long)(0x0c ^ pos) << (8 * slot) : (unsigned long long)(0x0c ^ slot) << (8 * pos); ++k; }
        *(LAS unsigned*)(buf + CODEC_LUT + m * 4) = (unsigned)sel; *(LAS unsigned*)(buf + CODEC_LUT + 1024 + m * 4) = (unsigned)(sel >> 32);
    }
    asm volatile("" ::: "memory");
}
__device__ __forceinline__ unsigned bfi(unsigned m, unsigned a, unsigned b) { return (a & m) | (b & ~m); }
__device__ __forceinline__ void split4(unsigned r0, unsigned r1, unsigned& m, unsigned& e) {
    const unsigned lo = __builtin_amdgcn_perm(r1, r0, 0x06040200u), hi = __builtin_amdgcn_perm(r1, r0, 0x07050301u);
    m = bfi(0x80808080u, hi, lo); e = bfi(0x01010101u, lo >> 7, hi << 1);
}
__device__ __forceinline__ u32x4 join8(unsigned m0, unsigned e0, unsigned m1, unsigned e1) {
    const unsigned h0 = bfi(0x80808080u, m0, e0 >> 1), l0 = bfi(0x7f7f7f7fu, m0, e0 << 7), h1 = bfi(0x80808080u, m1, e1 >> 1), l1 = bfi(0x7f7f7f7fu, m1, e1 << 7);
    return (u32x4){__builtin_amdgcn_perm(h0, l0, 0x05010400u), __builtin_amdgcn_perm(h0, l0, 0x07030602u), __builtin_amdgcn_perm(h1, l1, 0x05010400u), __builtin_amdgcn_perm(h1, l1, 0x07030602u)};
}

struct Item { bool isw; unsigned src, fin, cnt; };
__device__ __forceinline__ Item item_of(int d, int i) {
    const int chunk = (int)((d & 1) ? (unsigned)i / (unsigned)litems(1) : (unsigned)i / (unsigned)litems(0)), j = i - chunk * litems(d);
    Item it;
    it.isw = (d & 1) && j >= OH;
    if (it.isw) { it.src = (unsigned)((chunk * 128 + (j - OH) * 16) * BPITCH + (d >> 1) * 512); it.fin = (unsigned)(OFF_W2 + ((d >> 1) * NCH * 8 + chunk * 8 + j - OH) * 8192);
        it.cnt = (unsigned)(OFF_CNT + ((NSLOT + (d >> 1)) * NCH + chunk) * 64); }
    else { const int strip = (d & 1) ? EH + j : j, slot = (d & 1) ? 3 * EH + (d >> 1) * OH + j : ((d >> 1) - 1) * EH + j;
        it.src = (unsigned)(strip * STRIP_BYTES + chunk * 256); it.fin = (unsigned)(OFF_XA + slot * STRIP_BYTES + chunk * 8192);
        it.cnt = (unsigned)(OFF_CNT + (slot * NCH + chunk) * 64); }
    return it;
}
__device__ __forceinline__ void fetch(const char* src_x, const char* src_w, u32x4 (&nxt)[8], unsigned& tick, int d, int i, unsigned mask, const char* word, unsigned draw) {
    const int lane = threadIdx.x & 63;
    const Item x = item_of(d, i);
    const unsigned v = x.isw ? (lane >> 5) * BPITCH + (lane & 31) * 16 : (lane >> 4) * APITCH + (lane & 15) * 16, st = x.isw ? 2 * BPITCH : 4 * APITCH;
    asm volatile("s_mov_b32 exec_lo, %21\n\ts_mov_b32 exec_hi, %21\n\t"
                 "global_load_dwordx4 %0, %9, %17\n\tglobal_load_dwordx4 %1, %10, %17\n\tglobal_load_dwordx4 %2, %11, %17\n\tglobal_load_dwordx4 %3, %12, %17\n\t"
                 "global_load_dwordx4 %4, %13, %17\n\tglobal_load_dwordx4 %5, %14, %17\n\tglobal_load_dwordx4 %6, %15, %17\n\tglobal_load_dwordx4 %7, %16, %17\n\t"
                 "s_mov_b32 exec_lo, %22\n\ts_mov_b32 exec_hi, 0\n\tglobal_atomic_add %8, %18, %19, %20 sc0\n\ts_mov_b64 exec, -1"
                 : "=&v"(nxt[0]), "=&v"(nxt[1]), "=&v"(nxt[2]), "=&v"(nxt[3]), "=&v"(nxt[4]), "=&v"(nxt[5]), "=&v"(nxt[6]), "=&v"(nxt[7]), "=&v"(tick)
                 : "v"(v), "v"(v + st), "v"(v + 2 * st), "v"(v + 3 * st), "v"(v + 4 * st), "v"(v + 5 * st), "v"(v + 6 * st), "v"(v + 7 * st),
                   "s"((x.isw ? src_w : src_x) + x.src), "v"(0u), "v"(1u), "s"(word), "s"(mask), "s"(draw) : "memory");
}
#define WAIT_FETCH(nxt, tick) asm volatile("s_waitcnt vmcnt(0)" : "+v"(tick), "+v"(nxt[0]), "+v"(nxt[1]), "+v"(nxt[2]), "+v"(nxt[3]), "+v"(nxt[4]), "+v"(nxt[5]), "+v"(nxt[6]), "+v"(nxt[7]) :: "memory")
__device__ __forceinline__ void sender(const Params& P, LAS unsigned char* buf, volatile LAS unsigned* turn, int d, int first, unsigned call, u32x4 (&nxt)[8]) {
    const int lane = threadIdx.x & 63, nlink = NCH * litems(d), nwaves = SEND_WGS * 8;
    char* const queue = P.wsx[0] + OFF_QUEUE + d * 128;
    const char* const src_x = P.x; const char* const src_w = P.w;
    const unsigned lds0 = (unsigned)(size_t)buf;
    unsigned mine = (unsigned)__builtin_amdgcn_readfirstlane(threadIdx.x >> 6), tick;
    int cur = first;
    char* const peer = P.wsx[d & 7];
    char* const set = peer + (call & 1) * PAR_BYTES;
    char* const flags = peer + OFF_FLAG + (call & 1) * FLAG_BYTES;
    const unsigned qbase = (call - 1u) * nlink - 2 * nwaves;
    tick = qbase + first + nwaves;
    while (cur < nlink) {
        const int u = lbase(d) + cur;
        u32x4 raw[8];
#pragma unroll
        for (int v = 0; v < 8; ++v) raw[v] = nxt[v];
        const int next = (int)((unsigned)__builtin_amdgcn_readfirstlane(tick) - qbase);
        fetch(src_x, src_w, nxt, tick, d, next < nlink ? next : nlink - 1, ~0u, queue, 1u); SCHED;
        u32x4 pk[6]; unsigned ex[16], esum = 0;
#pragma unroll
        for (int g = 0; g < 16; ++g) {
            unsigned m; split4(raw[g >> 1][(g & 1) * 2], raw[g >> 1][(g & 1) * 2 + 1], m, ex[g]);
            pk[g >> 2][g & 3] = m;
            if ((g & 3) == 0) esum = __builtin_amdgcn_sad_u8(ex[g], 0u, esum);
        }
        *(LAS u32x4*)(buf + CODEC_STREAM + lane * 16) = (u32x4){0u, 0u, 0u, 0u};
        unsigned top = (((unsigned)__builtin_amdgcn_readlane(wave_scan(esum), 63) + 1023u + 174u) >> 10) + 2u;
        top = top < 18u ? 18u : top > 255u ? 255u : top;
        unsigned cg[8], xs[8], at[8], tot = 0;
#pragma unroll
        for (int v = 0; v < 8; ++v) {
            const unsigned ua = top * 0x01010101u - ex[2 * v], ub = top * 0x01010101u - ex[2 * v + 1];
            const unsigned ca = __builtin_amdgcn_perm(0u, 0x03020100u, (((ua >> 1) & 0x04040404u) | ua) & 0x07070707u);
            const unsigned cb = __builtin_amdgcn_perm(0u, 0x0c080400u, (((ub >> 1) & 0x04040404u) | ub) & 0x07070707u);
            cg[v] = ((ua | ub) & 0xf0f0f0f0u) != 0u ? 0u : ca | cb;
            const unsigned e = esc_bits(cg[v]);
            const u32x2 sel = lut_of(e, lds0 + CODEC_LUT);
            xs[v] = __builtin_amdgcn_perm(ub, ua, sel.x) | (__builtin_amdgcn_perm(ub, ua, sel.y) << 4);
            at[v] = tot; tot += __builtin_popcount(e);
        }
        const unsigned incl = wave_scan(tot);
        unsigned total = (unsigned)__builtin_amdgcn_readlane(incl, 63);
        if (total > 2048u) {
            total = 0;
#pragma unroll
            for (int v = 0; v < 8; ++v) cg[v] = 0u;
        } else {
            const unsigned n0 = incl - tot + (lds0 + CODEC_STREAM) * 2u;
            asm volatile("" ::: "memory");
#pragma unroll
            for (int v = 0; v < 8; ++v) {
                const unsigned n = n0 + at[v];
                const unsigned long long w = (unsigned long long)xs[v] << ((n * 4u) & 31u);
                LAS unsigned* q = (LAS unsigned*)(size_t)((n >> 1) & ~3u);
                __hip_atomic_fetch_or(q, (unsigned)w, __ATOMIC_RELAXED, __HIP_MEMORY_SCOPE_WAVEFRONT);
                __hip_atomic_fetch_or(q + 1, (unsigned)(w >> 32), __ATOMIC_RELAXED, __HIP_MEMORY_SCOPE_WAVEFRONT);
            }
        }
        asm volatile("" ::: "memory");
        pk[5] = *(const LAS u32x4*)(buf + CODEC_STREAM + lane * 16);
        unsigned least = cg[0];
#pragma unroll
        for (int v = 1; v < 8; ++v) least = cg[v] < least ? cg[v] : least;
#pragma unroll
        for (int c = 0; c < 4; ++c) pk[4][c] = cg[2 * c] | (cg[2 * c + 1] << 4);
        if (cur != first) mine = draw_turn(turn, lane);
        await_turn(turn, mine, SEND_SLOTS);
        if (__builtin_amdgcn_ballot_w64(least == 0u) != 0ull) {
            char* fin = set + item_of(d, cur).fin;
#pragma unroll
            for (int v = 0; v < 8; ++v) if (cg[v] == 0u) store16_sys(fin, v * 1024 + lane * 16, raw[v]);
        }
        const unsigned long long smask = __builtin_amdgcn_ballot_w64((unsigned)lane * 32u < total);
        asm volatile("s_mov_b64 exec, %8\n\tglobal_store_dwordx4 %0, %6, %7 offset:3072 sc0 sc1\n\ts_mov_b64 exec, -1\n\t"
                     "global_store_dwordx4 %0, %1, %7 offset:-2048 sc0 sc1\n\tglobal_store_dwordx4 %0, %2, %7 offset:-1024 sc0 sc1\n\t"
                     "global_store_dwordx4 %0, %3, %7 sc0 sc1\n\tglobal_store_dwordx4 %0, %4, %7 offset:1024 sc0 sc1\n\t"
                     "global_store_dwordx4 %0, %5, %7 offset:2048 sc0 sc1\n\ts_nop 1"
                     :: "v"(lane * 16), "v"(pk[0]), "v"(pk[1]), "v"(pk[2]), "v"(pk[3]), "v"(pk[4]), "v"(pk[5]), "s"(set + OFF_PK + (size_t)u * ITEM_PK + 2048), "s"(smask) : "memory");
        WAIT_FETCH(nxt, tick);
        if (lane == 0) {
            end_turn(turn);
            __hip_atomic_store((GAS unsigned*)(flags + (size_t)u * 64), (call << 8) | top, __ATOMIC_RELAXED, __HIP_MEMORY_SCOPE_SYSTEM);
        }
        DBG_AT(u * 24 + 16);
        cur = next;
    }
}

__device__ __forceinline__ void unpacker(const Params& P, LAS unsigned char* buf, int uw, unsigned call) {
    const int lane = threadIdx.x & 63;
    const int d = __builtin_amdgcn_readfirstlane(uw % 7 + 1), i0 = __builtin_amdgcn_readfirstlane(uw / 7);
    const int step = __builtin_amdgcn_readfirstlane(NUW / 7 + (d <= NUW % 7 ? 1 : 0)), nlink = NCH * litems(d);
    char* ws = P.wsx[0];
    char* set = ws + (call & 1) * PAR_BYTES;
    const char* pkb = set + OFF_PK + (size_t)lbase(d) * ITEM_PK + 2048;
    const char* flags = ws + OFF_FLAG + (call & 1) * FLAG_BYTES + (size_t)lbase(d) * 64;
    const unsigned lds0 = (unsigned)(size_t)buf;
    build_lut<false>(buf, lane);
    u32x4 nx[6]; unsigned nf = 0;
#pragma unroll
    for (int k = 0; k < 6; ++k) nx[k] = (u32x4){0u, 0u, 0u, 0u};
#define LOAD_ITEM(i, mask) asm volatile("s_mov_b32 exec_lo, %11\n\ts_mov_b32 exec_hi, %11\n\t" \
        "global_load_dwordx4 %0, %7, %8 offset:-2048 sc0 sc1\n\tglobal_load_dwordx4 %1, %7, %8 offset:-1024 sc0 sc1\n\tglobal_load_dwordx4 %2, %7, %8 sc0 sc1\n\t" \
        "global_load_dwordx4 %3, %7, %8 offset:1024 sc0 sc1\n\tglobal_load_dwordx4 %4, %7, %8 offset:2048 sc0 sc1\n\tglobal_load_dwordx4 %5, %7, %8 offset:3072 sc0 sc1\n\t" \
        "global_load_dword %6, %9, %10 sc0 sc1\n\ts_mov_b64 exec, -1" \
        : "+v"(nx[0]), "+v"(nx[1]), "+v"(nx[2]), "+v"(nx[3]), "+v"(nx[4]), "+v"(nx[5]), "+v"(nf) \
        : "v"(lane * 16), "s"(pkb + (size_t)(i) * ITEM_PK), "v"(0u), "s"(flags + (size_t)((i) + step < nlink ? (i) + step : (i)) * 64), "s"(mask) : "memory")
#define WAIT_ITEM(n) asm volatile("s_waitcnt vmcnt(" #n ")" : "+v"(nx[0]), "+v"(nx[1]), "+v"(nx[2]), "+v"(nx[3]), "+v"(nx[4]), "+v"(nx[5]), "+v"(nf) :: "memory")
#define SIGNAL() do { if (pending) { if (lane == 0) __hip_atomic_fetch_add((GAS unsigned*)(ws + pending), 1u, __ATOMIC_RELAXED, __HIP_MEMORY_SCOPE_AGENT); \
        DBG_AT((lbase(d) + pend_i) * 24 + 8); pending = 0; } } while (0)
    unsigned f = 0, pending = 0; bool have = false;
    int pend_i = 0; (void)pend_i;
    for (int i = i0; i < nlink; i += step) {
        if (!have) {
            WAIT_V(0); SIGNAL();
            Wait wait;
            bool dead = false;
            while (((f = (unsigned)__builtin_amdgcn_readfirstlane(ld_sys(flags + (size_t)i * 64))) >> 8) < call)
                if (wait.give_up(ws, call)) { dead = true; break; }
            if (dead) break;
            LOAD_ITEM(i, ~0u); WAIT_ITEM(0);
        }
        DBG_AT((lbase(d) + i) * 24);
        u32x4 pk[6];
#pragma unroll
        for (int k = 0; k < 6; ++k) pk[k] = nx[k];
        const unsigned top4 = (f & 255u) * 0x01010101u;
        f = (unsigned)__builtin_amdgcn_readfirstlane(nf);
        const unsigned ahead = below(i + step, nlink) & ~below((int)(f >> 8), (int)call);
        have = ahead != 0u;
        LOAD_ITEM(i + step < nlink ? i + step : i, ahead); SCHED;
        *(LAS u32x4*)(buf + CODEC_STREAM + lane * 16) = pk[5];
        unsigned cg[8], at[8], tot = 0; u32x2 sel[8];
#pragma unroll
        for (int v = 0; v < 8; ++v) {
            cg[v] = (pk[4][v >> 1] >> (4 * (v & 1))) & 0x0f0f0f0fu;
            const unsigned e = esc_bits(cg[v]);
            sel[v] = lut_of(e, lds0 + CODEC_LUT);
            at[v] = tot; tot += __builtin_popcount(e);
        }
        const unsigned n0 = wave_scan(tot) - tot + (lds0 + CODEC_STREAM) * 2u;
        asm volatile("" ::: "memory");
        const Item it = item_of(d, i);
        const unsigned sink = (unsigned)(OFF_SINK - (call & 1) * PAR_BYTES) - it.fin + uw * 1024 + lane * 16;
        unsigned xs[8];
#pragma unroll
        for (int v = 0; v < 8; ++v) {
            const unsigned n = n0 + at[v];
            const LAS unsigned* q = (const LAS unsigned*)(size_t)((n >> 1) & ~3u);
            xs[v] = __builtin_amdgcn_alignbit(q[1], q[0], n * 4u);
        }
#pragma unroll
        for (int v = 0; v < 8; ++v) {
            const unsigned xe = xs[v] & 0x0f0f0f0fu, xo = (xs[v] >> 4) & 0x0f0f0f0fu;
            const unsigned e0 = top4 - ((cg[v] & 0x03030303u) | __builtin_amdgcn_perm(xo, xe, sel[v].x));
            const unsigned e1 = top4 - (((cg[v] >> 2) & 0x03030303u) | __builtin_amdgcn_perm(xo, xe, sel[v].y));
            const u32x4 r = join8(pk[v >> 1][(v & 1) * 2], e0, pk[v >> 1][(v & 1) * 2 + 1], e1);
            store16_sys(set + it.fin, cg[v] != 0u ? v * 1024 + lane * 16 : sink, r);
        }
        WAIT_ITEM(8);
        SIGNAL();
        pending = it.cnt; pend_i = i;
        if (i == i0) STAMP(1);
    }
    WAIT_V(0); SIGNAL();
}

__device__ __forceinline__ int strip_of(int pos, bool remote, int q) {
    if (!remote) return pos < 32 + NSLOT ? pos : -1;
    if (pos < EH) return pos;
    if (pos < EH + 3 * OH) { const int k = (pos - EH) / OH; return 32 + 3 * EH + (k + (k >= q ? 1 : 0)) * OH + (pos - EH) % OH; }
    return (unsigned)(pos - NTE * SR) < 3u * EH ? 32 + pos - NTE * SR : -1;
}
__device__ __forceinline__ int out_row_of(int id, int rank) {
    if (id < 32) return renum(rank) * 1024 + id * 32;
    const int slot = id - 32;
    if (slot < 3 * EH) return renum(rank ^ (2 * (slot / EH + 1))) * 1024 + (slot % EH) * 32;
    const int o = slot - 3 * EH;
    return renum(rank ^ (2 * (o / OH) + 1)) * 1024 + (EH + o % OH) * 32;
}

constexpr int OUT_M = 256, OUT_LDS = 4160;
template <bool PACK> __device__ __forceinline__ u32x4 out_lut(int m) {
    unsigned long long sm = 0x0c0c0c0c0c0c0c0cull, sn = sm; int k = 0;
#pragma unroll
    for (int b = 0; b < 8; ++b) if (m >> b & 1) { const int slot = (k >> 1) | ((k & 1) << 2);
        sm ^= PACK ? (unsigned long long)(0x0c ^ b) << (8 * k) : (unsigned long long)(0x0c ^ k) << (8 * b);
        sn ^= PACK ? (unsigned long long)(0x0c ^ b) << (8 * slot) : (unsigned long long)(0x0c ^ slot) << (8 * b); ++k; }
    return (u32x4){(unsigned)sm, (unsigned)(sm >> 32), (unsigned)sn, (unsigned)(sn >> 32)};
}
struct OutItem { int o, q, w, f0, ida, idb; };
__device__ __forceinline__ OutItem out_item(int n) {
    const bool late = n >= NTE * HPT * 32;
    const int n1 = late ? n - NTE * HPT * 32 : n, per = (late ? NTR - NTE : NTE) * 32, h = late ? n1 / ((NTR - NTE) * 32) : n1 / (NTE * 32), rt = (late ? NTE : 0) + ((n1 - h * per) >> 5);
    OutItem t; t.w = (n1 >> 2) & 7; t.q = n1 & 3; t.f0 = (t.w >> 2) * SR + 2 * h;
    t.ida = n < NOI ? strip_of(rt * SR + (t.f0 >> 1), true, t.q) : -1; t.idb = 2 * h + 1 < SR ? strip_of(rt * SR + ((t.f0 + 1) >> 1), true, t.q) : -1;
    t.o = ((t.q * NTR + rt) * 8 + t.w) * HPT + h;
    return t;
}
__device__ __forceinline__ void expander(const Params& P, LAS unsigned char* buf, int xw, unsigned call) {
    const int lane = threadIdx.x & 63;
    char* ws = P.wsx[0];
    const unsigned lds0 = (unsigned)(size_t)buf;
    for (int m = lane; m < 256; m += 64) *(LAS u32x4*)(buf + OUT_LDS + m * 16) = out_lut<false>(m);
    const OutItem mine = out_item(xw + (lane & 7) * NXW);
    unsigned long long todo = __builtin_amdgcn_ballot_w64(lane < 8 && mine.ida >= 0);
    const char* watch = ws + OFF_OFLAG + (mine.ida >= 0 ? mine.o : 0) * 64;
    while (todo) {
        unsigned long long fl, ready; bool dead = false;
        { Wait wait;
          while ((ready = __builtin_amdgcn_ballot_w64((unsigned)((fl = __hip_atomic_load((const GAS unsigned long long*)watch, __ATOMIC_RELAXED, __HIP_MEMORY_SCOPE_SYSTEM)) >> 32) >= call) & todo) == 0ull) {
              if (wait.give_up(ws, call)) { dead = true; ready = todo; break; }
          } }
        const int k = __builtin_ctzll(ready);
        todo &= ~(1ull << k);
        const unsigned f = (unsigned)__builtin_amdgcn_readlane((unsigned)fl, k);
        const OutItem t = out_item(xw + k * NXW);
#define UNI(x) __builtin_amdgcn_readfirstlane(x)
        const int o = UNI(t.o), q = UNI(t.q), w = UNI(t.w), f0 = UNI(t.f0), ida = UNI(t.ida), idb = UNI(t.idb);
        DBG_AT(DBG_OUT + o * 32 + 8);
        u32x4 r[4];
        unsigned tmo = call;
        if (!dead) {
            u32x4 t[4];
            asm volatile("global_load_dwordx4 %0, %5, %6 sc0 sc1\n\tglobal_load_dwordx4 %1, %5, %6 offset:1024 sc0 sc1\n\tglobal_load_dwordx4 %2, %5, %6 offset:2048 sc0 sc1\n\t"
                         "global_load_dwordx4 %3, %5, %6 offset:3072 sc0 sc1\n\tglobal_load_dword %4, %7, %8 sc1\n\ts_waitcnt vmcnt(0)"
                         : "=&v"(t[0]), "=&v"(t[1]), "=&v"(t[2]), "=&v"(t[3]), "=&v"(tmo)
                         : "v"(lane * 16), "s"(ws + OFF_OPK + (size_t)o * OITEM), "v"(0u), "s"(ws + OFF_TMO) : "memory");
            if (!(f & 0x100000u)) {
#pragma unroll
                for (int k = 0; k < 4; ++k) *(LAS u32x4*)(buf + lane * 16 + k * 1024) = t[k];
                asm volatile("" ::: "memory");
                const unsigned bits = *(const LAS unsigned*)(buf + lane * 4), out_n = OUT_M + (((f & 0xfffu) + 15u) & 0xff0u);
                const unsigned tot = __builtin_popcount(bits), base4 = ((f >> 12) & 255u) * 0x01010101u;
                unsigned n = (wave_scan(tot) - tot) & 2047u;
                asm volatile("" ::: "memory");
#pragma unroll
                for (int k = 0; k < 4; ++k) {
                    const unsigned m = (bits >> (8 * k)) & 255u;
                    const u32x4 sel = *(const LAS u32x4*)(buf + OUT_LDS + m * 16);
                    const LAS unsigned* qm = (const LAS unsigned*)(size_t)(lds0 + OUT_M + (n & ~3u));
                    const LAS unsigned* qn = (const LAS unsigned*)(size_t)(lds0 + out_n + ((n >> 1) & ~3u));
                    const unsigned m0 = __builtin_amdgcn_alignbyte(qm[1], qm[0], n), m1 = __builtin_amdgcn_alignbyte(qm[2], qm[1], n), x = __builtin_amdgcn_alignbit(qn[1], qn[0], n * 4u);
                    const unsigned xe = x & 0x0f0f0f0fu, xo = (x >> 4) & 0x0f0f0f0fu;
                    const unsigned e0 = (base4 - __builtin_amdgcn_perm(xo, xe, sel.z)) & __builtin_amdgcn_perm(~0u, ~0u, sel.x);
                    const unsigned e1 = (base4 - __builtin_amdgcn_perm(xo, xe, sel.w)) & __builtin_amdgcn_perm(~0u, ~0u, sel.y);
                    t[k] = join8(__builtin_amdgcn_perm(m1, m0, sel.x), e0, __builtin_amdgcn_perm(m1, m0, sel.y), e1);
                    n += __builtin_popcount(m);
                }
                asm volatile("" ::: "memory");
            }
#pragma unroll
            for (int k = 0; k < 8; ++k) {
                const int row = (k >> 2) * 16 + (lane & 15), ch = (k & 3) * 2 + (lane >> 5);
                *(LAS u32x2*)(buf + row * 128 + ((ch ^ (row & 7)) * 16) + ((lane >> 4) & 1) * 8) = (u32x2){t[k >> 1][(k & 1) * 2], t[k >> 1][(k & 1) * 2 + 1]};
            }
            asm volatile("" ::: "memory");
#pragma unroll
            for (int v = 0; v < 4; ++v) { const int row = v * 8 + (lane >> 3); r[v] = *(const LAS u32x4*)(buf + row * 128 + (((lane & 7) ^ (row & 7)) * 16)); }
        }
        if (tmo == call) {
#pragma unroll
            for (int v = 0; v < 4; ++v) r[v] = (u32x4){0x7fc07fc0u, 0x7fc07fc0u, 0x7fc07fc0u, 0x7fc07fc0u};
        }
        const int src = P.rank ^ (2 * q + 1);
        const unsigned ooff = (lane >> 3) * OPITCH + q * 512 + ((lane >> 2) & 1) * 256 + (w & 3) * 64 + (lane & 3) * 16;
        { char* O = P.out + (size_t)(out_row_of(ida, src) + (f0 & 1) * 16) * OPITCH; store16_sys(O, ooff, r[0]); store16_sys(O, ooff + 8 * OPITCH, r[1]); }
        if (idb >= 0) { char* O = P.out + (size_t)(out_row_of(idb, src) + ((f0 + 1) & 1) * 16) * OPITCH; store16_sys(O, ooff, r[2]); store16_sys(O, ooff + 8 * OPITCH, r[3]); }
        DBG_AT(DBG_OUT + o * 32 + 16);
    }
}

__device__ __forceinline__ unsigned cvt_pk_bf16(float lo, float hi) { unsigned r; asm("v_cvt_pk_bf16_f32 %0, %1, %2" : "=v"(r) : "v"(lo), "v"(hi)); return r; }
template <int OFF> __device__ __forceinline__ s16x4 tr_read(unsigned addr) { s16x4 r; asm volatile("ds_read_b64_tr_b16 %0, %1 offset:%2" : "=v"(r) : "v"(addr), "i"(OFF) : "memory"); return r; }
template <int OFF> __device__ __forceinline__ bf16x8 row_read(unsigned addr) { bf16x8 r; asm volatile("ds_read_b128 %0, %1 offset:%2" : "=v"(r) : "v"(addr), "i"(OFF) : "memory"); return r; }
__device__ __forceinline__ void glds(unsigned ldsaddr, unsigned voff, const char* g) {
    asm volatile("s_mov_b32 m0, %0\n\ts_nop 0\n\tglobal_load_lds_dwordx4 %1, %2" GLF :: "s"(ldsaddr), "v"(voff), "s"(g) : "memory");
}
__device__ __forceinline__ void glds_masked(unsigned ldsaddr, unsigned voff, const char* g, unsigned mask) {
    asm volatile("s_mov_b32 m0, %0\n\ts_mov_b32 exec_lo, %3\n\ts_mov_b32 exec_hi, %3\n\tglobal_load_lds_dwordx4 %1, %2" GLF "\n\ts_mov_b64 exec, -1" :: "s"(ldsaddr), "v"(voff), "s"(g), "s"(mask) : "memory");
}

template <int S> struct Frag { bf16x8 a[S], b[4]; };
template <int S> __device__ __forceinline__ void wait_step(Frag<S>& F, unsigned& pre) {
    asm volatile("s_waitcnt vmcnt(6) lgkmcnt(0)" : "+v"(F.a[0]), "+v"(F.a[1]), "+v"(F.a[2]), "+v"(F.a[3]), "+v"(F.a[S - 1]), "+v"(F.b[0]), "+v"(F.b[1]), "+v"(F.b[2]), "+v"(F.b[3]), "+v"(pre) :: "memory");
    if constexpr (S > 5) asm volatile("" : "+v"(F.a[4]));
    if constexpr (S > 6) asm volatile("" : "+v"(F.a[5]));
    if constexpr (S > 7) asm volatile("" : "+v"(F.a[6]));
}
template <int S, bool remote> __device__ __forceinline__ void gemm_tile(const Params& P, LAS unsigned char* lds, int rt, int ct, unsigned call) {
    constexpr int SLOT_A = S * 2048, SLOT = SLOT_A + 16384;
    static_assert(S >= 5 && S <= 8 && 4 * SLOT <= LDS_STAGE && S * 16384 <= LDS_STAGE, "tile");
    int wid = __builtin_amdgcn_readfirstlane(threadIdx.x >> 6), lane = threadIdx.x & 63, wr = wid >> 2, wc = wid & 3, fr = lane & 15, fq = lane >> 4;
    char* ws = P.wsx[0];
    const char* set = ws + (call & 1) * PAR_BYTES;
    const char* const own_x = P.x;
    const unsigned ldst = (unsigned)(size_t)lds + wid * 1024;
    constexpr int BP = remote ? 512 : BPITCH;
    auto strip_at = [&](int k) { const int id = k < S ? strip_of(rt * S + k, remote, ct) : -1; return id < 0 ? strip_of(rt * S, remote, ct) : id; };
    const int id0 = strip_at(wid >> 1), id1 = strip_at(4 + (wid >> 1));
    const char* const ga0 = id0 < 32 ? own_x + id0 * STRIP_BYTES : set + OFF_XA + (id0 - 32) * STRIP_BYTES;
    const char* const ga1 = id1 < 32 ? own_x + id1 * STRIP_BYTES : set + OFF_XA + (id1 - 32) * STRIP_BYTES;
    const unsigned cs0 = id0 < 32 ? 256 : 8192, cs1 = id1 < 32 ? 256 : 8192;
    const char* const gb = remote ? set + OFF_W2 + ct * (W2_BYTES / 4) : P.w + ct * 512;
    unsigned voffA0, voffA1, voffB;
    { const int k0 = lane >> 4, ch = lane & 15; voffB = (unsigned)((wid * 4 + k0) * BP + (((ch >> 1) ^ (k0 | ((wid & 2) << 1))) * 32) + (ch & 1) * 16);
      const int sb = lane * 16, swz = sb ^ (((sb >> 9) & 1) << 5), row = (wid & 1) * 16 + (swz >> 6);
      voffA0 = (unsigned)(row * (id0 < 32 ? APITCH : 256) + (swz & 63)); voffA1 = (unsigned)(row * (id1 < 32 ? APITCH : 256) + (swz & 63)); }
    const unsigned amask = below(wid, 2 * S - 8);
    unsigned aoff[2], boff[2][2];
    { const unsigned lbase = (unsigned)(size_t)lds; const int ob = fr * 64 + fq * 16; aoff[0] = lbase + wr * S * 1024 + (ob ^ (((ob >> 9) & 1) << 5)); aoff[1] = aoff[0] + 2 * SLOT;
      const int q = (lane & 15) >> 2, p = lane & 3, f = q | ((fq & 1) << 2);
      for (int n = 0; n < 2; ++n) { boff[0][n] = lbase + SLOT_A + (8 * fq + q) * 256 + p * 8 + (((wc * 2 + n) ^ f) * 32); boff[1][n] = boff[0][n] + 2 * SLOT; } }
    unsigned coff, cstep, need;
    { const int id = strip_of(rt * S + (lane < S ? lane : 0), remote, ct);
      const bool watch = wid == 0 && (lane < S ? id >= 32 : (lane == 8 && remote));
      coff = watch ? (unsigned)(OFF_CNT + (lane < S ? id - 32 : NSLOT + ct) * NCH * 64) : (unsigned)OFF_TMO;
      cstep = watch ? 64u : 0u;
      need = watch ? call * (lane < S ? 1u : 8u) : 0u; }
    const unsigned w0 = below(wid, 1);
    unsigned pre = 0;
#define SLOW_WAIT() do { Wait wait; \
        while (__builtin_amdgcn_ballot_w64(ld_agent(ws + coff) < need) != 0ull) if (wait.give_up(ws, call)) break; \
        asm volatile("" ::: "memory"); } while (0)
#define PRE_LOAD(mask) do { coff += cstep; \
        asm volatile("s_mov_b32 exec_lo, %3\n\ts_mov_b32 exec_hi, 0\n\tglobal_load_dword %0, %1, %2 sc1\n\ts_mov_b64 exec, -1" : "+v"(pre) : "v"(coff), "s"(ws), "s"((mask) & w0) : "memory"); } while (0)
#define STAGE_B(h, sl, ch) glds(ldst + (sl) * SLOT + SLOT_A + (h) * 8192, voffB, gb + (h) * 256 + (size_t)((ch) * 4 + (sl)) * (32 * BP))
#define STAGE_A0(sl, ch) glds(ldst + (sl) * SLOT, voffA0, ga0 + (ch) * cs0 + (sl) * 64)
#define STAGE_A1(sl, ch) glds_masked(ldst + (sl) * SLOT + 8192, voffA1, ga1 + (ch) * cs1 + (sl) * 64, amask)
#define STAGE_ALL(sl, ch) do { STAGE_B(0, sl, ch); STAGE_B(1, sl, ch); STAGE_A0(sl, ch); STAGE_A1(sl, ch); } while (0)
#define RDA(F, m, sl) F.a[m] = row_read<((sl) & 1) * SLOT + (m) * 1024>(aoff[(sl) >> 1])
#define RDB(F, j, sl) do { const s16x4 _lo = tr_read<((sl) & 1) * SLOT + ((j) >> 1) * 8192>(boff[(sl) >> 1][(j) & 1]), _hi = tr_read<((sl) & 1) * SLOT + ((j) >> 1) * 8192 + 1024>(boff[(sl) >> 1][(j) & 1]); \
        F.b[j] = __builtin_shufflevector(_lo, _hi, 0, 1, 2, 3, 4, 5, 6, 7); } while (0)
#define MM(F, i) acc[(i) >> 2][(i) & 3] = __builtin_amdgcn_mfma_f32_16x16x32_bf16(F.b[(i) & 3], F.a[(i) >> 2], acc[(i) >> 2][(i) & 3], 0, 0, 0)
#define PAIR(C, i) do { MM(C, i); MM(C, (i) + 1); SCHED; } while (0)
#define STEP(C, N, sl, st, GATE) do { constexpr int nx = ((sl) + 1) & 3; \
        RDB(N, 0, nx); PAIR(C, 0); RDB(N, 1, nx); PAIR(C, 2); RDA(N, 0, nx); RDA(N, 1, nx); PAIR(C, 4); RDB(N, 2, nx); PAIR(C, 6); RDB(N, 3, nx); \
        STAGE_B(0, sl, st); PAIR(C, 8); RDA(N, 2, nx); RDA(N, 3, nx); STAGE_B(1, sl, st); PAIR(C, 10); \
        RDA(N, 4, nx); if constexpr (S > 5) RDA(N, 5, nx); STAGE_A0(sl, st); PAIR(C, 12); \
        if constexpr (S > 6) RDA(N, 6, nx); if constexpr (S > 7) RDA(N, 7, nx); STAGE_A1(sl, st); \
        _Pragma("unroll") for (int i = 14; i < 4 * S - 4; i += 2) PAIR(C, i); \
        wait_step(N, pre); GATE; BAR; SCHED; PAIR(C, 4 * S - 4); PAIR(C, 4 * S - 2); } while (0)
    f32x4 acc[S][4];
#pragma unroll
    for (int m = 0; m < S; ++m)
#pragma unroll
        for (int j = 0; j < 4; ++j) acc[m][j] = (f32x4){0.f, 0.f, 0.f, 0.f};
    Frag<S> F0, F1;
    constexpr int OUT_LUT = LDS_STAGE - 4096;
    static_assert(!remote || (4 * SLOT <= OUT_LUT && 8 * OUT_LDS <= OUT_LUT), "pack tables");
    if (remote && threadIdx.x >= 256) *(LAS u32x4*)(lds + OUT_LUT + (threadIdx.x - 256) * 16) = out_lut<true>(threadIdx.x - 256);
    if (wid == 0) SLOW_WAIT();
    BAR;
    STAMP(1); CSTAMP(6);
    PRE_LOAD(0x3ffu);
    STAGE_ALL(0, 0u); STAGE_ALL(1, 0u); STAGE_ALL(2, 0u); STAGE_ALL(3, 0u);
    WAIT_V(6); BAR;
    RDB(F0, 0, 0); RDB(F0, 1, 0); RDB(F0, 2, 0); RDB(F0, 3, 0);
    RDA(F0, 0, 0); RDA(F0, 1, 0); RDA(F0, 2, 0); RDA(F0, 3, 0); RDA(F0, 4, 0);
    if constexpr (S > 5) RDA(F0, 5, 0);
    if constexpr (S > 6) RDA(F0, 6, 0);
    if constexpr (S > 7) RDA(F0, 7, 0);
    wait_step(F0, pre);
    if (__builtin_amdgcn_ballot_w64(pre < need) != 0ull) SLOW_WAIT();
    BAR;
    for (int c = 0; c < NCH; ++c) {
        const unsigned st = c < NCH - 1 ? c + 1 : c;
        STEP(F0, F1, 0, st, (void)0);
        STEP(F1, F0, 1, st, (void)0);
        PRE_LOAD(c < NCH - 2 ? 0x3ffu : c < NCH - 1 ? 0x200u : 0u);
        STEP(F0, F1, 2, st, (void)0);
        STEP(F1, F0, 3, st, { TS_NOW(ts0, c); if (c < NCH - 2 && __builtin_amdgcn_ballot_w64(pre < need) != 0ull) SLOW_WAIT(); TS_NOW(ts1, c); });
    }
    volatile LAS unsigned* turn = (volatile LAS unsigned*)(lds + LDS_STAGE + 4);
    if (threadIdx.x == 9) { *(LAS unsigned*)(lds + LDS_STAGE) = pre; turn[0] = 0u; turn[1] = 0u; }
    asm volatile("s_waitcnt vmcnt(0) lgkmcnt(0)" ::: "memory");
    BAR;
    asm volatile("" ::: "memory");
    const bool bad = *(const LAS unsigned*)(lds + LDS_STAGE) == call;
    STAMP(2); CSTAMP(7);
    int tz = threadIdx.x; asm volatile("" : "+v"(tz));
    lane = tz & 63; fr = lane & 15; fq = lane >> 4;
    auto relu_pack = [&](int m, int j) {
        f32x4 v = acc[m][j];
#pragma unroll
        for (int e = 0; e < 4; ++e) v[e] = v[e] < 0.f ? 0.f : v[e];
        u32x2 w; w.x = cvt_pk_bf16(v[0], v[1]); w.y = cvt_pk_bf16(v[2], v[3]);
        if (bad) { w.x = 0x7fc07fc0u; w.y = 0x7fc07fc0u; }
        return w;
    };
    if constexpr (!remote) {
#pragma unroll
        for (int m = 0; m < S; ++m) {
            const int row = wr * S * 16 + m * 16 + fr;
#pragma unroll
            for (int j = 0; j < 4; ++j) {
                const int ch = (j >> 1) * 16 + wc * 4 + (j & 1) * 2 + (fq >> 1);
                *(LAS u32x2*)(lds + row * 512 + ((ch ^ (fr & 7)) * 16) + (fq & 1) * 8) = relu_pack(m, j);
            }
        }
        WAIT_L(0); BAR; asm volatile("" ::: "memory");
        u32x4 ov[S][2];
#pragma unroll
        for (int k = 0; k < S; ++k)
#pragma unroll
            for (int i = 0; i < 2; ++i) { const int row = k * 32 + wid * 4 + i * 2 + (lane >> 5); ov[k][i] = *(const LAS u32x4*)(lds + row * 512 + (((lane & 31) ^ (row & 7)) * 16)); }
        const unsigned ooff = (wid * 4 + (lane >> 5)) * OPITCH + ct * 512 + (lane & 31) * 16;
#pragma unroll
        for (int k = 0; k < S; ++k) {
            const int id = strip_of(rt * S + k, false, 0);
            if (id >= 0) { char* O = P.out + (size_t)out_row_of(id, P.rank) * OPITCH; store16_sys(O, ooff, ov[k][0]); store16_sys(O, ooff + 2 * OPITCH, ov[k][1]); }
        }
    } else {
        char* const peer = P.wsx[(2 * ct + 1) & 7];
        LAS unsigned char* buf = lds + wid * OUT_LDS;
        const unsigned lds0 = (unsigned)(size_t)buf;
        const int o0 = ((ct * NTR + rt) * 8 + wid) * HPT;
        int sent = -1; unsigned sent_meta = 0;
#define PUBLISH_OUT() do { WAIT_V(0); if (sent >= 0) { if (lane == 0) { end_turn(turn); \
            __hip_atomic_store((GAS unsigned long long*)(peer + OFF_OFLAG + sent * 64), ((unsigned long long)call << 32) | sent_meta, __ATOMIC_RELAXED, __HIP_MEMORY_SCOPE_SYSTEM); } \
            DBG_AT(DBG_OUT + sent * 32 + 24); sent = -1; } } while (0)
#pragma unroll
        for (int h = 0; h < HPT; ++h) {
            const int f0 = wr * S + 2 * h;
            if (strip_of(rt * S + (f0 >> 1), true, ct) < 0) break;
            const bool second = 2 * h + 1 < S && strip_of(rt * S + ((f0 + 1) >> 1), true, ct) >= 0;
            u32x4 t[4]; u16x2 mx = {0, 0};
#pragma unroll
            for (int k = 0; k < 8; ++k) {
                u32x2 w = relu_pack(2 * h + (k >> 2) < S ? 2 * h + (k >> 2) : 0, k & 3);
                if (k >= 4 && !second) w = (u32x2){0u, 0u};
                t[k >> 1][(k & 1) * 2] = w.x; t[k >> 1][(k & 1) * 2 + 1] = w.y;
                mx = __builtin_elementwise_max(mx, __builtin_elementwise_max(__builtin_bit_cast(u16x2, w.x & 0x7fff7fffu), __builtin_bit_cast(u16x2, w.y & 0x7fff7fffu)));
            }
#pragma unroll
            for (int k = 0; k < 3; ++k) *(LAS u32x4*)(buf + OUT_M + lane * 16 + k * 1024) = (u32x4){0u, 0u, 0u, 0u};
            const unsigned emax = (unsigned)__builtin_amdgcn_readlane(wave_max(mx.x > mx.y ? mx.x : mx.y), 63) >> 7;
            unsigned bits = 0, wide = 0, tot = 0, at[4], cm0[4], cm1[4], cn[4];
#pragma unroll
            for (int k = 0; k < 4; ++k) {
                unsigned z[4], m0, e0, m1, e1;
#pragma unroll
                for (int c = 0; c < 4; ++c) z[c] = nonzero2(t[k][c]);
                const unsigned m = __builtin_amdgcn_udot4(__builtin_amdgcn_perm(z[3], z[2], 0x06040200u), 0x80402010u, __builtin_amdgcn_udot4(__builtin_amdgcn_perm(z[1], z[0], 0x06040200u), 0x08040201u, 0u, false), false);
                const u32x4 sel = *(const LAS u32x4*)(lds + OUT_LUT + m * 16);
                split4(t[k].x, t[k].y, m0, e0); split4(t[k].z, t[k].w, m1, e1);
                e0 = emax * 0x01010101u - e0; e1 = emax * 0x01010101u - e1;
                cm0[k] = __builtin_amdgcn_perm(m1, m0, sel.x); cm1[k] = __builtin_amdgcn_perm(m1, m0, sel.y);
                const unsigned de = __builtin_amdgcn_perm(e1, e0, sel.z), dd = __builtin_amdgcn_perm(e1, e0, sel.w);
                wide |= de | dd; cn[k] = de | (dd << 4);
                bits |= m << (8 * k); at[k] = tot; tot += __builtin_popcount(m);
            }
            const unsigned incl = wave_scan(tot), count = (unsigned)__builtin_amdgcn_readlane(incl, 63);
            const bool rawout = __builtin_amdgcn_ballot_w64((wide & 0xf0f0f0f0u) != 0u) != 0ull;
            char* dst = peer + OFF_OPK + (size_t)(o0 + h) * OITEM;
            const unsigned out_n = OUT_M + ((count + 15u) & ~15u), total = rawout ? 4096u : out_n + ((count + 1u) >> 1);
            if (!rawout) {
                *(LAS unsigned*)(buf + lane * 4) = bits;
                asm volatile("" ::: "memory");
#pragma unroll
                for (int k = 0; k < 4; ++k) {
                    const unsigned n = incl - tot + at[k];
                    const unsigned long long a = (unsigned long long)cm0[k] << ((n & 3u) * 8u), b = (unsigned long long)cm1[k] << ((n & 3u) * 8u), x = (unsigned long long)cn[k] << ((n * 4u) & 31u);
                    LAS unsigned* qm = (LAS unsigned*)(size_t)(lds0 + OUT_M + (n & ~3u));
                    LAS unsigned* qn = (LAS unsigned*)(size_t)(lds0 + out_n + ((n >> 1) & ~3u));
#define LDS_OR(p, v) __hip_atomic_fetch_or(p, v, __ATOMIC_RELAXED, __HIP_MEMORY_SCOPE_WAVEFRONT)
                    LDS_OR(qm, (unsigned)a); LDS_OR(qm + 1, (unsigned)(a >> 32) | (unsigned)b); LDS_OR(qm + 2, (unsigned)(b >> 32));
                    LDS_OR(qn, (unsigned)x); LDS_OR(qn + 1, (unsigned)(x >> 32));
                }
                asm volatile("" ::: "memory");
#pragma unroll
                for (int k = 0; k < 4; ++k) t[k] = *(const LAS u32x4*)(buf + lane * 16 + k * 1024);
                asm volatile("" : "+v"(t[0]), "+v"(t[1]), "+v"(t[2]), "+v"(t[3]) :: "memory");
            }
            PUBLISH_OUT();
            await_turn(turn, draw_turn(turn, lane), OUT_SLOTS);
#pragma unroll
            for (int k = 0; k < 4; ++k) if (lane * 16u + k * 1024u < total) store16_sys(dst, lane * 16 + k * 1024, t[k]);
            sent = o0 + h; sent_meta = (rawout ? 0x100000u : 0u) | (emax << 12) | count;
        }
        PUBLISH_OUT();
    }
    STAMP(3);
}

__global__ __launch_bounds__(NTHR, 2) void ag_gemm_kernel(const Params P) {
    extern __shared__ __attribute__((aligned(16))) unsigned char shm[];
    LAS unsigned char* const lds = (LAS unsigned char*)shm;
    const int wid = __builtin_amdgcn_readfirstlane(threadIdx.x >> 6);
    char* const tick = P.wsx[0] + OFF_TICK;
    const char* const src_x = P.x; const char* const src_w = P.w;
    volatile LAS unsigned* const word = (volatile LAS unsigned*)(lds + LDS_STAGE);
    const int asked = (int)(__builtin_amdgcn_s_getreg((3 << 11) | 20) & 7u) * 32 + (int)(blockIdx.x >> 3);
    auto sends = [](int r) { return (r >> 5) != 3 && (r & 31) < SEND_WGS; };
    u32x4 nxt[8]; unsigned t = 0, c;
    asm volatile("s_mov_b32 exec_lo, %4\n\ts_mov_b32 exec_hi, 0\n\tglobal_atomic_add %0, %1, %2, %3 sc0\n\ts_mov_b64 exec, -1" : "+v"(t) : "v"(blockIdx.x * 64u), "v"(1u), "s"(tick), "s"(below(wid, 1) & 1u) : "memory");
    fetch(src_x, src_w, nxt, c, link_of(asked >> 5), wid * SEND_WGS + (asked & 31), below(asked & 31, SEND_WGS) & ~(below(asked >> 5, 4) & below(2, asked >> 5)), tick + (256 + asked) * 64, below(wid, 1) & 1u);
    STAMP(0);
    if (sends(asked)) build_lut<true>(lds + wid * WAVE_LDS, threadIdx.x & 63);
    asm volatile("" : "+v"(t));
    WAIT_FETCH(nxt, c);
    asm volatile("" : "+v"(t));
    if (threadIdx.x == 0) {
        int r = asked;
        if (c != t) {
            __hip_atomic_fetch_add((GAS unsigned*)(tick + (256 + r) * 64), ~0u, __ATOMIC_RELAXED, __HIP_MEMORY_SCOPE_AGENT);
            for (int k = 1; k < 256 * 64; ++k) {
                unsigned expect = t; r = (asked + k) & 255;
                if (__hip_atomic_compare_exchange_strong((GAS unsigned*)(tick + (256 + r) * 64), &expect, t + 1u, __ATOMIC_RELAXED, __ATOMIC_RELAXED, __HIP_MEMORY_SCOPE_AGENT)) break;
            }
        }
        word[0] = t + 1u; word[1] = (unsigned)r; word[2] = 8u; word[3] = 0u;
    }
    WAIT_L(0); BAR; asm volatile("" ::: "memory");
    struct { unsigned call; int role; } claim = {(unsigned)__builtin_amdgcn_readfirstlane(word[0]), __builtin_amdgcn_readfirstlane((int)word[1]) & 255};
    const int xcd = claim.role >> 5, slot = claim.role & 31;
#define PICK(f) __builtin_amdgcn_readfirstlane(xcd == 0 ? BeforeAt<0>::f : xcd == 1 ? BeforeAt<1>::f : xcd == 2 ? BeforeAt<2>::f : xcd == 3 ? BeforeAt<3>::f : \
                                               xcd == 4 ? BeforeAt<4>::f : xcd == 5 ? BeforeAt<5>::f : xcd == 6 ? BeforeAt<6>::f : BeforeAt<7>::f)
    const int ns = nsend(xcd), nc = ncomm(xcd), ub = PICK(unp), sb = PICK(snd);
    const int xw = __builtin_amdgcn_readfirstlane(slot < ns ? NUW + sb + wid * ns + slot : ub + (slot - ns) * 8 + wid);
    if (slot < ns) {
        if (claim.role != asked) { build_lut<true>(lds + wid * WAVE_LDS, threadIdx.x & 63); fetch(src_x, src_w, nxt, c, link_of(xcd), wid * SEND_WGS + slot, ~0u, tick, 0u); WAIT_FETCH(nxt, c); }
        sender(P, lds + wid * WAVE_LDS, word + 2, link_of(xcd), wid * SEND_WGS + slot, claim.call, nxt);
    }
    if (slot < nc) {
        if (slot >= ns) unpacker(P, lds + wid * WAVE_LDS, xw, claim.call);
        STAMP(2);
        expander(P, lds + wid * WAVE_LDS, xw, claim.call);
        STAMP(3);
    } else {
        const int g = slot - nc, rt = (g >> 2) * 4 + 3 - (xcd & 3);
        if (xcd < 4) gemm_tile<SL, false>(P, lds, rt, g & 3, claim.call);
        else gemm_tile<SR, true>(P, lds, rt, g & 3, claim.call);
    }
}
}

extern "C" void kernel_launch(int rank, int nranks, ncclComm_t comm, hipStream_t stream,
                              void* const* d_in, const int* in_sizes, int n_in,
                              void* d_out, int out_size, const int* in_dtypes, int out_dtype,
                              void* const* d_out_all, void* const* d_ws_all, size_t ws_bytes) {
    (void)comm; (void)in_dtypes; (void)out_dtype; (void)d_out_all;
    if (nranks != NR || (unsigned)rank >= (unsigned)NR || n_in != 2 || in_sizes[0] != 1024 * KDIM || in_sizes[1] != KDIM * 1024 || out_size != KDIM * 1024 || ws_bytes < WS_NEED) {
        fprintf(stderr, "kernel_launch: unexpected problem geometry\n");
        return;
    }
    static bool attr_set = false;
    if (!attr_set) { HIP_CHECK(hipFuncSetAttribute((const void*)ag_gemm_kernel, hipFuncAttributeMaxDynamicSharedMemorySize, LDS_BYTES)); attr_set = true; }
    Params P;
    P.x = (const char*)d_in[0]; P.w = (const char*)d_in[1]; P.out = (char*)d_out;
    for (int d = 0; d < NR; ++d) P.wsx[d] = (char*)d_ws_all[renum(renum(rank) ^ d)];
    P.rank = renum(rank); P.pad = 0;
    hipLaunchKernelGGL(ag_gemm_kernel, dim3(GRID), dim3(NTHR), LDS_BYTES, stream, P);
    HIP_CHECK(hipGetLastError());
}
```

```cpp
#include <hip/hip_runtime.h>
#include <rccl/rccl.h>
#include <cstdio>

#define HIP_CHECK(expr) do { hipError_t _e = (expr); if (_e != hipSuccess) { \
    fprintf(stderr, "HIP error %s at %s:%d\n", hipGetErrorString(_e), __FILE__, __LINE__); \
    return; } } while (0)

namespace {
#define LAS __attribute__((address_space(3)))
#define GAS __attribute__((address_space(1)))
typedef short bf16x8 __attribute__((ext_vector_type(8)));
typedef short s16x4 __attribute__((ext_vector_type(4)));
typedef float f32x4 __attribute__((ext_vector_type(4)));
typedef unsigned u32x4 __attribute__((ext_vector_type(4)));
typedef unsigned u32x2 __attribute__((ext_vector_type(2)));
typedef unsigned short u16x2 __attribute__((ext_vector_type(2)));

constexpr int NR = 8;
constexpr int KDIM = 8192;
constexpr int APITCH = KDIM * 2, BPITCH = 2048, OPITCH = 2048;
constexpr int E_STRIPS = 21;
constexpr int EH = E_STRIPS, OH = 32 - EH;
constexpr int NSLOT = 3 * EH + 4 * OH;
constexpr int SO = 7;
constexpr int NTO = 4;
constexpr int SL = 7;
constexpr int SE = 5;
constexpr int SR = 5;
constexpr int NTL = NTO + (32 + NSLOT - NTO * SO + SL - 1) / SL, NTE = (EH + 3 * OH + SE - 1) / SE, NTR = NTE + (3 * EH + SR - 1) / SR, HPT = 3;
static_assert(NTO * SO <= 32 && SE <= 6 && SR <= 6, "tile lists");
__host__ __device__ constexpr int first_pos(int rt, bool remote) { return remote ? (rt < NTE ? rt * SE : NTE * SE + (rt - NTE) * SR) : (rt < NTO ? rt * SO : NTO * SO + (rt - NTO) * SL); }
constexpr int NCH = 64;
__host__ __device__ constexpr int litems(int d) { return (d & 1) ? OH + 8 : EH; }
__host__ __device__ constexpr int lbase(int d) { return NCH * ((d >> 1) * litems(1) + ((d - 1) >> 1) * litems(0)); }
constexpr int NU = lbase(8);
constexpr int ITEM_PK = 6144;
constexpr size_t STRIP_BYTES = (size_t)32 * APITCH;
constexpr size_t XA_BYTES = NSLOT * STRIP_BYTES, W2_BYTES = (size_t)KDIM * BPITCH, PK_BYTES = (size_t)NU * ITEM_PK;
constexpr size_t OFF_XA = 0, OFF_W2 = XA_BYTES, OFF_PK = XA_BYTES + W2_BYTES, PAR_BYTES = OFF_PK + PK_BYTES;
constexpr size_t OFF_FLAG = 2 * PAR_BYTES, FLAG_BYTES = (size_t)NU * 64;
constexpr size_t OFF_CNT = OFF_FLAG + 2 * FLAG_BYTES, CNT_BYTES = (size_t)(NSLOT + 4) * NCH * 64;
constexpr size_t OFF_TICK = OFF_CNT + CNT_BYTES, TICK_BYTES = 512 * 64;
constexpr int OITEM = 4096, NOI = 4 * NTR * 8 * HPT;
constexpr size_t OFF_OPK = OFF_TICK + TICK_BYTES, OFF_OFLAG = OFF_OPK + (size_t)NOI * OITEM;
constexpr size_t OFF_TMO = OFF_OFLAG + (size_t)NOI * 64, OFF_QUEUE = OFF_TMO + 128;
constexpr size_t WS_NEED = OFF_QUEUE + 8 * 128;
static_assert(WS_NEED <= ((size_t)257 << 20), "workspace");
constexpr int GRID = 256, NTHR = 512;
constexpr int SEND_WGS = 6;
constexpr int OUT_SLOTS = 6;
constexpr int SEND_SLOTS = 3;
constexpr int TURN_SLEEP = 8;
constexpr int POLL_SLEEP = 2;
__host__ __device__ constexpr int ncomm(int x) { return 32 - 4 * (((x < 4 ? NTL : NTR) + x % 4) / 4); }
__host__ __device__ constexpr int nsend(int x) { return x == 3 ? 0 : SEND_WGS; }
__host__ __device__ constexpr int link_of(int x) { return x < 3 ? 2 * x + 2 : x < 4 ? 1 : 2 * x - 7; }
struct Before { int unp[9], snd[9]; };
constexpr Before before() { Before b = {}; for (int x = 0; x < 8; ++x) { b.unp[x + 1] = b.unp[x] + (ncomm(x) - nsend(x)) * 8; b.snd[x + 1] = b.snd[x] + nsend(x) * 8; } return b; }
template <int X> struct BeforeAt { static constexpr int unp = before().unp[X], snd = before().snd[X]; };
constexpr int NUW = BeforeAt<8>::unp, NXW = NUW + BeforeAt<8>::snd;
constexpr unsigned long long DEADLINE = 50000000ull;
constexpr int LDS_STAGE = 135168, LDS_BYTES = LDS_STAGE + 16;
static_assert(NOI <= 8 * NXW && ncomm(2) >= nsend(2) && ncomm(3) >= 0 && ncomm(7) >= nsend(7), "roles");

__host__ __device__ constexpr int renum(int r) { return r == 3 ? 4 : r == 4 ? 3 : r; }
struct Params {
    const char* x; const char* w;
    char* out;
    char* wsx[NR];
    int rank; int pad;
};

#define WAIT_V(n) asm volatile("s_waitcnt vmcnt(" #n ")" ::: "memory")
#define WAIT_L(n) asm volatile("s_waitcnt lgkmcnt(" #n ")" ::: "memory")
#define BAR __builtin_amdgcn_s_barrier()
#define SCHED __builtin_amdgcn_sched_barrier(0)

__device__ __forceinline__ unsigned ld_agent(const char* p) { return __hip_atomic_load((const GAS unsigned*)p, __ATOMIC_RELAXED, __HIP_MEMORY_SCOPE_AGENT); }
__device__ __forceinline__ unsigned ld_sys(const char* p) { return __hip_atomic_load((const GAS unsigned*)p, __ATOMIC_RELAXED, __HIP_MEMORY_SCOPE_SYSTEM); }
struct Wait {
    unsigned long long t0; unsigned polls;
    __device__ __forceinline__ Wait() : t0(__builtin_amdgcn_s_memrealtime()), polls(0) {}
    __device__ __forceinline__ bool give_up(char* ws, unsigned call) {
        if ((++polls & 15u) == 0u && ((unsigned)__builtin_amdgcn_readfirstlane(ld_agent(ws + OFF_TMO)) == call || __builtin_amdgcn_s_memrealtime() - t0 > DEADLINE)) {
            __hip_atomic_store((GAS unsigned*)(ws + OFF_TMO), call, __ATOMIC_RELAXED, __HIP_MEMORY_SCOPE_AGENT);
            return true;
        }
        __builtin_amdgcn_s_sleep(POLL_SLEEP);
        return false;
    }
};
__device__ __forceinline__ void store16_sys(char* base, unsigned voff, u32x4 v) {
    asm volatile("global_store_dwordx4 %0, %1, %2 sc0 sc1\n\ts_nop 1" :: "v"(voff), "v"(v), "s"(base) : "memory");
}

template <int CTRL, int ROWS> __device__ __forceinline__ unsigned max_dpp(unsigned x) { const unsigned y = (unsigned)__builtin_amdgcn_update_dpp(0u, x, CTRL, ROWS, 0xf, false); return x > y ? x : y; }
__device__ __forceinline__ unsigned wave_max(unsigned x) {
    return max_dpp<0x143, 0xc>(max_dpp<0x142, 0xa>(max_dpp<0x118, 0xf>(max_dpp<0x114, 0xf>(max_dpp<0x112, 0xf>(max_dpp<0x111, 0xf>(x))))));
}
__device__ __forceinline__ unsigned nonzero2(unsigned w) { unsigned r; asm("v_pk_min_u16 %0, %1, %2" : "=v"(r) : "v"(w), "s"(0x00010001u)); return r; }
__device__ __forceinline__ unsigned below(int a, int b) { unsigned r; asm("s_ashr_i32 %0, %1, 31" : "=s"(r) : "s"(a - b) : "scc"); return r; }
__device__ __forceinline__ unsigned wave_scan(unsigned x) {
    x += __builtin_amdgcn_update_dpp(0u, x, 0x111, 0xf, 0xf, false); x += __builtin_amdgcn_update_dpp(0u, x, 0x112, 0xf, 0xf, false);
    x += __builtin_amdgcn_update_dpp(0u, x, 0x114, 0xf, 0xf, false); x += __builtin_amdgcn_update_dpp(0u, x, 0x118, 0xf, 0xf, false);
    x += __builtin_amdgcn_update_dpp(0u, x, 0x142, 0xa, 0xf, false); x += __builtin_amdgcn_update_dpp(0u, x, 0x143, 0xc, 0xf, false);
    return x;
}

__device__ __forceinline__ unsigned draw_turn(volatile LAS unsigned* turn, int lane) {
    unsigned t = 0;
    if (lane == 0) t = __hip_atomic_fetch_add((LAS unsigned*)turn, 1u, __ATOMIC_RELAXED, __HIP_MEMORY_SCOPE_WORKGROUP);
    return (unsigned)__builtin_amdgcn_readfirstlane(t);
}
__device__ __forceinline__ void await_turn(volatile LAS unsigned* turn, unsigned mine, unsigned slots) {
    while (mine >= (unsigned)__builtin_amdgcn_readfirstlane(turn[1]) + slots) __builtin_amdgcn_s_sleep(TURN_SLEEP);
}
__device__ __forceinline__ void end_turn(volatile LAS unsigned* turn) { __hip_atomic_fetch_add((LAS unsigned*)turn + 1, 1u, __ATOMIC_RELAXED, __HIP_MEMORY_SCOPE_WORKGROUP); }

constexpr int CODEC_LUT = 0, CODEC_STREAM = 2048, CODEC_LDS = CODEC_STREAM + 1088;
constexpr int WAVE_LDS = 8448;
static_assert(CODEC_LDS <= WAVE_LDS && 8 * WAVE_LDS + 16 <= LDS_STAGE, "per-wave LDS");
__device__ __forceinline__ unsigned esc_bits(unsigned c) { return c == 0u ? 0u : ~(c | (c >> 1)) & 0x05050505u; }
__device__ __forceinline__ u32x2 lut_of(unsigned e, unsigned lut) {
    const LAS unsigned* q = (const LAS unsigned*)(size_t)__builtin_amdgcn_udot4(e, 0x80400804u, lut, false);
    return (u32x2){q[0], q[256]};
}
template <bool PACK> __device__ __forceinline__ void build_lut(LAS unsigned char* buf, int lane) {
    for (int m = lane; m < 256; m += 64) {
        unsigned long long sel = 0x0c0c0c0c0c0c0c0cull; int k = 0;
#pragma unroll
        for (int b = 0; b < 8; ++b) if (m >> b & 1) { const int pos = (b & 1) | ((b & 2) << 1) | ((b & 4) >> 1), slot = (k >> 1) | ((k & 1) << 2);
            sel ^= PACK ? (unsigned long long)(0x0c ^ pos) << (8 * slot) : (unsigned long long)(0x0c ^ slot) << (8 * pos); ++k; }
        *(LAS unsigned*)(buf + CODEC_LUT + m * 4) = (unsigned)sel; *(LAS unsigned*)(buf + CODEC_LUT + 1024 + m * 4) = (unsigned)(sel >> 32);
    }
    asm volatile("" ::: "memory");
}
__device__ __forceinline__ unsigned bfi(unsigned m, unsigned a, unsigned b) { return (a & m) | (b & ~m); }
__device__ __forceinline__ void split4(unsigned r0, unsigned r1, unsigned& m, unsigned& e) {
    const unsigned lo = __builtin_amdgcn_perm(r1, r0, 0x06040200u), hi = __builtin_amdgcn_perm(r1, r0, 0x07050301u);
    m = bfi(0x80808080u, hi, lo); e = bfi(0x01010101u, lo >> 7, hi << 1);
}
__device__ __forceinline__ u32x4 join8(unsigned m0, unsigned e0, unsigned m1, unsigned e1) {
    const unsigned h0 = bfi(0x80808080u, m0, e0 >> 1), l0 = bfi(0x7f7f7f7fu, m0, e0 << 7), h1 = bfi(0x80808080u, m1, e1 >> 1), l1 = bfi(0x7f7f7f7fu, m1, e1 << 7);
    return (u32x4){__builtin_amdgcn_perm(h0, l0, 0x05010400u), __builtin_amdgcn_perm(h0, l0, 0x07030602u), __builtin_amdgcn_perm(h1, l1, 0x05010400u), __builtin_amdgcn_perm(h1, l1, 0x07030602u)};
}

struct Item { bool isw; unsigned src, fin, cnt; };
__device__ __forceinline__ Item item_of(int d, int i) {
    const int chunk = (int)((d & 1) ? (unsigned)i / (unsigned)litems(1) : (unsigned)i / (unsigned)litems(0)), j = i - chunk * litems(d);
    Item it;
    it.isw = (d & 1) && j >= OH;
    if (it.isw) { it.src = (unsigned)((chunk * 128 + (j - OH) * 16) * BPITCH + (d >> 1) * 512); it.fin = (unsigned)(OFF_W2 + ((d >> 1) * NCH * 8 + chunk * 8 + j - OH) * 8192);
        it.cnt = (unsigned)(OFF_CNT + ((NSLOT + (d >> 1)) * NCH + chunk) * 64); }
    else { const int strip = (d & 1) ? EH + j : j, slot = (d & 1) ? 3 * EH + (d >> 1) * OH + j : ((d >> 1) - 1) * EH + j;
        it.src = (unsigned)(strip * STRIP_BYTES + chunk * 256); it.fin = (unsigned)(OFF_XA + slot * STRIP_BYTES + chunk * 8192);
        it.cnt = (unsigned)(OFF_CNT + (slot * NCH + chunk) * 64); }
    return it;
}
__device__ __forceinline__ void fetch(const char* src_x, const char* src_w, u32x4 (&nxt)[8], unsigned& tick, int d, int i, unsigned mask, const char* word, unsigned draw) {
    const int lane = threadIdx.x & 63;
    const Item x = item_of(d, i);
    const unsigned v = x.isw ? (lane >> 5) * BPITCH + (lane & 31) * 16 : (lane >> 4) * APITCH + (lane & 15) * 16, st = x.isw ? 2 * BPITCH : 4 * APITCH;
    asm volatile("s_mov_b32 exec_lo, %21\n\ts_mov_b32 exec_hi, %21\n\t"
                 "global_load_dwordx4 %0, %9, %17\n\tglobal_load_dwordx4 %1, %10, %17\n\tglobal_load_dwordx4 %2, %11, %17\n\tglobal_load_dwordx4 %3, %12, %17\n\t"
                 "global_load_dwordx4 %4, %13, %17\n\tglobal_load_dwordx4 %5, %14, %17\n\tglobal_load_dwordx4 %6, %15, %17\n\tglobal_load_dwordx4 %7, %16, %17\n\t"
                 "s_mov_b32 exec_lo, %22\n\ts_mov_b32 exec_hi, 0\n\tglobal_atomic_add %8, %18, %19, %20 sc0\n\ts_mov_b64 exec, -1"
                 : "=&v"(nxt[0]), "=&v"(nxt[1]), "=&v"(nxt[2]), "=&v"(nxt[3]), "=&v"(nxt[4]), "=&v"(nxt[5]), "=&v"(nxt[6]), "=&v"(nxt[7]), "=&v"(tick)
                 : "v"(v), "v"(v + st), "v"(v + 2 * st), "v"(v + 3 * st), "v"(v + 4 * st), "v"(v + 5 * st), "v"(v + 6 * st), "v"(v + 7 * st),
                   "s"((x.isw ? src_w : src_x) + x.src), "v"(0u), "v"(1u), "s"(word), "s"(mask), "s"(draw) : "memory");
}
#define WAIT_FETCH(nxt, tick) asm volatile("s_waitcnt vmcnt(0)" : "+v"(tick), "+v"(nxt[0]), "+v"(nxt[1]), "+v"(nxt[2]), "+v"(nxt[3]), "+v"(nxt[4]), "+v"(nxt[5]), "+v"(nxt[6]), "+v"(nxt[7]) :: "memory")
__device__ __forceinline__ void sender(const Params& P, LAS unsigned char* buf, volatile LAS unsigned* turn, int d, int first, unsigned call, u32x4 (&nxt)[8]) {
    const int lane = threadIdx.x & 63, nlink = NCH * litems(d), nwaves = SEND_WGS * 8;
    char* const queue = P.wsx[0] + OFF_QUEUE + d * 128;
    const char* const src_x = P.x; const char* const src_w = P.w;
    const unsigned lds0 = (unsigned)(size_t)buf;
    unsigned mine = (unsigned)__builtin_amdgcn_readfirstlane(threadIdx.x >> 6), tick;
    int cur = first;
    char* const peer = P.wsx[d & 7];
    char* const set = peer + (call & 1) * PAR_BYTES;
    char* const flags = peer + OFF_FLAG + (call & 1) * FLAG_BYTES;
    const unsigned qbase = (call - 1u) * nlink - 2 * nwaves;
    tick = qbase + first + nwaves;
    while (cur < nlink) {
        const int u = lbase(d) + cur;
        const int next = (int)((unsigned)__builtin_amdgcn_readfirstlane(tick) - qbase);
        u32x4 pk[6]; unsigned ex[16], esum = 0;
#pragma unroll
        for (int g = 0; g < 16; ++g) {
            unsigned m; split4(nxt[g >> 1][(g & 1) * 2], nxt[g >> 1][(g & 1) * 2 + 1], m, ex[g]);
            pk[g >> 2][g & 3] = m;
            if ((g & 3) == 0) esum = __builtin_amdgcn_sad_u8(ex[g], 0u, esum);
        }
        SCHED; fetch(src_x, src_w, nxt, tick, d, next < nlink ? next : nlink - 1, ~0u, queue, 1u); SCHED;
        *(LAS u32x4*)(buf + CODEC_STREAM + lane * 16) = (u32x4){0u, 0u, 0u, 0u};
        unsigned top = (((unsigned)__builtin_amdgcn_readlane(wave_scan(esum), 63) + 1023u + 174u) >> 10) + 2u;
        top = top < 18u ? 18u : top > 255u ? 255u : top;
        unsigned cg[8], xs[8], at[8], tot = 0;
#pragma unroll
        for (int v = 0; v < 8; ++v) {
            const unsigned ua = top * 0x01010101u - ex[2 * v], ub = top * 0x01010101u - ex[2 * v + 1];
            const unsigned ca = __builtin_amdgcn_perm(0u, 0x03020100u, (((ua >> 1) & 0x04040404u) | ua) & 0x07070707u);
            const unsigned cb = __builtin_amdgcn_perm(0u, 0x0c080400u, (((ub >> 1) & 0x04040404u) | ub) & 0x07070707u);
            cg[v] = ((ua | ub) & 0xf0f0f0f0u) != 0u ? 0u : ca | cb;
            const unsigned e = esc_bits(cg[v]);
            const u32x2 sel = lut_of(e, lds0 + CODEC_LUT);
            xs[v] = __builtin_amdgcn_perm(ub, ua, sel.x) | (__builtin_amdgcn_perm(ub, ua, sel.y) << 4);
            at[v] = tot; tot += __builtin_popcount(e);
        }
        const unsigned incl = wave_scan(tot);
        unsigned total = (unsigned)__builtin_amdgcn_readlane(incl, 63);
        if (total > 2048u) {
            total = 0;
#pragma unroll
            for (int v = 0; v < 8; ++v) cg[v] = 0u;
        } else {
            const unsigned n0 = incl - tot + (lds0 + CODEC_STREAM) * 2u;
            asm volatile("" ::: "memory");
#pragma unroll
            for (int v = 0; v < 8; ++v) {
                const unsigned n = n0 + at[v];
                const unsigned long long w = (unsigned long long)xs[v] << ((n * 4u) & 31u);
                LAS unsigned* q = (LAS unsigned*)(size_t)((n >> 1) & ~3u);
                __hip_atomic_fetch_or(q, (unsigned)w, __ATOMIC_RELAXED, __HIP_MEMORY_SCOPE_WAVEFRONT);
                __hip_atomic_fetch_or(q + 1, (unsigned)(w >> 32), __ATOMIC_RELAXED, __HIP_MEMORY_SCOPE_WAVEFRONT);
            }
        }
        asm volatile("" ::: "memory");
        pk[5] = *(const LAS u32x4*)(buf + CODEC_STREAM + lane * 16);
        unsigned least = cg[0];
#pragma unroll
        for (int v = 1; v < 8; ++v) least = cg[v] < least ? cg[v] : least;
#pragma unroll
        for (int c = 0; c < 4; ++c) pk[4][c] = cg[2 * c] | (cg[2 * c + 1] << 4);
        if (cur != first) mine = draw_turn(turn, lane);
        await_turn(turn, mine, SEND_SLOTS);
        if (__builtin_amdgcn_ballot_w64(least == 0u) != 0ull) {
            char* fin = set + item_of(d, cur).fin;
#pragma unroll
            for (int v = 0; v < 8; ++v) if (cg[v] == 0u) store16_sys(fin, v * 1024 + lane * 16, join8(pk[v >> 1][(v & 1) * 2], ex[2 * v], pk[v >> 1][(v & 1) * 2 + 1], ex[2 * v + 1]));
        }
        const unsigned long long smask = __builtin_amdgcn_ballot_w64((unsigned)lane * 32u < total);
        asm volatile("s_mov_b64 exec, %8\n\tglobal_store_dwordx4 %0, %6, %7 offset:3072 sc0 sc1\n\ts_mov_b64 exec, -1\n\t"
                     "global_store_dwordx4 %0, %1, %7 offset:-2048 sc0 sc1\n\tglobal_store_dwordx4 %0, %2, %7 offset:-1024 sc0 sc1\n\t"
                     "global_store_dwordx4 %0, %3, %7 sc0 sc1\n\tglobal_store_dwordx4 %0, %4, %7 offset:1024 sc0 sc1\n\t"
                     "global_store_dwordx4 %0, %5, %7 offset:2048 sc0 sc1\n\ts_nop 1"
                     :: "v"(lane * 16), "v"(pk[0]), "v"(pk[1]), "v"(pk[2]), "v"(pk[3]), "v"(pk[4]), "v"(pk[5]), "s"(set + OFF_PK + (size_t)u * ITEM_PK + 2048), "s"(smask) : "memory");
        WAIT_FETCH(nxt, tick);
        if (lane == 0) {
            end_turn(turn);
            __hip_atomic_store((GAS unsigned*)(flags + (size_t)u * 64), (call << 8) | top, __ATOMIC_RELAXED, __HIP_MEMORY_SCOPE_SYSTEM);
        }
        cur = next;
    }
}

__device__ __forceinline__ void unpacker(const Params& P, LAS unsigned char* buf, int uw, unsigned call) {
    const int lane = threadIdx.x & 63;
    const int d = __builtin_amdgcn_readfirstlane(uw % 7 + 1), i0 = __builtin_amdgcn_readfirstlane(uw / 7);
    const int step = __builtin_amdgcn_readfirstlane(NUW / 7 + (d <= NUW % 7 ? 1 : 0)), nlink = NCH * litems(d);
    char* ws = P.wsx[0];
    char* set = ws + (call & 1) * PAR_BYTES;
    const char* pkb = set + OFF_PK + (size_t)lbase(d) * ITEM_PK + 2048;
    const char* flags = ws + OFF_FLAG + (call & 1) * FLAG_BYTES + (size_t)lbase(d) * 64;
    const unsigned lds0 = (unsigned)(size_t)buf;
    build_lut<false>(buf, lane);
    u32x4 pk[6]; unsigned nf = 0;
#pragma unroll
    for (int k = 0; k < 6; ++k) pk[k] = (u32x4){0u, 0u, 0u, 0u};
#define LOAD_ITEM(i, mask) asm volatile("s_mov_b32 exec_lo, %11\n\ts_mov_b32 exec_hi, %11\n\t" \
        "global_load_dwordx4 %0, %7, %8 offset:-2048 sc0 sc1\n\tglobal_load_dwordx4 %1, %7, %8 offset:-1024 sc0 sc1\n\tglobal_load_dwordx4 %2, %7, %8 sc0 sc1\n\t" \
        "global_load_dwordx4 %3, %7, %8 offset:1024 sc0 sc1\n\tglobal_load_dwordx4 %4, %7, %8 offset:2048 sc0 sc1\n\tglobal_load_dwordx4 %5, %7, %8 offset:3072 sc0 sc1\n\t" \
        "global_load_dword %6, %9, %10 sc0 sc1\n\ts_mov_b64 exec, -1\n\ts_waitcnt vmcnt(0)" \
        : "+v"(pk[0]), "+v"(pk[1]), "+v"(pk[2]), "+v"(pk[3]), "+v"(pk[4]), "+v"(pk[5]), "+v"(nf) \
        : "v"(lane * 16), "s"(pkb + (size_t)(i) * ITEM_PK), "v"(0u), "s"(flags + (size_t)((i) + step < nlink ? (i) + step : (i)) * 64), "s"(mask) : "memory")
    unsigned f = 0, have = 0;
    for (int i = i0; i < nlink; i += step) {
        if (!have) {
            Wait wait;
            bool dead = false;
            while (((f = (unsigned)__builtin_amdgcn_readfirstlane(ld_sys(flags + (size_t)i * 64))) >> 8) < call)
                if (wait.give_up(ws, call)) { dead = true; break; }
            if (dead) break;
            LOAD_ITEM(i, ~0u);
        }
        const unsigned top4 = (f & 255u) * 0x01010101u;
        f = (unsigned)__builtin_amdgcn_readfirstlane(nf);
        have = below(i + step, nlink) & ~below((int)(f >> 8), (int)call);
        *(LAS u32x4*)(buf + CODEC_STREAM + lane * 16) = pk[5];
        unsigned cg[8], at[8], tot = 0; u32x2 sel[8];
#pragma unroll
        for (int v = 0; v < 8; ++v) {
            cg[v] = (pk[4][v >> 1] >> (4 * (v & 1))) & 0x0f0f0f0fu;
            const unsigned e = esc_bits(cg[v]);
            sel[v] = lut_of(e, lds0 + CODEC_LUT);
            at[v] = tot; tot += __builtin_popcount(e);
        }
        const unsigned n0 = wave_scan(tot) - tot + (lds0 + CODEC_STREAM) * 2u;
        asm volatile("" ::: "memory");
        const Item it = item_of(d, i);
        unsigned xs[8];
#pragma unroll
        for (int v = 0; v < 8; ++v) {
            const unsigned n = n0 + at[v];
            const LAS unsigned* q = (const LAS unsigned*)(size_t)((n >> 1) & ~3u);
            xs[v] = __builtin_amdgcn_alignbit(q[1], q[0], n * 4u);
        }
#pragma unroll
        for (int v = 0; v < 8; ++v) {
            const unsigned xe = xs[v] & 0x0f0f0f0fu, xo = (xs[v] >> 4) & 0x0f0f0f0fu;
            const unsigned e0 = top4 - ((cg[v] & 0x03030303u) | __builtin_amdgcn_perm(xo, xe, sel[v].x));
            const unsigned e1 = top4 - (((cg[v] >> 2) & 0x03030303u) | __builtin_amdgcn_perm(xo, xe, sel[v].y));
            const u32x4 r = join8(pk[v >> 1][(v & 1) * 2], e0, pk[v >> 1][(v & 1) * 2 + 1], e1);
            if (cg[v] != 0u) store16_sys(set + it.fin, v * 1024 + lane * 16, r);
        }
        LOAD_ITEM(i + step < nlink ? i + step : i, have);
        if (lane == 0) __hip_atomic_fetch_add((GAS unsigned*)(ws + it.cnt), 1u, __ATOMIC_RELAXED, __HIP_MEMORY_SCOPE_AGENT);
    }
}

__device__ __forceinline__ int strip_of(int pos, bool remote, int q) {
    if (!remote) return pos < 32 + NSLOT ? pos : -1;
    if (pos < EH) return pos;
    if (pos < EH + 3 * OH) { const int k = (pos - EH) / OH; return 32 + 3 * EH + (k + (k >= q ? 1 : 0)) * OH + (pos - EH) % OH; }
    return (unsigned)(pos - NTE * SE) < 3u * EH ? 32 + pos - NTE * SE : -1;
}
__device__ __forceinline__ int out_row_of(int id, int rank) {
    if (id < 32) return renum(rank) * 1024 + id * 32;
    const int slot = id - 32;
    if (slot < 3 * EH) return renum(rank ^ (2 * (slot / EH + 1))) * 1024 + (slot % EH) * 32;
    const int o = slot - 3 * EH;
    return renum(rank ^ (2 * (o / OH) + 1)) * 1024 + (EH + o % OH) * 32;
}

constexpr int OUT_M = 256, OUT_LDS = 4160;
template <bool PACK> __device__ __forceinline__ u32x4 out_lut(int m) {
    unsigned long long sm = 0x0c0c0c0c0c0c0c0cull, sn = sm; int k = 0;
#pragma unroll
    for (int b = 0; b < 8; ++b) if (m >> b & 1) { const int slot = (k >> 1) | ((k & 1) << 2);
        sm ^= PACK ? (unsigned long long)(0x0c ^ b) << (8 * k) : (unsigned long long)(0x0c ^ k) << (8 * b);
        sn ^= PACK ? (unsigned long long)(0x0c ^ b) << (8 * slot) : (unsigned long long)(0x0c ^ slot) << (8 * b); ++k; }
    return (u32x4){(unsigned)sm, (unsigned)(sm >> 32), (unsigned)sn, (unsigned)(sn >> 32)};
}
struct OutItem { int o, q, w, f0, ida, idb; };
__device__ __forceinline__ OutItem out_item(int n) {
    const bool late = n >= NTE * HPT * 32;
    const int n1 = late ? n - NTE * HPT * 32 : n, per = (late ? NTR - NTE : NTE) * 32, h = late ? n1 / ((NTR - NTE) * 32) : n1 / (NTE * 32), rt = (late ? NTE : 0) + ((n1 - h * per) >> 5);
    const int S = late ? SR : SE, p0 = first_pos(rt, true);
    OutItem t; t.w = (n1 >> 2) & 7; t.q = n1 & 3; t.f0 = (t.w >> 2) * S + 2 * h;
    t.ida = n < NOI ? strip_of(p0 + (t.f0 >> 1), true, t.q) : -1; t.idb = 2 * h + 1 < S ? strip_of(p0 + ((t.f0 + 1) >> 1), true, t.q) : -1;
    t.o = ((t.q * NTR + rt) * 8 + t.w) * HPT + h;
    return t;
}
__device__ __forceinline__ void expander(const Params& P, LAS unsigned char* buf, int xw, unsigned call) {
    const int lane = threadIdx.x & 63;
    char* ws = P.wsx[0];
    const unsigned lds0 = (unsigned)(size_t)buf;
    for (int m = lane; m < 256; m += 64) *(LAS u32x4*)(buf + OUT_LDS + m * 16) = out_lut<false>(m);
    const OutItem mine = out_item(xw + (lane & 7) * NXW);
    unsigned long long todo = __builtin_amdgcn_ballot_w64(lane < 8 && mine.ida >= 0);
    const char* watch = ws + OFF_OFLAG + (mine.ida >= 0 ? mine.o : 0) * 64;
    while (todo) {
        unsigned long long fl, ready; bool dead = false;
        { Wait wait;
          while ((ready = __builtin_amdgcn_ballot_w64((unsigned)((fl = __hip_atomic_load((const GAS unsigned long long*)watch, __ATOMIC_RELAXED, __HIP_MEMORY_SCOPE_SYSTEM)) >> 32) >= call) & todo) == 0ull) {
              if (wait.give_up(ws, call)) { dead = true; ready = todo; break; }
          } }
        const int k = __builtin_ctzll(ready);
        todo &= ~(1ull << k);
        const unsigned f = (unsigned)__builtin_amdgcn_readlane((unsigned)fl, k);
        const OutItem t = out_item(xw + k * NXW);
#define UNI(x) __builtin_amdgcn_readfirstlane(x)
        const int o = UNI(t.o), q = UNI(t.q), w = UNI(t.w), f0 = UNI(t.f0), ida = UNI(t.ida), idb = UNI(t.idb);
        u32x4 r[4];
        unsigned tmo = call;
        if (!dead) {
            u32x4 t[4];
            asm volatile("global_load_dwordx4 %0, %5, %6 sc0 sc1\n\tglobal_load_dwordx4 %1, %5, %6 offset:1024 sc0 sc1\n\tglobal_load_dwordx4 %2, %5, %6 offset:2048 sc0 sc1\n\t"
                         "global_load_dwordx4 %3, %5, %6 offset:3072 sc0 sc1\n\tglobal_load_dword %4, %7, %8 sc1\n\ts_waitcnt vmcnt(0)"
                         : "=&v"(t[0]), "=&v"(t[1]), "=&v"(t[2]), "=&v"(t[3]), "=&v"(tmo)
                         : "v"(lane * 16), "s"(ws + OFF_OPK + (size_t)o * OITEM), "v"(0u), "s"(ws + OFF_TMO) : "memory");
            if (!(f & 0x100000u)) {
#pragma unroll
                for (int k = 0; k < 4; ++k) *(LAS u32x4*)(buf + lane * 16 + k * 1024) = t[k];
                asm volatile("" ::: "memory");
                const unsigned bits = *(const LAS unsigned*)(buf + lane * 4), out_n = OUT_M + (((f & 0xfffu) + 15u) & 0xff0u);
                const unsigned tot = __builtin_popcount(bits), base4 = ((f >> 12) & 255u) * 0x01010101u;
                unsigned n = (wave_scan(tot) - tot) & 2047u;
                asm volatile("" ::: "memory");
#pragma unroll
                for (int k = 0; k < 4; ++k) {
                    const unsigned m = (bits >> (8 * k)) & 255u;
                    const u32x4 sel = *(const LAS u32x4*)(buf + OUT_LDS + m * 16);
                    const LAS unsigned* qm = (const LAS unsigned*)(size_t)(lds0 + OUT_M + (n & ~3u));
                    const LAS unsigned* qn = (const LAS unsigned*)(size_t)(lds0 + out_n + ((n >> 1) & ~3u));
                    const unsigned m0 = __builtin_amdgcn_alignbyte(qm[1], qm[0], n), m1 = __builtin_amdgcn_alignbyte(qm[2], qm[1], n), x = __builtin_amdgcn_alignbit(qn[1], qn[0], n * 4u);
                    const unsigned xe = x & 0x0f0f0f0fu, xo = (x >> 4) & 0x0f0f0f0fu;
                    const unsigned e0 = (base4 - __builtin_amdgcn_perm(xo, xe, sel.z)) & __builtin_amdgcn_perm(~0u, ~0u, sel.x);
                    const unsigned e1 = (base4 - __builtin_amdgcn_perm(xo, xe, sel.w)) & __builtin_amdgcn_perm(~0u, ~0u, sel.y);
                    t[k] = join8(__builtin_amdgcn_perm(m1, m0, sel.x), e0, __builtin_amdgcn_perm(m1, m0, sel.y), e1);
                    n += __builtin_popcount(m);
                }
                asm volatile("" ::: "memory");
            }
#pragma unroll
            for (int k = 0; k < 8; ++k) {
                const int row = (k >> 2) * 16 + (lane & 15), ch = (k & 3) * 2 + (lane >> 5);
                *(LAS u32x2*)(buf + row * 128 + ((ch ^ (row & 7)) * 16) + ((lane >> 4) & 1) * 8) = (u32x2){t[k >> 1][(k & 1) * 2], t[k >> 1][(k & 1) * 2 + 1]};
            }
            asm volatile("" ::: "memory");
#pragma unroll
            for (int v = 0; v < 4; ++v) { const int row = v * 8 + (lane >> 3); r[v] = *(const LAS u32x4*)(buf + row * 128 + (((lane & 7) ^ (row & 7)) * 16)); }
        }
        if (tmo == call) {
#pragma unroll
            for (int v = 0; v < 4; ++v) r[v] = (u32x4){0x7fc07fc0u, 0x7fc07fc0u, 0x7fc07fc0u, 0x7fc07fc0u};
        }
        const int src = P.rank ^ (2 * q + 1);
        const unsigned ooff = (lane >> 3) * OPITCH + q * 512 + ((lane >> 2) & 1) * 256 + (w & 3) * 64 + (lane & 3) * 16;
        { char* O = P.out + (size_t)(out_row_of(ida, src) + (f0 & 1) * 16) * OPITCH; store16_sys(O, ooff, r[0]); store16_sys(O, ooff + 8 * OPITCH, r[1]); }
        if (idb >= 0) { char* O = P.out + (size_t)(out_row_of(idb, src) + ((f0 + 1) & 1) * 16) * OPITCH; store16_sys(O, ooff, r[2]); store16_sys(O, ooff + 8 * OPITCH, r[3]); }
    }
}

__device__ __forceinline__ unsigned cvt_pk_bf16(float lo, float hi) { unsigned r; asm("v_cvt_pk_bf16_f32 %0, %1, %2" : "=v"(r) : "v"(lo), "v"(hi)); return r; }
template <int OFF> __device__ __forceinline__ s16x4 tr_read(unsigned addr) { s16x4 r; asm volatile("ds_read_b64_tr_b16 %0, %1 offset:%2" : "=v"(r) : "v"(addr), "i"(OFF) : "memory"); return r; }
template <int OFF> __device__ __forceinline__ bf16x8 row_read(unsigned addr) { bf16x8 r; asm volatile("ds_read_b128 %0, %1 offset:%2" : "=v"(r) : "v"(addr), "i"(OFF) : "memory"); return r; }
__device__ __forceinline__ void glds(unsigned ldsaddr, unsigned voff, const char* g) {
    asm volatile("s_mov_b32 m0, %0\n\ts_nop 0\n\tglobal_load_lds_dwordx4 %1, %2 sc0 sc1" :: "s"(ldsaddr), "v"(voff), "s"(g) : "memory");
}
__device__ __forceinline__ void glds_masked(unsigned ldsaddr, unsigned voff, const char* g, unsigned mask) {
    asm volatile("s_mov_b32 m0, %0\n\ts_mov_b32 exec_lo, %3\n\ts_mov_b32 exec_hi, %3\n\tglobal_load_lds_dwordx4 %1, %2 sc0 sc1\n\ts_mov_b64 exec, -1" :: "s"(ldsaddr), "v"(voff), "s"(g), "s"(mask) : "memory");
}

template <int S> struct Frag { bf16x8 a[S], b[4]; };
template <int S> __device__ __forceinline__ void wait_step(Frag<S>& F, unsigned& pre) {
    asm volatile("s_waitcnt vmcnt(6) lgkmcnt(0)" : "+v"(F.a[0]), "+v"(F.a[1]), "+v"(F.a[2]), "+v"(F.a[3]), "+v"(F.a[S - 1]), "+v"(F.b[0]), "+v"(F.b[1]), "+v"(F.b[2]), "+v"(F.b[3]), "+v"(pre) :: "memory");
    if constexpr (S > 5) asm volatile("" : "+v"(F.a[4]));
    if constexpr (S > 6) asm volatile("" : "+v"(F.a[5]));
    if constexpr (S > 7) asm volatile("" : "+v"(F.a[6]));
}
template <int S, bool remote> __device__ __forceinline__ void gemm_tile(const Params& P, LAS unsigned char* lds, int rt, int ct, unsigned call) {
    constexpr int SLOT_A = S * 2048, SLOT = SLOT_A + 16384;
    static_assert(S >= 5 && S <= 8 && 4 * SLOT <= LDS_STAGE && S * 16384 <= LDS_STAGE, "tile");
    int wid = __builtin_amdgcn_readfirstlane(threadIdx.x >> 6), lane = threadIdx.x & 63, wr = wid >> 2, wc = wid & 3, fr = lane & 15, fq = lane >> 4;
    const int p0 = first_pos(rt, remote);
    char* ws = P.wsx[0];
    const char* set = ws + (call & 1) * PAR_BYTES;
    const char* const own_x = P.x;
    const unsigned ldst = (unsigned)(size_t)lds + wid * 1024;
    constexpr int BP = remote ? 512 : BPITCH;
    auto strip_at = [&](int k) { const int id = k < S ? strip_of(p0 + k, remote, ct) : -1; return id < 0 ? strip_of(p0, remote, ct) : id; };
    const int id0 = strip_at(wid >> 1), id1 = strip_at(4 + (wid >> 1));
    const char* const ga0 = id0 < 32 ? own_x + id0 * STRIP_BYTES : set + OFF_XA + (id0 - 32) * STRIP_BYTES;
    const char* const ga1 = id1 < 32 ? own_x + id1 * STRIP_BYTES : set + OFF_XA + (id1 - 32) * STRIP_BYTES;
    const unsigned cs0 = id0 < 32 ? 256 : 8192, cs1 = id1 < 32 ? 256 : 8192;
    const char* const gb = remote ? set + OFF_W2 + ct * (W2_BYTES / 4) : P.w + ct * 512;
    unsigned voffA0, voffA1, voffB;
    { const int k0 = lane >> 4, ch = lane & 15; voffB = (unsigned)((wid * 4 + k0) * BP + (((ch >> 1) ^ (k0 | ((wid & 2) << 1))) * 32) + (ch & 1) * 16);
      const int sb = lane * 16, swz = sb ^ (((sb >> 9) & 1) << 5), row = (wid & 1) * 16 + (swz >> 6);
      voffA0 = (unsigned)(row * (id0 < 32 ? APITCH : 256) + (swz & 63)); voffA1 = (unsigned)(row * (id1 < 32 ? APITCH : 256) + (swz & 63)); }
    const unsigned amask = below(wid, 2 * S - 8);
    unsigned aoff[2], boff[2][2];
    { const unsigned lbase = (unsigned)(size_t)lds; const int ob = fr * 64 + fq * 16; aoff[0] = lbase + wr * S * 1024 + (ob ^ (((ob >> 9) & 1) << 5)); aoff[1] = aoff[0] + 2 * SLOT;
      const int q = (lane & 15) >> 2, p = lane & 3, f = q | ((fq & 1) << 2);
      for (int n = 0; n < 2; ++n) { boff[0][n] = lbase + SLOT_A + (8 * fq + q) * 256 + p * 8 + (((wc * 2 + n) ^ f) * 32); boff[1][n] = boff[0][n] + 2 * SLOT; } }
    unsigned coff, cstep, need;
    { const int id = strip_of(p0 + (lane < S ? lane : 0), remote, ct);
      const bool watch = wid == 0 && (lane < S ? id >= 32 : (lane == 8 && remote));
      coff = watch ? (unsigned)(OFF_CNT + (lane < S ? id - 32 : NSLOT + ct) * NCH * 64) : (unsigned)OFF_TMO;
      cstep = watch ? 64u : 0u;
      need = watch ? call * (lane < S ? 1u : 8u) : 0u; }
    const unsigned w0 = below(wid, 1);
    unsigned pre = 0;
#define SLOW_WAIT() do { Wait wait; \
        while (__builtin_amdgcn_ballot_w64(ld_agent(ws + coff) < need) != 0ull) if (wait.give_up(ws, call)) break; \
        asm volatile("" ::: "memory"); } while (0)
#define PRE_LOAD(mask) do { coff += cstep; \
        asm volatile("s_mov_b32 exec_lo, %3\n\ts_mov_b32 exec_hi, 0\n\tglobal_load_dword %0, %1, %2 sc1\n\ts_mov_b64 exec, -1" : "+v"(pre) : "v"(coff), "s"(ws), "s"((mask) & w0) : "memory"); } while (0)
#define STAGE_B(h, k, ch) glds(ldst + (k) * SLOT + SLOT_A + (h) * 8192, voffB, gb + (h) * 256 + (size_t)((ch) * 4 + (k)) * (32 * BP))
#define STAGE_A0(k, ch) glds(ldst + (k) * SLOT, voffA0, ga0 + (ch) * cs0 + (k) * 64)
#define STAGE_A1(k, ch) glds_masked(ldst + (k) * SLOT + 8192, voffA1, ga1 + (ch) * cs1 + (k) * 64, amask)
#define STAGE_ALL(k, ch) do { STAGE_B(0, k, ch); STAGE_B(1, k, ch); STAGE_A0(k, ch); STAGE_A1(k, ch); } while (0)
#define RDA(F, m, sl) F.a[m] = row_read<((sl) & 1) * SLOT + (m) * 1024>(aoff[(sl) >> 1])
#define RDB(F, j, sl) do { const s16x4 _lo = tr_read<((sl) & 1) * SLOT + ((j) >> 1) * 8192>(boff[(sl) >> 1][(j) & 1]), _hi = tr_read<((sl) & 1) * SLOT + ((j) >> 1) * 8192 + 1024>(boff[(sl) >> 1][(j) & 1]); \
        F.b[j] = __builtin_shufflevector(_lo, _hi, 0, 1, 2, 3, 4, 5, 6, 7); } while (0)
#define MM(F, i) acc[(i) >> 2][(i) & 3] = __builtin_amdgcn_mfma_f32_16x16x32_bf16(F.b[(i) & 3], F.a[(i) >> 2], acc[(i) >> 2][(i) & 3], 0, 0, 0)
#define PAIR(C, i) do { MM(C, i); MM(C, (i) + 1); SCHED; } while (0)
#define STEP(C, N, k) do { constexpr int nx = ((k) + 1) & 3; \
        if constexpr ((k) == 2) PRE_LOAD(c < NCH - 2 ? 0x3ffu : c < NCH - 1 ? 0x200u : 0u); \
        RDB(N, 0, nx); PAIR(C, 0); RDB(N, 1, nx); PAIR(C, 2); RDA(N, 0, nx); RDA(N, 1, nx); PAIR(C, 4); RDB(N, 2, nx); PAIR(C, 6); RDB(N, 3, nx); \
        STAGE_B(0, k, c1); PAIR(C, 8); RDA(N, 2, nx); RDA(N, 3, nx); STAGE_B(1, k, c1); PAIR(C, 10); \
        RDA(N, 4, nx); if constexpr (S > 5) RDA(N, 5, nx); STAGE_A0(k, c1); PAIR(C, 12); \
        if constexpr (S > 6) RDA(N, 6, nx); if constexpr (S > 7) RDA(N, 7, nx); STAGE_A1(k, c1); \
        _Pragma("unroll") for (int i = 14; i < 4 * S - 4; i += 2) PAIR(C, i); \
        wait_step(N, pre); \
        if constexpr ((k) == 3) { if (c < NCH - 2 && __builtin_amdgcn_ballot_w64(pre < need) != 0ull) SLOW_WAIT(); } \
        BAR; SCHED; PAIR(C, 4 * S - 4); PAIR(C, 4 * S - 2); } while (0)
    f32x4 acc[S][4];
#pragma unroll
    for (int m = 0; m < S; ++m)
#pragma unroll
        for (int j = 0; j < 4; ++j) acc[m][j] = (f32x4){0.f, 0.f, 0.f, 0.f};
    Frag<S> F0, F1;
    constexpr int OUT_LUT = LDS_STAGE - 4096;
    static_assert(!remote || (4 * SLOT <= OUT_LUT && 8 * OUT_LDS <= OUT_LUT), "pack tables");
    if (remote && threadIdx.x >= 256) *(LAS u32x4*)(lds + OUT_LUT + (threadIdx.x - 256) * 16) = out_lut<true>(threadIdx.x - 256);
    if (wid == 0) SLOW_WAIT();
    BAR;
    PRE_LOAD(0x3ffu);
    STAGE_ALL(0, 0u); STAGE_ALL(1, 0u); STAGE_ALL(2, 0u); STAGE_ALL(3, 0u);
    WAIT_V(6); BAR;
    RDB(F0, 0, 0); RDB(F0, 1, 0); RDB(F0, 2, 0); RDB(F0, 3, 0); RDA(F0, 0, 0); RDA(F0, 1, 0); RDA(F0, 2, 0); RDA(F0, 3, 0); RDA(F0, 4, 0);
    if constexpr (S > 5) RDA(F0, 5, 0);
    if constexpr (S > 6) RDA(F0, 6, 0);
    if constexpr (S > 7) RDA(F0, 7, 0);
    wait_step(F0, pre);
    if (__builtin_amdgcn_ballot_w64(pre < need) != 0ull) SLOW_WAIT();
    BAR;
    for (int c = 0; c < NCH; ++c) {
        const unsigned c1 = c < NCH - 1 ? c + 1 : c;
        STEP(F0, F1, 0); STEP(F1, F0, 1); STEP(F0, F1, 2); STEP(F1, F0, 3);
    }
    volatile LAS unsigned* turn = (volatile LAS unsigned*)(lds + LDS_STAGE + 4);
    if (threadIdx.x == 9) { *(LAS unsigned*)(lds + LDS_STAGE) = pre; turn[0] = 0u; turn[1] = 0u; }
    asm volatile("s_waitcnt vmcnt(0) lgkmcnt(0)" ::: "memory");
    BAR;
    asm volatile("" ::: "memory");
    const bool bad = *(const LAS unsigned*)(lds + LDS_STAGE) == call;
    int tz = threadIdx.x; asm volatile("" : "+v"(tz));
    lane = tz & 63; fr = lane & 15; fq = lane >> 4;
    auto relu_pack = [&](int m, int j) {
        f32x4 v = acc[m][j];
#pragma unroll
        for (int e = 0; e < 4; ++e) v[e] = v[e] < 0.f ? 0.f : v[e];
        u32x2 w; w.x = cvt_pk_bf16(v[0], v[1]); w.y = cvt_pk_bf16(v[2], v[3]);
        if (bad) { w.x = 0x7fc07fc0u; w.y = 0x7fc07fc0u; }
        return w;
    };
    if constexpr (!remote) {
        LAS unsigned char* buf = lds + wid * (S * 2048);
#pragma unroll
        for (int m = 0; m < S; ++m)
#pragma unroll
            for (int j = 0; j < 4; ++j) *(LAS u32x2*)(buf + m * 2048 + fr * 128 + (((j * 2 + (fq >> 1)) ^ (fr & 7)) * 16) + (fq & 1) * 8) = relu_pack(m, j);
        asm volatile("" ::: "memory");
        const unsigned ooff = (lane >> 3) * OPITCH + ct * 512 + ((lane >> 2) & 1) * 256 + wc * 64 + (lane & 3) * 16;
#pragma unroll
        for (int m = 0; m < S; ++m) {
            const int f = wr * S + m, id = strip_of(p0 + (f >> 1), false, 0);
            const u32x4 a = *(const LAS u32x4*)(buf + m * 2048 + (lane >> 3) * 128 + (((lane & 7) ^ (lane >> 3)) * 16));
            const u32x4 b = *(const LAS u32x4*)(buf + m * 2048 + 1024 + (lane >> 3) * 128 + (((lane & 7) ^ (lane >> 3)) * 16));
            if (id >= 0) { char* O = P.out + (size_t)(out_row_of(id, P.rank) + (f & 1) * 16) * OPITCH; store16_sys(O, ooff, a); store16_sys(O, ooff + 8 * OPITCH, b); }
        }
    } else {
        char* const peer = P.wsx[(2 * ct + 1) & 7];
        LAS unsigned char* buf = lds + wid * OUT_LDS;
        const unsigned lds0 = (unsigned)(size_t)buf;
        const int o0 = ((ct * NTR + rt) * 8 + wid) * HPT;
        int sent = -1; unsigned sent_meta = 0;
#define PUBLISH_OUT() do { WAIT_V(0); if (sent >= 0) { if (lane == 0) { end_turn(turn); \
            __hip_atomic_store((GAS unsigned long long*)(peer + OFF_OFLAG + sent * 64), ((unsigned long long)call << 32) | sent_meta, __ATOMIC_RELAXED, __HIP_MEMORY_SCOPE_SYSTEM); } \
            sent = -1; } } while (0)
#pragma unroll
        for (int h = 0; h < HPT; ++h) {
            const int f0 = wr * S + 2 * h;
            if (strip_of(p0 + (f0 >> 1), true, ct) < 0) break;
            const bool second = 2 * h + 1 < S && strip_of(p0 + ((f0 + 1) >> 1), true, ct) >= 0;
            u32x4 t[4]; u16x2 mx = {0, 0};
#pragma unroll
            for (int k = 0; k < 8; ++k) {
                u32x2 w = relu_pack(2 * h + (k >> 2) < S ? 2 * h + (k >> 2) : 0, k & 3);
                if (k >= 4 && !second) w = (u32x2){0u, 0u};
                t[k >> 1][(k & 1) * 2] = w.x; t[k >> 1][(k & 1) * 2 + 1] = w.y;
                mx = __builtin_elementwise_max(mx, __builtin_elementwise_max(__builtin_bit_cast(u16x2, w.x & 0x7fff7fffu), __builtin_bit_cast(u16x2, w.y & 0x7fff7fffu)));
            }
#pragma unroll
            for (int k = 0; k < 3; ++k) *(LAS u32x4*)(buf + OUT_M + lane * 16 + k * 1024) = (u32x4){0u, 0u, 0u, 0u};
            const unsigned emax = (unsigned)__builtin_amdgcn_readlane(wave_max(mx.x > mx.y ? mx.x : mx.y), 63) >> 7;
            unsigned bits = 0, wide = 0, tot = 0, at[4], cm0[4], cm1[4], cn[4];
#pragma unroll
            for (int k = 0; k < 4; ++k) {
                unsigned z[4], m0, e0, m1, e1;
#pragma unroll
                for (int c = 0; c < 4; ++c) z[c] = nonzero2(t[k][c]);
                const unsigned m = __builtin_amdgcn_udot4(__builtin_amdgcn_perm(z[3], z[2], 0x06040200u), 0x80402010u, __builtin_amdgcn_udot4(__builtin_amdgcn_perm(z[1], z[0], 0x06040200u), 0x08040201u, 0u, false), false);
                const u32x4 sel = *(const LAS u32x4*)(lds + OUT_LUT + m * 16);
                split4(t[k].x, t[k].y, m0, e0); split4(t[k].z, t[k].w, m1, e1);
                e0 = emax * 0x01010101u - e0; e1 = emax * 0x01010101u - e1;
                cm0[k] = __builtin_amdgcn_perm(m1, m0, sel.x); cm1[k] = __builtin_amdgcn_perm(m1, m0, sel.y);
                const unsigned de = __builtin_amdgcn_perm(e1, e0, sel.z), dd = __builtin_amdgcn_perm(e1, e0, sel.w);
                wide |= de | dd; cn[k] = de | (dd << 4);
                bits |= m << (8 * k); at[k] = tot; tot += __builtin_popcount(m);
            }
            const unsigned incl = wave_scan(tot), count = (unsigned)__builtin_amdgcn_readlane(incl, 63);
            const bool rawout = __builtin_amdgcn_ballot_w64((wide & 0xf0f0f0f0u) != 0u) != 0ull;
            char* dst = peer + OFF_OPK + (size_t)(o0 + h) * OITEM;
            const unsigned out_n = OUT_M + ((count + 15u) & ~15u), total = rawout ? 4096u : out_n + ((count + 1u) >> 1);
            if (!rawout) {
                *(LAS unsigned*)(buf + lane * 4) = bits;
                asm volatile("" ::: "memory");
#pragma unroll
                for (int k = 0; k < 4; ++k) {
                    const unsigned n = incl - tot + at[k];
                    const unsigned long long a = (unsigned long long)cm0[k] << ((n & 3u) * 8u), b = (unsigned long long)cm1[k] << ((n & 3u) * 8u), x = (unsigned long long)cn[k] << ((n * 4u) & 31u);
                    LAS unsigned* qm = (LAS unsigned*)(size_t)(lds0 + OUT_M + (n & ~3u));
                    LAS unsigned* qn = (LAS unsigned*)(size_t)(lds0 + out_n + ((n >> 1) & ~3u));
#define LDS_OR(p, v) __hip_atomic_fetch_or(p, v, __ATOMIC_RELAXED, __HIP_MEMORY_SCOPE_WAVEFRONT)
                    LDS_OR(qm, (unsigned)a); LDS_OR(qm + 1, (unsigned)(a >> 32) | (unsigned)b); LDS_OR(qm + 2, (unsigned)(b >> 32));
                    LDS_OR(qn, (unsigned)x); LDS_OR(qn + 1, (unsigned)(x >> 32));
                }
                asm volatile("" ::: "memory");
#pragma unroll
                for (int k = 0; k < 4; ++k) t[k] = *(const LAS u32x4*)(buf + lane * 16 + k * 1024);
                asm volatile("" : "+v"(t[0]), "+v"(t[1]), "+v"(t[2]), "+v"(t[3]) :: "memory");
            }
            PUBLISH_OUT();
            await_turn(turn, draw_turn(turn, lane), OUT_SLOTS);
#pragma unroll
            for (int k = 0; k < 4; ++k) if (lane * 16u + k * 1024u < total) store16_sys(dst, lane * 16 + k * 1024, t[k]);
            sent = o0 + h; sent_meta = (rawout ? 0x100000u : 0u) | (emax << 12) | count;
        }
        PUBLISH_OUT();
    }
}

__global__ __launch_bounds__(NTHR, 2) void ag_gemm_kernel(const Params P) {
    extern __shared__ __attribute__((aligned(16))) unsigned char shm[];
    LAS unsigned char* const lds = (LAS unsigned char*)shm;
    const int wid = __builtin_amdgcn_readfirstlane(threadIdx.x >> 6);
    char* const tick = P.wsx[0] + OFF_TICK;
    const char* const src_x = P.x; const char* const src_w = P.w;
    volatile LAS unsigned* const word = (volatile LAS unsigned*)(lds + LDS_STAGE);
    const int asked = (int)(__builtin_amdgcn_s_getreg((3 << 11) | 20) & 7u) * 32 + (int)(blockIdx.x >> 3);
    auto sends = [](int r) { return (r >> 5) != 3 && (r & 31) < SEND_WGS; };
    u32x4 nxt[8]; unsigned t = 0, c;
    asm volatile("s_mov_b32 exec_lo, %4\n\ts_mov_b32 exec_hi, 0\n\tglobal_atomic_add %0, %1, %2, %3 sc0\n\ts_mov_b64 exec, -1" : "+v"(t) : "v"(blockIdx.x * 64u), "v"(1u), "s"(tick), "s"(below(wid, 1) & 1u) : "memory");
    fetch(src_x, src_w, nxt, c, link_of(asked >> 5), wid * SEND_WGS + (asked & 31), below(asked & 31, SEND_WGS) & ~(below(asked >> 5, 4) & below(2, asked >> 5)), tick + (256 + asked) * 64, below(wid, 1) & 1u);
    if (sends(asked)) build_lut<true>(lds + wid * WAVE_LDS, threadIdx.x & 63);
    asm volatile("" : "+v"(t));
    WAIT_FETCH(nxt, c);
    asm volatile("" : "+v"(t));
    if (threadIdx.x == 0) {
        int r = asked;
        if (c != t) {
            __hip_atomic_fetch_add((GAS unsigned*)(tick + (256 + r) * 64), ~0u, __ATOMIC_RELAXED, __HIP_MEMORY_SCOPE_AGENT);
            for (int k = 1; k < 256 * 64; ++k) {
                unsigned expect = t; r = (asked + k) & 255;
                if (__hip_atomic_compare_exchange_strong((GAS unsigned*)(tick + (256 + r) * 64), &expect, t + 1u, __ATOMIC_RELAXED, __ATOMIC_RELAXED, __HIP_MEMORY_SCOPE_AGENT)) break;
            }
        }
        word[0] = t + 1u; word[1] = (unsigned)r; word[2] = 8u; word[3] = 0u;
    }
    WAIT_L(0); BAR; asm volatile("" ::: "memory");
    struct { unsigned call; int role; } claim = {(unsigned)__builtin_amdgcn_readfirstlane(word[0]), __builtin_amdgcn_readfirstlane((int)word[1]) & 255};
    const int xcd = claim.role >> 5, slot = claim.role & 31;
#define PICK(f) __builtin_amdgcn_readfirstlane(xcd == 0 ? BeforeAt<0>::f : xcd == 1 ? BeforeAt<1>::f : xcd == 2 ? BeforeAt<2>::f : xcd == 3 ? BeforeAt<3>::f : \
                                               xcd == 4 ? BeforeAt<4>::f : xcd == 5 ? BeforeAt<5>::f : xcd == 6 ? BeforeAt<6>::f : BeforeAt<7>::f)
    const int ns = nsend(xcd), nc = ncomm(xcd), ub = PICK(unp), sb = PICK(snd);
    const int xw = __builtin_amdgcn_readfirstlane(slot < ns ? NUW + sb + wid * ns + slot : ub + (slot - ns) * 8 + wid);
    if (slot < ns) {
        if (claim.role != asked) { build_lut<true>(lds + wid * WAVE_LDS, threadIdx.x & 63); fetch(src_x, src_w, nxt, c, link_of(xcd), wid * SEND_WGS + slot, ~0u, tick, 0u); WAIT_FETCH(nxt, c); }
        sender(P, lds + wid * WAVE_LDS, word + 2, link_of(xcd), wid * SEND_WGS + slot, claim.call, nxt);
    }
    if (slot < nc) {
        if (slot >= ns) unpacker(P, lds + wid * WAVE_LDS, xw, claim.call);
        expander(P, lds + wid * WAVE_LDS, xw, claim.call);
    } else {
        const int g = slot - nc, rt = (g >> 2) * 4 + 3 - (xcd & 3);
        if (xcd < 4) { if (SO != SL && rt < NTO) gemm_tile<SO, false>(P, lds, rt, g & 3, claim.call); else gemm_tile<SL, false>(P, lds, rt, g & 3, claim.call); }
        else { if (SE != SR && rt < NTE) gemm_tile<SE, true>(P, lds, rt, g & 3, claim.call); else gemm_tile<SR, true>(P, lds, rt, g & 3, claim.call); }
    }
}
}

extern "C" void kernel_launch(int rank, int nranks, ncclComm_t comm, hipStream_t stream,
                              void* const* d_in, const int* in_sizes, int n_in,
                              void* d_out, int out_size, const int* in_dtypes, int out_dtype,
                              void* const* d_out_all, void* const* d_ws_all, size_t ws_bytes) {
    (void)comm; (void)in_dtypes; (void)out_dtype; (void)d_out_all;
    if (nranks != NR || (unsigned)rank >= (unsigned)NR || n_in != 2 || in_sizes[0] != 1024 * KDIM || in_sizes[1] != KDIM * 1024 || out_size != KDIM * 1024 || ws_bytes < WS_NEED) {
        fprintf(stderr, "kernel_launch: unexpected problem geometry\n");
        return;
    }
    static bool attr_set = false;
    if (!attr_set) { HIP_CHECK(hipFuncSetAttribute((const void*)ag_gemm_kernel, hipFuncAttributeMaxDynamicSharedMemorySize, LDS_BYTES)); attr_set = true; }
    Params P;
    P.x = (const char*)d_in[0]; P.w = (const char*)d_in[1]; P.out = (char*)d_out;
    for (int d = 0; d < NR; ++d) P.wsx[d] = (char*)d_ws_all[renum(renum(rank) ^ d)];
    P.rank = renum(rank); P.pad = 0;
    hipLaunchKernelGGL(ag_gemm_kernel, dim3(GRID), dim3(NTHR), LDS_BYTES, stream, P);
    HIP_CHECK(hipGetLastError());
}
```

```cpp
#include <hip/hip_runtime.h>
#include <rccl/rccl.h>
#include <cstdio>

#define HIP_CHECK(expr) do { hipError_t _e = (expr); if (_e != hipSuccess) { \
    fprintf(stderr, "HIP error %s at %s:%d\n", hipGetErrorString(_e), __FILE__, __LINE__); \
    return; } } while (0)

namespace {
#define LAS __attribute__((address_space(3)))
#define GAS __attribute__((address_space(1)))
typedef short bf16x8 __attribute__((ext_vector_type(8)));
typedef short s16x4 __attribute__((ext_vector_type(4)));
typedef float f32x4 __attribute__((ext_vector_type(4)));
typedef unsigned u32x4 __attribute__((ext_vector_type(4)));
typedef unsigned u32x2 __attribute__((ext_vector_type(2)));
typedef unsigned short u16x2 __attribute__((ext_vector_type(2)));

constexpr int NR = 8;
constexpr int KDIM = 8192;
constexpr int APITCH = KDIM * 2, BPITCH = 2048, OPITCH = 2048;
constexpr int E_STRIPS = 21;
constexpr int EH = E_STRIPS, OH = 32 - EH;
constexpr int NSLOT = 3 * EH + 4 * OH;
constexpr int SL = 7;
constexpr int SR = 5;
constexpr int NTL = (32 + NSLOT + SL - 1) / SL, NTE = (EH + 3 * OH + SR - 1) / SR, NTR = NTE + (3 * EH + SR - 1) / SR, HPT = (SR + 1) / 2;
constexpr int NCH = 64;
__host__ __device__ constexpr int litems(int d) { return (d & 1) ? OH + 8 : EH; }
__host__ __device__ constexpr int lbase(int d) { return NCH * ((d >> 1) * litems(1) + ((d - 1) >> 1) * litems(0)); }
constexpr int NU = lbase(8);
constexpr int ITEM_PK = 6144;
constexpr size_t STRIP_BYTES = (size_t)32 * APITCH;
constexpr size_t XA_BYTES = NSLOT * STRIP_BYTES, W2_BYTES = (size_t)KDIM * BPITCH, PK_BYTES = (size_t)NU * ITEM_PK;
constexpr size_t OFF_XA = 0, OFF_W2 = XA_BYTES, OFF_PK = XA_BYTES + W2_BYTES, PAR_BYTES = OFF_PK + PK_BYTES;
constexpr size_t OFF_FLAG = 2 * PAR_BYTES, FLAG_BYTES = (size_t)NU * 64;
constexpr size_t OFF_CNT = OFF_FLAG + 2 * FLAG_BYTES, CNT_BYTES = (size_t)(NSLOT + 4) * NCH * 64;
constexpr size_t OFF_TICK = OFF_CNT + CNT_BYTES, TICK_BYTES = 512 * 64;
constexpr int OITEM = 4096, NOI = 4 * NTR * 8 * HPT;
constexpr size_t OFF_OPK = OFF_TICK + TICK_BYTES, OFF_OFLAG = OFF_OPK + (size_t)NOI * OITEM;
constexpr size_t OFF_TMO = OFF_OFLAG + (size_t)NOI * 64, OFF_QUEUE = OFF_TMO + 128;
constexpr size_t WS_NEED = OFF_QUEUE + 8 * 128;
static_assert(WS_NEED <= ((size_t)257 << 20), "workspace");
constexpr int GRID = 256, NTHR = 512;
constexpr int SEND_WGS = 6;
constexpr int OUT_SLOTS = 6;
constexpr int SEND_SLOTS = 3;
constexpr int TURN_SLEEP = 8;
constexpr int POLL_SLEEP = 2;
__host__ __device__ constexpr int ncomm(int x) { return x < 4 ? (x == 1 ? 16 : x == 2 ? 8 : 12) : 32 - 4 * ((NTR + x % 4) / 4); }
__host__ __device__ constexpr int row_tile(int x, int k) { return x >= 4 ? k * 4 + 7 - x : (x == 1 ? 0 : x == 0 ? 4 : x == 2 ? 9 : 15) + k; }
static_assert(NTL == 20, "row_tile");
__host__ __device__ constexpr int nsend(int x) { return x == 3 ? 0 : SEND_WGS; }
__host__ __device__ constexpr int link_of(int x) { return x < 3 ? 2 * x + 2 : x < 4 ? 1 : 2 * x - 7; }
struct Before { int unp[9], snd[9]; };
constexpr Before before() { Before b = {}; for (int x = 0; x < 8; ++x) { b.unp[x + 1] = b.unp[x] + (ncomm(x) - nsend(x)) * 8; b.snd[x + 1] = b.snd[x] + nsend(x) * 8; } return b; }
template <int X> struct BeforeAt { static constexpr int unp = before().unp[X], snd = before().snd[X]; };
constexpr int NUW = BeforeAt<8>::unp, NXW = NUW + BeforeAt<8>::snd;
constexpr unsigned long long DEADLINE = 50000000ull;
constexpr int LDS_STAGE = 135168, LDS_BYTES = LDS_STAGE + 16;
static_assert(NOI <= 8 * NXW && ncomm(2) >= nsend(2) && ncomm(3) >= 0 && ncomm(7) >= nsend(7), "roles");

__host__ __device__ constexpr int renum(int r) { return r == 3 ? 4 : r == 4 ? 3 : r; }
struct Params {
    const char* x; const char* w;
    char* out;
    char* wsx[NR];
    int rank; int pad;
};

#define WAIT_V(n) asm volatile("s_waitcnt vmcnt(" #n ")" ::: "memory")
#define WAIT_L(n) asm volatile("s_waitcnt lgkmcnt(" #n ")" ::: "memory")
#define BAR __builtin_amdgcn_s_barrier()
#define SCHED __builtin_amdgcn_sched_barrier(0)

__device__ __forceinline__ unsigned ld_agent(const char* p) { return __hip_atomic_load((const GAS unsigned*)p, __ATOMIC_RELAXED, __HIP_MEMORY_SCOPE_AGENT); }
__device__ __forceinline__ unsigned ld_sys(const char* p) { return __hip_atomic_load((const GAS unsigned*)p, __ATOMIC_RELAXED, __HIP_MEMORY_SCOPE_SYSTEM); }
struct Wait {
    unsigned long long t0; unsigned polls;
    __device__ __forceinline__ Wait() : t0(__builtin_amdgcn_s_memrealtime()), polls(0) {}
    __device__ __forceinline__ bool give_up(char* ws, unsigned call) {
        if ((++polls & 15u) == 0u && ((unsigned)__builtin_amdgcn_readfirstlane(ld_agent(ws + OFF_TMO)) == call || __builtin_amdgcn_s_memrealtime() - t0 > DEADLINE)) {
            __hip_atomic_store((GAS unsigned*)(ws + OFF_TMO), call, __ATOMIC_RELAXED, __HIP_MEMORY_SCOPE_AGENT);
            return true;
        }
        __builtin_amdgcn_s_sleep(POLL_SLEEP);
        return false;
    }
};
__device__ __forceinline__ void store16_sys(char* base, unsigned voff, u32x4 v) {
    asm volatile("global_store_dwordx4 %0, %1, %2 sc0 sc1\n\ts_nop 1" :: "v"(voff), "v"(v), "s"(base) : "memory");
}

template <int CTRL, int ROWS> __device__ __forceinline__ unsigned max_dpp(unsigned x) { const unsigned y = (unsigned)__builtin_amdgcn_update_dpp(0u, x, CTRL, ROWS, 0xf, false); return x > y ? x : y; }
__device__ __forceinline__ unsigned wave_max(unsigned x) {
    return max_dpp<0x143, 0xc>(max_dpp<0x142, 0xa>(max_dpp<0x118, 0xf>(max_dpp<0x114, 0xf>(max_dpp<0x112, 0xf>(max_dpp<0x111, 0xf>(x))))));
}
__device__ __forceinline__ unsigned nonzero2(unsigned w) { unsigned r; asm("v_pk_min_u16 %0, %1, %2" : "=v"(r) : "v"(w), "s"(0x00010001u)); return r; }
__device__ __forceinline__ unsigned below(int a, int b) { unsigned r; asm("s_ashr_i32 %0, %1, 31" : "=s"(r) : "s"(a - b) : "scc"); return r; }
__device__ __forceinline__ unsigned wave_scan(unsigned x) {
    x += __builtin_amdgcn_update_dpp(0u, x, 0x111, 0xf, 0xf, false); x += __builtin_amdgcn_update_dpp(0u, x, 0x112, 0xf, 0xf, false);
    x += __builtin_amdgcn_update_dpp(0u, x, 0x114, 0xf, 0xf, false); x += __builtin_amdgcn_update_dpp(0u, x, 0x118, 0xf, 0xf, false);
    x += __builtin_amdgcn_update_dpp(0u, x, 0x142, 0xa, 0xf, false); x += __builtin_amdgcn_update_dpp(0u, x, 0x143, 0xc, 0xf, false);
    return x;
}

__device__ __forceinline__ unsigned draw_turn(volatile LAS unsigned* turn, int lane) {
    unsigned t = 0;
    if (lane == 0) t = __hip_atomic_fetch_add((LAS unsigned*)turn, 1u, __ATOMIC_RELAXED, __HIP_MEMORY_SCOPE_WORKGROUP);
    return (unsigned)__builtin_amdgcn_readfirstlane(t);
}
__device__ __forceinline__ void await_turn(volatile LAS unsigned* turn, unsigned mine, unsigned slots) {
    while (mine >= (unsigned)__builtin_amdgcn_readfirstlane(turn[1]) + slots) __builtin_amdgcn_s_sleep(TURN_SLEEP);
}
__device__ __forceinline__ void end_turn(volatile LAS unsigned* turn) { __hip_atomic_fetch_add((LAS unsigned*)turn + 1, 1u, __ATOMIC_RELAXED, __HIP_MEMORY_SCOPE_WORKGROUP); }

constexpr int CODEC_LUT = 0, CODEC_STREAM = 2048, CODEC_LDS = CODEC_STREAM + 1088;
constexpr int WAVE_LDS = 8448;
static_assert(CODEC_LDS <= WAVE_LDS && 8 * WAVE_LDS + 16 <= LDS_STAGE, "per-wave LDS");
__device__ __forceinline__ unsigned esc_bits(unsigned c) { return c == 0u ? 0u : ~(c | (c >> 1)) & 0x05050505u; }
__device__ __forceinline__ u32x2 lut_of(unsigned e, unsigned lut) {
    const LAS unsigned* q = (const LAS unsigned*)(size_t)__builtin_amdgcn_udot4(e, 0x80400804u, lut, false);
    return (u32x2){q[0], q[256]};
}
template <bool PACK> __device__ __forceinline__ void build_lut(LAS unsigned char* buf, int lane) {
    for (int m = lane; m < 256; m += 64) {
        unsigned long long sel = 0x0c0c0c0c0c0c0c0cull; int k = 0;
#pragma unroll
        for (int b = 0; b < 8; ++b) if (m >> b & 1) { const int pos = (b & 1) | ((b & 2) << 1) | ((b & 4) >> 1), slot = (k >> 1) | ((k & 1) << 2);
            sel ^= PACK ? (unsigned long long)(0x0c ^ pos) << (8 * slot) : (unsigned long long)(0x0c ^ slot) << (8 * pos); ++k; }
        *(LAS unsigned*)(buf + CODEC_LUT + m * 4) = (unsigned)sel; *(LAS unsigned*)(buf + CODEC_LUT + 1024 + m * 4) = (unsigned)(sel >> 32);
    }
    asm volatile("" ::: "memory");
}
__device__ __forceinline__ unsigned bfi(unsigned m, unsigned a, unsigned b) { return (a & m) | (b & ~m); }
__device__ __forceinline__ void split4(unsigned r0, unsigned r1, unsigned& m, unsigned& e) {
    const unsigned lo = __builtin_amdgcn_perm(r1, r0, 0x06040200u), hi = __builtin_amdgcn_perm(r1, r0, 0x07050301u);
    m = bfi(0x80808080u, hi, lo); e = bfi(0x01010101u, lo >> 7, hi << 1);
}
__device__ __forceinline__ u32x4 join8(unsigned m0, unsigned e0, unsigned m1, unsigned e1) {
    const unsigned h0 = bfi(0x80808080u, m0, e0 >> 1), l0 = bfi(0x7f7f7f7fu, m0, e0 << 7), h1 = bfi(0x80808080u, m1, e1 >> 1), l1 = bfi(0x7f7f7f7fu, m1, e1 << 7);
    return (u32x4){__builtin_amdgcn_perm(h0, l0, 0x05010400u), __builtin_amdgcn_perm(h0, l0, 0x07030602u), __builtin_amdgcn_perm(h1, l1, 0x05010400u), __builtin_amdgcn_perm(h1, l1, 0x07030602u)};
}

struct Item { bool isw; unsigned src, fin, cnt; };
__device__ __forceinline__ Item item_of(int d, int i) {
    const int chunk = (int)((d & 1) ? (unsigned)i / (unsigned)litems(1) : (unsigned)i / (unsigned)litems(0)), j = i - chunk * litems(d);
    Item it;
    it.isw = (d & 1) && j >= OH;
    if (it.isw) { it.src = (unsigned)((chunk * 128 + (j - OH) * 16) * BPITCH + (d >> 1) * 512); it.fin = (unsigned)(OFF_W2 + ((d >> 1) * NCH * 8 + chunk * 8 + j - OH) * 8192);
        it.cnt = (unsigned)(OFF_CNT + ((NSLOT + (d >> 1)) * NCH + chunk) * 64); }
    else { const int strip = (d & 1) ? EH + j : j, slot = (d & 1) ? 3 * EH + (d >> 1) * OH + j : ((d >> 1) - 1) * EH + j;
        it.src = (unsigned)(strip * STRIP_BYTES + chunk * 256); it.fin = (unsigned)(OFF_XA + slot * STRIP_BYTES + chunk * 8192);
        it.cnt = (unsigned)(OFF_CNT + (slot * NCH + chunk) * 64); }
    return it;
}
__device__ __forceinline__ void fetch(const char* src_x, const char* src_w, u32x4 (&nxt)[8], unsigned& tick, int d, int i, unsigned mask, const char* word, unsigned draw) {
    const int lane = threadIdx.x & 63;
    const Item x = item_of(d, i);
    const unsigned v = x.isw ? (lane >> 5) * BPITCH + (lane & 31) * 16 : (lane >> 4) * APITCH + (lane & 15) * 16, st = x.isw ? 2 * BPITCH : 4 * APITCH;
    asm volatile("s_mov_b32 exec_lo, %21\n\ts_mov_b32 exec_hi, %21\n\t"
                 "global_load_dwordx4 %0, %9, %17\n\tglobal_load_dwordx4 %1, %10, %17\n\tglobal_load_dwordx4 %2, %11, %17\n\tglobal_load_dwordx4 %3, %12, %17\n\t"
                 "global_load_dwordx4 %4, %13, %17\n\tglobal_load_dwordx4 %5, %14, %17\n\tglobal_load_dwordx4 %6, %15, %17\n\tglobal_load_dwordx4 %7, %16, %17\n\t"
                 "s_mov_b32 exec_lo, %22\n\ts_mov_b32 exec_hi, 0\n\tglobal_atomic_add %8, %18, %19, %20 sc0\n\ts_mov_b64 exec, -1"
                 : "=&v"(nxt[0]), "=&v"(nxt[1]), "=&v"(nxt[2]), "=&v"(nxt[3]), "=&v"(nxt[4]), "=&v"(nxt[5]), "=&v"(nxt[6]), "=&v"(nxt[7]), "=&v"(tick)
                 : "v"(v), "v"(v + st), "v"(v + 2 * st), "v"(v + 3 * st), "v"(v + 4 * st), "v"(v + 5 * st), "v"(v + 6 * st), "v"(v + 7 * st),
                   "s"((x.isw ? src_w : src_x) + x.src), "v"(0u), "v"(1u), "s"(word), "s"(mask), "s"(draw) : "memory");
}
#define WAIT_FETCH(nxt, tick) asm volatile("s_waitcnt vmcnt(0)" : "+v"(tick), "+v"(nxt[0]), "+v"(nxt[1]), "+v"(nxt[2]), "+v"(nxt[3]), "+v"(nxt[4]), "+v"(nxt[5]), "+v"(nxt[6]), "+v"(nxt[7]) :: "memory")
__device__ __forceinline__ void sender(const Params& P, LAS unsigned char* buf, volatile LAS unsigned* turn, int d, int first, unsigned call, u32x4 (&nxt)[8]) {
    const int lane = threadIdx.x & 63, nlink = NCH * litems(d), nwaves = SEND_WGS * 8;
    char* const queue = P.wsx[0] + OFF_QUEUE + d * 128;
    const char* const src_x = P.x; const char* const src_w = P.w;
    const unsigned lds0 = (unsigned)(size_t)buf;
    unsigned mine = (unsigned)__builtin_amdgcn_readfirstlane(threadIdx.x >> 6), tick;
    int cur = first;
    char* const peer = P.wsx[d & 7];
    char* const set = peer + (call & 1) * PAR_BYTES;
    char* const flags = peer + OFF_FLAG + (call & 1) * FLAG_BYTES;
    const unsigned qbase = (call - 1u) * nlink - 2 * nwaves;
    tick = qbase + first + nwaves;
    while (cur < nlink) {
        const int u = lbase(d) + cur;
        const int next = (int)((unsigned)__builtin_amdgcn_readfirstlane(tick) - qbase);
        u32x4 pk[6]; unsigned ex[16], esum = 0;
#pragma unroll
        for (int g = 0; g < 16; ++g) {
            unsigned m; split4(nxt[g >> 1][(g & 1) * 2], nxt[g >> 1][(g & 1) * 2 + 1], m, ex[g]);
            pk[g >> 2][g & 3] = m;
            if ((g & 3) == 0) esum = __builtin_amdgcn_sad_u8(ex[g], 0u, esum);
        }
        SCHED; fetch(src_x, src_w, nxt, tick, d, next < nlink ? next : nlink - 1, ~0u, queue, 1u); SCHED;
        *(LAS u32x4*)(buf + CODEC_STREAM + lane * 16) = (u32x4){0u, 0u, 0u, 0u};
        unsigned top = (((unsigned)__builtin_amdgcn_readlane(wave_scan(esum), 63) + 1023u + 174u) >> 10) + 2u;
        top = top < 18u ? 18u : top > 255u ? 255u : top;
        unsigned cg[8], xs[8], at[8], tot = 0;
#pragma unroll
        for (int v = 0; v < 8; ++v) {
            const unsigned ua = top * 0x01010101u - ex[2 * v], ub = top * 0x01010101u - ex[2 * v + 1];
            const unsigned ca = __builtin_amdgcn_perm(0u, 0x03020100u, (((ua >> 1) & 0x04040404u) | ua) & 0x07070707u);
            const unsigned cb = __builtin_amdgcn_perm(0u, 0x0c080400u, (((ub >> 1) & 0x04040404u) | ub) & 0x07070707u);
            cg[v] = ((ua | ub) & 0xf0f0f0f0u) != 0u ? 0u : ca | cb;
            const unsigned e = esc_bits(cg[v]);
            const u32x2 sel = lut_of(e, lds0 + CODEC_LUT);
            xs[v] = __builtin_amdgcn_perm(ub, ua, sel.x) | (__builtin_amdgcn_perm(ub, ua, sel.y) << 4);
            at[v] = tot; tot += __builtin_popcount(e);
        }
        const unsigned incl = wave_scan(tot);
        unsigned total = (unsigned)__builtin_amdgcn_readlane(incl, 63);
        if (total > 2048u) {
            total = 0;
#pragma unroll
            for (int v = 0; v < 8; ++v) cg[v] = 0u;
        } else {
            const unsigned n0 = incl - tot + (lds0 + CODEC_STREAM) * 2u;
            asm volatile("" ::: "memory");
#pragma unroll
            for (int v = 0; v < 8; ++v) {
                const unsigned n = n0 + at[v];
                const unsigned long long w = (unsigned long long)xs[v] << ((n * 4u) & 31u);
                LAS unsigned* q = (LAS unsigned*)(size_t)((n >> 1) & ~3u);
                __hip_atomic_fetch_or(q, (unsigned)w, __ATOMIC_RELAXED, __HIP_MEMORY_SCOPE_WAVEFRONT);
                __hip_atomic_fetch_or(q + 1, (unsigned)(w >> 32), __ATOMIC_RELAXED, __HIP_MEMORY_SCOPE_WAVEFRONT);
            }
        }
        asm volatile("" ::: "memory");
        pk[5] = *(const LAS u32x4*)(buf + CODEC_STREAM + lane * 16);
        unsigned least = cg[0];
#pragma unroll
        for (int v = 1; v < 8; ++v) least = cg[v] < least ? cg[v] : least;
#pragma unroll
        for (int c = 0; c < 4; ++c) pk[4][c] = cg[2 * c] | (cg[2 * c + 1] << 4);
        if (cur != first) mine = draw_turn(turn, lane);
        await_turn(turn, mine, SEND_SLOTS);
        if (__builtin_amdgcn_ballot_w64(least == 0u) != 0ull) {
            char* fin = set + item_of(d, cur).fin;
#pragma unroll
            for (int v = 0; v < 8; ++v) if (cg[v] == 0u) store16_sys(fin, v * 1024 + lane * 16, join8(pk[v >> 1][(v & 1) * 2], ex[2 * v], pk[v >> 1][(v & 1) * 2 + 1], ex[2 * v + 1]));
        }
        const unsigned long long smask = __builtin_amdgcn_ballot_w64((unsigned)lane * 32u < total);
        asm volatile("s_mov_b64 exec, %8\n\tglobal_store_dwordx4 %0, %6, %7 offset:3072 sc0 sc1\n\ts_mov_b64 exec, -1\n\t"
                     "global_store_dwordx4 %0, %1, %7 offset:-2048 sc0 sc1\n\tglobal_store_dwordx4 %0, %2, %7 offset:-1024 sc0 sc1\n\t"
                     "global_store_dwordx4 %0, %3, %7 sc0 sc1\n\tglobal_store_dwordx4 %0, %4, %7 offset:1024 sc0 sc1\n\t"
                     "global_store_dwordx4 %0, %5, %7 offset:2048 sc0 sc1\n\ts_nop 1"
                     :: "v"(lane * 16), "v"(pk[0]), "v"(pk[1]), "v"(pk[2]), "v"(pk[3]), "v"(pk[4]), "v"(pk[5]), "s"(set + OFF_PK + (size_t)u * ITEM_PK + 2048), "s"(smask) : "memory");
        WAIT_FETCH(nxt, tick);
        if (lane == 0) {
            end_turn(turn);
            __hip_atomic_store((GAS unsigned*)(flags + (size_t)u * 64), (call << 8) | top, __ATOMIC_RELAXED, __HIP_MEMORY_SCOPE_SYSTEM);
        }
        cur = next;
    }
}

__device__ __forceinline__ void unpacker(const Params& P, LAS unsigned char* buf, int uw, unsigned call) {
    const int lane = threadIdx.x & 63;
    const int d = __builtin_amdgcn_readfirstlane(uw % 7 + 1), i0 = __builtin_amdgcn_readfirstlane(uw / 7);
    const int step = __builtin_amdgcn_readfirstlane(NUW / 7 + (d <= NUW % 7 ? 1 : 0)), nlink = NCH * litems(d);
    char* ws = P.wsx[0];
    char* set = ws + (call & 1) * PAR_BYTES;
    const char* pkb = set + OFF_PK + (size_t)lbase(d) * ITEM_PK + 2048;
    const char* flags = ws + OFF_FLAG + (call & 1) * FLAG_BYTES + (size_t)lbase(d) * 64;
    const unsigned lds0 = (unsigned)(size_t)buf;
    build_lut<false>(buf, lane);
    u32x4 pk[6]; unsigned nf = 0;
#pragma unroll
    for (int k = 0; k < 6; ++k) pk[k] = (u32x4){0u, 0u, 0u, 0u};
#define LOAD_ITEM(i, mask) asm volatile("s_mov_b32 exec_lo, %11\n\ts_mov_b32 exec_hi, %11\n\t" \
        "global_load_dwordx4 %0, %7, %8 offset:-2048 sc0 sc1\n\tglobal_load_dwordx4 %1, %7, %8 offset:-1024 sc0 sc1\n\tglobal_load_dwordx4 %2, %7, %8 sc0 sc1\n\t" \
        "global_load_dwordx4 %3, %7, %8 offset:1024 sc0 sc1\n\tglobal_load_dwordx4 %4, %7, %8 offset:2048 sc0 sc1\n\tglobal_load_dwordx4 %5, %7, %8 offset:3072 sc0 sc1\n\t" \
        "global_load_dword %6, %9, %10 sc0 sc1\n\ts_mov_b64 exec, -1\n\ts_waitcnt vmcnt(0)" \
        : "+v"(pk[0]), "+v"(pk[1]), "+v"(pk[2]), "+v"(pk[3]), "+v"(pk[4]), "+v"(pk[5]), "+v"(nf) \
        : "v"(lane * 16), "s"(pkb + (size_t)(i) * ITEM_PK), "v"(0u), "s"(flags + (size_t)((i) + step < nlink ? (i) + step : (i)) * 64), "s"(mask) : "memory")
    unsigned f = 0, have = 0;
    for (int i = i0; i < nlink; i += step) {
        if (!have) {
            Wait wait;
            bool dead = false;
            while (((f = (unsigned)__builtin_amdgcn_readfirstlane(ld_sys(flags + (size_t)i * 64))) >> 8) < call)
                if (wait.give_up(ws, call)) { dead = true; break; }
            if (dead) break;
            LOAD_ITEM(i, ~0u);
        }
        const unsigned top4 = (f & 255u) * 0x01010101u;
        f = (unsigned)__builtin_amdgcn_readfirstlane(nf);
        have = below(i + step, nlink) & ~below((int)(f >> 8), (int)call);
        *(LAS u32x4*)(buf + CODEC_STREAM + lane * 16) = pk[5];
        unsigned cg[8], at[8], tot = 0; u32x2 sel[8];
#pragma unroll
        for (int v = 0; v < 8; ++v) {
            cg[v] = (pk[4][v >> 1] >> (4 * (v & 1))) & 0x0f0f0f0fu;
            const unsigned e = esc_bits(cg[v]);
            sel[v] = lut_of(e, lds0 + CODEC_LUT);
            at[v] = tot; tot += __builtin_popcount(e);
        }
        const unsigned n0 = wave_scan(tot) - tot + (lds0 + CODEC_STREAM) * 2u;
        asm volatile("" ::: "memory");
        const Item it = item_of(d, i);
        unsigned xs[8];
#pragma unroll
        for (int v = 0; v < 8; ++v) {
            const unsigned n = n0 + at[v];
            const LAS unsigned* q = (const LAS unsigned*)(size_t)((n >> 1) & ~3u);
            xs[v] = __builtin_amdgcn_alignbit(q[1], q[0], n * 4u);
        }
#pragma unroll
        for (int v = 0; v < 8; ++v) {
            const unsigned xe = xs[v] & 0x0f0f0f0fu, xo = (xs[v] >> 4) & 0x0f0f0f0fu;
            const unsigned e0 = top4 - ((cg[v] & 0x03030303u) | __builtin_amdgcn_perm(xo, xe, sel[v].x));
            const unsigned e1 = top4 - (((cg[v] >> 2) & 0x03030303u) | __builtin_amdgcn_perm(xo, xe, sel[v].y));
            const u32x4 r = join8(pk[v >> 1][(v & 1) * 2], e0, pk[v >> 1][(v & 1) * 2 + 1], e1);
            if (cg[v] != 0u) store16_sys(set + it.fin, v * 1024 + lane * 16, r);
        }
        LOAD_ITEM(i + step < nlink ? i + step : i, have);
        if (lane == 0) __hip_atomic_fetch_add((GAS unsigned*)(ws + it.cnt), 1u, __ATOMIC_RELAXED, __HIP_MEMORY_SCOPE_AGENT);
    }
}

__device__ __forceinline__ int strip_of(int pos, bool remote, int q) {
    if (!remote) return pos < 32 + NSLOT ? pos : -1;
    if (pos < EH) return pos;
    if (pos < EH + 3 * OH) { const int k = (pos - EH) / OH; return 32 + 3 * EH + (k + (k >= q ? 1 : 0)) * OH + (pos - EH) % OH; }
    return (unsigned)(pos - NTE * SR) < 3u * EH ? 32 + pos - NTE * SR : -1;
}
__device__ __forceinline__ int out_row_of(int id, int rank) {
    if (id < 32) return renum(rank) * 1024 + id * 32;
    const int slot = id - 32;
    if (slot < 3 * EH) return renum(rank ^ (2 * (slot / EH + 1))) * 1024 + (slot % EH) * 32;
    const int o = slot - 3 * EH;
    return renum(rank ^ (2 * (o / OH) + 1)) * 1024 + (EH + o % OH) * 32;
}

constexpr int OUT_M = 256, OUT_LDS = 4160;
template <bool PACK> __device__ __forceinline__ u32x4 out_lut(int m) {
    unsigned long long sm = 0x0c0c0c0c0c0c0c0cull, sn = sm; int k = 0;
#pragma unroll
    for (int b = 0; b < 8; ++b) if (m >> b & 1) { const int slot = (k >> 1) | ((k & 1) << 2);
        sm ^= PACK ? (unsigned long long)(0x0c ^ b) << (8 * k) : (unsigned long long)(0x0c ^ k) << (8 * b);
        sn ^= PACK ? (unsigned long long)(0x0c ^ b) << (8 * slot) : (unsigned long long)(0x0c ^ slot) << (8 * b); ++k; }
    return (u32x4){(unsigned)sm, (unsigned)(sm >> 32), (unsigned)sn, (unsigned)(sn >> 32)};
}
struct OutItem { int o; unsigned at0, at1; };
__device__ __forceinline__ OutItem out_item(int n, int rank) {
    const bool late = n >= NTE * HPT * 32;
    const int n1 = late ? n - NTE * HPT * 32 : n, per = (late ? NTR - NTE : NTE) * 32, h = late ? n1 / ((NTR - NTE) * 32) : n1 / (NTE * 32), rt = (late ? NTE : 0) + ((n1 - h * per) >> 5);
    const int p0 = rt * SR, w = (n1 >> 2) & 7, q = n1 & 3, f0 = (w >> 2) * SR + 2 * h;
    const int ida = n < NOI ? strip_of(p0 + (f0 >> 1), true, q) : -1, idb = 2 * h + 1 < SR ? strip_of(p0 + ((f0 + 1) >> 1), true, q) : -1, src = rank ^ (2 * q + 1);
    OutItem t;
    t.o = ida < 0 ? -1 : ((q * NTR + rt) * 8 + w) * HPT + h;
    t.at0 = (unsigned)(out_row_of(ida, src) + (f0 & 1) * 16) * OPITCH + q * 512 + (w & 3) * 64;
    t.at1 = idb < 0 ? ~0u : (unsigned)(out_row_of(idb, src) + ((f0 + 1) & 1) * 16) * OPITCH + q * 512 + (w & 3) * 64;
    return t;
}
__device__ __forceinline__ void expander(const Params& P, LAS unsigned char* buf, int xw, unsigned call) {
    const int lane = threadIdx.x & 63;
    char* ws = P.wsx[0];
    const unsigned lds0 = (unsigned)(size_t)buf;
    for (int m = lane; m < 256; m += 64) *(LAS u32x4*)(buf + OUT_LDS + m * 16) = out_lut<false>(m);
    const OutItem mine = out_item(xw + (lane & 7) * NXW, P.rank);
    char* const out = P.out;
    unsigned long long todo = __builtin_amdgcn_ballot_w64(lane < 8 && mine.o >= 0);
    const char* watch = ws + OFF_OFLAG + (mine.o >= 0 ? mine.o : 0) * 64;
    while (todo) {
        unsigned long long fl, ready; bool dead = false;
        { Wait wait;
          while ((ready = __builtin_amdgcn_ballot_w64((unsigned)((fl = __hip_atomic_load((const GAS unsigned long long*)watch, __ATOMIC_RELAXED, __HIP_MEMORY_SCOPE_SYSTEM)) >> 32) >= call) & todo) == 0ull) {
              if (wait.give_up(ws, call)) { dead = true; ready = todo; break; }
          } }
        const int k = __builtin_ctzll(ready);
        todo &= ~(1ull << k);
        const unsigned f = (unsigned)__builtin_amdgcn_readlane((unsigned)fl, k);
        const int o = __builtin_amdgcn_readlane(mine.o, k);
        const unsigned at0 = (unsigned)__builtin_amdgcn_readlane(mine.at0, k), at1 = (unsigned)__builtin_amdgcn_readlane(mine.at1, k);
        u32x4 r[4];
        unsigned tmo = call;
        if (!dead) {
            u32x4 t[4];
            asm volatile("global_load_dwordx4 %0, %5, %6 sc0 sc1\n\tglobal_load_dwordx4 %1, %5, %6 offset:1024 sc0 sc1\n\tglobal_load_dwordx4 %2, %5, %6 offset:2048 sc0 sc1\n\t"
                         "global_load_dwordx4 %3, %5, %6 offset:3072 sc0 sc1\n\tglobal_load_dword %4, %7, %8 sc1\n\ts_waitcnt vmcnt(0)"
                         : "=&v"(t[0]), "=&v"(t[1]), "=&v"(t[2]), "=&v"(t[3]), "=&v"(tmo)
                         : "v"(lane * 16), "s"(ws + OFF_OPK + (size_t)o * OITEM), "v"(0u), "s"(ws + OFF_TMO) : "memory");
            if (!(f & 0x100000u)) {
#pragma unroll
                for (int k = 0; k < 4; ++k) *(LAS u32x4*)(buf + lane * 16 + k * 1024) = t[k];
                asm volatile("" ::: "memory");
                const unsigned bits = *(const LAS unsigned*)(buf + lane * 4), out_n = OUT_M + (((f & 0xfffu) + 15u) & 0xff0u);
                const unsigned tot = __builtin_popcount(bits), base4 = ((f >> 12) & 255u) * 0x01010101u;
                unsigned n = (wave_scan(tot) - tot) & 2047u;
                asm volatile("" ::: "memory");
#pragma unroll
                for (int k = 0; k < 4; ++k) {
                    const unsigned m = (bits >> (8 * k)) & 255u;
                    const u32x4 sel = *(const LAS u32x4*)(buf + OUT_LDS + m * 16);
                    const LAS unsigned* qm = (const LAS unsigned*)(size_t)(lds0 + OUT_M + (n & ~3u));
                    const LAS unsigned* qn = (const LAS unsigned*)(size_t)(lds0 + out_n + ((n >> 1) & ~3u));
                    const unsigned m0 = __builtin_amdgcn_alignbyte(qm[1], qm[0], n), m1 = __builtin_amdgcn_alignbyte(qm[2], qm[1], n), x = __builtin_amdgcn_alignbit(qn[1], qn[0], n * 4u);
                    const unsigned xe = x & 0x0f0f0f0fu, xo = (x >> 4) & 0x0f0f0f0fu;
                    const unsigned e0 = (base4 - __builtin_amdgcn_perm(xo, xe, sel.z)) & __builtin_amdgcn_perm(~0u, ~0u, sel.x);
                    const unsigned e1 = (base4 - __builtin_amdgcn_perm(xo, xe, sel.w)) & __builtin_amdgcn_perm(~0u, ~0u, sel.y);
                    t[k] = join8(__builtin_amdgcn_perm(m1, m0, sel.x), e0, __builtin_amdgcn_perm(m1, m0, sel.y), e1);
                    n += __builtin_popcount(m);
                }
                asm volatile("" ::: "memory");
            }
#pragma unroll
            for (int k = 0; k < 8; ++k) {
                const int row = (k >> 2) * 16 + (lane & 15), ch = (k & 3) * 2 + (lane >> 5);
                *(LAS u32x2*)(buf + row * 128 + ((ch ^ (row & 7)) * 16) + ((lane >> 4) & 1) * 8) = (u32x2){t[k >> 1][(k & 1) * 2], t[k >> 1][(k & 1) * 2 + 1]};
            }
            asm volatile("" ::: "memory");
#pragma unroll
            for (int v = 0; v < 4; ++v) { const int row = v * 8 + (lane >> 3); r[v] = *(const LAS u32x4*)(buf + row * 128 + (((lane & 7) ^ (row & 7)) * 16)); }
        }
        if (tmo == call) {
#pragma unroll
            for (int v = 0; v < 4; ++v) r[v] = (u32x4){0x7fc07fc0u, 0x7fc07fc0u, 0x7fc07fc0u, 0x7fc07fc0u};
        }
        const unsigned ooff = (lane >> 3) * OPITCH + ((lane >> 2) & 1) * 256 + (lane & 3) * 16;
        store16_sys(out, ooff + at0, r[0]); store16_sys(out, ooff + at0 + 8 * OPITCH, r[1]);
        if (at1 != ~0u) { store16_sys(out, ooff + at1, r[2]); store16_sys(out, ooff + at1 + 8 * OPITCH, r[3]); }
    }
}

__device__ __forceinline__ unsigned cvt_pk_bf16(float lo, float hi) { unsigned r; asm("v_cvt_pk_bf16_f32 %0, %1, %2" : "=v"(r) : "v"(lo), "v"(hi)); return r; }
template <int OFF> __device__ __forceinline__ s16x4 tr_read(unsigned addr) { s16x4 r; asm volatile("ds_read_b64_tr_b16 %0, %1 offset:%2" : "=v"(r) : "v"(addr), "i"(OFF) : "memory"); return r; }
template <int OFF> __device__ __forceinline__ bf16x8 row_read(unsigned addr) { bf16x8 r; asm volatile("ds_read_b128 %0, %1 offset:%2" : "=v"(r) : "v"(addr), "i"(OFF) : "memory"); return r; }
template <int OFF> __device__ __forceinline__ void glds(unsigned ldsaddr, unsigned voff, const char* g) {
    asm volatile("s_mov_b32 m0, %0\n\ts_nop 0\n\tglobal_load_lds_dwordx4 %1, %2 offset:%3 sc0 sc1" :: "s"(ldsaddr - OFF), "v"(voff), "s"(g), "i"(OFF) : "memory");
}
template <int OFF> __device__ __forceinline__ void glds_masked(unsigned ldsaddr, unsigned voff, const char* g, unsigned mask) {
    asm volatile("s_mov_b32 m0, %0\n\ts_mov_b32 exec_lo, %3\n\ts_mov_b32 exec_hi, %3\n\tglobal_load_lds_dwordx4 %1, %2 offset:%4 sc0 sc1\n\ts_mov_b64 exec, -1"
                 :: "s"(ldsaddr - OFF), "v"(voff), "s"(g), "s"(mask), "i"(OFF) : "memory");
}

template <int S> struct Frag { bf16x8 a[S], b[4]; };
template <int S> __device__ __forceinline__ void wait_step(Frag<S>& F, unsigned& pre) {
    asm volatile("s_waitcnt vmcnt(6) lgkmcnt(0)" : "+v"(F.a[0]), "+v"(F.a[1]), "+v"(F.a[2]), "+v"(F.a[3]), "+v"(F.a[S - 1]), "+v"(F.b[0]), "+v"(F.b[1]), "+v"(F.b[2]), "+v"(F.b[3]), "+v"(pre) :: "memory");
    if constexpr (S > 5) asm volatile("" : "+v"(F.a[4]));
    if constexpr (S > 6) asm volatile("" : "+v"(F.a[5]));
    if constexpr (S > 7) asm volatile("" : "+v"(F.a[6]));
}
template <int S, bool remote> __device__ __forceinline__ void gemm_tile(const Params& P, LAS unsigned char* lds, int rt, int ct, unsigned call) {
    constexpr int SLOT_A = S * 2048, SLOT = SLOT_A + 16384;
    static_assert(S >= 5 && S <= 8 && 4 * SLOT <= LDS_STAGE && S * 16384 <= LDS_STAGE, "tile");
    int wid = __builtin_amdgcn_readfirstlane(threadIdx.x >> 6), lane = threadIdx.x & 63, wr = wid >> 2, wc = wid & 3, fr = lane & 15, fq = lane >> 4;
    const int p0 = rt * S;
    char* ws = P.wsx[0];
    const char* set = ws + (call & 1) * PAR_BYTES;
    const char* const own_x = P.x;
    const unsigned ldst = (unsigned)(size_t)lds + wid * 1024;
    constexpr int BP = remote ? 512 : BPITCH;
    auto strip_at = [&](int k) { const int id = k < S ? strip_of(p0 + k, remote, ct) : -1; return id < 0 ? strip_of(p0, remote, ct) : id; };
    const int id0 = strip_at(wid >> 1), id1 = strip_at(4 + (wid >> 1));
    const char* const ga0 = id0 < 32 ? own_x + id0 * STRIP_BYTES : set + OFF_XA + (id0 - 32) * STRIP_BYTES;
    const char* const ga1 = id1 < 32 ? own_x + id1 * STRIP_BYTES : set + OFF_XA + (id1 - 32) * STRIP_BYTES;
    const unsigned cs0 = id0 < 32 ? 256 : 8192, cs1 = id1 < 32 ? 256 : 8192;
    const char* const gb = remote ? set + OFF_W2 + ct * (W2_BYTES / 4) : P.w + ct * 512;
    unsigned voffA0, voffA1, voffB;
    { const int k0 = lane >> 4, ch = lane & 15; voffB = (unsigned)((wid * 4 + k0) * BP + (((ch >> 1) ^ (k0 | ((wid & 2) << 1))) * 32) + (ch & 1) * 16);
      const int sb = lane * 16, swz = sb ^ (((sb >> 9) & 1) << 5), row = (wid & 1) * 16 + (swz >> 6);
      voffA0 = (unsigned)(row * (id0 < 32 ? APITCH : 256) + (swz & 63)); voffA1 = (unsigned)(row * (id1 < 32 ? APITCH : 256) + (swz & 63)); }
    const unsigned amask = below(wid, 2 * S - 8);
    unsigned aoff[2], boff[2][2];
    { const unsigned lbase = (unsigned)(size_t)lds; const int ob = fr * 64 + fq * 16; aoff[0] = lbase + wr * S * 1024 + (ob ^ (((ob >> 9) & 1) << 5)); aoff[1] = aoff[0] + 2 * SLOT;
      const int q = (lane & 15) >> 2, p = lane & 3, f = q | ((fq & 1) << 2);
      for (int n = 0; n < 2; ++n) { boff[0][n] = lbase + SLOT_A + (8 * fq + q) * 256 + p * 8 + (((wc * 2 + n) ^ f) * 32); boff[1][n] = boff[0][n] + 2 * SLOT; } }
    unsigned coff, cstep, need;
    { const int id = strip_of(p0 + (lane < S ? lane : 0), remote, ct);
      const bool watch = wid == 0 && (lane < S ? id >= 32 : (lane == 8 && remote));
      coff = watch ? (unsigned)(OFF_CNT + (lane < S ? id - 32 : NSLOT + ct) * NCH * 64) : (unsigned)OFF_TMO;
      cstep = watch ? 64u : 0u;
      need = watch ? call * (lane < S ? 1u : 8u) : 0u; }
    unsigned orow = 0;
    if constexpr (!remote) { const int f = wr * S + (lane & 7), id = strip_of(p0 + (f >> 1), false, 0); orow = (unsigned)(out_row_of(id, P.rank) + (f & 1) * 16) * OPITCH + ct * 512; }
    char* const out = remote ? P.wsx[(2 * ct + 1) & 7] : P.out;
    const int nfrag = __builtin_popcountll(__builtin_amdgcn_ballot_w64(lane < S && strip_of(p0 + ((wr * S + lane) >> 1), remote, ct) >= 0));
    const unsigned w0 = below(wid, 1);
    unsigned pre = 0;
#define SLOW_WAIT() do { Wait wait; \
        while (__builtin_amdgcn_ballot_w64(ld_agent(ws + coff) < need) != 0ull) if (wait.give_up(ws, call)) break; \
        asm volatile("" ::: "memory"); } while (0)
#define PRE_LOAD(mask) do { coff += cstep; \
        asm volatile("s_mov_b32 exec_lo, %3\n\ts_mov_b32 exec_hi, 0\n\tglobal_load_dword %0, %1, %2 sc1\n\ts_mov_b64 exec, -1" : "+v"(pre) : "v"(coff), "s"(ws), "s"((mask) & w0) : "memory"); } while (0)
#define STAGE_B(h, k, ch) glds<(h) * 256>(ldst + (k) * SLOT + SLOT_A + (h) * 8192, voffB, gb + (size_t)((ch) * 4 + (k)) * (32 * BP))
#define STAGE_A0(k, ch) glds<(k) * 64>(ldst + (k) * SLOT, voffA0, ga0 + (ch) * cs0)
#define STAGE_A1(k, ch) glds_masked<(k) * 64>(ldst + (k) * SLOT + 8192, voffA1, ga1 + (ch) * cs1, amask)
#define STAGE_ALL(k, ch) do { STAGE_B(0, k, ch); STAGE_B(1, k, ch); STAGE_A0(k, ch); STAGE_A1(k, ch); } while (0)
#define RDA(F, m, sl) F.a[m] = row_read<((sl) & 1) * SLOT + (m) * 1024>(aoff[(sl) >> 1])
#define RDB(F, j, sl) do { const s16x4 _lo = tr_read<((sl) & 1) * SLOT + ((j) >> 1) * 8192>(boff[(sl) >> 1][(j) & 1]), _hi = tr_read<((sl) & 1) * SLOT + ((j) >> 1) * 8192 + 1024>(boff[(sl) >> 1][(j) & 1]); \
        F.b[j] = __builtin_shufflevector(_lo, _hi, 0, 1, 2, 3, 4, 5, 6, 7); } while (0)
#define MM(F, i) acc[(i) >> 2][(i) & 3] = __builtin_amdgcn_mfma_f32_16x16x32_bf16(F.b[(i) & 3], F.a[(i) >> 2], acc[(i) >> 2][(i) & 3], 0, 0, 0)
#define M1(C, i) do { MM(C, i); SCHED; } while (0)
#define STEP(C, N, k) do { constexpr int nx = ((k) + 1) & 3; \
        if constexpr ((k) == 2) PRE_LOAD(c < NCH - 2 ? 0x3ffu : c < NCH - 1 ? 0x200u : 0u); \
        M1(C, 0); STAGE_B(0, k, c1); M1(C, 1); RDB(N, 0, nx); M1(C, 2); STAGE_B(1, k, c1); M1(C, 3); RDB(N, 1, nx); \
        M1(C, 4); STAGE_A0(k, c1); M1(C, 5); RDB(N, 2, nx); M1(C, 6); STAGE_A1(k, c1); M1(C, 7); RDB(N, 3, nx); \
        M1(C, 8); RDA(N, 0, nx); M1(C, 9); RDA(N, 1, nx); M1(C, 10); RDA(N, 2, nx); M1(C, 11); RDA(N, 3, nx); M1(C, 12); RDA(N, 4, nx); \
        M1(C, 13); if constexpr (S > 5) RDA(N, 5, nx); M1(C, 14); if constexpr (S > 6) RDA(N, 6, nx); if constexpr (S > 7) RDA(N, 7, nx); \
        _Pragma("unroll") for (int i = 15; i < 4 * S - 4; ++i) M1(C, i); \
        wait_step(N, pre); \
        if constexpr ((k) == 3) { if (c < NCH - 2 && __builtin_amdgcn_ballot_w64(pre < need) != 0ull) SLOW_WAIT(); } \
        BAR; SCHED; M1(C, 4 * S - 4); M1(C, 4 * S - 3); M1(C, 4 * S - 2); M1(C, 4 * S - 1); } while (0)
    f32x4 acc[S][4];
#pragma unroll
    for (int m = 0; m < S; ++m)
#pragma unroll
        for (int j = 0; j < 4; ++j) acc[m][j] = (f32x4){0.f, 0.f, 0.f, 0.f};
    Frag<S> F0, F1;
    constexpr int OUT_LUT = LDS_STAGE - 4096;
    static_assert(!remote || (4 * SLOT <= OUT_LUT && 8 * OUT_LDS <= OUT_LUT), "pack tables");
    if (remote && threadIdx.x >= 256) *(LAS u32x4*)(lds + OUT_LUT + (threadIdx.x - 256) * 16) = out_lut<true>(threadIdx.x - 256);
    if (wid == 0) SLOW_WAIT();
    BAR;
    PRE_LOAD(0x3ffu);
    STAGE_ALL(0, 0u); STAGE_ALL(1, 0u); STAGE_ALL(2, 0u); STAGE_ALL(3, 0u);
    WAIT_V(6); BAR;
    RDB(F0, 0, 0); RDB(F0, 1, 0); RDB(F0, 2, 0); RDB(F0, 3, 0); RDA(F0, 0, 0); RDA(F0, 1, 0); RDA(F0, 2, 0); RDA(F0, 3, 0); RDA(F0, 4, 0);
    if constexpr (S > 5) RDA(F0, 5, 0);
    if constexpr (S > 6) RDA(F0, 6, 0);
    if constexpr (S > 7) RDA(F0, 7, 0);
    wait_step(F0, pre);
    if (__builtin_amdgcn_ballot_w64(pre < need) != 0ull) SLOW_WAIT();
    BAR;
    for (int c = 0; c < NCH; ++c) {
        const unsigned c1 = c < NCH - 1 ? c + 1 : c;
        STEP(F0, F1, 0); STEP(F1, F0, 1); STEP(F0, F1, 2); STEP(F1, F0, 3);
    }
    volatile LAS unsigned* turn = (volatile LAS unsigned*)(lds + LDS_STAGE + 4);
    if (threadIdx.x == 9) { *(LAS unsigned*)(lds + LDS_STAGE) = pre; turn[0] = 0u; turn[1] = 0u; }
    asm volatile("s_waitcnt vmcnt(0) lgkmcnt(0)" ::: "memory");
    BAR;
    asm volatile("" ::: "memory");
    const bool bad = *(const LAS unsigned*)(lds + LDS_STAGE) == call;
    int tz = threadIdx.x; asm volatile("" : "+v"(tz));
    lane = tz & 63; fr = lane & 15; fq = lane >> 4;
    if (__builtin_amdgcn_readfirstlane(bad)) {
#pragma unroll
        for (int m = 0; m < S; ++m)
#pragma unroll
            for (int j = 0; j < 4; ++j) acc[m][j] = (f32x4){__builtin_nanf(""), __builtin_nanf(""), __builtin_nanf(""), __builtin_nanf("")};
    }
    auto relu_pack = [&](int m, int j) {
        const f32x4 v = __builtin_elementwise_maximum(acc[m][j], (f32x4){0.f, 0.f, 0.f, 0.f});
        return (u32x2){cvt_pk_bf16(v[0], v[1]), cvt_pk_bf16(v[2], v[3])};
    };
    if constexpr (!remote) {
        LAS unsigned char* buf = lds + wid * (S * 2048);
#pragma unroll
        for (int m = 0; m < S; ++m)
#pragma unroll
            for (int j = 0; j < 4; ++j) *(LAS u32x2*)(buf + m * 2048 + fr * 128 + (((j * 2 + (fq >> 1)) ^ (fr & 7)) * 16) + (fq & 1) * 8) = relu_pack(m, j);
        asm volatile("" ::: "memory");
        u32x4 ov[S][2];
#pragma unroll
        for (int m = 0; m < S; ++m)
#pragma unroll
            for (int i = 0; i < 2; ++i) ov[m][i] = *(const LAS u32x4*)(buf + m * 2048 + i * 1024 + (lane >> 3) * 128 + (((lane & 7) ^ (lane >> 3)) * 16));
        const unsigned ooff = (lane >> 3) * OPITCH + ((lane >> 2) & 1) * 256 + wc * 64 + (lane & 3) * 16;
#pragma unroll
        for (int m = 0; m < S; ++m) {
            const unsigned at = (unsigned)__builtin_amdgcn_readlane(orow, m);
            if (m < nfrag) { store16_sys(out, ooff + at, ov[m][0]); store16_sys(out, ooff + at + 8 * OPITCH, ov[m][1]); }
        }
    } else {
        char* const peer = out;
        LAS unsigned char* buf = lds + wid * OUT_LDS;
        const unsigned lds0 = (unsigned)(size_t)buf;
        const int o0 = ((ct * NTR + rt) * 8 + wid) * HPT;
        int sent = -1; unsigned sent_meta = 0;
#define PUBLISH_OUT() do { WAIT_V(0); if (sent >= 0) { if (lane == 0) { end_turn(turn); \
            __hip_atomic_store((GAS unsigned long long*)(peer + OFF_OFLAG + sent * 64), ((unsigned long long)call << 32) | sent_meta, __ATOMIC_RELAXED, __HIP_MEMORY_SCOPE_SYSTEM); } \
            sent = -1; } } while (0)
#pragma unroll
        for (int h = 0; h < HPT; ++h) {
            if (2 * h >= nfrag) break;
            const bool second = 2 * h + 1 < nfrag;
            u32x4 t[4]; u16x2 mx = {0, 0};
#pragma unroll
            for (int k = 0; k < 8; ++k) {
                u32x2 w = relu_pack(2 * h + (k >> 2) < S ? 2 * h + (k >> 2) : 0, k & 3);
                if (k >= 4 && !second) w = (u32x2){0u, 0u};
                t[k >> 1][(k & 1) * 2] = w.x; t[k >> 1][(k & 1) * 2 + 1] = w.y;
                mx = __builtin_elementwise_max(mx, __builtin_elementwise_max(__builtin_bit_cast(u16x2, w.x & 0x7fff7fffu), __builtin_bit_cast(u16x2, w.y & 0x7fff7fffu)));
            }
#pragma unroll
            for (int k = 0; k < 3; ++k) *(LAS u32x4*)(buf + OUT_M + lane * 16 + k * 1024) = (u32x4){0u, 0u, 0u, 0u};
            const unsigned emax = (unsigned)__builtin_amdgcn_readlane(wave_max(mx.x > mx.y ? mx.x : mx.y), 63) >> 7;
            unsigned bits = 0, wide = 0, tot = 0, at[4], cm0[4], cm1[4], cn[4];
#pragma unroll
            for (int k = 0; k < 4; ++k) {
                unsigned z[4], m0, e0, m1, e1;
#pragma unroll
                for (int c = 0; c < 4; ++c) z[c] = nonzero2(t[k][c]);
                const unsigned m = __builtin_amdgcn_udot4(__builtin_amdgcn_perm(z[3], z[2], 0x06040200u), 0x80402010u, __builtin_amdgcn_udot4(__builtin_amdgcn_perm(z[1], z[0], 0x06040200u), 0x08040201u, 0u, false), false);
                const u32x4 sel = *(const LAS u32x4*)(lds + OUT_LUT + m * 16);
                split4(t[k].x, t[k].y, m0, e0); split4(t[k].z, t[k].w, m1, e1);
                e0 = emax * 0x01010101u - e0; e1 = emax * 0x01010101u - e1;
                cm0[k] = __builtin_amdgcn_perm(m1, m0, sel.x); cm1[k] = __builtin_amdgcn_perm(m1, m0, sel.y);
                const unsigned de = __builtin_amdgcn_perm(e1, e0, sel.z), dd = __builtin_amdgcn_perm(e1, e0, sel.w);
                wide |= de | dd; cn[k] = de | (dd << 4);
                bits |= m << (8 * k); at[k] = tot; tot += __builtin_popcount(m);
            }
            const unsigned incl = wave_scan(tot), count = (unsigned)__builtin_amdgcn_readlane(incl, 63);
            const bool rawout = __builtin_amdgcn_ballot_w64((wide & 0xf0f0f0f0u) != 0u) != 0ull;
            char* dst = peer + OFF_OPK + (size_t)(o0 + h) * OITEM;
            const unsigned out_n = OUT_M + ((count + 15u) & ~15u), total = rawout ? 4096u : out_n + ((count + 1u) >> 1);
            if (!rawout) {
                *(LAS unsigned*)(buf + lane * 4) = bits;
                asm volatile("" ::: "memory");
#pragma unroll
                for (int k = 0; k < 4; ++k) {
                    const unsigned n = incl - tot + at[k];
                    const unsigned long long a = (unsigned long long)cm0[k] << ((n & 3u) * 8u), b = (unsigned long long)cm1[k] << ((n & 3u) * 8u), x = (unsigned long long)cn[k] << ((n * 4u) & 31u);
                    LAS unsigned* qm = (LAS unsigned*)(size_t)(lds0 + OUT_M + (n & ~3u));
                    LAS unsigned* qn = (LAS unsigned*)(size_t)(lds0 + out_n + ((n >> 1) & ~3u));
#define LDS_OR(p, v) __hip_atomic_fetch_or(p, v, __ATOMIC_RELAXED, __HIP_MEMORY_SCOPE_WAVEFRONT)
                    LDS_OR(qm, (unsigned)a); LDS_OR(qm + 1, (unsigned)(a >> 32) | (unsigned)b); LDS_OR(qm + 2, (unsigned)(b >> 32));
                    LDS_OR(qn, (unsigned)x); LDS_OR(qn + 1, (unsigned)(x >> 32));
                }
                asm volatile("" ::: "memory");
#pragma unroll
                for (int k = 0; k < 4; ++k) t[k] = *(const LAS u32x4*)(buf + lane * 16 + k * 1024);
                asm volatile("" : "+v"(t[0]), "+v"(t[1]), "+v"(t[2]), "+v"(t[3]) :: "memory");
            }
            PUBLISH_OUT();
            await_turn(turn, draw_turn(turn, lane), OUT_SLOTS);
#pragma unroll
            for (int k = 0; k < 4; ++k) if (lane * 16u + k * 1024u < total) store16_sys(dst, lane * 16 + k * 1024, t[k]);
            sent = o0 + h; sent_meta = (rawout ? 0x100000u : 0u) | (emax << 12) | count;
        }
        PUBLISH_OUT();
    }
}

__global__ __launch_bounds__(NTHR, 2) void ag_gemm_kernel(const Params P) {
    extern __shared__ __attribute__((aligned(16))) unsigned char shm[];
    LAS unsigned char* const lds = (LAS unsigned char*)shm;
    const int wid = __builtin_amdgcn_readfirstlane(threadIdx.x >> 6);
    char* const tick = P.wsx[0] + OFF_TICK;
    const char* const src_x = P.x; const char* const src_w = P.w;
    volatile LAS unsigned* const word = (volatile LAS unsigned*)(lds + LDS_STAGE);
    const int asked = (int)(__builtin_amdgcn_s_getreg((3 << 11) | 20) & 7u) * 32 + (int)(blockIdx.x >> 3);
    auto sends = [](int r) { return (r >> 5) != 3 && (r & 31) < SEND_WGS; };
    u32x4 nxt[8]; unsigned t = 0, c;
    asm volatile("s_mov_b32 exec_lo, %4\n\ts_mov_b32 exec_hi, 0\n\tglobal_atomic_add %0, %1, %2, %3 sc0\n\ts_mov_b64 exec, -1" : "+v"(t) : "v"(blockIdx.x * 64u), "v"(1u), "s"(tick), "s"(below(wid, 1) & 1u) : "memory");
    fetch(src_x, src_w, nxt, c, link_of(asked >> 5), wid * SEND_WGS + (asked & 31), below(asked & 31, SEND_WGS) & ~(below(asked >> 5, 4) & below(2, asked >> 5)), tick + (256 + asked) * 64, below(wid, 1) & 1u);
    if (sends(asked)) build_lut<true>(lds + wid * WAVE_LDS, threadIdx.x & 63);
    asm volatile("" : "+v"(t));
    WAIT_FETCH(nxt, c);
    asm volatile("" : "+v"(t));
    if (threadIdx.x == 0) {
        int r = asked;
        if (c != t) {
            __hip_atomic_fetch_add((GAS unsigned*)(tick + (256 + r) * 64), ~0u, __ATOMIC_RELAXED, __HIP_MEMORY_SCOPE_AGENT);
            for (int k = 1; k < 256 * 64; ++k) {
                unsigned expect = t; r = (asked + k) & 255;
                if (__hip_atomic_compare_exchange_strong((GAS unsigned*)(tick + (256 + r) * 64), &expect, t + 1u, __ATOMIC_RELAXED, __ATOMIC_RELAXED, __HIP_MEMORY_SCOPE_AGENT)) break;
            }
        }
        word[0] = t + 1u; word[1] = (unsigned)r; word[2] = 8u; word[3] = 0u;
    }
    WAIT_L(0); BAR; asm volatile("" ::: "memory");
    struct { unsigned call; int role; } claim = {(unsigned)__builtin_amdgcn_readfirstlane(word[0]), __builtin_amdgcn_readfirstlane((int)word[1]) & 255};
    const int xcd = claim.role >> 5, slot = claim.role & 31;
#define PICK(f) __builtin_amdgcn_readfirstlane(xcd == 0 ? BeforeAt<0>::f : xcd == 1 ? BeforeAt<1>::f : xcd == 2 ? BeforeAt<2>::f : xcd == 3 ? BeforeAt<3>::f : \
                                               xcd == 4 ? BeforeAt<4>::f : xcd == 5 ? BeforeAt<5>::f : xcd == 6 ? BeforeAt<6>::f : BeforeAt<7>::f)
    const int ns = nsend(xcd), nc = ncomm(xcd), ub = PICK(unp), sb = PICK(snd);
    const int xw = __builtin_amdgcn_readfirstlane(slot < ns ? NUW + sb + wid * ns + slot : ub + (slot - ns) * 8 + wid);
    if (slot < ns) {
        if (claim.role != asked) { build_lut<true>(lds + wid * WAVE_LDS, threadIdx.x & 63); fetch(src_x, src_w, nxt, c, link_of(xcd), wid * SEND_WGS + slot, ~0u, tick, 0u); WAIT_FETCH(nxt, c); }
        sender(P, lds + wid * WAVE_LDS, word + 2, link_of(xcd), wid * SEND_WGS + slot, claim.call, nxt);
    }
    if (slot < nc) {
        if (slot >= ns) unpacker(P, lds + wid * WAVE_LDS, xw, claim.call);
        expander(P, lds + wid * WAVE_LDS, xw, claim.call);
    } else {
        const int g = slot - nc, rt = row_tile(xcd, g >> 2);
        if (xcd < 4) gemm_tile<SL, false>(P, lds, rt, g & 3, claim.call);
        else gemm_tile<SR, true>(P, lds, rt, g & 3, claim.call);
    }
}
}

extern "C" void kernel_launch(int rank, int nranks, ncclComm_t comm, hipStream_t stream,
                              void* const* d_in, const int* in_sizes, int n_in,
                              void* d_out, int out_size, const int* in_dtypes, int out_dtype,
                              void* const* d_out_all, void* const* d_ws_all, size_t ws_bytes) {
    (void)comm; (void)in_dtypes; (void)out_dtype; (void)d_out_all;
    if (nranks != NR || (unsigned)rank >= (unsigned)NR || n_in != 2 || in_sizes[0] != 1024 * KDIM || in_sizes[1] != KDIM * 1024 || out_size != KDIM * 1024 || ws_bytes < WS_NEED) {
        fprintf(stderr, "kernel_launch: unexpected problem geometry\n");
        return;
    }
    static bool attr_set = false;
    if (!attr_set) { HIP_CHECK(hipFuncSetAttribute((const void*)ag_gemm_kernel, hipFuncAttributeMaxDynamicSharedMemorySize, LDS_BYTES)); attr_set = true; }
    Params P;
    P.x = (const char*)d_in[0]; P.w = (const char*)d_in[1]; P.out = (char*)d_out;
    for (int d = 0; d < NR; ++d) P.wsx[d] = (char*)d_ws_all[renum(renum(rank) ^ d)];
    P.rank = renum(rank); P.pad = 0;
    hipLaunchKernelGGL(ag_gemm_kernel, dim3(GRID), dim3(NTHR), LDS_BYTES, stream, P);
    HIP_CHECK(hipGetLastError());
}
```

```cpp
#include <hip/hip_runtime.h>
#include <rccl/rccl.h>
#include <cstdio>

#define HIP_CHECK(expr) do { hipError_t _e = (expr); if (_e != hipSuccess) { \
    fprintf(stderr, "HIP error %s at %s:%d\n", hipGetErrorString(_e), __FILE__, __LINE__); \
    return; } } while (0)

namespace {
#define LAS __attribute__((address_space(3)))
#define GAS __attribute__((address_space(1)))
typedef short bf16x8 __attribute__((ext_vector_type(8)));
typedef short s16x4 __attribute__((ext_vector_type(4)));
typedef float f32x4 __attribute__((ext_vector_type(4)));
typedef unsigned u32x4 __attribute__((ext_vector_type(4)));
typedef unsigned u32x2 __attribute__((ext_vector_type(2)));
typedef unsigned short u16x2 __attribute__((ext_vector_type(2)));

constexpr int NR = 8;
constexpr int KDIM = 8192;
constexpr int APITCH = KDIM * 2, BPITCH = 2048, OPITCH = 2048;
constexpr int E_STRIPS = 21;
constexpr int EH = E_STRIPS, OH = 32 - EH;
constexpr int NSLOT = 3 * EH + 4 * OH;
constexpr int SL = 7;
constexpr int SR = 5;
constexpr int NTL = (32 + NSLOT + SL - 1) / SL, NTE = (EH + 3 * OH + SR - 1) / SR, NTR = NTE + (3 * EH + SR - 1) / SR, HPT = (SR + 1) / 2;
constexpr int NCH = 64;
__host__ __device__ constexpr int litems(int d) { return (d & 1) ? OH + 8 : EH; }
__host__ __device__ constexpr int lbase(int d) { return NCH * ((d >> 1) * litems(1) + ((d - 1) >> 1) * litems(0)); }
constexpr int NU = lbase(8);
constexpr int ITEM_PK = 6144;
constexpr size_t STRIP_BYTES = (size_t)32 * APITCH;
constexpr size_t XA_BYTES = NSLOT * STRIP_BYTES, W2_BYTES = (size_t)KDIM * BPITCH, PK_BYTES = (size_t)NU * ITEM_PK;
constexpr size_t OFF_XA = 0, OFF_W2 = XA_BYTES, OFF_PK = XA_BYTES + W2_BYTES, PAR_BYTES = OFF_PK + PK_BYTES;
constexpr size_t OFF_FLAG = 2 * PAR_BYTES, FLAG_BYTES = (size_t)NU * 64;
constexpr size_t OFF_CNT = OFF_FLAG + 2 * FLAG_BYTES, CNT_BYTES = (size_t)(NSLOT + 4) * NCH * 64;
constexpr size_t OFF_TICK = OFF_CNT + CNT_BYTES, TICK_BYTES = 512 * 64;
constexpr int OITEM = 4096, NOI = 4 * NTR * 8 * HPT;
constexpr size_t OFF_OPK = OFF_TICK + TICK_BYTES, OFF_OFLAG = OFF_OPK + (size_t)NOI * OITEM;
constexpr size_t OFF_TMO = OFF_OFLAG + (size_t)NOI * 64, OFF_QUEUE = OFF_TMO + 128;
constexpr size_t WS_NEED = OFF_QUEUE + 8 * 128;
static_assert(WS_NEED <= ((size_t)257 << 20), "workspace");
constexpr int GRID = 256, NTHR = 512;
constexpr int SEND_WGS = 6;
constexpr int OUT_SLOTS = 6;
constexpr int SEND_SLOTS = 3;
constexpr int TURN_SLEEP = 8;
constexpr int POLL_SLEEP = 2;
__host__ __device__ constexpr int ncomm(int x) { return x < 4 ? (x == 1 ? 16 : x == 2 ? 8 : 12) : 32 - 4 * ((NTR + x % 4) / 4); }
__host__ __device__ constexpr int row_tile(int x, int k) { return x >= 4 ? k * 4 + 7 - x : (x == 1 ? 0 : x == 0 ? 4 : x == 2 ? 9 : 15) + k; }
static_assert(NTL == 20, "row_tile");
__host__ __device__ constexpr int nsend(int x) { return x == 3 ? 0 : SEND_WGS; }
__host__ __device__ constexpr int link_of(int x) { return x < 3 ? 2 * x + 2 : x < 4 ? 1 : 2 * x - 7; }
struct Before { int unp[9], snd[9]; };
constexpr Before before() { Before b = {}; for (int x = 0; x < 8; ++x) { b.unp[x + 1] = b.unp[x] + (ncomm(x) - nsend(x)) * 8; b.snd[x + 1] = b.snd[x] + nsend(x) * 8; } return b; }
template <int X> struct BeforeAt { static constexpr int unp = before().unp[X], snd = before().snd[X]; };
constexpr int NUW = BeforeAt<8>::unp, NXW = NUW + BeforeAt<8>::snd;
constexpr unsigned long long DEADLINE = 50000000ull;
constexpr int LDS_STAGE = 135168, LDS_BYTES = LDS_STAGE + 16;
static_assert(NOI <= 8 * NXW && ncomm(2) >= nsend(2) && ncomm(3) >= 0 && ncomm(7) >= nsend(7), "roles");

__host__ __device__ constexpr int renum(int r) { return r == 3 ? 4 : r == 4 ? 3 : r; }
struct Params {
    const char* x; const char* w;
    char* out;
    char* wsx[NR];
    int rank; int pad;
};

#define WAIT_V(n) asm volatile("s_waitcnt vmcnt(" #n ")" ::: "memory")
#define WAIT_L(n) asm volatile("s_waitcnt lgkmcnt(" #n ")" ::: "memory")
#define BAR __builtin_amdgcn_s_barrier()
#define SCHED __builtin_amdgcn_sched_barrier(0)

__device__ __forceinline__ unsigned ld_agent(const char* p) { return __hip_atomic_load((const GAS unsigned*)p, __ATOMIC_RELAXED, __HIP_MEMORY_SCOPE_AGENT); }
__device__ __forceinline__ unsigned ld_sys(const char* p) { return __hip_atomic_load((const GAS unsigned*)p, __ATOMIC_RELAXED, __HIP_MEMORY_SCOPE_SYSTEM); }
struct Wait {
    unsigned long long t0; unsigned polls;
    __device__ __forceinline__ Wait() : t0(__builtin_amdgcn_s_memrealtime()), polls(0) {}
    __device__ __forceinline__ bool give_up(char* ws, unsigned call) {
        if ((++polls & 15u) == 0u && ((unsigned)__builtin_amdgcn_readfirstlane(ld_agent(ws + OFF_TMO)) == call || __builtin_amdgcn_s_memrealtime() - t0 > DEADLINE)) {
            __hip_atomic_store((GAS unsigned*)(ws + OFF_TMO), call, __ATOMIC_RELAXED, __HIP_MEMORY_SCOPE_AGENT);
            return true;
        }
        __builtin_amdgcn_s_sleep(POLL_SLEEP);
        return false;
    }
};
__device__ __forceinline__ void store16_sys(char* base, unsigned voff, u32x4 v) {
    asm volatile("global_store_dwordx4 %0, %1, %2 sc0 sc1\n\ts_nop 1" :: "v"(voff), "v"(v), "s"(base) : "memory");
}

template <int CTRL, int ROWS> __device__ __forceinline__ unsigned max_dpp(unsigned x) { const unsigned y = (unsigned)__builtin_amdgcn_update_dpp(0u, x, CTRL, ROWS, 0xf, false); return x > y ? x : y; }
__device__ __forceinline__ unsigned wave_max(unsigned x) {
    return max_dpp<0x143, 0xc>(max_dpp<0x142, 0xa>(max_dpp<0x118, 0xf>(max_dpp<0x114, 0xf>(max_dpp<0x112, 0xf>(max_dpp<0x111, 0xf>(x))))));
}
__device__ __forceinline__ unsigned nonzero2(unsigned w) { unsigned r; asm("v_pk_min_u16 %0, %1, %2" : "=v"(r) : "v"(w), "s"(0x00010001u)); return r; }
__device__ __forceinline__ unsigned below(int a, int b) { unsigned r; asm("s_ashr_i32 %0, %1, 31" : "=s"(r) : "s"(a - b) : "scc"); return r; }
__device__ __forceinline__ unsigned wave_scan(unsigned x) {
    x += __builtin_amdgcn_update_dpp(0u, x, 0x111, 0xf, 0xf, false); x += __builtin_amdgcn_update_dpp(0u, x, 0x112, 0xf, 0xf, false);
    x += __builtin_amdgcn_update_dpp(0u, x, 0x114, 0xf, 0xf, false); x += __builtin_amdgcn_update_dpp(0u, x, 0x118, 0xf, 0xf, false);
    x += __builtin_amdgcn_update_dpp(0u, x, 0x142, 0xa, 0xf, false); x += __builtin_amdgcn_update_dpp(0u, x, 0x143, 0xc, 0xf, false);
    return x;
}

__device__ __forceinline__ unsigned draw_turn(volatile LAS unsigned* turn, int lane) {
    unsigned t = 0;
    if (lane == 0) t = __hip_atomic_fetch_add((LAS unsigned*)turn, 1u, __ATOMIC_RELAXED, __HIP_MEMORY_SCOPE_WORKGROUP);
    return (unsigned)__builtin_amdgcn_readfirstlane(t);
}
__device__ __forceinline__ void await_turn(volatile LAS unsigned* turn, unsigned mine, unsigned slots) {
    while (mine >= (unsigned)__builtin_amdgcn_readfirstlane(turn[1]) + slots) __builtin_amdgcn_s_sleep(TURN_SLEEP);
}
__device__ __forceinline__ void end_turn(volatile LAS unsigned* turn) { __hip_atomic_fetch_add((LAS unsigned*)turn + 1, 1u, __ATOMIC_RELAXED, __HIP_MEMORY_SCOPE_WORKGROUP); }

constexpr int CODEC_LUT = 0, CODEC_STREAM = 2048, CODEC_LDS = CODEC_STREAM + 1088;
constexpr int WAVE_LDS = 8448, PACK_LUT = 8 * WAVE_LDS;
static_assert(CODEC_LDS <= WAVE_LDS && PACK_LUT + 2048 <= LDS_STAGE, "per-wave LDS");
__device__ __forceinline__ unsigned esc_bits(unsigned c) { return c == 0u ? 0u : ~(c | (c >> 1)) & 0x05050505u; }
__device__ __forceinline__ u32x2 lut_of(unsigned e, unsigned lut) {
    const LAS unsigned* q = (const LAS unsigned*)(size_t)__builtin_amdgcn_udot4(e, 0x80400804u, lut, false);
    return (u32x2){q[0], q[256]};
}
template <bool PACK> __device__ __forceinline__ void lut_entry(LAS unsigned char* lut, int m) {
    unsigned long long sel = 0x0c0c0c0c0c0c0c0cull; int k = 0;
#pragma unroll
    for (int b = 0; b < 8; ++b) if (m >> b & 1) { const int pos = (b & 1) | ((b & 2) << 1) | ((b & 4) >> 1), slot = (k >> 1) | ((k & 1) << 2);
        sel ^= PACK ? (unsigned long long)(0x0c ^ pos) << (8 * slot) : (unsigned long long)(0x0c ^ slot) << (8 * pos); ++k; }
    *(LAS unsigned*)(lut + m * 4) = (unsigned)sel; *(LAS unsigned*)(lut + 1024 + m * 4) = (unsigned)(sel >> 32);
}
__device__ __forceinline__ unsigned bfi(unsigned m, unsigned a, unsigned b) { return (a & m) | (b & ~m); }
__device__ __forceinline__ void split4(unsigned r0, unsigned r1, unsigned& m, unsigned& e) {
    const unsigned lo = __builtin_amdgcn_perm(r1, r0, 0x06040200u), hi = __builtin_amdgcn_perm(r1, r0, 0x07050301u);
    m = bfi(0x80808080u, hi, lo); e = bfi(0x01010101u, lo >> 7, hi << 1);
}
__device__ __forceinline__ u32x4 join8(unsigned m0, unsigned e0, unsigned m1, unsigned e1) {
    const unsigned h0 = bfi(0x80808080u, m0, e0 >> 1), l0 = bfi(0x7f7f7f7fu, m0, e0 << 7), h1 = bfi(0x80808080u, m1, e1 >> 1), l1 = bfi(0x7f7f7f7fu, m1, e1 << 7);
    return (u32x4){__builtin_amdgcn_perm(h0, l0, 0x05010400u), __builtin_amdgcn_perm(h0, l0, 0x07030602u), __builtin_amdgcn_perm(h1, l1, 0x05010400u), __builtin_amdgcn_perm(h1, l1, 0x07030602u)};
}

struct Item { bool isw; unsigned src, fin, cnt; };
__device__ __forceinline__ Item item_of(int d, int i) {
    const int chunk = (int)((d & 1) ? (unsigned)i / (unsigned)litems(1) : (unsigned)i / (unsigned)litems(0)), j = i - chunk * litems(d);
    Item it;
    it.isw = (d & 1) && j >= OH;
    if (it.isw) { it.src = (unsigned)((chunk * 128 + (j - OH) * 16) * BPITCH + (d >> 1) * 512); it.fin = (unsigned)(OFF_W2 + ((d >> 1) * NCH * 8 + chunk * 8 + j - OH) * 8192);
        it.cnt = (unsigned)(OFF_CNT + ((NSLOT + (d >> 1)) * NCH + chunk) * 64); }
    else { const int strip = (d & 1) ? EH + j : j, slot = (d & 1) ? 3 * EH + (d >> 1) * OH + j : ((d >> 1) - 1) * EH + j;
        it.src = (unsigned)(strip * STRIP_BYTES + chunk * 256); it.fin = (unsigned)(OFF_XA + slot * STRIP_BYTES + chunk * 8192);
        it.cnt = (unsigned)(OFF_CNT + (slot * NCH + chunk) * 64); }
    return it;
}
__device__ __forceinline__ void fetch(const char* src_x, const char* src_w, u32x4 (&nxt)[8], unsigned& tick, int d, int i, unsigned mask, const char* word, unsigned draw) {
    const int lane = threadIdx.x & 63;
    const Item x = item_of(d, i);
    const unsigned v = x.isw ? (lane >> 5) * BPITCH + (lane & 31) * 16 : (lane >> 4) * APITCH + (lane & 15) * 16, st = x.isw ? 2 * BPITCH : 4 * APITCH;
    asm volatile("s_mov_b32 exec_lo, %21\n\ts_mov_b32 exec_hi, %21\n\t"
                 "global_load_dwordx4 %0, %9, %17\n\tglobal_load_dwordx4 %1, %10, %17\n\tglobal_load_dwordx4 %2, %11, %17\n\tglobal_load_dwordx4 %3, %12, %17\n\t"
                 "global_load_dwordx4 %4, %13, %17\n\tglobal_load_dwordx4 %5, %14, %17\n\tglobal_load_dwordx4 %6, %15, %17\n\tglobal_load_dwordx4 %7, %16, %17\n\t"
                 "s_mov_b32 exec_lo, %22\n\ts_mov_b32 exec_hi, 0\n\tglobal_atomic_add %8, %18, %19, %20 sc0\n\ts_mov_b64 exec, -1"
                 : "=&v"(nxt[0]), "=&v"(nxt[1]), "=&v"(nxt[2]), "=&v"(nxt[3]), "=&v"(nxt[4]), "=&v"(nxt[5]), "=&v"(nxt[6]), "=&v"(nxt[7]), "=&v"(tick)
                 : "v"(v), "v"(v + st), "v"(v + 2 * st), "v"(v + 3 * st), "v"(v + 4 * st), "v"(v + 5 * st), "v"(v + 6 * st), "v"(v + 7 * st),
                   "s"((x.isw ? src_w : src_x) + x.src), "v"(0u), "v"(1u), "s"(word), "s"(mask), "s"(draw) : "memory");
}
#define WAIT_FETCH(nxt, tick) asm volatile("s_waitcnt vmcnt(0)" : "+v"(tick), "+v"(nxt[0]), "+v"(nxt[1]), "+v"(nxt[2]), "+v"(nxt[3]), "+v"(nxt[4]), "+v"(nxt[5]), "+v"(nxt[6]), "+v"(nxt[7]) :: "memory")
__device__ __forceinline__ void sender(const Params& P, char* peer, LAS unsigned char* buf, unsigned lut, volatile LAS unsigned* turn, int d, int first, unsigned call, u32x4 (&nxt)[8]) {
    const int lane = threadIdx.x & 63, nlink = NCH * litems(d), nwaves = SEND_WGS * 8;
    char* const queue = P.wsx[0] + OFF_QUEUE + d * 128;
    const char* const src_x = P.x; const char* const src_w = P.w;
    const unsigned lds0 = (unsigned)(size_t)buf;
    unsigned mine = (unsigned)__builtin_amdgcn_readfirstlane(threadIdx.x >> 6), tick;
    int cur = first;
    char* const set = peer + (call & 1) * PAR_BYTES;
    char* const flags = peer + OFF_FLAG + (call & 1) * FLAG_BYTES;
    const unsigned qbase = (call - 1u) * nlink - 2 * nwaves;
    tick = qbase + first + nwaves;
    unsigned early = below((int)mine, SEND_SLOTS), late = below(SEND_SLOTS - 1, (int)mine);
    while (cur < nlink) {
        const int u = lbase(d) + cur;
        const int next = (int)((unsigned)__builtin_amdgcn_readfirstlane(tick) - qbase);
        u32x4 pk[6]; unsigned ex[16], esum = 0;
#pragma unroll
        for (int g = 0; g < 16; ++g) {
            unsigned m; split4(nxt[g >> 1][(g & 1) * 2], nxt[g >> 1][(g & 1) * 2 + 1], m, ex[g]);
            pk[g >> 2][g & 3] = m;
            if ((g & 3) == 0) esum = __builtin_amdgcn_sad_u8(ex[g], 0u, esum);
        }
        char* const dst = set + OFF_PK + (size_t)u * ITEM_PK + 2048;
#define STORE_PLANES(mask) asm volatile("s_mov_b32 exec_lo, %6\n\ts_mov_b32 exec_hi, %6\n\tglobal_store_dwordx4 %0, %1, %5 offset:-2048 sc0 sc1\n\tglobal_store_dwordx4 %0, %2, %5 offset:-1024 sc0 sc1\n\t" \
                     "global_store_dwordx4 %0, %3, %5 sc0 sc1\n\tglobal_store_dwordx4 %0, %4, %5 offset:1024 sc0 sc1\n\ts_mov_b64 exec, -1\n\ts_nop 0" \
                     :: "v"(lane * 16), "v"(pk[0]), "v"(pk[1]), "v"(pk[2]), "v"(pk[3]), "s"(dst), "s"(mask) : "memory")
        STORE_PLANES(early);
        SCHED; fetch(src_x, src_w, nxt, tick, d, next < nlink ? next : nlink - 1, ~0u, queue, 1u); SCHED;
        *(LAS u32x4*)(buf + CODEC_STREAM + lane * 16) = (u32x4){0u, 0u, 0u, 0u};
        unsigned top = (((unsigned)__builtin_amdgcn_readlane(wave_scan(esum), 63) + 1023u + 174u) >> 10) + 2u;
        top = top < 18u ? 18u : top > 255u ? 255u : top;
        unsigned cg[8], xs[8], at[8], tot = 0;
#pragma unroll
        for (int v = 0; v < 8; ++v) {
            const unsigned ua = top * 0x01010101u - ex[2 * v], ub = top * 0x01010101u - ex[2 * v + 1];
            const unsigned ca = __builtin_amdgcn_perm(0u, 0x03020100u, (((ua >> 1) & 0x04040404u) | ua) & 0x07070707u);
            const unsigned cb = __builtin_amdgcn_perm(0u, 0x0c080400u, (((ub >> 1) & 0x04040404u) | ub) & 0x07070707u);
            cg[v] = ((ua | ub) & 0xf0f0f0f0u) != 0u ? 0u : ca | cb;
            const unsigned e = esc_bits(cg[v]);
            const u32x2 sel = lut_of(e, lut);
            xs[v] = __builtin_amdgcn_perm(ub, ua, sel.x) | (__builtin_amdgcn_perm(ub, ua, sel.y) << 4);
            at[v] = tot; tot += __builtin_popcount(e);
        }
        const unsigned incl = wave_scan(tot);
        unsigned total = (unsigned)__builtin_amdgcn_readlane(incl, 63);
        if (total > 2048u) {
            total = 0;
#pragma unroll
            for (int v = 0; v < 8; ++v) cg[v] = 0u;
        } else {
            const unsigned n0 = incl - tot + (lds0 + CODEC_STREAM) * 2u;
            asm volatile("" ::: "memory");
#pragma unroll
            for (int v = 0; v < 8; ++v) {
                const unsigned n = n0 + at[v];
                const unsigned long long w = (unsigned long long)xs[v] << ((n * 4u) & 31u);
                LAS unsigned* q = (LAS unsigned*)(size_t)((n >> 1) & ~3u);
                __hip_atomic_fetch_or(q, (unsigned)w, __ATOMIC_RELAXED, __HIP_MEMORY_SCOPE_WAVEFRONT);
                __hip_atomic_fetch_or(q + 1, (unsigned)(w >> 32), __ATOMIC_RELAXED, __HIP_MEMORY_SCOPE_WAVEFRONT);
            }
        }
        asm volatile("" ::: "memory");
        pk[5] = *(const LAS u32x4*)(buf + CODEC_STREAM + lane * 16);
        unsigned least = cg[0];
#pragma unroll
        for (int v = 1; v < 8; ++v) least = cg[v] < least ? cg[v] : least;
#pragma unroll
        for (int c = 0; c < 4; ++c) pk[4][c] = cg[2 * c] | (cg[2 * c + 1] << 4);
        if (cur != first) mine = draw_turn(turn, lane);
        await_turn(turn, mine, SEND_SLOTS);
        if (__builtin_amdgcn_ballot_w64(least == 0u) != 0ull) {
            char* fin = set + item_of(d, cur).fin;
#pragma unroll
            for (int v = 0; v < 8; ++v) if (cg[v] == 0u) store16_sys(fin, v * 1024 + lane * 16, join8(pk[v >> 1][(v & 1) * 2], ex[2 * v], pk[v >> 1][(v & 1) * 2 + 1], ex[2 * v + 1]));
        }
        const unsigned long long smask = __builtin_amdgcn_ballot_w64((unsigned)lane * 32u < total);
        STORE_PLANES(late);
        asm volatile("global_store_dwordx4 %0, %1, %3 offset:2048 sc0 sc1\n\ts_mov_b64 exec, %4\n\tglobal_store_dwordx4 %0, %2, %3 offset:3072 sc0 sc1\n\ts_mov_b64 exec, -1\n\ts_nop 0"
                     :: "v"(lane * 16), "v"(pk[4]), "v"(pk[5]), "s"(dst), "s"(smask) : "memory");
        early = 0u; late = ~0u;
        WAIT_FETCH(nxt, tick);
        if (lane == 0) {
            end_turn(turn);
            __hip_atomic_store((GAS unsigned*)(flags + (size_t)u * 64), (call << 8) | top, __ATOMIC_RELAXED, __HIP_MEMORY_SCOPE_SYSTEM);
        }
        cur = next;
    }
}

__device__ __forceinline__ void unpacker(const Params& P, LAS unsigned char* buf, int uw, unsigned call) {
    const int lane = threadIdx.x & 63;
    const int d = __builtin_amdgcn_readfirstlane(uw % 7 + 1), i0 = __builtin_amdgcn_readfirstlane(uw / 7);
    const int step = __builtin_amdgcn_readfirstlane(NUW / 7 + (d <= NUW % 7 ? 1 : 0)), nlink = NCH * litems(d);
    char* ws = P.wsx[0];
    char* set = ws + (call & 1) * PAR_BYTES;
    const char* pkb = set + OFF_PK + (size_t)lbase(d) * ITEM_PK + 2048;
    const char* flags = ws + OFF_FLAG + (call & 1) * FLAG_BYTES + (size_t)lbase(d) * 64;
    const unsigned lds0 = (unsigned)(size_t)buf;
    for (int m = lane; m < 256; m += 64) lut_entry<false>(buf + CODEC_LUT, m);
    asm volatile("" ::: "memory");
    u32x4 pk[6]; unsigned nf = 0;
#pragma unroll
    for (int k = 0; k < 6; ++k) pk[k] = (u32x4){0u, 0u, 0u, 0u};
#define LOAD_ITEM(i, mask) asm volatile("s_mov_b32 exec_lo, %11\n\ts_mov_b32 exec_hi, %11\n\t" \
        "global_load_dwordx4 %0, %7, %8 offset:-2048 sc0 sc1\n\tglobal_load_dwordx4 %1, %7, %8 offset:-1024 sc0 sc1\n\tglobal_load_dwordx4 %2, %7, %8 sc0 sc1\n\t" \
        "global_load_dwordx4 %3, %7, %8 offset:1024 sc0 sc1\n\tglobal_load_dwordx4 %4, %7, %8 offset:2048 sc0 sc1\n\tglobal_load_dwordx4 %5, %7, %8 offset:3072 sc0 sc1\n\t" \
        "global_load_dword %6, %9, %10 sc0 sc1\n\ts_mov_b64 exec, -1\n\ts_waitcnt vmcnt(0)" \
        : "+v"(pk[0]), "+v"(pk[1]), "+v"(pk[2]), "+v"(pk[3]), "+v"(pk[4]), "+v"(pk[5]), "+v"(nf) \
        : "v"(lane * 16), "s"(pkb + (size_t)(i) * ITEM_PK), "v"(0u), "s"(flags + (size_t)((i) + step < nlink ? (i) + step : (i)) * 64), "s"(mask) : "memory")
    unsigned f = 0, have = 0;
    for (int i = i0; i < nlink; i += step) {
        if (!have) {
            Wait wait;
            bool dead = false;
            while (((f = (unsigned)__builtin_amdgcn_readfirstlane(ld_sys(flags + (size_t)i * 64))) >> 8) < call)
                if (wait.give_up(ws, call)) { dead = true; break; }
            if (dead) break;
            LOAD_ITEM(i, ~0u);
        }
        const unsigned top4 = (f & 255u) * 0x01010101u;
        f = (unsigned)__builtin_amdgcn_readfirstlane(nf);
        have = below(i + step, nlink) & ~below((int)(f >> 8), (int)call);
        *(LAS u32x4*)(buf + CODEC_STREAM + lane * 16) = pk[5];
        unsigned cg[8], at[8], tot = 0; u32x2 sel[8];
#pragma unroll
        for (int v = 0; v < 8; ++v) {
            cg[v] = (pk[4][v >> 1] >> (4 * (v & 1))) & 0x0f0f0f0fu;
            const unsigned e = esc_bits(cg[v]);
            sel[v] = lut_of(e, lds0 + CODEC_LUT);
            at[v] = tot; tot += __builtin_popcount(e);
        }
        const unsigned n0 = wave_scan(tot) - tot + (lds0 + CODEC_STREAM) * 2u;
        asm volatile("" ::: "memory");
        const Item it = item_of(d, i);
        unsigned xs[8];
#pragma unroll
        for (int v = 0; v < 8; ++v) {
            const unsigned n = n0 + at[v];
            const LAS unsigned* q = (const LAS unsigned*)(size_t)((n >> 1) & ~3u);
            xs[v] = __builtin_amdgcn_alignbit(q[1], q[0], n * 4u);
        }
#pragma unroll
        for (int v = 0; v < 8; ++v) {
            const unsigned xe = xs[v] & 0x0f0f0f0fu, xo = (xs[v] >> 4) & 0x0f0f0f0fu;
            const unsigned e0 = top4 - ((cg[v] & 0x03030303u) | __builtin_amdgcn_perm(xo, xe, sel[v].x));
            const unsigned e1 = top4 - (((cg[v] >> 2) & 0x03030303u) | __builtin_amdgcn_perm(xo, xe, sel[v].y));
            const u32x4 r = join8(pk[v >> 1][(v & 1) * 2], e0, pk[v >> 1][(v & 1) * 2 + 1], e1);
            if (cg[v] != 0u) store16_sys(set + it.fin, v * 1024 + lane * 16, r);
        }
        LOAD_ITEM(i + step < nlink ? i + step : i, have);
        if (lane == 0) __hip_atomic_fetch_add((GAS unsigned*)(ws + it.cnt), 1u, __ATOMIC_RELAXED, __HIP_MEMORY_SCOPE_AGENT);
    }
}

__device__ __forceinline__ int strip_of(int pos, bool remote, int q) {
    if (!remote) return pos < 32 + NSLOT ? pos : -1;
    if (pos < EH) return pos;
    if (pos < EH + 3 * OH) { const int k = (pos - EH) / OH; return 32 + 3 * EH + (k + (k >= q ? 1 : 0)) * OH + (pos - EH) % OH; }
    return (unsigned)(pos - NTE * SR) < 3u * EH ? 32 + pos - NTE * SR : -1;
}
__device__ __forceinline__ int out_row_of(int id, int rank) {
    if (id < 32) return renum(rank) * 1024 + id * 32;
    const int slot = id - 32;
    if (slot < 3 * EH) return renum(rank ^ (2 * (slot / EH + 1))) * 1024 + (slot % EH) * 32;
    const int o = slot - 3 * EH;
    return renum(rank ^ (2 * (o / OH) + 1)) * 1024 + (EH + o % OH) * 32;
}

constexpr int OUT_M = 256, OUT_LDS = 4160;
template <bool PACK> __device__ __forceinline__ u32x4 out_lut(int m) {
    unsigned long long sm = 0x0c0c0c0c0c0c0c0cull, sn = sm; int k = 0;
#pragma unroll
    for (int b = 0; b < 8; ++b) if (m >> b & 1) { const int slot = (k >> 1) | ((k & 1) << 2);
        sm ^= PACK ? (unsigned long long)(0x0c ^ b) << (8 * k) : (unsigned long long)(0x0c ^ k) << (8 * b);
        sn ^= PACK ? (unsigned long long)(0x0c ^ b) << (8 * slot) : (unsigned long long)(0x0c ^ slot) << (8 * b); ++k; }
    return (u32x4){(unsigned)sm, (unsigned)(sm >> 32), (unsigned)sn, (unsigned)(sn >> 32)};
}
struct OutItem { int o; unsigned at0, at1; };
__device__ __forceinline__ OutItem out_item(int n, int rank) {
    const bool late = n >= NTE * HPT * 32;
    const int n1 = late ? n - NTE * HPT * 32 : n, per = (late ? NTR - NTE : NTE) * 32, h = late ? n1 / ((NTR - NTE) * 32) : n1 / (NTE * 32), rt = (late ? NTE : 0) + ((n1 - h * per) >> 5);
    const int p0 = rt * SR, w = (n1 >> 2) & 7, q = n1 & 3, f0 = (w >> 2) * SR + 2 * h;
    const int ida = n < NOI ? strip_of(p0 + (f0 >> 1), true, q) : -1, idb = 2 * h + 1 < SR ? strip_of(p0 + ((f0 + 1) >> 1), true, q) : -1, src = rank ^ (2 * q + 1);
    OutItem t;
    t.o = ida < 0 ? -1 : ((q * NTR + rt) * 8 + w) * HPT + h;
    t.at0 = (unsigned)(out_row_of(ida, src) + (f0 & 1) * 16) * OPITCH + q * 512 + (w & 3) * 64;
    t.at1 = idb < 0 ? ~0u : (unsigned)(out_row_of(idb, src) + ((f0 + 1) & 1) * 16) * OPITCH + q * 512 + (w & 3) * 64;
    return t;
}
__device__ __forceinline__ void expander(const Params& P, LAS unsigned char* buf, int xw, unsigned call) {
    const int lane = threadIdx.x & 63;
    char* ws = P.wsx[0];
    const unsigned lds0 = (unsigned)(size_t)buf;
    for (int m = lane; m < 256; m += 64) *(LAS u32x4*)(buf + OUT_LDS + m * 16) = out_lut<false>(m);
    const OutItem mine = out_item(xw + (lane & 7) * NXW, P.rank);
    char* const out = P.out;
    unsigned long long todo = __builtin_amdgcn_ballot_w64(lane < 8 && mine.o >= 0);
    const char* watch = ws + OFF_OFLAG + (mine.o >= 0 ? mine.o : 0) * 64;
    while (todo) {
        unsigned long long fl, ready; bool dead = false;
        { Wait wait;
          while ((ready = __builtin_amdgcn_ballot_w64((unsigned)((fl = __hip_atomic_load((const GAS unsigned long long*)watch, __ATOMIC_RELAXED, __HIP_MEMORY_SCOPE_SYSTEM)) >> 32) >= call) & todo) == 0ull) {
              if (wait.give_up(ws, call)) { dead = true; ready = todo; break; }
          } }
        const int k = __builtin_ctzll(ready);
        todo &= ~(1ull << k);
        const unsigned f = (unsigned)__builtin_amdgcn_readlane((unsigned)fl, k);
        const int o = __builtin_amdgcn_readlane(mine.o, k);
        const unsigned at0 = (unsigned)__builtin_amdgcn_readlane(mine.at0, k), at1 = (unsigned)__builtin_amdgcn_readlane(mine.at1, k);
        u32x4 r[4];
        unsigned tmo = call;
        if (!dead) {
            u32x4 t[4];
            asm volatile("global_load_dwordx4 %0, %5, %6 sc0 sc1\n\tglobal_load_dwordx4 %1, %5, %6 offset:1024 sc0 sc1\n\tglobal_load_dwordx4 %2, %5, %6 offset:2048 sc0 sc1\n\t"
                         "global_load_dwordx4 %3, %5, %6 offset:3072 sc0 sc1\n\tglobal_load_dword %4, %7, %8 sc1\n\ts_waitcnt vmcnt(0)"
                         : "=&v"(t[0]), "=&v"(t[1]), "=&v"(t[2]), "=&v"(t[3]), "=&v"(tmo)
                         : "v"(lane * 16), "s"(ws + OFF_OPK + (size_t)o * OITEM), "v"(0u), "s"(ws + OFF_TMO) : "memory");
            if (!(f & 0x100000u)) {
#pragma unroll
                for (int k = 0; k < 4; ++k) *(LAS u32x4*)(buf + lane * 16 + k * 1024) = t[k];
                asm volatile("" ::: "memory");
                const unsigned bits = *(const LAS unsigned*)(buf + lane * 4), out_n = OUT_M + (((f & 0xfffu) + 15u) & 0xff0u);
                const unsigned tot = __builtin_popcount(bits), base4 = ((f >> 12) & 255u) * 0x01010101u;
                unsigned n = (wave_scan(tot) - tot) & 2047u;
                asm volatile("" ::: "memory");
#pragma unroll
                for (int k = 0; k < 4; ++k) {
                    const unsigned m = (bits >> (8 * k)) & 255u;
                    const u32x4 sel = *(const LAS u32x4*)(buf + OUT_LDS + m * 16);
                    const LAS unsigned* qm = (const LAS unsigned*)(size_t)(lds0 + OUT_M + (n & ~3u));
                    const LAS unsigned* qn = (const LAS unsigned*)(size_t)(lds0 + out_n + ((n >> 1) & ~3u));
                    const unsigned m0 = __builtin_amdgcn_alignbyte(qm[1], qm[0], n), m1 = __builtin_amdgcn_alignbyte(qm[2], qm[1], n), x = __builtin_amdgcn_alignbit(qn[1], qn[0], n * 4u);
                    const unsigned xe = x & 0x0f0f0f0fu, xo = (x >> 4) & 0x0f0f0f0fu;
                    const unsigned e0 = (base4 - __builtin_amdgcn_perm(xo, xe, sel.z)) & __builtin_amdgcn_perm(~0u, ~0u, sel.x);
                    const unsigned e1 = (base4 - __builtin_amdgcn_perm(xo, xe, sel.w)) & __builtin_amdgcn_perm(~0u, ~0u, sel.y);
                    t[k] = join8(__builtin_amdgcn_perm(m1, m0, sel.x), e0, __builtin_amdgcn_perm(m1, m0, sel.y), e1);
                    n += __builtin_popcount(m);
                }
                asm volatile("" ::: "memory");
            }
#pragma unroll
            for (int k = 0; k < 8; ++k) {
                const int row = (k >> 2) * 16 + (lane & 15), ch = (k & 3) * 2 + (lane >> 5);
                *(LAS u32x2*)(buf + row * 128 + ((ch ^ (row & 7)) * 16) + ((lane >> 4) & 1) * 8) = (u32x2){t[k >> 1][(k & 1) * 2], t[k >> 1][(k & 1) * 2 + 1]};
            }
            asm volatile("" ::: "memory");
#pragma unroll
            for (int v = 0; v < 4; ++v) { const int row = v * 8 + (lane >> 3); r[v] = *(const LAS u32x4*)(buf + row * 128 + (((lane & 7) ^ (row & 7)) * 16)); }
        }
        if (tmo == call) {
#pragma unroll
            for (int v = 0; v < 4; ++v) r[v] = (u32x4){0x7fc07fc0u, 0x7fc07fc0u, 0x7fc07fc0u, 0x7fc07fc0u};
        }
        const unsigned ooff = (lane >> 3) * OPITCH + ((lane >> 2) & 1) * 256 + (lane & 3) * 16;
        store16_sys(out, ooff + at0, r[0]); store16_sys(out, ooff + at0 + 8 * OPITCH, r[1]);
        if (at1 != ~0u) { store16_sys(out, ooff + at1, r[2]); store16_sys(out, ooff + at1 + 8 * OPITCH, r[3]); }
    }
}

__device__ __forceinline__ unsigned cvt_pk_bf16(float lo, float hi) { unsigned r; asm("v_cvt_pk_bf16_f32 %0, %1, %2" : "=v"(r) : "v"(lo), "v"(hi)); return r; }
template <int OFF> __device__ __forceinline__ s16x4 tr_read(unsigned addr) { s16x4 r; asm volatile("ds_read_b64_tr_b16 %0, %1 offset:%2" : "=v"(r) : "v"(addr), "i"(OFF) : "memory"); return r; }
template <int OFF> __device__ __forceinline__ bf16x8 row_read(unsigned addr) { bf16x8 r; asm volatile("ds_read_b128 %0, %1 offset:%2" : "=v"(r) : "v"(addr), "i"(OFF) : "memory"); return r; }
template <int OFF> __device__ __forceinline__ void glds(unsigned ldsaddr, unsigned voff, const char* g) {
    asm volatile("s_mov_b32 m0, %0\n\ts_nop 0\n\tglobal_load_lds_dwordx4 %1, %2 offset:%3 sc0 sc1" :: "s"(ldsaddr - OFF), "v"(voff), "s"(g), "i"(OFF) : "memory");
}
template <int OFF> __device__ __forceinline__ void glds_masked(unsigned ldsaddr, unsigned voff, const char* g, unsigned mask) {
    asm volatile("s_mov_b32 m0, %0\n\ts_mov_b32 exec_lo, %3\n\ts_mov_b32 exec_hi, %3\n\tglobal_load_lds_dwordx4 %1, %2 offset:%4 sc0 sc1\n\ts_mov_b64 exec, -1"
                 :: "s"(ldsaddr - OFF), "v"(voff), "s"(g), "s"(mask), "i"(OFF) : "memory");
}

template <int S> struct Frag { bf16x8 a[S], b[4]; };
template <int S> __device__ __forceinline__ void wait_step(Frag<S>& F, unsigned& pre) {
    asm volatile("s_waitcnt vmcnt(6) lgkmcnt(0)" : "+v"(F.a[0]), "+v"(F.a[1]), "+v"(F.a[2]), "+v"(F.a[3]), "+v"(F.a[S - 1]), "+v"(F.b[0]), "+v"(F.b[1]), "+v"(F.b[2]), "+v"(F.b[3]), "+v"(pre) :: "memory");
    if constexpr (S > 5) asm volatile("" : "+v"(F.a[4]));
    if constexpr (S > 6) asm volatile("" : "+v"(F.a[5]));
    if constexpr (S > 7) asm volatile("" : "+v"(F.a[6]));
}
template <int S, bool remote> __device__ __forceinline__ void gemm_tile(const Params& P, LAS unsigned char* lds, int rt, int ct, unsigned call) {
    constexpr int SLOT_A = S * 2048, SLOT = SLOT_A + 16384;
    static_assert(S >= 5 && S <= 8 && 4 * SLOT <= LDS_STAGE && S * 16384 <= LDS_STAGE, "tile");
    int wid = __builtin_amdgcn_readfirstlane(threadIdx.x >> 6), lane = threadIdx.x & 63, wr = wid >> 2, wc = wid & 3, fr = lane & 15, fq = lane >> 4;
    const int p0 = rt * S;
    char* ws = P.wsx[0];
    const char* set = ws + (call & 1) * PAR_BYTES;
    const char* const own_x = P.x;
    const unsigned ldst = (unsigned)(size_t)lds + wid * 1024;
    constexpr int BP = remote ? 512 : BPITCH;
    auto strip_at = [&](int k) { const int id = k < S ? strip_of(p0 + k, remote, ct) : -1; return id < 0 ? strip_of(p0, remote, ct) : id; };
    const int id0 = strip_at(wid >> 1), id1 = strip_at(4 + (wid >> 1));
    const char* const ga0 = id0 < 32 ? own_x + id0 * STRIP_BYTES : set + OFF_XA + (id0 - 32) * STRIP_BYTES;
    const char* const ga1 = id1 < 32 ? own_x + id1 * STRIP_BYTES : set + OFF_XA + (id1 - 32) * STRIP_BYTES;
    const unsigned cs0 = id0 < 32 ? 256 : 8192, cs1 = id1 < 32 ? 256 : 8192;
    const char* const gb = remote ? set + OFF_W2 + ct * (W2_BYTES / 4) : P.w + ct * 512;
    unsigned voffA0, voffA1, voffB;
    { const int k0 = lane >> 4, ch = lane & 15; voffB = (unsigned)((wid * 4 + k0) * BP + (((ch >> 1) ^ (k0 | ((wid & 2) << 1))) * 32) + (ch & 1) * 16);
      const int sb = lane * 16, swz = sb ^ (((sb >> 9) & 1) << 5), row = (wid & 1) * 16 + (swz >> 6);
      voffA0 = (unsigned)(row * (id0 < 32 ? APITCH : 256) + (swz & 63)); voffA1 = (unsigned)(row * (id1 < 32 ? APITCH : 256) + (swz & 63)); }
    const unsigned amask = below(wid, 2 * S - 8);
    unsigned aoff[2], boff[2][2];
    { const unsigned lbase = (unsigned)(size_t)lds; const int ob = fr * 64 + fq * 16; aoff[0] = lbase + wr * S * 1024 + (ob ^ (((ob >> 9) & 1) << 5)); aoff[1] = aoff[0] + 2 * SLOT;
      const int q = (lane & 15) >> 2, p = lane & 3, f = q | ((fq & 1) << 2);
      for (int n = 0; n < 2; ++n) { boff[0][n] = lbase + SLOT_A + (8 * fq + q) * 256 + p * 8 + (((wc * 2 + n) ^ f) * 32); boff[1][n] = boff[0][n] + 2 * SLOT; } }
    unsigned coff, cstep, need;
    { const int id = strip_of(p0 + (lane < S ? lane : 0), remote, ct);
      const bool watch = wid == 0 && (lane < S ? id >= 32 : (lane == 8 && remote));
      coff = watch ? (unsigned)(OFF_CNT + (lane < S ? id - 32 : NSLOT + ct) * NCH * 64) : (unsigned)OFF_TMO;
      cstep = watch ? 64u : 0u;
      need = watch ? call * (lane < S ? 1u : 8u) : 0u; }
    unsigned orow = 0;
    if constexpr (!remote) { const int f = wr * S + (lane & 7), id = strip_of(p0 + (f >> 1), false, 0); orow = (unsigned)(out_row_of(id, P.rank) + (f & 1) * 16) * OPITCH + ct * 512; }
    char* const out = remote ? P.wsx[(2 * ct + 1) & 7] : P.out;
    const int nfrag = __builtin_popcountll(__builtin_amdgcn_ballot_w64(lane < S && strip_of(p0 + ((wr * S + lane) >> 1), remote, ct) >= 0));
    const unsigned w0 = below(wid, 1);
    unsigned pre = 0;
#define SLOW_WAIT() do { Wait wait; \
        while (__builtin_amdgcn_ballot_w64(ld_agent(ws + coff) < need) != 0ull) if (wait.give_up(ws, call)) break; \
        asm volatile("" ::: "memory"); } while (0)
#define PRE_LOAD(mask) do { coff += cstep; \
        asm volatile("s_mov_b32 exec_lo, %3\n\ts_mov_b32 exec_hi, 0\n\tglobal_load_dword %0, %1, %2 sc1\n\ts_mov_b64 exec, -1" : "+v"(pre) : "v"(coff), "s"(ws), "s"((mask) & w0) : "memory"); } while (0)
#define STAGE_B(h, k, ch) glds<(h) * 256>(ldst + (k) * SLOT + SLOT_A + (h) * 8192, voffB, gb + (size_t)((ch) * 4 + (k)) * (32 * BP))
#define STAGE_A0(k, ch) glds<(k) * 64>(ldst + (k) * SLOT, voffA0, ga0 + (ch) * cs0)
#define STAGE_A1(k, ch) glds_masked<(k) * 64>(ldst + (k) * SLOT + 8192, voffA1, ga1 + (ch) * cs1, amask)
#define STAGE_ALL(k, ch) do { STAGE_B(0, k, ch); STAGE_B(1, k, ch); STAGE_A0(k, ch); STAGE_A1(k, ch); } while (0)
#define RDA(F, m, sl) F.a[m] = row_read<((sl) & 1) * SLOT + (m) * 1024>(aoff[(sl) >> 1])
#define RDB(F, j, sl) do { const s16x4 _lo = tr_read<((sl) & 1) * SLOT + ((j) >> 1) * 8192>(boff[(sl) >> 1][(j) & 1]), _hi = tr_read<((sl) & 1) * SLOT + ((j) >> 1) * 8192 + 1024>(boff[(sl) >> 1][(j) & 1]); \
        F.b[j] = __builtin_shufflevector(_lo, _hi, 0, 1, 2, 3, 4, 5, 6, 7); } while (0)
#define MM(F, i) acc[(i) >> 2][(i) & 3] = __builtin_amdgcn_mfma_f32_16x16x32_bf16(F.b[(i) & 3], F.a[(i) >> 2], acc[(i) >> 2][(i) & 3], 0, 0, 0)
#define M1(C, i) do { MM(C, i); SCHED; } while (0)
#define STEP(C, N, k) do { constexpr int nx = ((k) + 1) & 3; \
        if constexpr ((k) == 2) PRE_LOAD(c < NCH - 2 ? 0x3ffu : c < NCH - 1 ? 0x200u : 0u); \
        M1(C, 0); STAGE_B(0, k, c1); M1(C, 1); RDB(N, 0, nx); M1(C, 2); STAGE_B(1, k, c1); M1(C, 3); RDB(N, 1, nx); \
        M1(C, 4); STAGE_A0(k, c1); M1(C, 5); RDB(N, 2, nx); M1(C, 6); STAGE_A1(k, c1); M1(C, 7); RDB(N, 3, nx); \
        M1(C, 8); RDA(N, 0, nx); M1(C, 9); RDA(N, 1, nx); M1(C, 10); RDA(N, 2, nx); M1(C, 11); RDA(N, 3, nx); M1(C, 12); RDA(N, 4, nx); \
        M1(C, 13); if constexpr (S > 5) RDA(N, 5, nx); M1(C, 14); if constexpr (S > 6) RDA(N, 6, nx); if constexpr (S > 7) RDA(N, 7, nx); \
        _Pragma("unroll") for (int i = 15; i < 4 * S - 4; ++i) M1(C, i); \
        wait_step(N, pre); \
        if constexpr ((k) == 3) { if (c < NCH - 2 && __builtin_amdgcn_ballot_w64(pre < need) != 0ull) SLOW_WAIT(); } \
        BAR; SCHED; M1(C, 4 * S - 4); M1(C, 4 * S - 3); M1(C, 4 * S - 2); M1(C, 4 * S - 1); } while (0)
    f32x4 acc[S][4];
#pragma unroll
    for (int m = 0; m < S; ++m)
#pragma unroll
        for (int j = 0; j < 4; ++j) acc[m][j] = (f32x4){0.f, 0.f, 0.f, 0.f};
    Frag<S> F0, F1;
    constexpr int OUT_LUT = LDS_STAGE - 4096;
    static_assert(!remote || (4 * SLOT <= OUT_LUT && 8 * OUT_LDS <= OUT_LUT), "pack tables");
    if (remote && threadIdx.x >= 256) *(LAS u32x4*)(lds + OUT_LUT + (threadIdx.x - 256) * 16) = out_lut<true>(threadIdx.x - 256);
    if (wid == 0) SLOW_WAIT();
    BAR;
    PRE_LOAD(0x3ffu);
    STAGE_ALL(0, 0u); STAGE_ALL(1, 0u); STAGE_ALL(2, 0u); STAGE_ALL(3, 0u);
    WAIT_V(6); BAR;
    RDB(F0, 0, 0); RDB(F0, 1, 0); RDB(F0, 2, 0); RDB(F0, 3, 0); RDA(F0, 0, 0); RDA(F0, 1, 0); RDA(F0, 2, 0); RDA(F0, 3, 0); RDA(F0, 4, 0);
    if constexpr (S > 5) RDA(F0, 5, 0);
    if constexpr (S > 6) RDA(F0, 6, 0);
    if constexpr (S > 7) RDA(F0, 7, 0);
    wait_step(F0, pre);
    if (__builtin_amdgcn_ballot_w64(pre < need) != 0ull) SLOW_WAIT();
    BAR;
    for (int c = 0; c < NCH; ++c) {
        const unsigned c1 = c < NCH - 1 ? c + 1 : c;
        STEP(F0, F1, 0); STEP(F1, F0, 1); STEP(F0, F1, 2); STEP(F1, F0, 3);
    }
    volatile LAS unsigned* turn = (volatile LAS unsigned*)(lds + LDS_STAGE + 4);
    if (threadIdx.x == 9) { *(LAS unsigned*)(lds + LDS_STAGE) = pre; turn[0] = 0u; turn[1] = 0u; }
    asm volatile("s_waitcnt vmcnt(0) lgkmcnt(0)" ::: "memory");
    BAR;
    asm volatile("" ::: "memory");
    const bool bad = *(const LAS unsigned*)(lds + LDS_STAGE) == call;
    int tz = threadIdx.x; asm volatile("" : "+v"(tz));
    lane = tz & 63; fr = lane & 15; fq = lane >> 4;
    if (__builtin_amdgcn_readfirstlane(bad)) {
#pragma unroll
        for (int m = 0; m < S; ++m)
#pragma unroll
            for (int j = 0; j < 4; ++j) acc[m][j] = (f32x4){__builtin_nanf(""), __builtin_nanf(""), __builtin_nanf(""), __builtin_nanf("")};
    }
    auto relu_pack = [&](int m, int j) {
        const f32x4 v = __builtin_elementwise_maximum(acc[m][j], (f32x4){0.f, 0.f, 0.f, 0.f});
        return (u32x2){cvt_pk_bf16(v[0], v[1]), cvt_pk_bf16(v[2], v[3])};
    };
    if constexpr (!remote) {
        LAS unsigned char* buf = lds + wid * (S * 2048);
#pragma unroll
        for (int m = 0; m < S; ++m)
#pragma unroll
            for (int j = 0; j < 4; ++j) *(LAS u32x2*)(buf + m * 2048 + fr * 128 + (((j * 2 + (fq >> 1)) ^ (fr & 7)) * 16) + (fq & 1) * 8) = relu_pack(m, j);
        asm volatile("" ::: "memory");
        u32x4 ov[S][2];
#pragma unroll
        for (int m = 0; m < S; ++m)
#pragma unroll
            for (int i = 0; i < 2; ++i) ov[m][i] = *(const LAS u32x4*)(buf + m * 2048 + i * 1024 + (lane >> 3) * 128 + (((lane & 7) ^ (lane >> 3)) * 16));
        const unsigned ooff = (lane >> 3) * OPITCH + ((lane >> 2) & 1) * 256 + wc * 64 + (lane & 3) * 16;
#pragma unroll
        for (int m = 0; m < S; ++m) {
            const unsigned at = (unsigned)__builtin_amdgcn_readlane(orow, m);
            if (m < nfrag) { store16_sys(out, ooff + at, ov[m][0]); store16_sys(out, ooff + at + 8 * OPITCH, ov[m][1]); }
        }
    } else {
        char* const peer = out;
        LAS unsigned char* buf = lds + wid * OUT_LDS;
        const unsigned lds0 = (unsigned)(size_t)buf;
        const int o0 = ((ct * NTR + rt) * 8 + wid) * HPT;
        int sent = -1; unsigned sent_meta = 0;
#define PUBLISH_OUT() do { WAIT_V(0); if (sent >= 0) { if (lane == 0) { end_turn(turn); \
            __hip_atomic_store((GAS unsigned long long*)(peer + OFF_OFLAG + sent * 64), ((unsigned long long)call << 32) | sent_meta, __ATOMIC_RELAXED, __HIP_MEMORY_SCOPE_SYSTEM); } \
            sent = -1; } } while (0)
#pragma unroll
        for (int h = 0; h < HPT; ++h) {
            if (2 * h >= nfrag) break;
            const bool second = 2 * h + 1 < nfrag;
            u32x4 t[4]; u16x2 mx = {0, 0};
#pragma unroll
            for (int k = 0; k < 8; ++k) {
                u32x2 w = relu_pack(2 * h + (k >> 2) < S ? 2 * h + (k >> 2) : 0, k & 3);
                if (k >= 4 && !second) w = (u32x2){0u, 0u};
                t[k >> 1][(k & 1) * 2] = w.x; t[k >> 1][(k & 1) * 2 + 1] = w.y;
                mx = __builtin_elementwise_max(mx, __builtin_elementwise_max(__builtin_bit_cast(u16x2, w.x & 0x7fff7fffu), __builtin_bit_cast(u16x2, w.y & 0x7fff7fffu)));
            }
#pragma unroll
            for (int k = 0; k < 3; ++k) *(LAS u32x4*)(buf + OUT_M + lane * 16 + k * 1024) = (u32x4){0u, 0u, 0u, 0u};
            const unsigned emax = (unsigned)__builtin_amdgcn_readlane(wave_max(mx.x > mx.y ? mx.x : mx.y), 63) >> 7;
            unsigned bits = 0, wide = 0, tot = 0, at[4], cm0[4], cm1[4], cn[4];
#pragma unroll
            for (int k = 0; k < 4; ++k) {
                unsigned z[4], m0, e0, m1, e1;
#pragma unroll
                for (int c = 0; c < 4; ++c) z[c] = nonzero2(t[k][c]);
                const unsigned m = __builtin_amdgcn_udot4(__builtin_amdgcn_perm(z[3], z[2], 0x06040200u), 0x80402010u, __builtin_amdgcn_udot4(__builtin_amdgcn_perm(z[1], z[0], 0x06040200u), 0x08040201u, 0u, false), false);
                const u32x4 sel = *(const LAS u32x4*)(lds + OUT_LUT + m * 16);
                split4(t[k].x, t[k].y, m0, e0); split4(t[k].z, t[k].w, m1, e1);
                e0 = emax * 0x01010101u - e0; e1 = emax * 0x01010101u - e1;
                cm0[k] = __builtin_amdgcn_perm(m1, m0, sel.x); cm1[k] = __builtin_amdgcn_perm(m1, m0, sel.y);
                const unsigned de = __builtin_amdgcn_perm(e1, e0, sel.z), dd = __builtin_amdgcn_perm(e1, e0, sel.w);
                wide |= de | dd; cn[k] = de | (dd << 4);
                bits |= m << (8 * k); at[k] = tot; tot += __builtin_popcount(m);
            }
            const unsigned incl = wave_scan(tot), count = (unsigned)__builtin_amdgcn_readlane(incl, 63);
            const bool rawout = __builtin_amdgcn_ballot_w64((wide & 0xf0f0f0f0u) != 0u) != 0ull;
            char* dst = peer + OFF_OPK + (size_t)(o0 + h) * OITEM;
            const unsigned out_n = OUT_M + ((count + 15u) & ~15u), total = rawout ? 4096u : out_n + ((count + 1u) >> 1);
            if (!rawout) {
                *(LAS unsigned*)(buf + lane * 4) = bits;
                asm volatile("" ::: "memory");
#pragma unroll
                for (int k = 0; k < 4; ++k) {
                    const unsigned n = incl - tot + at[k];
                    const unsigned long long a = (unsigned long long)cm0[k] << ((n & 3u) * 8u), b = (unsigned long long)cm1[k] << ((n & 3u) * 8u), x = (unsigned long long)cn[k] << ((n * 4u) & 31u);
                    LAS unsigned* qm = (LAS unsigned*)(size_t)(lds0 + OUT_M + (n & ~3u));
                    LAS unsigned* qn = (LAS unsigned*)(size_t)(lds0 + out_n + ((n >> 1) & ~3u));
#define LDS_OR(p, v) __hip_atomic_fetch_or(p, v, __ATOMIC_RELAXED, __HIP_MEMORY_SCOPE_WAVEFRONT)
                    LDS_OR(qm, (unsigned)a); LDS_OR(qm + 1, (unsigned)(a >> 32) | (unsigned)b); LDS_OR(qm + 2, (unsigned)(b >> 32));
                    LDS_OR(qn, (unsigned)x); LDS_OR(qn + 1, (unsigned)(x >> 32));
                }
                asm volatile("" ::: "memory");
#pragma unroll
                for (int k = 0; k < 4; ++k) t[k] = *(const LAS u32x4*)(buf + lane * 16 + k * 1024);
                asm volatile("" : "+v"(t[0]), "+v"(t[1]), "+v"(t[2]), "+v"(t[3]) :: "memory");
            }
            PUBLISH_OUT();
            await_turn(turn, draw_turn(turn, lane), OUT_SLOTS);
#pragma unroll
            for (int k = 0; k < 4; ++k) if (lane * 16u + k * 1024u < total) store16_sys(dst, lane * 16 + k * 1024, t[k]);
            sent = o0 + h; sent_meta = (rawout ? 0x100000u : 0u) | (emax << 12) | count;
        }
        PUBLISH_OUT();
    }
}

__global__ __launch_bounds__(NTHR, 2) void ag_gemm_kernel(const Params P) {
    extern __shared__ __attribute__((aligned(16))) unsigned char shm[];
    LAS unsigned char* const lds = (LAS unsigned char*)shm;
    const int wid = __builtin_amdgcn_readfirstlane(threadIdx.x >> 6);
    char* const tick = P.wsx[0] + OFF_TICK;
    const char* const src_x = P.x; const char* const src_w = P.w;
    volatile LAS unsigned* const word = (volatile LAS unsigned*)(lds + LDS_STAGE);
    const int asked = (int)(__builtin_amdgcn_s_getreg((3 << 11) | 20) & 7u) * 32 + (int)(blockIdx.x >> 3);
    u32x4 nxt[8]; unsigned t = 0, c;
    asm volatile("s_mov_b32 exec_lo, %4\n\ts_mov_b32 exec_hi, 0\n\tglobal_atomic_add %0, %1, %2, %3 sc0\n\ts_mov_b64 exec, -1" : "+v"(t) : "v"(blockIdx.x * 64u), "v"(1u), "s"(tick), "s"(below(wid, 1) & 1u) : "memory");
    fetch(src_x, src_w, nxt, c, link_of(asked >> 5), wid * SEND_WGS + (asked & 31), below(asked & 31, SEND_WGS) & ~(below(asked >> 5, 4) & below(2, asked >> 5)), tick + (256 + asked) * 64, below(wid, 1) & 1u);
    char* peer = P.wsx[link_of(asked >> 5) & 7];
    if (threadIdx.x >= 256) lut_entry<true>(lds + PACK_LUT, threadIdx.x - 256);
    asm volatile("" : "+v"(t));
    WAIT_FETCH(nxt, c);
    asm volatile("" : "+v"(t));
    if (threadIdx.x == 0) {
        int r = asked;
        if (c != t) {
            __hip_atomic_fetch_add((GAS unsigned*)(tick + (256 + r) * 64), ~0u, __ATOMIC_RELAXED, __HIP_MEMORY_SCOPE_AGENT);
            for (int k = 1; k < 256 * 64; ++k) {
                unsigned expect = t; r = (asked + k) & 255;
                if (__hip_atomic_compare_exchange_strong((GAS unsigned*)(tick + (256 + r) * 64), &expect, t + 1u, __ATOMIC_RELAXED, __ATOMIC_RELAXED, __HIP_MEMORY_SCOPE_AGENT)) break;
            }
        }
        word[0] = t + 1u; word[1] = (unsigned)r; word[2] = 8u; word[3] = 0u;
    }
    WAIT_L(0); BAR; asm volatile("" ::: "memory");
    struct { unsigned call; int role; } claim = {(unsigned)__builtin_amdgcn_readfirstlane(word[0]), __builtin_amdgcn_readfirstlane((int)word[1]) & 255};
    const int xcd = claim.role >> 5, slot = claim.role & 31;
#define PICK(f) __builtin_amdgcn_readfirstlane(xcd == 0 ? BeforeAt<0>::f : xcd == 1 ? BeforeAt<1>::f : xcd == 2 ? BeforeAt<2>::f : xcd == 3 ? BeforeAt<3>::f : \
                                               xcd == 4 ? BeforeAt<4>::f : xcd == 5 ? BeforeAt<5>::f : xcd == 6 ? BeforeAt<6>::f : BeforeAt<7>::f)
    const int ns = nsend(xcd), nc = ncomm(xcd), ub = PICK(unp), sb = PICK(snd);
    const int xw = __builtin_amdgcn_readfirstlane(slot < ns ? NUW + sb + wid * ns + slot : ub + (slot - ns) * 8 + wid);
    if (slot < ns) {
        if (claim.role != asked) { peer = P.wsx[link_of(xcd) & 7]; fetch(src_x, src_w, nxt, c, link_of(xcd), wid * SEND_WGS + slot, ~0u, tick, 0u); WAIT_FETCH(nxt, c); }
        sender(P, peer, lds + wid * WAVE_LDS, (unsigned)(size_t)lds + PACK_LUT, word + 2, link_of(xcd), wid * SEND_WGS + slot, claim.call, nxt);
    }
    if (slot < nc) {
        if (slot >= ns) unpacker(P, lds + wid * WAVE_LDS, xw, claim.call);
        expander(P, lds + wid * WAVE_LDS, xw, claim.call);
    } else {
        const int g = slot - nc, rt = row_tile(xcd, g >> 2);
        if (xcd < 4) gemm_tile<SL, false>(P, lds, rt, g & 3, claim.call);
        else gemm_tile<SR, true>(P, lds, rt, g & 3, claim.call);
    }
}
}

extern "C" void kernel_launch(int rank, int nranks, ncclComm_t comm, hipStream_t stream,
                              void* const* d_in, const int* in_sizes, int n_in,
                              void* d_out, int out_size, const int* in_dtypes, int out_dtype,
                              void* const* d_out_all, void* const* d_ws_all, size_t ws_bytes) {
    (void)comm; (void)in_dtypes; (void)out_dtype; (void)d_out_all;
    if (nranks != NR || (unsigned)rank >= (unsigned)NR || n_in != 2 || in_sizes[0] != 1024 * KDIM || in_sizes[1] != KDIM * 1024 || out_size != KDIM * 1024 || ws_bytes < WS_NEED) {
        fprintf(stderr, "kernel_launch: unexpected problem geometry\n");
        return;
    }
    static bool attr_set = false;
    if (!attr_set) { HIP_CHECK(hipFuncSetAttribute((const void*)ag_gemm_kernel, hipFuncAttributeMaxDynamicSharedMemorySize, LDS_BYTES)); attr_set = true; }
    Params P;
    P.x = (const char*)d_in[0]; P.w = (const char*)d_in[1]; P.out = (char*)d_out;
    for (int d = 0; d < NR; ++d) P.wsx[d] = (char*)d_ws_all[renum(renum(rank) ^ d)];
    P.rank = renum(rank); P.pad = 0;
    hipLaunchKernelGGL(ag_gemm_kernel, dim3(GRID), dim3(NTHR), LDS_BYTES, stream, P);
    HIP_CHECK(hipGetLastError());
}
```

```cpp
#include <hip/hip_runtime.h>
#include <rccl/rccl.h>
#include <cstdio>

#define HIP_CHECK(expr) do { hipError_t _e = (expr); if (_e != hipSuccess) { \
    fprintf(stderr, "HIP error %s at %s:%d\n", hipGetErrorString(_e), __FILE__, __LINE__); \
    return; } } while (0)

namespace {
#define LAS __attribute__((address_space(3)))
#define GAS __attribute__((address_space(1)))
typedef short bf16x8 __attribute__((ext_vector_type(8)));
typedef short s16x4 __attribute__((ext_vector_type(4)));
typedef float f32x4 __attribute__((ext_vector_type(4)));
typedef unsigned u32x4 __attribute__((ext_vector_type(4)));
typedef unsigned u32x2 __attribute__((ext_vector_type(2)));
typedef unsigned short u16x2 __attribute__((ext_vector_type(2)));

constexpr int NR = 8;
constexpr int KDIM = 8192;
constexpr int APITCH = KDIM * 2, BPITCH = 2048, OPITCH = 2048;
constexpr int E_STRIPS = 21;
constexpr int EH = E_STRIPS, OH = 32 - EH;
constexpr int NSLOT = 3 * EH + 4 * OH;
constexpr int SL = 7;
constexpr int SR = 5;
constexpr int NTL = (32 + NSLOT + SL - 1) / SL, NTE = (EH + 3 * OH + SR - 1) / SR, NTR = NTE + (3 * EH + SR - 1) / SR, HPT = (SR + 1) / 2;
constexpr int NCH = 64;
__host__ __device__ constexpr int litems(int d) { return (d & 1) ? OH + 8 : EH; }
__host__ __device__ constexpr int lbase(int d) { return NCH * ((d >> 1) * litems(1) + ((d - 1) >> 1) * litems(0)); }
constexpr int NU = lbase(8);
constexpr int ITEM_PK = 6144;
constexpr size_t STRIP_BYTES = (size_t)32 * APITCH;
constexpr size_t XA_BYTES = NSLOT * STRIP_BYTES, W2_BYTES = (size_t)KDIM * BPITCH, PK_BYTES = (size_t)NU * ITEM_PK;
constexpr size_t OFF_XA = 0, OFF_W2 = XA_BYTES, OFF_PK = XA_BYTES + W2_BYTES, PAR_BYTES = OFF_PK + PK_BYTES;
constexpr size_t OFF_FLAG = 2 * PAR_BYTES, FLAG_BYTES = (size_t)NU * 64;
constexpr size_t OFF_CNT = OFF_FLAG + 2 * FLAG_BYTES, CNT_BYTES = (size_t)(NSLOT + 4) * NCH * 64;
constexpr size_t OFF_TICK = OFF_CNT + CNT_BYTES, TICK_BYTES = 512 * 64;
constexpr int OITEM = 4096, NOI = 4 * NTR * 8 * HPT;
constexpr size_t OFF_OPK = OFF_TICK + TICK_BYTES, OFF_OFLAG = OFF_OPK + (size_t)NOI * OITEM;
constexpr size_t OFF_TMO = OFF_OFLAG + (size_t)NOI * 64, OFF_QUEUE = OFF_TMO + 128;
constexpr size_t WS_NEED = OFF_QUEUE + 8 * 128;
static_assert(WS_NEED <= ((size_t)257 << 20), "workspace");
constexpr int GRID = 256, NTHR = 512;
constexpr int SEND_WGS = 6;
constexpr int OUT_SLOTS = 6;
constexpr int SEND_SLOTS = 3;
constexpr int TURN_SLEEP = 8;
constexpr int POLL_SLEEP = 2;
__host__ __device__ constexpr int ncomm(int x) { return x < 4 ? (x == 1 ? 16 : x == 2 ? 8 : 12) : 32 - 4 * ((NTR + x % 4) / 4); }
__host__ __device__ constexpr int row_tile(int x, int k) { return x >= 4 ? k * 4 + 7 - x : (x == 1 ? 0 : x == 0 ? 4 : x == 2 ? 9 : 15) + k; }
static_assert(NTL == 20, "row_tile");
__host__ __device__ constexpr int nsend(int x) { return x == 3 ? 0 : SEND_WGS; }
__host__ __device__ constexpr int link_of(int x) { return x < 3 ? 2 * x + 2 : x < 4 ? 1 : 2 * x - 7; }
struct Before { int unp[9], snd[9]; };
constexpr Before before() { Before b = {}; for (int x = 0; x < 8; ++x) { b.unp[x + 1] = b.unp[x] + (ncomm(x) - nsend(x)) * 8; b.snd[x + 1] = b.snd[x] + nsend(x) * 8; } return b; }
template <int X> struct BeforeAt { static constexpr int unp = before().unp[X], snd = before().snd[X]; };
constexpr int NUW = BeforeAt<8>::unp, NXW = NUW + BeforeAt<8>::snd;
constexpr unsigned long long DEADLINE = 50000000ull;
constexpr int LDS_STAGE = 135168, LDS_BYTES = LDS_STAGE + 16;
static_assert(NOI <= 8 * NXW && ncomm(2) >= nsend(2) && ncomm(3) >= 0 && ncomm(7) >= nsend(7), "roles");

__host__ __device__ constexpr int renum(int r) { return r == 3 ? 4 : r == 4 ? 3 : r; }
struct Params {
    const char* x; const char* w;
    char* out;
    char* wsx[NR];
    int rank; int pad;
};

#define WAIT_V(n) asm volatile("s_waitcnt vmcnt(" #n ")" ::: "memory")
#define WAIT_L(n) asm volatile("s_waitcnt lgkmcnt(" #n ")" ::: "memory")
#define BAR __builtin_amdgcn_s_barrier()
#define SCHED __builtin_amdgcn_sched_barrier(0)

__device__ __forceinline__ unsigned ld_agent(const char* p) { return __hip_atomic_load((const GAS unsigned*)p, __ATOMIC_RELAXED, __HIP_MEMORY_SCOPE_AGENT); }
__device__ __forceinline__ unsigned ld_sys(const char* p) { return __hip_atomic_load((const GAS unsigned*)p, __ATOMIC_RELAXED, __HIP_MEMORY_SCOPE_SYSTEM); }
struct Wait {
    unsigned long long t0; unsigned polls;
    __device__ __forceinline__ Wait() : t0(__builtin_amdgcn_s_memrealtime()), polls(0) {}
    __device__ __forceinline__ bool give_up(char* ws, unsigned call) {
        if ((++polls & 15u) == 0u && ((unsigned)__builtin_amdgcn_readfirstlane(ld_agent(ws + OFF_TMO)) == call || __builtin_amdgcn_s_memrealtime() - t0 > DEADLINE)) {
            __hip_atomic_store((GAS unsigned*)(ws + OFF_TMO), call, __ATOMIC_RELAXED, __HIP_MEMORY_SCOPE_AGENT);
            return true;
        }
        __builtin_amdgcn_s_sleep(POLL_SLEEP);
        return false;
    }
};
__device__ __forceinline__ void store16_sys(char* base, unsigned voff, u32x4 v) {
    asm volatile("global_store_dwordx4 %0, %1, %2 sc0 sc1\n\ts_nop 1" :: "v"(voff), "v"(v), "s"(base) : "memory");
}

template <int CTRL, int ROWS> __device__ __forceinline__ unsigned max_dpp(unsigned x) { const unsigned y = (unsigned)__builtin_amdgcn_update_dpp(0u, x, CTRL, ROWS, 0xf, false); return x > y ? x : y; }
__device__ __forceinline__ unsigned wave_max(unsigned x) {
    return max_dpp<0x143, 0xc>(max_dpp<0x142, 0xa>(max_dpp<0x118, 0xf>(max_dpp<0x114, 0xf>(max_dpp<0x112, 0xf>(max_dpp<0x111, 0xf>(x))))));
}
__device__ __forceinline__ unsigned nonzero2(unsigned w) { unsigned r; asm("v_pk_min_u16 %0, %1, %2" : "=v"(r) : "v"(w), "s"(0x00010001u)); return r; }
__device__ __forceinline__ unsigned below(int a, int b) { unsigned r; asm("s_ashr_i32 %0, %1, 31" : "=s"(r) : "s"(a - b) : "scc"); return r; }
__device__ __forceinline__ unsigned wave_scan(unsigned x) {
    x += __builtin_amdgcn_update_dpp(0u, x, 0x111, 0xf, 0xf, false); x += __builtin_amdgcn_update_dpp(0u, x, 0x112, 0xf, 0xf, false);
    x += __builtin_amdgcn_update_dpp(0u, x, 0x114, 0xf, 0xf, false); x += __builtin_amdgcn_update_dpp(0u, x, 0x118, 0xf, 0xf, false);
    x += __builtin_amdgcn_update_dpp(0u, x, 0x142, 0xa, 0xf, false); x += __builtin_amdgcn_update_dpp(0u, x, 0x143, 0xc, 0xf, false);
    return x;
}

__device__ __forceinline__ unsigned draw_turn(volatile LAS unsigned* turn, int lane) {
    unsigned t = 0;
    if (lane == 0) t = __hip_atomic_fetch_add((LAS unsigned*)turn, 1u, __ATOMIC_RELAXED, __HIP_MEMORY_SCOPE_WORKGROUP);
    return (unsigned)__builtin_amdgcn_readfirstlane(t);
}
__device__ __forceinline__ void await_turn(volatile LAS unsigned* turn, unsigned mine, unsigned slots) {
    while (mine >= (unsigned)__builtin_amdgcn_readfirstlane(turn[1]) + slots) __builtin_amdgcn_s_sleep(TURN_SLEEP);
}
__device__ __forceinline__ void end_turn(volatile LAS unsigned* turn) { __hip_atomic_fetch_add((LAS unsigned*)turn + 1, 1u, __ATOMIC_RELAXED, __HIP_MEMORY_SCOPE_WORKGROUP); }

constexpr int CODEC_LUT = 0, CODEC_STREAM = 2048, CODEC_LDS = CODEC_STREAM + 1088;
constexpr int WAVE_LDS = 8448, PACK_LUT = 8 * WAVE_LDS;
static_assert(CODEC_LDS <= WAVE_LDS && PACK_LUT + 2048 <= LDS_STAGE, "per-wave LDS");
__device__ __forceinline__ unsigned esc_bits(unsigned c) { return c == 0u ? 0u : ~(c | (c >> 1)) & 0x05050505u; }
__device__ __forceinline__ u32x2 lut_of(unsigned e, unsigned lut) {
    const LAS unsigned* q = (const LAS unsigned*)(size_t)__builtin_amdgcn_udot4(e, 0x80400804u, lut, false);
    return (u32x2){q[0], q[256]};
}
template <bool PACK> __device__ __forceinline__ void lut_entry(LAS unsigned char* lut, int m) {
    unsigned long long sel = 0x0c0c0c0c0c0c0c0cull; int k = 0;
#pragma unroll
    for (int b = 0; b < 8; ++b) if (m >> b & 1) { const int pos = (b & 1) | ((b & 2) << 1) | ((b & 4) >> 1), slot = (k >> 1) | ((k & 1) << 2);
        sel ^= PACK ? (unsigned long long)(0x0c ^ pos) << (8 * slot) : (unsigned long long)(0x0c ^ slot) << (8 * pos); ++k; }
    *(LAS unsigned*)(lut + m * 4) = (unsigned)sel; *(LAS unsigned*)(lut + 1024 + m * 4) = (unsigned)(sel >> 32);
}
__device__ __forceinline__ unsigned bfi(unsigned m, unsigned a, unsigned b) { return (a & m) | (b & ~m); }
__device__ __forceinline__ void split4(unsigned r0, unsigned r1, unsigned& m, unsigned& e) {
    const unsigned lo = __builtin_amdgcn_perm(r1, r0, 0x06040200u), hi = __builtin_amdgcn_perm(r1, r0, 0x07050301u);
    m = bfi(0x80808080u, hi, lo); e = bfi(0x01010101u, lo >> 7, hi << 1);
}
__device__ __forceinline__ u32x4 join8(unsigned m0, unsigned e0, unsigned m1, unsigned e1) {
    const unsigned h0 = bfi(0x80808080u, m0, e0 >> 1), l0 = bfi(0x7f7f7f7fu, m0, e0 << 7), h1 = bfi(0x80808080u, m1, e1 >> 1), l1 = bfi(0x7f7f7f7fu, m1, e1 << 7);
    return (u32x4){__builtin_amdgcn_perm(h0, l0, 0x05010400u), __builtin_amdgcn_perm(h0, l0, 0x07030602u), __builtin_amdgcn_perm(h1, l1, 0x05010400u), __builtin_amdgcn_perm(h1, l1, 0x07030602u)};
}

struct Item { bool isw; unsigned src, fin, cnt; };
__device__ __forceinline__ Item item_of(int d, int i) {
    const int chunk = (int)((d & 1) ? (unsigned)i / (unsigned)litems(1) : (unsigned)i / (unsigned)litems(0)), j = i - chunk * litems(d);
    Item it;
    it.isw = (d & 1) && j < 8;
    if (it.isw) { it.src = (unsigned)((chunk * 128 + j * 16) * BPITCH + (d >> 1) * 512); it.fin = (unsigned)(OFF_W2 + ((d >> 1) * NCH * 8 + chunk * 8 + j) * 8192);
        it.cnt = (unsigned)(OFF_CNT + ((NSLOT + (d >> 1)) * NCH + chunk) * 64); }
    else { const int strip = (d & 1) ? EH + j - 8 : j, slot = (d & 1) ? 3 * EH + (d >> 1) * OH + j - 8 : ((d >> 1) - 1) * EH + j;
        it.src = (unsigned)(strip * STRIP_BYTES + chunk * 256); it.fin = (unsigned)(OFF_XA + slot * STRIP_BYTES + chunk * 8192);
        it.cnt = (unsigned)(OFF_CNT + (slot * NCH + chunk) * 64); }
    return it;
}
__device__ __forceinline__ void fetch(const char* src_x, const char* src_w, u32x4 (&nxt)[8], unsigned& tick, int d, int i, unsigned mask, const char* word, unsigned draw) {
    const int lane = threadIdx.x & 63;
    const Item x = item_of(d, i);
    const unsigned v = x.isw ? (lane >> 5) * BPITCH + (lane & 31) * 16 : (lane >> 4) * APITCH + (lane & 15) * 16, st = x.isw ? 2 * BPITCH : 4 * APITCH;
    asm volatile("s_mov_b32 exec_lo, %21\n\ts_mov_b32 exec_hi, %21\n\t"
                 "global_load_dwordx4 %0, %9, %17\n\tglobal_load_dwordx4 %1, %10, %17\n\tglobal_load_dwordx4 %2, %11, %17\n\tglobal_load_dwordx4 %3, %12, %17\n\t"
                 "global_load_dwordx4 %4, %13, %17\n\tglobal_load_dwordx4 %5, %14, %17\n\tglobal_load_dwordx4 %6, %15, %17\n\tglobal_load_dwordx4 %7, %16, %17\n\t"
                 "s_mov_b32 exec_lo, %22\n\ts_mov_b32 exec_hi, 0\n\tglobal_atomic_add %8, %18, %19, %20 sc0\n\ts_mov_b64 exec, -1"
                 : "=&v"(nxt[0]), "=&v"(nxt[1]), "=&v"(nxt[2]), "=&v"(nxt[3]), "=&v"(nxt[4]), "=&v"(nxt[5]), "=&v"(nxt[6]), "=&v"(nxt[7]), "=&v"(tick)
                 : "v"(v), "v"(v + st), "v"(v + 2 * st), "v"(v + 3 * st), "v"(v + 4 * st), "v"(v + 5 * st), "v"(v + 6 * st), "v"(v + 7 * st),
                   "s"((x.isw ? src_w : src_x) + x.src), "v"(0u), "v"(1u), "s"(word), "s"(mask), "s"(draw) : "memory");
}
#define WAIT_FETCH(nxt, tick) asm volatile("s_waitcnt vmcnt(0)" : "+v"(tick), "+v"(nxt[0]), "+v"(nxt[1]), "+v"(nxt[2]), "+v"(nxt[3]), "+v"(nxt[4]), "+v"(nxt[5]), "+v"(nxt[6]), "+v"(nxt[7]) :: "memory")
__device__ __forceinline__ void sender(const Params& P, char* peer, LAS unsigned char* buf, unsigned lut, volatile LAS unsigned* turn, int d, int first, unsigned call, u32x4 (&nxt)[8]) {
    const int lane = threadIdx.x & 63, nlink = NCH * litems(d), nwaves = SEND_WGS * 8;
    char* const queue = P.wsx[0] + OFF_QUEUE + d * 128;
    const char* const src_x = P.x; const char* const src_w = P.w;
    const unsigned lds0 = (unsigned)(size_t)buf;
    unsigned mine = (unsigned)__builtin_amdgcn_readfirstlane(threadIdx.x >> 6), tick;
    int cur = first;
    char* const set = peer + (call & 1) * PAR_BYTES;
    char* const flags = peer + OFF_FLAG + (call & 1) * FLAG_BYTES;
    const unsigned qbase = (call - 1u) * nlink - 2 * nwaves;
    tick = qbase + first + nwaves;
    unsigned early = below((int)mine, SEND_SLOTS), late = below(SEND_SLOTS - 1, (int)mine);
    while (cur < nlink) {
        const int u = lbase(d) + cur;
        const int next = (int)((unsigned)__builtin_amdgcn_readfirstlane(tick) - qbase);
        u32x4 pk[6]; unsigned ex[16], esum = 0;
#pragma unroll
        for (int g = 0; g < 16; ++g) {
            unsigned m; split4(nxt[g >> 1][(g & 1) * 2], nxt[g >> 1][(g & 1) * 2 + 1], m, ex[g]);
            pk[g >> 2][g & 3] = m;
            if ((g & 3) == 0) esum = __builtin_amdgcn_sad_u8(ex[g], 0u, esum);
        }
        char* const dst = set + OFF_PK + (size_t)u * ITEM_PK + 2048;
#define STORE_PLANES(mask) asm volatile("s_mov_b32 exec_lo, %6\n\ts_mov_b32 exec_hi, %6\n\tglobal_store_dwordx4 %0, %1, %5 offset:-2048 sc0 sc1\n\tglobal_store_dwordx4 %0, %2, %5 offset:-1024 sc0 sc1\n\t" \
                     "global_store_dwordx4 %0, %3, %5 sc0 sc1\n\tglobal_store_dwordx4 %0, %4, %5 offset:1024 sc0 sc1\n\ts_mov_b64 exec, -1\n\ts_nop 0" \
                     :: "v"(lane * 16), "v"(pk[0]), "v"(pk[1]), "v"(pk[2]), "v"(pk[3]), "s"(dst), "s"(mask) : "memory")
        STORE_PLANES(early);
        SCHED; fetch(src_x, src_w, nxt, tick, d, next < nlink ? next : nlink - 1, ~0u, queue, 1u); SCHED;
        *(LAS u32x4*)(buf + CODEC_STREAM + lane * 16) = (u32x4){0u, 0u, 0u, 0u};
        unsigned top = (((unsigned)__builtin_amdgcn_readlane(wave_scan(esum), 63) + 1023u + 174u) >> 10) + 2u;
        top = top < 18u ? 18u : top > 255u ? 255u : top;
        unsigned cg[8], xs[8], at[8], tot = 0;
#pragma unroll
        for (int v = 0; v < 8; ++v) {
            const unsigned ua = top * 0x01010101u - ex[2 * v], ub = top * 0x01010101u - ex[2 * v + 1];
            const unsigned ca = __builtin_amdgcn_perm(0u, 0x03020100u, (((ua >> 1) & 0x04040404u) | ua) & 0x07070707u);
            const unsigned cb = __builtin_amdgcn_perm(0u, 0x0c080400u, (((ub >> 1) & 0x04040404u) | ub) & 0x07070707u);
            cg[v] = ((ua | ub) & 0xf0f0f0f0u) != 0u ? 0u : ca | cb;
            const unsigned e = esc_bits(cg[v]);
            const u32x2 sel = lut_of(e, lut);
            xs[v] = __builtin_amdgcn_perm(ub, ua, sel.x) | (__builtin_amdgcn_perm(ub, ua, sel.y) << 4);
            at[v] = tot; tot += __builtin_popcount(e);
        }
        const unsigned incl = wave_scan(tot);
        unsigned total = (unsigned)__builtin_amdgcn_readlane(incl, 63);
        if (total > 2048u) {
            total = 0;
#pragma unroll
            for (int v = 0; v < 8; ++v) cg[v] = 0u;
        } else {
            const unsigned n0 = incl - tot + (lds0 + CODEC_STREAM) * 2u;
            asm volatile("" ::: "memory");
#pragma unroll
            for (int v = 0; v < 8; ++v) {
                const unsigned n = n0 + at[v];
                const unsigned long long w = (unsigned long long)xs[v] << ((n * 4u) & 31u);
                LAS unsigned* q = (LAS unsigned*)(size_t)((n >> 1) & ~3u);
                __hip_atomic_fetch_or(q, (unsigned)w, __ATOMIC_RELAXED, __HIP_MEMORY_SCOPE_WAVEFRONT);
                __hip_atomic_fetch_or(q + 1, (unsigned)(w >> 32), __ATOMIC_RELAXED, __HIP_MEMORY_SCOPE_WAVEFRONT);
            }
        }
        asm volatile("" ::: "memory");
        pk[5] = *(const LAS u32x4*)(buf + CODEC_STREAM + lane * 16);
        unsigned least = cg[0];
#pragma unroll
        for (int v = 1; v < 8; ++v) least = cg[v] < least ? cg[v] : least;
#pragma unroll
        for (int c = 0; c < 4; ++c) pk[4][c] = cg[2 * c] | (cg[2 * c + 1] << 4);
        if (cur != first) mine = draw_turn(turn, lane);
        await_turn(turn, mine, SEND_SLOTS);
        if (__builtin_amdgcn_ballot_w64(least == 0u) != 0ull) {
            char* fin = set + item_of(d, cur).fin;
#pragma unroll
            for (int v = 0; v < 8; ++v) if (cg[v] == 0u) store16_sys(fin, v * 1024 + lane * 16, join8(pk[v >> 1][(v & 1) * 2], ex[2 * v], pk[v >> 1][(v & 1) * 2 + 1], ex[2 * v + 1]));
        }
        const unsigned long long smask = __builtin_amdgcn_ballot_w64((unsigned)lane * 32u < total);
        STORE_PLANES(late);
        asm volatile("global_store_dwordx4 %0, %1, %3 offset:2048 sc0 sc1\n\ts_mov_b64 exec, %4\n\tglobal_store_dwordx4 %0, %2, %3 offset:3072 sc0 sc1\n\ts_mov_b64 exec, -1\n\ts_nop 0"
                     :: "v"(lane * 16), "v"(pk[4]), "v"(pk[5]), "s"(dst), "s"(smask) : "memory");
        early = 0u; late = ~0u;
        WAIT_FETCH(nxt, tick);
        if (lane == 0) {
            end_turn(turn);
            __hip_atomic_store((GAS unsigned*)(flags + (size_t)u * 64), (call << 8) | top, __ATOMIC_RELAXED, __HIP_MEMORY_SCOPE_SYSTEM);
        }
        cur = next;
    }
}

__device__ __forceinline__ void unpacker(const Params& P, LAS unsigned char* buf, int uw, unsigned call) {
    const int lane = threadIdx.x & 63;
    const int d = __builtin_amdgcn_readfirstlane(uw % 7 + 1), i0 = __builtin_amdgcn_readfirstlane(uw / 7);
    const int step = __builtin_amdgcn_readfirstlane(NUW / 7 + (d <= NUW % 7 ? 1 : 0)), nlink = NCH * litems(d);
    char* ws = P.wsx[0];
    char* set = ws + (call & 1) * PAR_BYTES;
    const char* pkb = set + OFF_PK + (size_t)lbase(d) * ITEM_PK + 2048;
    const char* flags = ws + OFF_FLAG + (call & 1) * FLAG_BYTES + (size_t)lbase(d) * 64;
    const unsigned lds0 = (unsigned)(size_t)buf;
    for (int m = lane; m < 256; m += 64) lut_entry<false>(buf + CODEC_LUT, m);
    asm volatile("" ::: "memory");
    u32x4 pk[6]; unsigned nf = 0;
#pragma unroll
    for (int k = 0; k < 6; ++k) pk[k] = (u32x4){0u, 0u, 0u, 0u};
#define LOAD_ITEM(i, mask) asm volatile("s_mov_b32 exec_lo, %11\n\ts_mov_b32 exec_hi, %11\n\t" \
        "global_load_dwordx4 %0, %7, %8 offset:-2048 sc0 sc1\n\tglobal_load_dwordx4 %1, %7, %8 offset:-1024 sc0 sc1\n\tglobal_load_dwordx4 %2, %7, %8 sc0 sc1\n\t" \
        "global_load_dwordx4 %3, %7, %8 offset:1024 sc0 sc1\n\tglobal_load_dwordx4 %4, %7, %8 offset:2048 sc0 sc1\n\tglobal_load_dwordx4 %5, %7, %8 offset:3072 sc0 sc1\n\t" \
        "global_load_dword %6, %9, %10 sc0 sc1\n\ts_mov_b64 exec, -1\n\ts_waitcnt vmcnt(0)" \
        : "+v"(pk[0]), "+v"(pk[1]), "+v"(pk[2]), "+v"(pk[3]), "+v"(pk[4]), "+v"(pk[5]), "+v"(nf) \
        : "v"(lane * 16), "s"(pkb + (size_t)(i) * ITEM_PK), "v"(0u), "s"(flags + (size_t)((i) + step < nlink ? (i) + step : (i)) * 64), "s"(mask) : "memory")
    unsigned f = 0, have = 0;
    for (int i = i0; i < nlink; i += step) {
        if (!have) {
            Wait wait;
            bool dead = false;
            while (((f = (unsigned)__builtin_amdgcn_readfirstlane(ld_sys(flags + (size_t)i * 64))) >> 8) < call)
                if (wait.give_up(ws, call)) { dead = true; break; }
            if (dead) break;
            LOAD_ITEM(i, ~0u);
        }
        const unsigned top4 = (f & 255u) * 0x01010101u;
        f = (unsigned)__builtin_amdgcn_readfirstlane(nf);
        have = below(i + step, nlink) & ~below((int)(f >> 8), (int)call);
        *(LAS u32x4*)(buf + CODEC_STREAM + lane * 16) = pk[5];
        unsigned cg[8], at[8], tot = 0; u32x2 sel[8];
#pragma unroll
        for (int v = 0; v < 8; ++v) {
            cg[v] = (pk[4][v >> 1] >> (4 * (v & 1))) & 0x0f0f0f0fu;
            const unsigned e = esc_bits(cg[v]);
            sel[v] = lut_of(e, lds0 + CODEC_LUT);
            at[v] = tot; tot += __builtin_popcount(e);
        }
        const unsigned n0 = wave_scan(tot) - tot + (lds0 + CODEC_STREAM) * 2u;
        asm volatile("" ::: "memory");
        const Item it = item_of(d, i);
        unsigned xs[8];
#pragma unroll
        for (int v = 0; v < 8; ++v) {
            const unsigned n = n0 + at[v];
            const LAS unsigned* q = (const LAS unsigned*)(size_t)((n >> 1) & ~3u);
            xs[v] = __builtin_amdgcn_alignbit(q[1], q[0], n * 4u);
        }
#pragma unroll
        for (int v = 0; v < 8; ++v) {
            const unsigned xe = xs[v] & 0x0f0f0f0fu, xo = (xs[v] >> 4) & 0x0f0f0f0fu;
            const unsigned e0 = top4 - ((cg[v] & 0x03030303u) | __builtin_amdgcn_perm(xo, xe, sel[v].x));
            const unsigned e1 = top4 - (((cg[v] >> 2) & 0x03030303u) | __builtin_amdgcn_perm(xo, xe, sel[v].y));
            const u32x4 r = join8(pk[v >> 1][(v & 1) * 2], e0, pk[v >> 1][(v & 1) * 2 + 1], e1);
            if (cg[v] != 0u) store16_sys(set + it.fin, v * 1024 + lane * 16, r);
        }
        LOAD_ITEM(i + step < nlink ? i + step : i, have);
        if (lane == 0) __hip_atomic_fetch_add((GAS unsigned*)(ws + it.cnt), 1u, __ATOMIC_RELAXED, __HIP_MEMORY_SCOPE_AGENT);
    }
}

__device__ __forceinline__ int strip_of(int pos, bool remote, int q) {
    if (!remote) return pos < 32 + NSLOT ? pos : -1;
    if (pos < EH) return pos;
    if (pos < EH + 3 * OH) { const int k = (pos - EH) / OH; return 32 + 3 * EH + (k + (k >= q ? 1 : 0)) * OH + (pos - EH) % OH; }
    return (unsigned)(pos - NTE * SR) < 3u * EH ? 32 + pos - NTE * SR : -1;
}
__device__ __forceinline__ int out_row_of(int id, int rank) {
    if (id < 32) return renum(rank) * 1024 + id * 32;
    const int slot = id - 32;
    if (slot < 3 * EH) return renum(rank ^ (2 * (slot / EH + 1))) * 1024 + (slot % EH) * 32;
    const int o = slot - 3 * EH;
    return renum(rank ^ (2 * (o / OH) + 1)) * 1024 + (EH + o % OH) * 32;
}

constexpr int OUT_M = 256, OUT_LDS = 4160;
template <bool PACK> __device__ __forceinline__ u32x4 out_lut(int m) {
    unsigned long long sm = 0x0c0c0c0c0c0c0c0cull, sn = sm; int k = 0;
#pragma unroll
    for (int b = 0; b < 8; ++b) if (m >> b & 1) { const int slot = (k >> 1) | ((k & 1) << 2);
        sm ^= PACK ? (unsigned long long)(0x0c ^ b) << (8 * k) : (unsigned long long)(0x0c ^ k) << (8 * b);
        sn ^= PACK ? (unsigned long long)(0x0c ^ b) << (8 * slot) : (unsigned long long)(0x0c ^ slot) << (8 * b); ++k; }
    return (u32x4){(unsigned)sm, (unsigned)(sm >> 32), (unsigned)sn, (unsigned)(sn >> 32)};
}
struct OutItem { int o; unsigned at0, at1; };
__device__ __forceinline__ OutItem out_item(int n, int rank) {
    const bool late = n >= NTE * HPT * 32;
    const int n1 = late ? n - NTE * HPT * 32 : n, per = (late ? NTR - NTE : NTE) * 32, h = late ? n1 / ((NTR - NTE) * 32) : n1 / (NTE * 32), rt = (late ? NTE : 0) + ((n1 - h * per) >> 5);
    const int p0 = rt * SR, w = (n1 >> 2) & 7, q = n1 & 3, f0 = (w >> 2) * SR + 2 * h;
    const int ida = n < NOI ? strip_of(p0 + (f0 >> 1), true, q) : -1, idb = 2 * h + 1 < SR ? strip_of(p0 + ((f0 + 1) >> 1), true, q) : -1, src = rank ^ (2 * q + 1);
    OutItem t;
    t.o = ida < 0 ? -1 : ((q * NTR + rt) * 8 + w) * HPT + h;
    t.at0 = (unsigned)(out_row_of(ida, src) + (f0 & 1) * 16) * OPITCH + q * 512 + (w & 3) * 64;
    t.at1 = idb < 0 ? ~0u : (unsigned)(out_row_of(idb, src) + ((f0 + 1) & 1) * 16) * OPITCH + q * 512 + (w & 3) * 64;
    return t;
}
__device__ __forceinline__ void expander(const Params& P, LAS unsigned char* buf, int xw, unsigned call) {
    const int lane = threadIdx.x & 63;
    char* ws = P.wsx[0];
    const unsigned lds0 = (unsigned)(size_t)buf;
    for (int m = lane; m < 256; m += 64) *(LAS u32x4*)(buf + OUT_LDS + m * 16) = out_lut<false>(m);
    const OutItem mine = out_item(xw + (lane & 7) * NXW, P.rank);
    char* const out = P.out;
    unsigned long long todo = __builtin_amdgcn_ballot_w64(lane < 8 && mine.o >= 0);
    const char* watch = ws + OFF_OFLAG + (mine.o >= 0 ? mine.o : 0) * 64;
    while (todo) {
        unsigned long long fl, ready; bool dead = false;
        { Wait wait;
          while ((ready = __builtin_amdgcn_ballot_w64((unsigned)((fl = __hip_atomic_load((const GAS unsigned long long*)watch, __ATOMIC_RELAXED, __HIP_MEMORY_SCOPE_SYSTEM)) >> 32) >= call) & todo) == 0ull) {
              if (wait.give_up(ws, call)) { dead = true; ready = todo; break; }
          } }
        const int k = __builtin_ctzll(ready);
        todo &= ~(1ull << k);
        const unsigned f = (unsigned)__builtin_amdgcn_readlane((unsigned)fl, k);
        const int o = __builtin_amdgcn_readlane(mine.o, k);
        const unsigned at0 = (unsigned)__builtin_amdgcn_readlane(mine.at0, k), at1 = (unsigned)__builtin_amdgcn_readlane(mine.at1, k);
        u32x4 r[4];
        unsigned tmo = call;
        if (!dead) {
            u32x4 t[4];
            asm volatile("global_load_dwordx4 %0, %5, %6 sc0 sc1\n\tglobal_load_dwordx4 %1, %5, %6 offset:1024 sc0 sc1\n\tglobal_load_dwordx4 %2, %5, %6 offset:2048 sc0 sc1\n\t"
                         "global_load_dwordx4 %3, %5, %6 offset:3072 sc0 sc1\n\tglobal_load_dword %4, %7, %8 sc1\n\ts_waitcnt vmcnt(0)"
                         : "=&v"(t[0]), "=&v"(t[1]), "=&v"(t[2]), "=&v"(t[3]), "=&v"(tmo)
                         : "v"(lane * 16), "s"(ws + OFF_OPK + (size_t)o * OITEM), "v"(0u), "s"(ws + OFF_TMO) : "memory");
            if (!(f & 0x100000u)) {
#pragma unroll
                for (int k = 0; k < 4; ++k) *(LAS u32x4*)(buf + lane * 16 + k * 1024) = t[k];
                asm volatile("" ::: "memory");
                const unsigned bits = *(const LAS unsigned*)(buf + lane * 4), out_n = OUT_M + (((f & 0xfffu) + 15u) & 0xff0u);
                const unsigned tot = __builtin_popcount(bits), base4 = ((f >> 12) & 255u) * 0x01010101u;
                unsigned n = (wave_scan(tot) - tot) & 2047u;
                asm volatile("" ::: "memory");
#pragma unroll
                for (int k = 0; k < 4; ++k) {
                    const unsigned m = (bits >> (8 * k)) & 255u;
                    const u32x4 sel = *(const LAS u32x4*)(buf + OUT_LDS + m * 16);
                    const LAS unsigned* qm = (const LAS unsigned*)(size_t)(lds0 + OUT_M + (n & ~3u));
                    const LAS unsigned* qn = (const LAS unsigned*)(size_t)(lds0 + out_n + ((n >> 1) & ~3u));
                    const unsigned m0 = __builtin_amdgcn_alignbyte(qm[1], qm[0], n), m1 = __builtin_amdgcn_alignbyte(qm[2], qm[1], n), x = __builtin_amdgcn_alignbit(qn[1], qn[0], n * 4u);
                    const unsigned xe = x & 0x0f0f0f0fu, xo = (x >> 4) & 0x0f0f0f0fu;
                    const unsigned e0 = (base4 - __builtin_amdgcn_perm(xo, xe, sel.z)) & __builtin_amdgcn_perm(~0u, ~0u, sel.x);
                    const unsigned e1 = (base4 - __builtin_amdgcn_perm(xo, xe, sel.w)) & __builtin_amdgcn_perm(~0u, ~0u, sel.y);
                    t[k] = join8(__builtin_amdgcn_perm(m1, m0, sel.x), e0, __builtin_amdgcn_perm(m1, m0, sel.y), e1);
                    n += __builtin_popcount(m);
                }
                asm volatile("" ::: "memory");
            }
#pragma unroll
            for (int k = 0; k < 8; ++k) {
                const int row = (k >> 2) * 16 + (lane & 15), ch = (k & 3) * 2 + (lane >> 5);
                *(LAS u32x2*)(buf + row * 128 + ((ch ^ (row & 7)) * 16) + ((lane >> 4) & 1) * 8) = (u32x2){t[k >> 1][(k & 1) * 2], t[k >> 1][(k & 1) * 2 + 1]};
            }
            asm volatile("" ::: "memory");
#pragma unroll
            for (int v = 0; v < 4; ++v) { const int row = v * 8 + (lane >> 3); r[v] = *(const LAS u32x4*)(buf + row * 128 + (((lane & 7) ^ (row & 7)) * 16)); }
        }
        if (tmo == call) {
#pragma unroll
            for (int v = 0; v < 4; ++v) r[v] = (u32x4){0x7fc07fc0u, 0x7fc07fc0u, 0x7fc07fc0u, 0x7fc07fc0u};
        }
        const unsigned ooff = (lane >> 3) * OPITCH + ((lane >> 2) & 1) * 256 + (lane & 3) * 16;
        store16_sys(out, ooff + at0, r[0]); store16_sys(out, ooff + at0 + 8 * OPITCH, r[1]);
        if (at1 != ~0u) { store16_sys(out, ooff + at1, r[2]); store16_sys(out, ooff + at1 + 8 * OPITCH, r[3]); }
    }
}

__device__ __forceinline__ unsigned cvt_pk_bf16(float lo, float hi) { unsigned r; asm("v_cvt_pk_bf16_f32 %0, %1, %2" : "=v"(r) : "v"(lo), "v"(hi)); return r; }
template <int OFF> __device__ __forceinline__ s16x4 tr_read(unsigned addr) { s16x4 r; asm volatile("ds_read_b64_tr_b16 %0, %1 offset:%2" : "=v"(r) : "v"(addr), "i"(OFF) : "memory"); return r; }
template <int OFF> __device__ __forceinline__ bf16x8 row_read(unsigned addr) { bf16x8 r; asm volatile("ds_read_b128 %0, %1 offset:%2" : "=v"(r) : "v"(addr), "i"(OFF) : "memory"); return r; }
template <int OFF> __device__ __forceinline__ void glds(unsigned ldsaddr, unsigned voff, const char* g) {
    asm volatile("s_mov_b32 m0, %0\n\ts_nop 0\n\tglobal_load_lds_dwordx4 %1, %2 offset:%3 sc0 sc1" :: "s"(ldsaddr - OFF), "v"(voff), "s"(g), "i"(OFF) : "memory");
}
template <int OFF> __device__ __forceinline__ void glds_masked(unsigned ldsaddr, unsigned voff, const char* g, unsigned mask) {
    asm volatile("s_mov_b32 m0, %0\n\ts_mov_b32 exec_lo, %3\n\ts_mov_b32 exec_hi, %3\n\tglobal_load_lds_dwordx4 %1, %2 offset:%4 sc0 sc1\n\ts_mov_b64 exec, -1"
                 :: "s"(ldsaddr - OFF), "v"(voff), "s"(g), "s"(mask), "i"(OFF) : "memory");
}

template <int S> struct Frag { bf16x8 a[S], b[4]; };
template <int S> __device__ __forceinline__ void wait_step(Frag<S>& F, unsigned& pre) {
    asm volatile("s_waitcnt vmcnt(6) lgkmcnt(0)" : "+v"(F.a[0]), "+v"(F.a[1]), "+v"(F.a[2]), "+v"(F.a[3]), "+v"(F.a[S - 1]), "+v"(F.b[0]), "+v"(F.b[1]), "+v"(F.b[2]), "+v"(F.b[3]), "+v"(pre) :: "memory");
    if constexpr (S > 5) asm volatile("" : "+v"(F.a[4]));
    if constexpr (S > 6) asm volatile("" : "+v"(F.a[5]));
    if constexpr (S > 7) asm volatile("" : "+v"(F.a[6]));
}
template <int S, bool remote> __device__ __forceinline__ void gemm_tile(const Params& P, LAS unsigned char* lds, int rt, int ct, unsigned call) {
    constexpr int SLOT_A = S * 2048, SLOT = SLOT_A + 16384;
    static_assert(S >= 5 && S <= 8 && 4 * SLOT <= LDS_STAGE && S * 16384 <= LDS_STAGE, "tile");
    int wid = __builtin_amdgcn_readfirstlane(threadIdx.x >> 6), lane = threadIdx.x & 63, wr = wid >> 2, wc = wid & 3, fr = lane & 15, fq = lane >> 4;
    const int p0 = rt * S;
    char* ws = P.wsx[0];
    const char* set = ws + (call & 1) * PAR_BYTES;
    const char* const own_x = P.x;
    const unsigned ldst = (unsigned)(size_t)lds + wid * 1024;
    constexpr int BP = remote ? 512 : BPITCH;
    auto strip_at = [&](int k) { const int id = k < S ? strip_of(p0 + k, remote, ct) : -1; return id < 0 ? strip_of(p0, remote, ct) : id; };
    const int id0 = strip_at(wid >> 1), id1 = strip_at(4 + (wid >> 1));
    const char* const ga0 = id0 < 32 ? own_x + id0 * STRIP_BYTES : set + OFF_XA + (id0 - 32) * STRIP_BYTES;
    const char* const ga1 = id1 < 32 ? own_x + id1 * STRIP_BYTES : set + OFF_XA + (id1 - 32) * STRIP_BYTES;
    const unsigned cs0 = id0 < 32 ? 256 : 8192, cs1 = id1 < 32 ? 256 : 8192;
    const char* const gb = remote ? set + OFF_W2 + ct * (W2_BYTES / 4) : P.w + ct * 512;
    unsigned voffA0, voffA1, voffB;
    { const int k0 = lane >> 4, ch = lane & 15; voffB = (unsigned)((wid * 4 + k0) * BP + (((ch >> 1) ^ (k0 | ((wid & 2) << 1))) * 32) + (ch & 1) * 16);
      const int sb = lane * 16, swz = sb ^ (((sb >> 9) & 1) << 5), row = (wid & 1) * 16 + (swz >> 6);
      voffA0 = (unsigned)(row * (id0 < 32 ? APITCH : 256) + (swz & 63)); voffA1 = (unsigned)(row * (id1 < 32 ? APITCH : 256) + (swz & 63)); }
    const unsigned amask = below(wid, 2 * S - 8);
    unsigned aoff[2], boff[2][2];
    { const unsigned lbase = (unsigned)(size_t)lds; const int ob = fr * 64 + fq * 16; aoff[0] = lbase + wr * S * 1024 + (ob ^ (((ob >> 9) & 1) << 5)); aoff[1] = aoff[0] + 2 * SLOT;
      const int q = (lane & 15) >> 2, p = lane & 3, f = q | ((fq & 1) << 2);
      for (int n = 0; n < 2; ++n) { boff[0][n] = lbase + SLOT_A + (8 * fq + q) * 256 + p * 8 + (((wc * 2 + n) ^ f) * 32); boff[1][n] = boff[0][n] + 2 * SLOT; } }
    unsigned coff, cstep, need;
    { const int id = strip_of(p0 + (lane < S ? lane : 0), remote, ct);
      const bool watch = wid == 0 && (lane < S ? id >= 32 : (lane == 8 && remote));
      coff = watch ? (unsigned)(OFF_CNT + (lane < S ? id - 32 : NSLOT + ct) * NCH * 64) : (unsigned)OFF_TMO;
      cstep = watch ? 64u : 0u;
      need = watch ? call * (lane < S ? 1u : 8u) : 0u; }
    unsigned orow = 0;
    if constexpr (!remote) { const int f = wr * S + (lane & 7), id = strip_of(p0 + (f >> 1), false, 0); orow = (unsigned)(out_row_of(id, P.rank) + (f & 1) * 16) * OPITCH + ct * 512; }
    char* const out = remote ? P.wsx[(2 * ct + 1) & 7] : P.out;
    const int nfrag = __builtin_popcountll(__builtin_amdgcn_ballot_w64(lane < S && strip_of(p0 + ((wr * S + lane) >> 1), remote, ct) >= 0));
    const unsigned w0 = below(wid, 1);
    unsigned pre = 0;
#define SLOW_WAIT() do { Wait wait; \
        while (__builtin_amdgcn_ballot_w64(ld_agent(ws + coff) < need) != 0ull) if (wait.give_up(ws, call)) break; \
        asm volatile("" ::: "memory"); } while (0)
#define PRE_LOAD(mask) do { coff += cstep; \
        asm volatile("s_mov_b32 exec_lo, %3\n\ts_mov_b32 exec_hi, 0\n\tglobal_load_dword %0, %1, %2 sc1\n\ts_mov_b64 exec, -1" : "+v"(pre) : "v"(coff), "s"(ws), "s"((mask) & w0) : "memory"); } while (0)
#define STAGE_B(h, k, ch) glds<(h) * 256>(ldst + (k) * SLOT + SLOT_A + (h) * 8192, voffB, gb + (size_t)((ch) * 4 + (k)) * (32 * BP))
#define STAGE_A0(k, ch) glds<(k) * 64>(ldst + (k) * SLOT, voffA0, ga0 + (ch) * cs0)
#define STAGE_A1(k, ch) glds_masked<(k) * 64>(ldst + (k) * SLOT + 8192, voffA1, ga1 + (ch) * cs1, amask)
#define STAGE_ALL(k, ch) do { STAGE_B(0, k, ch); STAGE_B(1, k, ch); STAGE_A0(k, ch); STAGE_A1(k, ch); } while (0)
#define RDA(F, m, sl) F.a[m] = row_read<((sl) & 1) * SLOT + (m) * 1024>(aoff[(sl) >> 1])
#define RDB(F, j, sl) do { const s16x4 _lo = tr_read<((sl) & 1) * SLOT + ((j) >> 1) * 8192>(boff[(sl) >> 1][(j) & 1]), _hi = tr_read<((sl) & 1) * SLOT + ((j) >> 1) * 8192 + 1024>(boff[(sl) >> 1][(j) & 1]); \
        F.b[j] = __builtin_shufflevector(_lo, _hi, 0, 1, 2, 3, 4, 5, 6, 7); } while (0)
#define MM(F, i) acc[(i) >> 2][(i) & 3] = __builtin_amdgcn_mfma_f32_16x16x32_bf16(F.b[(i) & 3], F.a[(i) >> 2], acc[(i) >> 2][(i) & 3], 0, 0, 0)
#define M1(C, i) do { MM(C, i); SCHED; } while (0)
#define STEP(C, N, k) do { constexpr int nx = ((k) + 1) & 3; \
        if constexpr ((k) == 2) PRE_LOAD(c < NCH - 2 ? 0x3ffu : c < NCH - 1 ? 0x200u : 0u); \
        if constexpr (S > 5) { \
            M1(C, 0); STAGE_B(0, k, c1); M1(C, 1); RDB(N, 0, nx); M1(C, 2); STAGE_B(1, k, c1); M1(C, 3); RDB(N, 1, nx); \
            M1(C, 4); STAGE_A0(k, c1); M1(C, 5); RDB(N, 2, nx); M1(C, 6); STAGE_A1(k, c1); M1(C, 7); RDB(N, 3, nx); \
            M1(C, 8); RDA(N, 0, nx); M1(C, 9); RDA(N, 1, nx); M1(C, 10); RDA(N, 2, nx); M1(C, 11); RDA(N, 3, nx); M1(C, 12); RDA(N, 4, nx); \
            M1(C, 13); RDA(N, 5, nx); M1(C, 14); if constexpr (S > 6) RDA(N, 6, nx); if constexpr (S > 7) RDA(N, 7, nx); \
        } else { \
            M1(C, 0); RDB(N, 0, nx); M1(C, 1); RDA(N, 0, nx); M1(C, 2); STAGE_B(0, k, c1); M1(C, 3); RDB(N, 1, nx); M1(C, 4); RDA(N, 1, nx); \
            M1(C, 5); RDA(N, 2, nx); M1(C, 6); STAGE_B(1, k, c1); M1(C, 7); RDB(N, 2, nx); M1(C, 8); RDA(N, 3, nx); M1(C, 9); RDA(N, 4, nx); \
            M1(C, 10); STAGE_A0(k, c1); M1(C, 11); RDB(N, 3, nx); M1(C, 12); M1(C, 13); M1(C, 14); STAGE_A1(k, c1); \
        } \
        _Pragma("unroll") for (int i = 15; i < 4 * S - 4; ++i) M1(C, i); \
        wait_step(N, pre); \
        if constexpr ((k) == 3) { if (c < NCH - 2 && __builtin_amdgcn_ballot_w64(pre < need) != 0ull) SLOW_WAIT(); } \
        BAR; SCHED; M1(C, 4 * S - 4); M1(C, 4 * S - 3); M1(C, 4 * S - 2); M1(C, 4 * S - 1); } while (0)
    f32x4 acc[S][4];
#pragma unroll
    for (int m = 0; m < S; ++m)
#pragma unroll
        for (int j = 0; j < 4; ++j) acc[m][j] = (f32x4){0.f, 0.f, 0.f, 0.f};
    Frag<S> F0, F1;
    constexpr int OUT_LUT = LDS_STAGE - 4096;
    static_assert(!remote || (4 * SLOT <= OUT_LUT && 8 * OUT_LDS <= OUT_LUT), "pack tables");
    if (remote && threadIdx.x >= 256) *(LAS u32x4*)(lds + OUT_LUT + (threadIdx.x - 256) * 16) = out_lut<true>(threadIdx.x - 256);
    if (wid == 0) SLOW_WAIT();
    BAR;
    PRE_LOAD(0x3ffu);
    STAGE_ALL(0, 0u); STAGE_ALL(1, 0u); STAGE_ALL(2, 0u); STAGE_ALL(3, 0u);
    WAIT_V(6); BAR;
    RDB(F0, 0, 0); RDB(F0, 1, 0); RDB(F0, 2, 0); RDB(F0, 3, 0); RDA(F0, 0, 0); RDA(F0, 1, 0); RDA(F0, 2, 0); RDA(F0, 3, 0); RDA(F0, 4, 0);
    if constexpr (S > 5) RDA(F0, 5, 0);
    if constexpr (S > 6) RDA(F0, 6, 0);
    if constexpr (S > 7) RDA(F0, 7, 0);
    wait_step(F0, pre);
    if (__builtin_amdgcn_ballot_w64(pre < need) != 0ull) SLOW_WAIT();
    BAR;
    for (int c = 0; c < NCH; ++c) {
        const unsigned c1 = c < NCH - 1 ? c + 1 : c;
        STEP(F0, F1, 0); STEP(F1, F0, 1); STEP(F0, F1, 2); STEP(F1, F0, 3);
    }
    volatile LAS unsigned* turn = (volatile LAS unsigned*)(lds + LDS_STAGE + 4);
    if (threadIdx.x == 9) { *(LAS unsigned*)(lds + LDS_STAGE) = pre; turn[0] = 0u; turn[1] = 0u; }
    asm volatile("s_waitcnt vmcnt(0) lgkmcnt(0)" ::: "memory");
    BAR;
    asm volatile("" ::: "memory");
    const bool bad = *(const LAS unsigned*)(lds + LDS_STAGE) == call;
    int tz = threadIdx.x; asm volatile("" : "+v"(tz));
    lane = tz & 63; fr = lane & 15; fq = lane >> 4;
    if (__builtin_amdgcn_readfirstlane(bad)) {
#pragma unroll
        for (int m = 0; m < S; ++m)
#pragma unroll
            for (int j = 0; j < 4; ++j) acc[m][j] = (f32x4){__builtin_nanf(""), __builtin_nanf(""), __builtin_nanf(""), __builtin_nanf("")};
    }
    auto relu_pack = [&](int m, int j) {
        const f32x4 v = __builtin_elementwise_maximum(acc[m][j], (f32x4){0.f, 0.f, 0.f, 0.f});
        return (u32x2){cvt_pk_bf16(v[0], v[1]), cvt_pk_bf16(v[2], v[3])};
    };
    if constexpr (!remote) {
        LAS unsigned char* buf = lds + wid * (S * 2048);
#pragma unroll
        for (int m = 0; m < S; ++m)
#pragma unroll
            for (int j = 0; j < 4; ++j) *(LAS u32x2*)(buf + m * 2048 + fr * 128 + (((j * 2 + (fq >> 1)) ^ (fr & 7)) * 16) + (fq & 1) * 8) = relu_pack(m, j);
        asm volatile("" ::: "memory");
        u32x4 ov[S][2];
#pragma unroll
        for (int m = 0; m < S; ++m)
#pragma unroll
            for (int i = 0; i < 2; ++i) ov[m][i] = *(const LAS u32x4*)(buf + m * 2048 + i * 1024 + (lane >> 3) * 128 + (((lane & 7) ^ (lane >> 3)) * 16));
        const unsigned ooff = (lane >> 3) * OPITCH + ((lane >> 2) & 1) * 256 + wc * 64 + (lane & 3) * 16;
#pragma unroll
        for (int m = 0; m < S; ++m) {
            const unsigned at = (unsigned)__builtin_amdgcn_readlane(orow, m);
            if (m < nfrag) { store16_sys(out, ooff + at, ov[m][0]); store16_sys(out, ooff + at + 8 * OPITCH, ov[m][1]); }
        }
    } else {
        char* const peer = out;
        LAS unsigned char* buf = lds + wid * OUT_LDS;
        const unsigned lds0 = (unsigned)(size_t)buf;
        const int o0 = ((ct * NTR + rt) * 8 + wid) * HPT;
        int sent = -1; unsigned sent_meta = 0;
#define PUBLISH_OUT() do { WAIT_V(0); if (sent >= 0) { if (lane == 0) { end_turn(turn); \
            __hip_atomic_store((GAS unsigned long long*)(peer + OFF_OFLAG + sent * 64), ((unsigned long long)call << 32) | sent_meta, __ATOMIC_RELAXED, __HIP_MEMORY_SCOPE_SYSTEM); } \
            sent = -1; } } while (0)
#pragma unroll
        for (int h = 0; h < HPT; ++h) {
            if (2 * h >= nfrag) break;
            const bool second = 2 * h + 1 < nfrag;
            u32x4 t[4]; u16x2 mx = {0, 0};
#pragma unroll
            for (int k = 0; k < 8; ++k) {
                u32x2 w = relu_pack(2 * h + (k >> 2) < S ? 2 * h + (k >> 2) : 0, k & 3);
                if (k >= 4 && !second) w = (u32x2){0u, 0u};
                t[k >> 1][(k & 1) * 2] = w.x; t[k >> 1][(k & 1) * 2 + 1] = w.y;
                mx = __builtin_elementwise_max(mx, __builtin_elementwise_max(__builtin_bit_cast(u16x2, w.x & 0x7fff7fffu), __builtin_bit_cast(u16x2, w.y & 0x7fff7fffu)));
            }
#pragma unroll
            for (int k = 0; k < 3; ++k) *(LAS u32x4*)(buf + OUT_M + lane * 16 + k * 1024) = (u32x4){0u, 0u, 0u, 0u};
            const unsigned emax = (unsigned)__builtin_amdgcn_readlane(wave_max(mx.x > mx.y ? mx.x : mx.y), 63) >> 7;
            unsigned bits = 0, wide = 0, tot = 0, at[4], cm0[4], cm1[4], cn[4];
#pragma unroll
            for (int k = 0; k < 4; ++k) {
                unsigned z[4], m0, e0, m1, e1;
#pragma unroll
                for (int c = 0; c < 4; ++c) z[c] = nonzero2(t[k][c]);
                const unsigned m = __builtin_amdgcn_udot4(__builtin_amdgcn_perm(z[3], z[2], 0x06040200u), 0x80402010u, __builtin_amdgcn_udot4(__builtin_amdgcn_perm(z[1], z[0], 0x06040200u), 0x08040201u, 0u, false), false);
                const u32x4 sel = *(const LAS u32x4*)(lds + OUT_LUT + m * 16);
                split4(t[k].x, t[k].y, m0, e0); split4(t[k].z, t[k].w, m1, e1);
                e0 = emax * 0x01010101u - e0; e1 = emax * 0x01010101u - e1;
                cm0[k] = __builtin_amdgcn_perm(m1, m0, sel.x); cm1[k] = __builtin_amdgcn_perm(m1, m0, sel.y);
                const unsigned de = __builtin_amdgcn_perm(e1, e0, sel.z), dd = __builtin_amdgcn_perm(e1, e0, sel.w);
                wide |= de | dd; cn[k] = de | (dd << 4);
                bits |= m << (8 * k); at[k] = tot; tot += __builtin_popcount(m);
            }
            const unsigned incl = wave_scan(tot), count = (unsigned)__builtin_amdgcn_readlane(incl, 63);
            const bool rawout = __builtin_amdgcn_ballot_w64((wide & 0xf0f0f0f0u) != 0u) != 0ull;
            char* dst = peer + OFF_OPK + (size_t)(o0 + h) * OITEM;
            const unsigned out_n = OUT_M + ((count + 15u) & ~15u), total = rawout ? 4096u : out_n + ((count + 1u) >> 1);
            if (!rawout) {
                *(LAS unsigned*)(buf + lane * 4) = bits;
                asm volatile("" ::: "memory");
#pragma unroll
                for (int k = 0; k < 4; ++k) {
                    const unsigned n = incl - tot + at[k];
                    const unsigned long long a = (unsigned long long)cm0[k] << ((n & 3u) * 8u), b = (unsigned long long)cm1[k] << ((n & 3u) * 8u), x = (unsigned long long)cn[k] << ((n * 4u) & 31u);
                    LAS unsigned* qm = (LAS unsigned*)(size_t)(lds0 + OUT_M + (n & ~3u));
                    LAS unsigned* qn = (LAS unsigned*)(size_t)(lds0 + out_n + ((n >> 1) & ~3u));
#define LDS_OR(p, v) __hip_atomic_fetch_or(p, v, __ATOMIC_RELAXED, __HIP_MEMORY_SCOPE_WAVEFRONT)
                    LDS_OR(qm, (unsigned)a); LDS_OR(qm + 1, (unsigned)(a >> 32) | (unsigned)b); LDS_OR(qm + 2, (unsigned)(b >> 32));
                    LDS_OR(qn, (unsigned)x); LDS_OR(qn + 1, (unsigned)(x >> 32));
                }
                asm volatile("" ::: "memory");
#pragma unroll
                for (int k = 0; k < 4; ++k) t[k] = *(const LAS u32x4*)(buf + lane * 16 + k * 1024);
                asm volatile("" : "+v"(t[0]), "+v"(t[1]), "+v"(t[2]), "+v"(t[3]) :: "memory");
            }
            PUBLISH_OUT();
            await_turn(turn, draw_turn(turn, lane), OUT_SLOTS);
#pragma unroll
            for (int k = 0; k < 4; ++k) if (lane * 16u + k * 1024u < total) store16_sys(dst, lane * 16 + k * 1024, t[k]);
            sent = o0 + h; sent_meta = (rawout ? 0x100000u : 0u) | (emax << 12) | count;
        }
        PUBLISH_OUT();
    }
}

__global__ __launch_bounds__(NTHR, 2) void ag_gemm_kernel(const Params P) {
    extern __shared__ __attribute__((aligned(16))) unsigned char shm[];
    LAS unsigned char* const lds = (LAS unsigned char*)shm;
    const int wid = __builtin_amdgcn_readfirstlane(threadIdx.x >> 6);
    char* const tick = P.wsx[0] + OFF_TICK;
    const char* const src_x = P.x; const char* const src_w = P.w;
    volatile LAS unsigned* const word = (volatile LAS unsigned*)(lds + LDS_STAGE);
    const int asked = (int)(__builtin_amdgcn_s_getreg((3 << 11) | 20) & 7u) * 32 + (int)(blockIdx.x >> 3);
    u32x4 nxt[8]; unsigned t = 0, c;
    asm volatile("s_mov_b32 exec_lo, %4\n\ts_mov_b32 exec_hi, 0\n\tglobal_atomic_add %0, %1, %2, %3 sc0\n\ts_mov_b64 exec, -1" : "+v"(t) : "v"(blockIdx.x * 64u), "v"(1u), "s"(tick), "s"(below(wid, 1) & 1u) : "memory");
    fetch(src_x, src_w, nxt, c, link_of(asked >> 5), wid * SEND_WGS + (asked & 31), below(asked & 31, SEND_WGS) & ~(below(asked >> 5, 4) & below(2, asked >> 5)), tick + (256 + asked) * 64, below(wid, 1) & 1u);
    char* peer = P.wsx[link_of(asked >> 5) & 7];
    if (threadIdx.x >= 256) lut_entry<true>(lds + PACK_LUT, threadIdx.x - 256);
    asm volatile("" : "+v"(t));
    WAIT_FETCH(nxt, c);
    asm volatile("" : "+v"(t));
    if (threadIdx.x == 0) {
        int r = asked;
        if (c != t) {
            __hip_atomic_fetch_add((GAS unsigned*)(tick + (256 + r) * 64), ~0u, __ATOMIC_RELAXED, __HIP_MEMORY_SCOPE_AGENT);
            for (int k = 1; k < 256 * 64; ++k) {
                unsigned expect = t; r = (asked + k) & 255;
                if (__hip_atomic_compare_exchange_strong((GAS unsigned*)(tick + (256 + r) * 64), &expect, t + 1u, __ATOMIC_RELAXED, __ATOMIC_RELAXED, __HIP_MEMORY_SCOPE_AGENT)) break;
            }
        }
        word[0] = t + 1u; word[1] = (unsigned)r; word[2] = 8u; word[3] = 0u;
    }
    WAIT_L(0); BAR; asm volatile("" ::: "memory");
    struct { unsigned call; int role; } claim = {(unsigned)__builtin_amdgcn_readfirstlane(word[0]), __builtin_amdgcn_readfirstlane((int)word[1]) & 255};
    const int xcd = claim.role >> 5, slot = claim.role & 31;
#define PICK(f) __builtin_amdgcn_readfirstlane(xcd == 0 ? BeforeAt<0>::f : xcd == 1 ? BeforeAt<1>::f : xcd == 2 ? BeforeAt<2>::f : xcd == 3 ? BeforeAt<3>::f : \
                                               xcd == 4 ? BeforeAt<4>::f : xcd == 5 ? BeforeAt<5>::f : xcd == 6 ? BeforeAt<6>::f : BeforeAt<7>::f)
    const int ns = nsend(xcd), nc = ncomm(xcd), ub = PICK(unp), sb = PICK(snd);
    const int xw = __builtin_amdgcn_readfirstlane(slot < ns ? NUW + sb + wid * ns + slot : ub + (slot - ns) * 8 + wid);
    if (slot < ns) {
        if (claim.role != asked) { peer = P.wsx[link_of(xcd) & 7]; fetch(src_x, src_w, nxt, c, link_of(xcd), wid * SEND_WGS + slot, ~0u, tick, 0u); WAIT_FETCH(nxt, c); }
        sender(P, peer, lds + wid * WAVE_LDS, (unsigned)(size_t)lds + PACK_LUT, word + 2, link_of(xcd), wid * SEND_WGS + slot, claim.call, nxt);
    }
    if (slot < nc) {
        if (slot >= ns) unpacker(P, lds + wid * WAVE_LDS, xw, claim.call);
        expander(P, lds + wid * WAVE_LDS, xw, claim.call);
    } else {
        const int g = slot - nc, rt = row_tile(xcd, g >> 2);
        if (xcd < 4) gemm_tile<SL, false>(P, lds, rt, g & 3, claim.call);
        else gemm_tile<SR, true>(P, lds, rt, g & 3, claim.call);
    }
}
}

extern "C" void kernel_launch(int rank, int nranks, ncclComm_t comm, hipStream_t stream,
                              void* const* d_in, const int* in_sizes, int n_in,
                              void* d_out, int out_size, const int* in_dtypes, int out_dtype,
                              void* const* d_out_all, void* const* d_ws_all, size_t ws_bytes) {
    (void)comm; (void)in_dtypes; (void)out_dtype; (void)d_out_all;
    if (nranks != NR || (unsigned)rank >= (unsigned)NR || n_in != 2 || in_sizes[0] != 1024 * KDIM || in_sizes[1] != KDIM * 1024 || out_size != KDIM * 1024 || ws_bytes < WS_NEED) {
        fprintf(stderr, "kernel_launch: unexpected problem geometry\n");
        return;
    }
    static bool attr_set = false;
    if (!attr_set) { HIP_CHECK(hipFuncSetAttribute((const void*)ag_gemm_kernel, hipFuncAttributeMaxDynamicSharedMemorySize, LDS_BYTES)); attr_set = true; }
    Params P;
    P.x = (const char*)d_in[0]; P.w = (const char*)d_in[1]; P.out = (char*)d_out;
    for (int d = 0; d < NR; ++d) P.wsx[d] = (char*)d_ws_all[renum(renum(rank) ^ d)];
    P.rank = renum(rank); P.pad = 0;
    hipLaunchKernelGGL(ag_gemm_kernel, dim3(GRID), dim3(NTHR), LDS_BYTES, stream, P);
    HIP_CHECK(hipGetLastError());
}
```

```cpp
#include <hip/hip_runtime.h>
#include <rccl/rccl.h>
#include <cstdio>

#define HIP_CHECK(expr) do { hipError_t _e = (expr); if (_e != hipSuccess) { \
    fprintf(stderr, "HIP error %s at %s:%d\n", hipGetErrorString(_e), __FILE__, __LINE__); \
    return; } } while (0)

namespace {
#define LAS __attribute__((address_space(3)))
#define GAS __attribute__((address_space(1)))
typedef short bf16x8 __attribute__((ext_vector_type(8)));
typedef short s16x4 __attribute__((ext_vector_type(4)));
typedef float f32x4 __attribute__((ext_vector_type(4)));
typedef unsigned u32x4 __attribute__((ext_vector_type(4)));
typedef unsigned u32x2 __attribute__((ext_vector_type(2)));
typedef unsigned short u16x2 __attribute__((ext_vector_type(2)));

constexpr int NR = 8;
constexpr int KDIM = 8192;
constexpr int APITCH = KDIM * 2, BPITCH = 2048, OPITCH = 2048;
constexpr int E_STRIPS = 21;
constexpr int EH = E_STRIPS, OH = 32 - EH;
constexpr int NSLOT = 3 * EH + 4 * OH;
constexpr int SL = 6;
constexpr int SR = 5;
constexpr int NTL = (32 + NSLOT + SL - 1) / SL, NTE = (EH + 3 * OH + SR - 1) / SR, NTR = NTE + (3 * EH + SR - 1) / SR, HPT = (SR + 1) / 2;
constexpr int NCH = 64;
__host__ __device__ constexpr int litems(int d) { return (d & 1) ? OH + 8 : EH; }
__host__ __device__ constexpr int lbase(int d) { return NCH * ((d >> 1) * litems(1) + ((d - 1) >> 1) * litems(0)); }
constexpr int NU = lbase(8);
constexpr int ITEM = 8192;
constexpr size_t STRIP_BYTES = (size_t)32 * APITCH;
constexpr size_t XA_BYTES = NSLOT * STRIP_BYTES, W2_BYTES = (size_t)KDIM * BPITCH;
constexpr size_t OFF_XA = 0, OFF_W2 = XA_BYTES;
constexpr size_t OFF_SET = XA_BYTES + W2_BYTES, SET_PK = 0, SET_IN = (size_t)NCH * 64 * ITEM, SET_ENT = SET_IN + (size_t)7 * 64 * ITEM, SET_BYTES = SET_ENT + (size_t)NU * 8;
constexpr size_t OFF_DESC = OFF_SET + 2 * SET_BYTES, OFF_PCNT = OFF_DESC + NCH * 64 * 4;
constexpr size_t OFF_CNT = OFF_PCNT + NCH * 64, CNT_BYTES = (size_t)(NSLOT + 4) * NCH * 64;
constexpr size_t OFF_TICK = OFF_CNT + CNT_BYTES, TICK_BYTES = 512 * 64;
constexpr int OITEM = 4096, NOI = 4 * NTR * 8 * HPT;
constexpr size_t OFF_OPK = OFF_TICK + TICK_BYTES, OFF_OFLAG = OFF_OPK + (size_t)NOI * OITEM;
constexpr size_t OFF_TMO = OFF_OFLAG + (size_t)NOI * 64, OFF_QUEUE = OFF_TMO + 128;
constexpr size_t WS_NEED = OFF_QUEUE + 8 * 128;
static_assert(WS_NEED <= ((size_t)257 << 20), "workspace");
constexpr int GRID = 256, NTHR = 512;
constexpr int PULL_WGS = 6;
constexpr int PULL_SLOTS = 2;
constexpr int BOOST = 10;
constexpr int BOOST_GAP = 16;
constexpr int PULL_EXTRA = 1;
constexpr int HEAD = PULL_WGS * 8 / EH;
constexpr int OUT_SLOTS = 6;
constexpr int SEND_SLOTS = 3;
constexpr int TURN_SLEEP = 0;
constexpr int POLL_SLEEP = 2;
__host__ __device__ constexpr int ncomm(int x) { return 32 - (x < 4 ? NTL : NTR); }
__host__ __device__ constexpr int row_tile(int x, int k) { return x >= 4 ? k * 4 + 7 - x : (x == 1 ? 0 : x == 0 ? 1 : x) * (NTL / 4) + k; }
static_assert(NTL % 4 == 0 && NTR % 4 == 0, "row_tile");
__host__ __device__ constexpr int npull(int x) { return x == 3 ? 0 : PULL_WGS; }
__host__ __device__ constexpr int link_of(int x) { return x < 3 ? 2 * x + 2 : x < 4 ? 1 : 2 * x - 7; }
struct Before { int pack[9], comm[9]; };
constexpr Before before() { Before b = {}; for (int x = 0; x < 8; ++x) { b.pack[x + 1] = b.pack[x] + (ncomm(x) - npull(x)) * 8; b.comm[x + 1] = b.comm[x] + ncomm(x) * 8; } return b; }
template <int X> struct BeforeAt { static constexpr int pack = before().pack[X], comm = before().comm[X]; };
constexpr int NPW = BeforeAt<8>::pack, NXW = BeforeAt<8>::comm;
constexpr unsigned long long DEADLINE = 50000000ull;
constexpr int LDS_STAGE = 135168, LDS_BYTES = LDS_STAGE + 16;
static_assert(NOI <= 8 * NXW && ncomm(2) >= npull(2) && ncomm(7) >= npull(7) && HEAD >= 1 && HEAD * EH <= 48 && BOOST * 64 <= 6 * 24 * 7 && ncomm(0) == 8 && ncomm(4) == 8, "roles");

__host__ __device__ constexpr int renum(int r) { return r == 3 ? 4 : r == 4 ? 3 : r; }
struct Params {
    const char* x; const char* w;
    char* out;
    char* wsx[NR];
    int rank; int pad;
};

#define WAIT_V(n) asm volatile("s_waitcnt vmcnt(" #n ")" ::: "memory")
#define WAIT_L(n) asm volatile("s_waitcnt lgkmcnt(" #n ")" ::: "memory")
#define BAR __builtin_amdgcn_s_barrier()
#define SCHED __builtin_amdgcn_sched_barrier(0)

__device__ __forceinline__ unsigned ld_agent(const char* p) { return __hip_atomic_load((const GAS unsigned*)p, __ATOMIC_RELAXED, __HIP_MEMORY_SCOPE_AGENT); }
__device__ __forceinline__ unsigned ld_sys(const char* p) { return __hip_atomic_load((const GAS unsigned*)p, __ATOMIC_RELAXED, __HIP_MEMORY_SCOPE_SYSTEM); }
struct Wait {
    unsigned long long t0; unsigned polls;
    __device__ __forceinline__ Wait() : t0(__builtin_amdgcn_s_memrealtime()), polls(0) {}
    __device__ __forceinline__ bool give_up(char* ws, unsigned call) {
        if ((++polls & 15u) == 0u && ((unsigned)__builtin_amdgcn_readfirstlane(ld_agent(ws + OFF_TMO)) == call || __builtin_amdgcn_s_memrealtime() - t0 > DEADLINE)) {
            __hip_atomic_store((GAS unsigned*)(ws + OFF_TMO), call, __ATOMIC_RELAXED, __HIP_MEMORY_SCOPE_AGENT);
            return true;
        }
        __builtin_amdgcn_s_sleep(POLL_SLEEP);
        return false;
    }
};
__device__ __forceinline__ void store16_sys(char* base, unsigned voff, u32x4 v) {
    asm volatile("global_store_dwordx4 %0, %1, %2 sc0 sc1\n\ts_nop 1" :: "v"(voff), "v"(v), "s"(base) : "memory");
}

template <int CTRL, int ROWS> __device__ __forceinline__ unsigned max_dpp(unsigned x) { const unsigned y = (unsigned)__builtin_amdgcn_update_dpp(0u, x, CTRL, ROWS, 0xf, false); return x > y ? x : y; }
__device__ __forceinline__ unsigned wave_max(unsigned x) {
    return max_dpp<0x143, 0xc>(max_dpp<0x142, 0xa>(max_dpp<0x118, 0xf>(max_dpp<0x114, 0xf>(max_dpp<0x112, 0xf>(max_dpp<0x111, 0xf>(x))))));
}
__device__ __forceinline__ unsigned nonzero2(unsigned w) { unsigned r; asm("v_pk_min_u16 %0, %1, %2" : "=v"(r) : "v"(w), "s"(0x00010001u)); return r; }
__device__ __forceinline__ unsigned below(int a, int b) { unsigned r; asm("s_ashr_i32 %0, %1, 31" : "=s"(r) : "s"(a - b) : "scc"); return r; }
__device__ __forceinline__ unsigned wave_scan(unsigned x) {
    x += __builtin_amdgcn_update_dpp(0u, x, 0x111, 0xf, 0xf, false); x += __builtin_amdgcn_update_dpp(0u, x, 0x112, 0xf, 0xf, false);
    x += __builtin_amdgcn_update_dpp(0u, x, 0x114, 0xf, 0xf, false); x += __builtin_amdgcn_update_dpp(0u, x, 0x118, 0xf, 0xf, false);
    x += __builtin_amdgcn_update_dpp(0u, x, 0x142, 0xa, 0xf, false); x += __builtin_amdgcn_update_dpp(0u, x, 0x143, 0xc, 0xf, false);
    return x;
}

__device__ __forceinline__ unsigned draw_turn(volatile LAS unsigned* turn, int lane) {
    unsigned t = 0;
    if (lane == 0) t = __hip_atomic_fetch_add((LAS unsigned*)turn, 1u, __ATOMIC_RELAXED, __HIP_MEMORY_SCOPE_WORKGROUP);
    return (unsigned)__builtin_amdgcn_readfirstlane(t);
}
__device__ __forceinline__ void await_turn(volatile LAS unsigned* turn, unsigned mine, unsigned slots) {
    while (mine >= (unsigned)__builtin_amdgcn_readfirstlane(turn[1]) + slots) __builtin_amdgcn_s_sleep(TURN_SLEEP);
}
__device__ __forceinline__ void end_turn(volatile LAS unsigned* turn) { __hip_atomic_fetch_add((LAS unsigned*)turn + 1, 1u, __ATOMIC_RELAXED, __HIP_MEMORY_SCOPE_WORKGROUP); }

constexpr int CODEC_STREAM = 0, CODEC_LDS = 3088;
constexpr int WAVE_LDS = 8448, PACK_LUT = 8 * WAVE_LDS;
static_assert(CODEC_LDS <= WAVE_LDS && PACK_LUT + 4096 <= LDS_STAGE, "per-wave LDS");
__device__ __forceinline__ unsigned esc_bits(unsigned c) { return c == 0u ? 0u : ~(c | (c >> 1)) & 0x05050505u; }
__device__ __forceinline__ u32x2 lut_of(unsigned e, unsigned lut) {
    const LAS unsigned* q = (const LAS unsigned*)(size_t)__builtin_amdgcn_udot4(e, 0x80400804u, lut, false);
    return (u32x2){q[0], q[256]};
}
template <bool PACK> __device__ __forceinline__ void lut_entry(LAS unsigned char* lut, int m) {
    unsigned long long sel = 0x0c0c0c0c0c0c0c0cull; int k = 0;
#pragma unroll
    for (int b = 0; b < 8; ++b) if (m >> b & 1) { const int pos = (b & 1) | ((b & 2) << 1) | ((b & 4) >> 1), slot = (k >> 1) | ((k & 1) << 2);
        sel ^= PACK ? (unsigned long long)(0x0c ^ pos) << (8 * slot) : (unsigned long long)(0x0c ^ slot) << (8 * pos); ++k; }
    *(LAS unsigned*)(lut + m * 4) = (unsigned)sel; *(LAS unsigned*)(lut + 1024 + m * 4) = (unsigned)(sel >> 32);
}
__device__ __forceinline__ unsigned bfi(unsigned m, unsigned a, unsigned b) { return (a & m) | (b & ~m); }
__device__ __forceinline__ void split4(unsigned r0, unsigned r1, unsigned& m, unsigned& e) {
    const unsigned lo = __builtin_amdgcn_perm(r1, r0, 0x06040200u), hi = __builtin_amdgcn_perm(r1, r0, 0x07050301u);
    m = bfi(0x80808080u, hi, lo); e = bfi(0x01010101u, lo >> 7, hi << 1);
}
__device__ __forceinline__ u32x4 join8(unsigned m0, unsigned e0, unsigned m1, unsigned e1) {
    const unsigned h0 = bfi(0x80808080u, m0, e0 >> 1), l0 = bfi(0x7f7f7f7fu, m0, e0 << 7), h1 = bfi(0x80808080u, m1, e1 >> 1), l1 = bfi(0x7f7f7f7fu, m1, e1 << 7);
    return (u32x4){__builtin_amdgcn_perm(h0, l0, 0x05010400u), __builtin_amdgcn_perm(h0, l0, 0x07030602u), __builtin_amdgcn_perm(h1, l1, 0x05010400u), __builtin_amdgcn_perm(h1, l1, 0x07030602u)};
}

struct Item { int c, p; unsigned fin, cnt; };
__device__ __forceinline__ Item item_of(int d, int i) {
    Item it;
    it.c = (int)((d & 1) ? (unsigned)i / (unsigned)litems(1) : (unsigned)i / (unsigned)litems(0));
    const int j = i - it.c * litems(d);
    if ((d & 1) && j < 8) { it.p = (d >> 1) * 8 + j; it.fin = (unsigned)(OFF_W2 + ((d >> 1) * NCH * 8 + it.c * 8 + j) * 8192);
        it.cnt = (unsigned)(OFF_CNT + ((NSLOT + (d >> 1)) * NCH + it.c) * 64); }
    else { const int slot = (d & 1) ? 3 * EH + (d >> 1) * OH + j - 8 : ((d >> 1) - 1) * EH + j;
        it.p = 32 + ((d & 1) ? EH + j - 8 : j); it.fin = (unsigned)(OFF_XA + slot * STRIP_BYTES + it.c * 8192);
        it.cnt = (unsigned)(OFF_CNT + (slot * NCH + it.c) * 64); }
    return it;
}
__device__ __forceinline__ void fetch(const char* src_x, const char* src_w, u32x4 (&nxt)[8], int c, int p, unsigned mask) {
    const int lane = threadIdx.x & 63;
    const bool isw = p < 32;
    const unsigned v = isw ? (lane >> 5) * BPITCH + (lane & 31) * 16 : (lane >> 4) * APITCH + (lane & 15) * 16, st = isw ? 2 * BPITCH : 4 * APITCH;
    const unsigned off = isw ? (unsigned)((c * 128 + (p & 7) * 16) * BPITCH + (p >> 3) * 512) : (unsigned)((p - 32) * STRIP_BYTES + c * 256);
    asm volatile("s_mov_b32 exec_lo, %17\n\ts_mov_b32 exec_hi, %17\n\t"
                 "global_load_dwordx4 %0, %8, %16\n\tglobal_load_dwordx4 %1, %9, %16\n\tglobal_load_dwordx4 %2, %10, %16\n\tglobal_load_dwordx4 %3, %11, %16\n\t"
                 "global_load_dwordx4 %4, %12, %16\n\tglobal_load_dwordx4 %5, %13, %16\n\tglobal_load_dwordx4 %6, %14, %16\n\tglobal_load_dwordx4 %7, %15, %16\n\ts_mov_b64 exec, -1"
                 : "=&v"(nxt[0]), "=&v"(nxt[1]), "=&v"(nxt[2]), "=&v"(nxt[3]), "=&v"(nxt[4]), "=&v"(nxt[5]), "=&v"(nxt[6]), "=&v"(nxt[7])
                 : "v"(v), "v"(v + st), "v"(v + 2 * st), "v"(v + 3 * st), "v"(v + 4 * st), "v"(v + 5 * st), "v"(v + 6 * st), "v"(v + 7 * st),
                   "s"((isw ? src_w : src_x) + off), "s"(mask) : "memory");
}
#define WAIT_FETCH(nxt) asm volatile("s_waitcnt vmcnt(0)" : "+v"(nxt[0]), "+v"(nxt[1]), "+v"(nxt[2]), "+v"(nxt[3]), "+v"(nxt[4]), "+v"(nxt[5]), "+v"(nxt[6]), "+v"(nxt[7]) :: "memory")
#define STORE4(a, dst, mask) asm volatile("s_mov_b32 exec_lo, %6\n\ts_mov_b32 exec_hi, %6\n\tglobal_store_dwordx4 %0, %1, %5 sc0 sc1\n\tglobal_store_dwordx4 %0, %2, %5 offset:1024 sc0 sc1\n\t" \
        "global_store_dwordx4 %0, %3, %5 offset:2048 sc0 sc1\n\tglobal_store_dwordx4 %0, %4, %5 offset:3072 sc0 sc1\n\ts_mov_b64 exec, -1\n\ts_nop 0" \
        :: "v"(lane * 16), "v"((a)[0]), "v"((a)[1]), "v"((a)[2]), "v"((a)[3]), "s"(dst), "s"(mask) : "memory")
template <class F, class G> __device__ __forceinline__ unsigned pack(u32x4 (&nxt)[8], LAS unsigned char* buf, unsigned lut, char* dst, unsigned early, unsigned late, F split, G turn) {
    const int lane = threadIdx.x & 63;
    const unsigned lds0 = (unsigned)(size_t)buf;
    u32x4 pk[8]; unsigned ex[16], esum = 0;
#pragma unroll
    for (int g = 0; g < 16; ++g) {
        unsigned m; split4(nxt[g >> 1][(g & 1) * 2], nxt[g >> 1][(g & 1) * 2 + 1], m, ex[g]);
        pk[g >> 2][g & 3] = m;
        if ((g & 3) == 0) esum = __builtin_amdgcn_sad_u8(ex[g], 0u, esum);
    }
    STORE4(pk, dst, early);
    SCHED; split(); SCHED;
    *(LAS u32x4*)(buf + CODEC_STREAM + lane * 16) = (u32x4){0u, 0u, 0u, 0u};
    *(LAS u32x4*)(buf + CODEC_STREAM + 1024 + lane * 16) = (u32x4){0u, 0u, 0u, 0u};
    unsigned top = (((unsigned)__builtin_amdgcn_readlane(wave_scan(esum), 63) + 1023u + 174u) >> 10) + 2u;
    top = top < 18u ? 18u : top > 255u ? 255u : top;
    unsigned cg[8], xs[8], at[8], tot = 0;
#pragma unroll
    for (int v = 0; v < 8; ++v) {
        const unsigned ua = top * 0x01010101u - ex[2 * v], ub = top * 0x01010101u - ex[2 * v + 1];
        const unsigned ca = __builtin_amdgcn_perm(0u, 0x03020100u, (((ua >> 1) & 0x04040404u) | ua) & 0x07070707u);
        const unsigned cb = __builtin_amdgcn_perm(0u, 0x0c080400u, (((ub >> 1) & 0x04040404u) | ub) & 0x07070707u);
        cg[v] = ((ua | ub) & 0xf0f0f0f0u) != 0u ? 0u : ca | cb;
        const unsigned e = esc_bits(cg[v]);
        const u32x2 sel = lut_of(e, lut);
        xs[v] = __builtin_amdgcn_perm(ub, ua, sel.x) | (__builtin_amdgcn_perm(ub, ua, sel.y) << 4);
        at[v] = tot; tot += __builtin_popcount(e);
    }
    const unsigned incl = wave_scan(tot), n0 = incl - tot + (lds0 + CODEC_STREAM) * 2u;
    unsigned len = ((unsigned)__builtin_amdgcn_readlane(incl, 63) + 31u) >> 5;
    asm volatile("" ::: "memory");
#pragma unroll
    for (int v = 0; v < 8; ++v) {
        const unsigned n = n0 + at[v];
        const unsigned long long w = (unsigned long long)xs[v] << ((n * 4u) & 31u);
        LAS unsigned* q = (LAS unsigned*)(size_t)((n >> 1) & ~3u);
        __hip_atomic_fetch_or(q, (unsigned)w, __ATOMIC_RELAXED, __HIP_MEMORY_SCOPE_WAVEFRONT);
        __hip_atomic_fetch_or(q + 1, (unsigned)(w >> 32), __ATOMIC_RELAXED, __HIP_MEMORY_SCOPE_WAVEFRONT);
    }
    unsigned least = cg[0];
#pragma unroll
    for (int v = 1; v < 8; ++v) least = cg[v] < least ? cg[v] : least;
#define RAW_GROUP(v) join8(pk[(v) >> 1][((v) & 1) * 2], ex[2 * (v)], pk[(v) >> 1][((v) & 1) * 2 + 1], ex[2 * (v) + 1])
    if (__builtin_amdgcn_ballot_w64(least == 0u) != 0ull) {
#pragma unroll
        for (int v = 0; v < 8; ++v) {
            const unsigned long long m = __builtin_amdgcn_ballot_w64(cg[v] == 0u);
            const unsigned r = len + __builtin_amdgcn_mbcnt_hi((unsigned)(m >> 32), __builtin_amdgcn_mbcnt_lo((unsigned)m, 0u));
            if (cg[v] == 0u && r < 192u) *(LAS u32x4*)(buf + CODEC_STREAM + r * 16) = RAW_GROUP(v);
            len += __builtin_popcountll(m);
        }
    }
    asm volatile("" ::: "memory");
    if (len > 192u) {
        WAIT_V(0);
        turn();
#pragma unroll
        for (int v = 0; v < 8; ++v) store16_sys(dst, v * 1024 + lane * 16, RAW_GROUP(v));
        return 0x10000u;
    }
#pragma unroll
    for (int c = 0; c < 4; ++c) pk[4][c] = cg[2 * c] | (cg[2 * c + 1] << 4);
    pk[5] = *(const LAS u32x4*)(buf + CODEC_STREAM + lane * 16);
    turn();
    STORE4(pk, dst, late);
    store16_sys(dst, 4096 + lane * 16, pk[4]);
    if (lane < len) store16_sys(dst, 5120 + lane * 16, pk[5]);
    if (len > 64u) {
#pragma unroll
        for (int k = 1; k < 3; ++k) if (lane + k * 64u < len) store16_sys(dst, 5120 + k * 1024 + lane * 16, *(const LAS u32x4*)(buf + CODEC_STREAM + k * 1024 + lane * 16));
    }
    return (len << 8) | top;
}
__device__ __forceinline__ void packer(const Params& P, LAS unsigned char* buf, unsigned lut, int first, int last, unsigned call, u32x4 (&nxt)[8]) {
    const int lane = threadIdx.x & 63;
    char* const ws = P.wsx[0];
    const char* const src_x = P.x; const char* const src_w = P.w;
    const size_t set = OFF_SET + (call & 1) * SET_BYTES;
    for (int g = first; g < last; g += NPW) {
        const int gn = g + NPW < last ? g + NPW : g, c = g >> 6;
        const unsigned ent = pack(nxt, buf, lut, ws + set + SET_PK + (size_t)g * ITEM, 0u, ~0u, [&] { fetch(src_x, src_w, nxt, gn >> 6, gn & 63, ~0u); }, [] {});
        if (lane == 0) __hip_atomic_store((GAS unsigned*)(ws + OFF_DESC + g * 4), ent, __ATOMIC_RELAXED, __HIP_MEMORY_SCOPE_AGENT);
        WAIT_FETCH(nxt);
        unsigned done = 0;
        if (lane == 0) done = __hip_atomic_fetch_add((GAS unsigned*)(ws + OFF_PCNT + c * 64), 1u, __ATOMIC_RELAXED, __HIP_MEMORY_SCOPE_AGENT);
        if ((unsigned)__builtin_amdgcn_readfirstlane(done) + 1u == call * 64u) {
            unsigned low = ld_agent(ws + OFF_DESC + (c * 64 + lane) * 4);
            asm volatile("" : "+v"(low));
            const unsigned long long e = ((unsigned long long)call << 32) | low;
            for (int d = 1; d < 8; ++d) {
                const int j = (d & 1) ? (lane < 32 ? lane - (d >> 1) * 8 : lane - 32 - EH + 8) : lane - 32;
                const bool mine = (d & 1) ? (lane < 32 ? (unsigned)j < 8u : j >= 8) : (unsigned)j < (unsigned)EH;
                if (mine) __hip_atomic_store((GAS unsigned long long*)(P.wsx[d] + set + SET_ENT + (size_t)(lbase(d) + c * litems(d) + j) * 8), e, __ATOMIC_RELAXED, __HIP_MEMORY_SCOPE_SYSTEM);
            }
        }
    }
}

__device__ __forceinline__ unsigned len_of(unsigned f) { return (f & 0x10000u) ? 192u : (f >> 8) & 255u; }
__device__ __forceinline__ void unpack(const u32x4 (&pk)[8], unsigned f, LAS unsigned char* buf, unsigned lut, char* fin) {
    const int lane = threadIdx.x & 63;
    const unsigned lds0 = (unsigned)(size_t)buf, len = len_of(f);
    if (f & 0x10000u) {
#pragma unroll
        for (int v = 0; v < 8; ++v) store16_sys(fin, v * 1024 + lane * 16, pk[v]);
    } else {
        const unsigned top4 = (f & 255u) * 0x01010101u;
        *(LAS u32x4*)(buf + CODEC_STREAM + lane * 16) = pk[5];
        if (len > 64u) { *(LAS u32x4*)(buf + CODEC_STREAM + 1024 + lane * 16) = pk[6]; *(LAS u32x4*)(buf + CODEC_STREAM + 2048 + lane * 16) = pk[7]; }
        unsigned cg[8], at[8], tot = 0; u32x2 sel[8];
#pragma unroll
        for (int v = 0; v < 8; ++v) {
            cg[v] = (pk[4][v >> 1] >> (4 * (v & 1))) & 0x0f0f0f0fu;
            const unsigned eb = esc_bits(cg[v]);
            sel[v] = lut_of(eb, lut + 2048);
            at[v] = tot; tot += __builtin_popcount(eb);
        }
        const unsigned incl = wave_scan(tot), n0 = incl - tot + (lds0 + CODEC_STREAM) * 2u;
        unsigned raw = ((unsigned)__builtin_amdgcn_readlane(incl, 63) + 31u) >> 5;
        asm volatile("" ::: "memory");
        unsigned xs[8];
#pragma unroll
        for (int v = 0; v < 8; ++v) {
            const unsigned n = n0 + at[v];
            const LAS unsigned* q = (const LAS unsigned*)(size_t)((n >> 1) & ~3u);
            xs[v] = __builtin_amdgcn_alignbit(q[1], q[0], n * 4u);
        }
#pragma unroll
        for (int v = 0; v < 8; ++v) {
            const unsigned xe = xs[v] & 0x0f0f0f0fu, xo = (xs[v] >> 4) & 0x0f0f0f0fu;
            const unsigned e0 = top4 - ((cg[v] & 0x03030303u) | __builtin_amdgcn_perm(xo, xe, sel[v].x));
            const unsigned e1 = top4 - (((cg[v] >> 2) & 0x03030303u) | __builtin_amdgcn_perm(xo, xe, sel[v].y));
            if (cg[v] != 0u) store16_sys(fin, v * 1024 + lane * 16, join8(pk[v >> 1][(v & 1) * 2], e0, pk[v >> 1][(v & 1) * 2 + 1], e1));
        }
        if (raw < len) {
#pragma unroll
            for (int v = 0; v < 8; ++v) {
                const unsigned long long m = __builtin_amdgcn_ballot_w64(cg[v] == 0u);
                const unsigned r = raw + __builtin_amdgcn_mbcnt_hi((unsigned)(m >> 32), __builtin_amdgcn_mbcnt_lo((unsigned)m, 0u));
                if (cg[v] == 0u) store16_sys(fin, v * 1024 + lane * 16, *(const LAS u32x4*)(buf + CODEC_STREAM + (r < 192u ? r : 0u) * 16));
                raw += __builtin_popcountll(m);
            }
        }
    }
}
#define LOAD8(v, s, m5, m6, m7) "global_load_dwordx4 %0, " v ", " s " offset:-4096 sc0 sc1\n\tglobal_load_dwordx4 %1, " v ", " s " offset:-3072 sc0 sc1\n\t" \
        "global_load_dwordx4 %2, " v ", " s " offset:-2048 sc0 sc1\n\tglobal_load_dwordx4 %3, " v ", " s " offset:-1024 sc0 sc1\n\tglobal_load_dwordx4 %4, " v ", " s " sc0 sc1\n\t" \
        "s_mov_b64 exec, " m5 "\n\tglobal_load_dwordx4 %5, " v ", " s " offset:1024 sc0 sc1\n\ts_mov_b64 exec, " m6 "\n\tglobal_load_dwordx4 %6, " v ", " s " offset:2048 sc0 sc1\n\t" \
        "s_mov_b64 exec, " m7 "\n\tglobal_load_dwordx4 %7, " v ", " s " offset:3072 sc0 sc1\n\t"
__device__ __forceinline__ bool await_entry(char* ws, const char* ent, unsigned call, unsigned& low) {
    Wait wait;
    unsigned long long w;
    while ((unsigned)__builtin_amdgcn_readfirstlane((unsigned)((w = __hip_atomic_load((const GAS unsigned long long*)ent, __ATOMIC_RELAXED, __HIP_MEMORY_SCOPE_SYSTEM)) >> 32)) < call)
        if (wait.give_up(ws, call)) return false;
    low = (unsigned)w;
    return true;
}
__device__ __forceinline__ void unpack_pushed(const Params& P, LAS unsigned char* buf, unsigned lut, int k, unsigned call) {
    const int lane = threadIdx.x & 63, d = k % 7 + 1, i = k / 7;
    char* const ws = P.wsx[0];
    const size_t set = OFF_SET + (call & 1) * SET_BYTES;
    unsigned low;
    if (i >= HEAD * litems(d) || !await_entry(ws, ws + set + SET_ENT + (size_t)(lbase(d) + i) * 8, call, low)) return;
    const unsigned f = (unsigned)__builtin_amdgcn_readfirstlane(low), len = len_of(f);
    const Item it = item_of(d, i);
    u32x4 pk[8];
    pk[5] = pk[6] = pk[7] = (u32x4){0u, 0u, 0u, 0u};
    asm volatile(LOAD8("%8", "%9", "%10", "%11", "%12") "s_mov_b64 exec, -1\n\ts_waitcnt vmcnt(0)"
                 : "=&v"(pk[0]), "=&v"(pk[1]), "=&v"(pk[2]), "=&v"(pk[3]), "=&v"(pk[4]), "+v"(pk[5]), "+v"(pk[6]), "+v"(pk[7])
                 : "v"(lane * 16), "s"(ws + set + SET_IN + (size_t)((d - 1) * 64 + i) * ITEM + 4096), "s"(__builtin_amdgcn_ballot_w64(lane < len)),
                   "s"(__builtin_amdgcn_ballot_w64(lane + 64u < len)), "s"(__builtin_amdgcn_ballot_w64(lane + 128u < len)) : "memory");
    unpack(pk, f, buf, lut, ws + it.fin);
    WAIT_V(0);
    if (lane == 0) __hip_atomic_fetch_add((GAS unsigned*)(ws + it.cnt), 1u, __ATOMIC_RELAXED, __HIP_MEMORY_SCOPE_AGENT);
}
__device__ __forceinline__ void puller(const Params& P, char* peer, LAS unsigned char* buf, unsigned lut, volatile LAS unsigned* turn, int d, int first, unsigned call, u32x4 (&nxt)[8]) {
    const int lane = threadIdx.x & 63, nlink = NCH * litems(d), nhead = HEAD * litems(d), wid = __builtin_amdgcn_readfirstlane(threadIdx.x >> 6);
    char* const ws = P.wsx[0];
    const size_t set = OFF_SET + (call & 1) * SET_BYTES;
    {
        unsigned ent = 0;
        if (first < nhead) {
            ent = pack(nxt, buf, lut, peer + set + SET_IN + (size_t)((d - 1) * 64 + first) * ITEM, below(wid, SEND_SLOTS), below(SEND_SLOTS - 1, wid), [] {}, [&] { await_turn(turn, wid, SEND_SLOTS); });
            WAIT_V(0);
        } else await_turn(turn, wid, SEND_SLOTS);
        if (lane == 0) {
            end_turn(turn);
            if (first < nhead) __hip_atomic_store((GAS unsigned long long*)(peer + set + SET_ENT + (size_t)(lbase(d) + first) * 8), ((unsigned long long)call << 32) | ent, __ATOMIC_RELAXED, __HIP_MEMORY_SCOPE_SYSTEM);
        }
    }
    const char* const ents = ws + set + SET_ENT + (size_t)lbase(d) * 8;
    char* const queue = ws + OFF_QUEUE + d * 128;
    const unsigned slots = PULL_SLOTS + (first % PULL_WGS < PULL_EXTRA ? 1 : 0);
    const unsigned qbase = (call - 1u) * (nlink - nhead) - 2 * PULL_WGS * 8 - nhead;
    u32x2 e = {0u, 0u};
    unsigned pending = 0;
#define SIGNAL() do { if (pending && lane == 0) __hip_atomic_fetch_add((GAS unsigned*)(ws + pending), 1u, __ATOMIC_RELAXED, __HIP_MEMORY_SCOPE_AGENT); pending = 0; } while (0)
    for (int i = nhead + first, next = i + PULL_WGS * 8; i < nlink;) {
        if ((unsigned)__builtin_amdgcn_readfirstlane(e.y) < call) {
            WAIT_V(0); SIGNAL();
            unsigned low;
            if (!await_entry(ws, ents + i * 8, call, low)) break;
            e.x = low;
        }
        const unsigned f = (unsigned)__builtin_amdgcn_readfirstlane(e.x), len = len_of(f);
        const Item it = item_of(d, i);
        const char* src = peer + set + SET_PK + (size_t)(it.c * 64 + it.p) * ITEM;
        u32x4 pk[8]; unsigned tick;
        pk[5] = pk[6] = pk[7] = (u32x4){0u, 0u, 0u, 0u};
        const unsigned mine = draw_turn(turn, lane);
        if (mine >= (unsigned)__builtin_amdgcn_readfirstlane(turn[1]) + slots) { WAIT_V(0); SIGNAL(); await_turn(turn, mine, slots); }
        asm volatile(LOAD8("%10", "%11", "%12", "%13", "%14") "s_mov_b64 exec, 1\n\tglobal_atomic_add %8, %15, %16, %17 sc0\n\tglobal_load_dwordx2 %9, %15, %18 sc0 sc1\n\t"
                     "s_mov_b64 exec, -1\n\ts_waitcnt vmcnt(0)"
                     : "=&v"(pk[0]), "=&v"(pk[1]), "=&v"(pk[2]), "=&v"(pk[3]), "=&v"(pk[4]), "+v"(pk[5]), "+v"(pk[6]), "+v"(pk[7]), "=&v"(tick), "=&v"(e)
                     : "v"(lane * 16), "s"(src + 4096), "s"(__builtin_amdgcn_ballot_w64(lane < len)), "s"(__builtin_amdgcn_ballot_w64(lane + 64u < len)),
                       "s"(__builtin_amdgcn_ballot_w64(lane + 128u < len)), "v"(0u), "v"(1u), "s"(queue), "s"(ents + (next < nlink ? next : i) * 8) : "memory");
        if (lane == 0) end_turn(turn);
        SIGNAL();
        unpack(pk, f, buf, lut, ws + it.fin);
        pending = it.cnt;
        i = next; next = (int)((unsigned)__builtin_amdgcn_readfirstlane(tick) - qbase);
    }
    WAIT_V(0); SIGNAL();
}

__device__ __forceinline__ int strip_of(int pos, bool remote, int q) {
    if (!remote) return pos < 32 + NSLOT ? pos : -1;
    if (pos < EH) return pos;
    if (pos < EH + 3 * OH) { const int k = (pos - EH) / OH; return 32 + 3 * EH + (k + (k >= q ? 1 : 0)) * OH + (pos - EH) % OH; }
    return (unsigned)(pos - NTE * SR) < 3u * EH ? 32 + pos - NTE * SR : -1;
}
__device__ __forceinline__ int out_row_of(int id, int rank) {
    if (id < 32) return renum(rank) * 1024 + id * 32;
    const int slot = id - 32;
    if (slot < 3 * EH) return renum(rank ^ (2 * (slot / EH + 1))) * 1024 + (slot % EH) * 32;
    const int o = slot - 3 * EH;
    return renum(rank ^ (2 * (o / OH) + 1)) * 1024 + (EH + o % OH) * 32;
}

constexpr int OUT_M = 256, OUT_LDS = 4160;
template <bool PACK> __device__ __forceinline__ u32x4 out_lut(int m) {
    unsigned long long sm = 0x0c0c0c0c0c0c0c0cull, sn = sm; int k = 0;
#pragma unroll
    for (int b = 0; b < 8; ++b) if (m >> b & 1) { const int slot = (k >> 1) | ((k & 1) << 2);
        sm ^= PACK ? (unsigned long long)(0x0c ^ b) << (8 * k) : (unsigned long long)(0x0c ^ k) << (8 * b);
        sn ^= PACK ? (unsigned long long)(0x0c ^ b) << (8 * slot) : (unsigned long long)(0x0c ^ slot) << (8 * b); ++k; }
    return (u32x4){(unsigned)sm, (unsigned)(sm >> 32), (unsigned)sn, (unsigned)(sn >> 32)};
}
struct OutItem { int o; unsigned at0, at1; };
__device__ __forceinline__ OutItem out_item(int n, int rank) {
    const bool late = n >= NTE * HPT * 32;
    const int n1 = late ? n - NTE * HPT * 32 : n, per = (late ? NTR - NTE : NTE) * 32, h = late ? n1 / ((NTR - NTE) * 32) : n1 / (NTE * 32), rt = (late ? NTE : 0) + ((n1 - h * per) >> 5);
    const int p0 = rt * SR, w = (n1 >> 2) & 7, q = n1 & 3, f0 = (w >> 2) * SR + 2 * h;
    const int ida = n < NOI ? strip_of(p0 + (f0 >> 1), true, q) : -1, idb = 2 * h + 1 < SR ? strip_of(p0 + ((f0 + 1) >> 1), true, q) : -1, src = rank ^ (2 * q + 1);
    OutItem t;
    t.o = ida < 0 ? -1 : ((q * NTR + rt) * 8 + w) * HPT + h;
    t.at0 = (unsigned)(out_row_of(ida, src) + (f0 & 1) * 16) * OPITCH + q * 512 + (w & 3) * 64;
    t.at1 = idb < 0 ? ~0u : (unsigned)(out_row_of(idb, src) + ((f0 + 1) & 1) * 16) * OPITCH + q * 512 + (w & 3) * 64;
    return t;
}
__device__ __forceinline__ void expander(const Params& P, LAS unsigned char* buf, int xw, unsigned call) {
    const int lane = threadIdx.x & 63;
    char* ws = P.wsx[0];
    const unsigned lds0 = (unsigned)(size_t)buf;
    for (int m = lane; m < 256; m += 64) *(LAS u32x4*)(buf + OUT_LDS + m * 16) = out_lut<false>(m);
    const OutItem mine = out_item(xw + (lane & 7) * NXW, P.rank);
    char* const out = P.out;
    unsigned long long todo = __builtin_amdgcn_ballot_w64(lane < 8 && mine.o >= 0);
    const char* watch = ws + OFF_OFLAG + (mine.o >= 0 ? mine.o : 0) * 64;
    while (todo) {
        unsigned long long fl, ready; bool dead = false;
        { Wait wait;
          while ((ready = __builtin_amdgcn_ballot_w64((unsigned)((fl = __hip_atomic_load((const GAS unsigned long long*)watch, __ATOMIC_RELAXED, __HIP_MEMORY_SCOPE_SYSTEM)) >> 32) >= call) & todo) == 0ull) {
              if (wait.give_up(ws, call)) { dead = true; ready = todo; break; }
          } }
        const int k = __builtin_ctzll(ready);
        todo &= ~(1ull << k);
        const unsigned f = (unsigned)__builtin_amdgcn_readlane((unsigned)fl, k);
        const int o = __builtin_amdgcn_readlane(mine.o, k);
        const unsigned at0 = (unsigned)__builtin_amdgcn_readlane(mine.at0, k), at1 = (unsigned)__builtin_amdgcn_readlane(mine.at1, k);
        u32x4 r[4];
        unsigned tmo = call;
        if (!dead) {
            u32x4 t[4];
            asm volatile("global_load_dwordx4 %0, %5, %6 sc0 sc1\n\tglobal_load_dwordx4 %1, %5, %6 offset:1024 sc0 sc1\n\tglobal_load_dwordx4 %2, %5, %6 offset:2048 sc0 sc1\n\t"
                         "global_load_dwordx4 %3, %5, %6 offset:3072 sc0 sc1\n\tglobal_load_dword %4, %7, %8 sc1\n\ts_waitcnt vmcnt(0)"
                         : "=&v"(t[0]), "=&v"(t[1]), "=&v"(t[2]), "=&v"(t[3]), "=&v"(tmo)
                         : "v"(lane * 16), "s"(ws + OFF_OPK + (size_t)o * OITEM), "v"(0u), "s"(ws + OFF_TMO) : "memory");
            if (!(f & 0x100000u)) {
#pragma unroll
                for (int k = 0; k < 4; ++k) *(LAS u32x4*)(buf + lane * 16 + k * 1024) = t[k];
                asm volatile("" ::: "memory");
                const unsigned bits = *(const LAS unsigned*)(buf + lane * 4), out_n = OUT_M + (((f & 0xfffu) + 15u) & 0xff0u);
                const unsigned tot = __builtin_popcount(bits), base4 = ((f >> 12) & 255u) * 0x01010101u;
                unsigned n = (wave_scan(tot) - tot) & 2047u;
                asm volatile("" ::: "memory");
#pragma unroll
                for (int k = 0; k < 4; ++k) {
                    const unsigned m = (bits >> (8 * k)) & 255u;
                    const u32x4 sel = *(const LAS u32x4*)(buf + OUT_LDS + m * 16);
                    const LAS unsigned* qm = (const LAS unsigned*)(size_t)(lds0 + OUT_M + (n & ~3u));
                    const LAS unsigned* qn = (const LAS unsigned*)(size_t)(lds0 + out_n + ((n >> 1) & ~3u));
                    const unsigned m0 = __builtin_amdgcn_alignbyte(qm[1], qm[0], n), m1 = __builtin_amdgcn_alignbyte(qm[2], qm[1], n), x = __builtin_amdgcn_alignbit(qn[1], qn[0], n * 4u);
                    const unsigned xe = x & 0x0f0f0f0fu, xo = (x >> 4) & 0x0f0f0f0fu;
                    const unsigned e0 = (base4 - __builtin_amdgcn_perm(xo, xe, sel.z)) & __builtin_amdgcn_perm(~0u, ~0u, sel.x);
                    const unsigned e1 = (base4 - __builtin_amdgcn_perm(xo, xe, sel.w)) & __builtin_amdgcn_perm(~0u, ~0u, sel.y);
                    t[k] = join8(__builtin_amdgcn_perm(m1, m0, sel.x), e0, __builtin_amdgcn_perm(m1, m0, sel.y), e1);
                    n += __builtin_popcount(m);
                }
                asm volatile("" ::: "memory");
            }
#pragma unroll
            for (int k = 0; k < 8; ++k) {
                const int row = (k >> 2) * 16 + (lane & 15), ch = (k & 3) * 2 + (lane >> 5);
                *(LAS u32x2*)(buf + row * 128 + ((ch ^ (row & 7)) * 16) + ((lane >> 4) & 1) * 8) = (u32x2){t[k >> 1][(k & 1) * 2], t[k >> 1][(k & 1) * 2 + 1]};
            }
            asm volatile("" ::: "memory");
#pragma unroll
            for (int v = 0; v < 4; ++v) { const int row = v * 8 + (lane >> 3); r[v] = *(const LAS u32x4*)(buf + row * 128 + (((lane & 7) ^ (row & 7)) * 16)); }
        }
        if (tmo == call) {
#pragma unroll
            for (int v = 0; v < 4; ++v) r[v] = (u32x4){0x7fc07fc0u, 0x7fc07fc0u, 0x7fc07fc0u, 0x7fc07fc0u};
        }
        const unsigned ooff = (lane >> 3) * OPITCH + ((lane >> 2) & 1) * 256 + (lane & 3) * 16;
        store16_sys(out, ooff + at0, r[0]); store16_sys(out, ooff + at0 + 8 * OPITCH, r[1]);
        if (at1 != ~0u) { store16_sys(out, ooff + at1, r[2]); store16_sys(out, ooff + at1 + 8 * OPITCH, r[3]); }
    }
}

__device__ __forceinline__ unsigned cvt_pk_bf16(float lo, float hi) { unsigned r; asm("v_cvt_pk_bf16_f32 %0, %1, %2" : "=v"(r) : "v"(lo), "v"(hi)); return r; }
template <int OFF> __device__ __forceinline__ s16x4 tr_read(unsigned addr) { s16x4 r; asm volatile("ds_read_b64_tr_b16 %0, %1 offset:%2" : "=v"(r) : "v"(addr), "i"(OFF) : "memory"); return r; }
template <int OFF> __device__ __forceinline__ bf16x8 row_read(unsigned addr) { bf16x8 r; asm volatile("ds_read_b128 %0, %1 offset:%2" : "=v"(r) : "v"(addr), "i"(OFF) : "memory"); return r; }
template <int OFF> __device__ __forceinline__ void glds(unsigned ldsaddr, unsigned voff, const char* g) {
    asm volatile("s_mov_b32 m0, %0\n\ts_nop 0\n\tglobal_load_lds_dwordx4 %1, %2 offset:%3 sc0 sc1" :: "s"(ldsaddr - OFF), "v"(voff), "s"(g), "i"(OFF) : "memory");
}
template <int OFF> __device__ __forceinline__ void glds_masked(unsigned ldsaddr, unsigned voff, const char* g, unsigned mask) {
    asm volatile("s_mov_b32 m0, %0\n\ts_mov_b32 exec_lo, %3\n\ts_mov_b32 exec_hi, %3\n\tglobal_load_lds_dwordx4 %1, %2 offset:%4 sc0 sc1\n\ts_mov_b64 exec, -1"
                 :: "s"(ldsaddr - OFF), "v"(voff), "s"(g), "s"(mask), "i"(OFF) : "memory");
}

template <int S> struct Frag { bf16x8 a[S], b[4]; };
template <int S> __device__ __forceinline__ void wait_step(Frag<S>& F, unsigned& pre) {
    asm volatile("s_waitcnt vmcnt(6) lgkmcnt(0)" : "+v"(F.a[0]), "+v"(F.a[1]), "+v"(F.a[2]), "+v"(F.a[3]), "+v"(F.a[S - 1]), "+v"(F.b[0]), "+v"(F.b[1]), "+v"(F.b[2]), "+v"(F.b[3]), "+v"(pre) :: "memory");
    if constexpr (S > 5) asm volatile("" : "+v"(F.a[4]));
    if constexpr (S > 6) asm volatile("" : "+v"(F.a[5]));
    if constexpr (S > 7) asm volatile("" : "+v"(F.a[6]));
}
template <int S, bool remote> __device__ __forceinline__ void gemm_tile(const Params& P, LAS unsigned char* lds, int rt, int ct, unsigned call) {
    constexpr int SLOT_A = S * 2048, SLOT = SLOT_A + 16384;
    static_assert(S >= 5 && S <= 8 && 4 * SLOT <= LDS_STAGE && S * 16384 <= LDS_STAGE, "tile");
    int wid = __builtin_amdgcn_readfirstlane(threadIdx.x >> 6), lane = threadIdx.x & 63, wr = wid >> 2, wc = wid & 3, fr = lane & 15, fq = lane >> 4;
    const int p0 = rt * S;
    char* ws = P.wsx[0];
    const char* const set = ws;
    const char* const own_x = P.x;
    const unsigned ldst = (unsigned)(size_t)lds + wid * 1024;
    constexpr int BP = remote ? 512 : BPITCH;
    auto strip_at = [&](int k) { const int id = k < S ? strip_of(p0 + k, remote, ct) : -1; return id < 0 ? strip_of(p0, remote, ct) : id; };
    const int id0 = strip_at(wid >> 1), id1 = strip_at(4 + (wid >> 1));
    const char* const ga0 = id0 < 32 ? own_x + id0 * STRIP_BYTES : set + OFF_XA + (id0 - 32) * STRIP_BYTES;
    const char* const ga1 = id1 < 32 ? own_x + id1 * STRIP_BYTES : set + OFF_XA + (id1 - 32) * STRIP_BYTES;
    const unsigned cs0 = id0 < 32 ? 256 : 8192, cs1 = id1 < 32 ? 256 : 8192;
    const char* const gb = remote ? set + OFF_W2 + ct * (W2_BYTES / 4) : P.w + ct * 512;
    unsigned voffA0, voffA1, voffB;
    { const int k0 = lane >> 4, ch = lane & 15; voffB = (unsigned)((wid * 4 + k0) * BP + (((ch >> 1) ^ (k0 | ((wid & 2) << 1))) * 32) + (ch & 1) * 16);
      const int sb = lane * 16, swz = sb ^ (((sb >> 9) & 1) << 5), row = (wid & 1) * 16 + (swz >> 6);
      voffA0 = (unsigned)(row * (id0 < 32 ? APITCH : 256) + (swz & 63)); voffA1 = (unsigned)(row * (id1 < 32 ? APITCH : 256) + (swz & 63)); }
    const unsigned amask = below(wid, 2 * S - 8);
    unsigned aoff[2], boff[2][2];
    { const unsigned lbase = (unsigned)(size_t)lds; const int ob = fr * 64 + fq * 16; aoff[0] = lbase + wr * S * 1024 + (ob ^ (((ob >> 9) & 1) << 5)); aoff[1] = aoff[0] + 2 * SLOT;
      const int q = (lane & 15) >> 2, p = lane & 3, f = q | ((fq & 1) << 2);
      for (int n = 0; n < 2; ++n) { boff[0][n] = lbase + SLOT_A + (8 * fq + q) * 256 + p * 8 + (((wc * 2 + n) ^ f) * 32); boff[1][n] = boff[0][n] + 2 * SLOT; } }
    unsigned coff, cstep, need;
    { const int id = strip_of(p0 + (lane < S ? lane : 0), remote, ct);
      const bool watch = wid == 0 && (lane < S ? id >= 32 : (lane == 8 && remote));
      coff = watch ? (unsigned)(OFF_CNT + (lane < S ? id - 32 : NSLOT + ct) * NCH * 64) : (unsigned)OFF_TMO;
      cstep = watch ? 64u : 0u;
      need = watch ? call * (lane < S ? 1u : 8u) : 0u; }
    unsigned orow = 0;
    if constexpr (!remote) { const int f = wr * S + (lane & 7), id = strip_of(p0 + (f >> 1), false, 0); orow = (unsigned)(out_row_of(id, P.rank) + (f & 1) * 16) * OPITCH + ct * 512; }
    char* const out = remote ? P.wsx[(2 * ct + 1) & 7] : P.out;
    const int nfrag = __builtin_popcountll(__builtin_amdgcn_ballot_w64(lane < S && strip_of(p0 + ((wr * S + lane) >> 1), remote, ct) >= 0));
    const unsigned w0 = below(wid, 1);
    unsigned pre = 0;
#define SLOW_WAIT() do { Wait wait; \
        while (__builtin_amdgcn_ballot_w64(ld_agent(ws + coff) < need) != 0ull) if (wait.give_up(ws, call)) break; \
        asm volatile("" ::: "memory"); } while (0)
#define PRE_LOAD(mask) do { coff += cstep; \
        asm volatile("s_mov_b32 exec_lo, %3\n\ts_mov_b32 exec_hi, 0\n\tglobal_load_dword %0, %1, %2 sc1\n\ts_mov_b64 exec, -1" : "+v"(pre) : "v"(coff), "s"(ws), "s"((mask) & w0) : "memory"); } while (0)
#define STAGE_B(h, k, ch) glds<(h) * 256>(ldst + (k) * SLOT + SLOT_A + (h) * 8192, voffB, gb + (size_t)((ch) * 4 + (k)) * (32 * BP))
#define STAGE_A0(k, ch) glds<(k) * 64>(ldst + (k) * SLOT, voffA0, ga0 + (ch) * cs0)
#define STAGE_A1(k, ch) glds_masked<(k) * 64>(ldst + (k) * SLOT + 8192, voffA1, ga1 + (ch) * cs1, amask)
#define STAGE_ALL(k, ch) do { STAGE_B(0, k, ch); STAGE_B(1, k, ch); STAGE_A0(k, ch); STAGE_A1(k, ch); } while (0)
#define RDA(F, m, sl) F.a[m] = row_read<((sl) & 1) * SLOT + (m) * 1024>(aoff[(sl) >> 1])
#define RDB(F, j, sl) do { const s16x4 _lo = tr_read<((sl) & 1) * SLOT + ((j) >> 1) * 8192>(boff[(sl) >> 1][(j) & 1]), _hi = tr_read<((sl) & 1) * SLOT + ((j) >> 1) * 8192 + 1024>(boff[(sl) >> 1][(j) & 1]); \
        F.b[j] = __builtin_shufflevector(_lo, _hi, 0, 1, 2, 3, 4, 5, 6, 7); } while (0)
#define MM(F, i) acc[(i) >> 2][(i) & 3] = __builtin_amdgcn_mfma_f32_16x16x32_bf16(F.b[(i) & 3], F.a[(i) >> 2], acc[(i) >> 2][(i) & 3], 0, 0, 0)
#define M1(C, i) do { MM(C, i); SCHED; } while (0)
#define STEP(C, N, k) do { constexpr int nx = ((k) + 1) & 3; \
        if constexpr ((k) == 2) PRE_LOAD(c < NCH - 2 ? 0x3ffu : c < NCH - 1 ? 0x200u : 0u); \
        if constexpr (S > 5) { \
            M1(C, 0); STAGE_B(0, k, c1); M1(C, 1); RDB(N, 0, nx); M1(C, 2); STAGE_B(1, k, c1); M1(C, 3); RDB(N, 1, nx); \
            M1(C, 4); STAGE_A0(k, c1); M1(C, 5); RDB(N, 2, nx); M1(C, 6); STAGE_A1(k, c1); M1(C, 7); RDB(N, 3, nx); \
            M1(C, 8); RDA(N, 0, nx); M1(C, 9); RDA(N, 1, nx); M1(C, 10); RDA(N, 2, nx); M1(C, 11); RDA(N, 3, nx); M1(C, 12); RDA(N, 4, nx); \
            M1(C, 13); RDA(N, 5, nx); M1(C, 14); if constexpr (S > 6) RDA(N, 6, nx); if constexpr (S > 7) RDA(N, 7, nx); \
        } else { \
            M1(C, 0); RDB(N, 0, nx); M1(C, 1); RDA(N, 0, nx); M1(C, 2); STAGE_B(0, k, c1); M1(C, 3); RDB(N, 1, nx); M1(C, 4); RDA(N, 1, nx); \
            M1(C, 5); RDA(N, 2, nx); M1(C, 6); STAGE_B(1, k, c1); M1(C, 7); RDB(N, 2, nx); M1(C, 8); RDA(N, 3, nx); M1(C, 9); RDA(N, 4, nx); \
            M1(C, 10); STAGE_A0(k, c1); M1(C, 11); RDB(N, 3, nx); M1(C, 12); M1(C, 13); M1(C, 14); STAGE_A1(k, c1); \
        } \
        _Pragma("unroll") for (int i = 15; i < 4 * S - 4; ++i) M1(C, i); \
        wait_step(N, pre); \
        if constexpr ((k) == 3) { if (c < NCH - 2 && __builtin_amdgcn_ballot_w64(pre < need) != 0ull) SLOW_WAIT(); } \
        BAR; SCHED; M1(C, 4 * S - 4); M1(C, 4 * S - 3); M1(C, 4 * S - 2); M1(C, 4 * S - 1); } while (0)
    f32x4 acc[S][4];
#pragma unroll
    for (int m = 0; m < S; ++m)
#pragma unroll
        for (int j = 0; j < 4; ++j) acc[m][j] = (f32x4){0.f, 0.f, 0.f, 0.f};
    Frag<S> F0, F1;
    constexpr int OUT_LUT = LDS_STAGE - 4096;
    static_assert(!remote || (4 * SLOT <= OUT_LUT && 8 * OUT_LDS <= OUT_LUT), "pack tables");
    if (remote && threadIdx.x >= 256) *(LAS u32x4*)(lds + OUT_LUT + (threadIdx.x - 256) * 16) = out_lut<true>(threadIdx.x - 256);
    if (wid == 0) SLOW_WAIT();
    BAR;
    PRE_LOAD(0x3ffu);
    STAGE_ALL(0, 0u); STAGE_ALL(1, 0u); STAGE_ALL(2, 0u); STAGE_ALL(3, 0u);
    WAIT_V(6); BAR;
    RDB(F0, 0, 0); RDB(F0, 1, 0); RDB(F0, 2, 0); RDB(F0, 3, 0); RDA(F0, 0, 0); RDA(F0, 1, 0); RDA(F0, 2, 0); RDA(F0, 3, 0); RDA(F0, 4, 0);
    if constexpr (S > 5) RDA(F0, 5, 0);
    if constexpr (S > 6) RDA(F0, 6, 0);
    if constexpr (S > 7) RDA(F0, 7, 0);
    wait_step(F0, pre);
    if (__builtin_amdgcn_ballot_w64(pre < need) != 0ull) SLOW_WAIT();
    BAR;
    for (int c = 0; c < NCH; ++c) {
        const unsigned c1 = c < NCH - 1 ? c + 1 : c;
        STEP(F0, F1, 0); STEP(F1, F0, 1); STEP(F0, F1, 2); STEP(F1, F0, 3);
    }
    volatile LAS unsigned* turn = (volatile LAS unsigned*)(lds + LDS_STAGE + 4);
    if (threadIdx.x == 9) { *(LAS unsigned*)(lds + LDS_STAGE) = pre; turn[0] = 0u; turn[1] = 0u; }
    asm volatile("s_waitcnt vmcnt(0) lgkmcnt(0)" ::: "memory");
    BAR;
    asm volatile("" ::: "memory");
    const bool bad = *(const LAS unsigned*)(lds + LDS_STAGE) == call;
    int tz = threadIdx.x; asm volatile("" : "+v"(tz));
    lane = tz & 63; fr = lane & 15; fq = lane >> 4;
    if (__builtin_amdgcn_readfirstlane(bad)) {
#pragma unroll
        for (int m = 0; m < S; ++m)
#pragma unroll
            for (int j = 0; j < 4; ++j) acc[m][j] = (f32x4){__builtin_nanf(""), __builtin_nanf(""), __builtin_nanf(""), __builtin_nanf("")};
    }
    auto relu_pack = [&](int m, int j) {
        const f32x4 v = __builtin_elementwise_maximum(acc[m][j], (f32x4){0.f, 0.f, 0.f, 0.f});
        return (u32x2){cvt_pk_bf16(v[0], v[1]), cvt_pk_bf16(v[2], v[3])};
    };
    if constexpr (!remote) {
        LAS unsigned char* buf = lds + wid * (S * 2048);
#pragma unroll
        for (int m = 0; m < S; ++m)
#pragma unroll
            for (int j = 0; j < 4; ++j) *(LAS u32x2*)(buf + m * 2048 + fr * 128 + (((j * 2 + (fq >> 1)) ^ (fr & 7)) * 16) + (fq & 1) * 8) = relu_pack(m, j);
        asm volatile("" ::: "memory");
        u32x4 ov[S][2];
#pragma unroll
        for (int m = 0; m < S; ++m)
#pragma unroll
            for (int i = 0; i < 2; ++i) ov[m][i] = *(const LAS u32x4*)(buf + m * 2048 + i * 1024 + (lane >> 3) * 128 + (((lane & 7) ^ (lane >> 3)) * 16));
        const unsigned ooff = (lane >> 3) * OPITCH + ((lane >> 2) & 1) * 256 + wc * 64 + (lane & 3) * 16;
#pragma unroll
        for (int m = 0; m < S; ++m) {
            const unsigned at = (unsigned)__builtin_amdgcn_readlane(orow, m);
            if (m < nfrag) { store16_sys(out, ooff + at, ov[m][0]); store16_sys(out, ooff + at + 8 * OPITCH, ov[m][1]); }
        }
    } else {
        char* const peer = out;
        LAS unsigned char* buf = lds + wid * OUT_LDS;
        const unsigned lds0 = (unsigned)(size_t)buf;
        const int o0 = ((ct * NTR + rt) * 8 + wid) * HPT;
        int sent = -1; unsigned sent_meta = 0;
#define PUBLISH_OUT() do { WAIT_V(0); if (sent >= 0) { if (lane == 0) { end_turn(turn); \
            __hip_atomic_store((GAS unsigned long long*)(peer + OFF_OFLAG + sent * 64), ((unsigned long long)call << 32) | sent_meta, __ATOMIC_RELAXED, __HIP_MEMORY_SCOPE_SYSTEM); } \
            sent = -1; } } while (0)
#pragma unroll
        for (int h = 0; h < HPT; ++h) {
            if (2 * h >= nfrag) break;
            const bool second = 2 * h + 1 < nfrag;
            u32x4 t[4]; u16x2 mx = {0, 0};
#pragma unroll
            for (int k = 0; k < 8; ++k) {
                u32x2 w = relu_pack(2 * h + (k >> 2) < S ? 2 * h + (k >> 2) : 0, k & 3);
                if (k >= 4 && !second) w = (u32x2){0u, 0u};
                t[k >> 1][(k & 1) * 2] = w.x; t[k >> 1][(k & 1) * 2 + 1] = w.y;
                mx = __builtin_elementwise_max(mx, __builtin_elementwise_max(__builtin_bit_cast(u16x2, w.x & 0x7fff7fffu), __builtin_bit_cast(u16x2, w.y & 0x7fff7fffu)));
            }
#pragma unroll
            for (int k = 0; k < 3; ++k) *(LAS u32x4*)(buf + OUT_M + lane * 16 + k * 1024) = (u32x4){0u, 0u, 0u, 0u};
            const unsigned emax = (unsigned)__builtin_amdgcn_readlane(wave_max(mx.x > mx.y ? mx.x : mx.y), 63) >> 7;
            unsigned bits = 0, wide = 0, tot = 0, at[4], cm0[4], cm1[4], cn[4];
#pragma unroll
            for (int k = 0; k < 4; ++k) {
                unsigned z[4], m0, e0, m1, e1;
#pragma unroll
                for (int c = 0; c < 4; ++c) z[c] = nonzero2(t[k][c]);
                const unsigned m = __builtin_amdgcn_udot4(__builtin_amdgcn_perm(z[3], z[2], 0x06040200u), 0x80402010u, __builtin_amdgcn_udot4(__builtin_amdgcn_perm(z[1], z[0], 0x06040200u), 0x08040201u, 0u, false), false);
                const u32x4 sel = *(const LAS u32x4*)(lds + OUT_LUT + m * 16);
                split4(t[k].x, t[k].y, m0, e0); split4(t[k].z, t[k].w, m1, e1);
                e0 = emax * 0x01010101u - e0; e1 = emax * 0x01010101u - e1;
                cm0[k] = __builtin_amdgcn_perm(m1, m0, sel.x); cm1[k] = __builtin_amdgcn_perm(m1, m0, sel.y);
                const unsigned de = __builtin_amdgcn_perm(e1, e0, sel.z), dd = __builtin_amdgcn_perm(e1, e0, sel.w);
                wide |= de | dd; cn[k] = de | (dd << 4);
                bits |= m << (8 * k); at[k] = tot; tot += __builtin_popcount(m);
            }
            const unsigned incl = wave_scan(tot), count = (unsigned)__builtin_amdgcn_readlane(incl, 63);
            const bool rawout = __builtin_amdgcn_ballot_w64((wide & 0xf0f0f0f0u) != 0u) != 0ull;
            char* dst = peer + OFF_OPK + (size_t)(o0 + h) * OITEM;
            const unsigned out_n = OUT_M + ((count + 15u) & ~15u), total = rawout ? 4096u : out_n + ((count + 1u) >> 1);
            if (!rawout) {
                *(LAS unsigned*)(buf + lane * 4) = bits;
                asm volatile("" ::: "memory");
#pragma unroll
                for (int k = 0; k < 4; ++k) {
                    const unsigned n = incl - tot + at[k];
                    const unsigned long long a = (unsigned long long)cm0[k] << ((n & 3u) * 8u), b = (unsigned long long)cm1[k] << ((n & 3u) * 8u), x = (unsigned long long)cn[k] << ((n * 4u) & 31u);
                    LAS unsigned* qm = (LAS unsigned*)(size_t)(lds0 + OUT_M + (n & ~3u));
                    LAS unsigned* qn = (LAS unsigned*)(size_t)(lds0 + out_n + ((n >> 1) & ~3u));
#define LDS_OR(p, v) __hip_atomic_fetch_or(p, v, __ATOMIC_RELAXED, __HIP_MEMORY_SCOPE_WAVEFRONT)
                    LDS_OR(qm, (unsigned)a); LDS_OR(qm + 1, (unsigned)(a >> 32) | (unsigned)b); LDS_OR(qm + 2, (unsigned)(b >> 32));
                    LDS_OR(qn, (unsigned)x); LDS_OR(qn + 1, (unsigned)(x >> 32));
                }
                asm volatile("" ::: "memory");
#pragma unroll
                for (int k = 0; k < 4; ++k) t[k] = *(const LAS u32x4*)(buf + lane * 16 + k * 1024);
                asm volatile("" : "+v"(t[0]), "+v"(t[1]), "+v"(t[2]), "+v"(t[3]) :: "memory");
            }
            PUBLISH_OUT();
            await_turn(turn, draw_turn(turn, lane), OUT_SLOTS);
#pragma unroll
            for (int k = 0; k < 4; ++k) if (lane * 16u + k * 1024u < total) store16_sys(dst, lane * 16 + k * 1024, t[k]);
            sent = o0 + h; sent_meta = (rawout ? 0x100000u : 0u) | (emax << 12) | count;
        }
        PUBLISH_OUT();
    }
}

__global__ __launch_bounds__(NTHR, 2) void ag_gemm_kernel(const Params P) {
    extern __shared__ __attribute__((aligned(16))) unsigned char shm[];
    LAS unsigned char* const lds = (LAS unsigned char*)shm;
    const int wid = __builtin_amdgcn_readfirstlane(threadIdx.x >> 6);
    char* const tick = P.wsx[0] + OFF_TICK;
    const char* const src_x = P.x; const char* const src_w = P.w;
    volatile LAS unsigned* const word = (volatile LAS unsigned*)(lds + LDS_STAGE);
    const int asked = (int)(__builtin_amdgcn_s_getreg((3 << 11) | 20) & 7u) * 32 + (int)(blockIdx.x >> 3);
#define PICK(x, f) __builtin_amdgcn_readfirstlane(x == 0 ? BeforeAt<0>::f : x == 1 ? BeforeAt<1>::f : x == 2 ? BeforeAt<2>::f : x == 3 ? BeforeAt<3>::f : \
                                                  x == 4 ? BeforeAt<4>::f : x == 5 ? BeforeAt<5>::f : x == 6 ? BeforeAt<6>::f : BeforeAt<7>::f)
    u32x4 nxt[8]; unsigned t = 0, c = 0;
    auto first_piece = [&](int x, int k) {
        if (k < ncomm(x)) return (HEAD + BOOST) * 64 + PICK(x, pack) + (k - npull(x)) * 8 + wid;
        const int n = (wid * 24 + k - ncomm(x)) * 7 + (x ? x - 1 : 0);
        return x != 1 && wid < 6 && n < BOOST * 64 ? HEAD * 64 + n : NCH * 64;
    };
    auto fetch_first = [&](int role, unsigned now) {
        const int x = role >> 5, k = role & 31, g = __builtin_amdgcn_readfirstlane(first_piece(x, k));
        const Item it = item_of(link_of(x), wid * PULL_WGS + k);
        fetch(src_x, src_w, nxt, k < npull(x) ? it.c : (g >> 6) & 63, k < npull(x) ? it.p : g & 63, below(g, NCH * 64) & now);
    };
    asm volatile("s_mov_b32 exec_lo, %6\n\ts_mov_b32 exec_hi, 0\n\tglobal_atomic_add %0, %2, %4, %5 sc0\n\tglobal_atomic_add %1, %3, %4, %5 sc0\n\ts_mov_b64 exec, -1"
                 : "+v"(t), "+v"(c) : "v"(blockIdx.x * 64u), "v"((256u + asked) * 64u), "v"(1u), "s"(tick), "s"(below(wid, 1) & 1u) : "memory");
    fetch_first(asked, below(asked & 24, 1) | below(wid, 1));
    char* peer = P.wsx[link_of(asked >> 5) & 7];
    if (threadIdx.x >= 256) { lut_entry<true>(lds + PACK_LUT, threadIdx.x - 256); lut_entry<false>(lds + PACK_LUT + 2048, threadIdx.x - 256); }
    asm volatile("" : "+v"(t), "+v"(c));
    WAIT_FETCH(nxt);
    asm volatile("" : "+v"(t), "+v"(c));
    if (threadIdx.x == 0) {
        int r = asked;
        if (c != t) {
            __hip_atomic_fetch_add((GAS unsigned*)(tick + (256 + r) * 64), ~0u, __ATOMIC_RELAXED, __HIP_MEMORY_SCOPE_AGENT);
            for (int k = 1; k < 256 * 64; ++k) {
                unsigned expect = t; r = (asked + k) & 255;
                if (__hip_atomic_compare_exchange_strong((GAS unsigned*)(tick + (256 + r) * 64), &expect, t + 1u, __ATOMIC_RELAXED, __ATOMIC_RELAXED, __HIP_MEMORY_SCOPE_AGENT)) break;
            }
            __builtin_amdgcn_s_waitcnt(0x0f70);
        }
        word[0] = t + 1u; word[1] = (unsigned)r; word[2] = 8u; word[3] = 0u;
    }
    WAIT_L(0); BAR; asm volatile("" ::: "memory");
    struct { unsigned call; int role; } claim = {(unsigned)__builtin_amdgcn_readfirstlane(word[0]), __builtin_amdgcn_readfirstlane((int)word[1]) & 255};
    const int xcd = claim.role >> 5, slot = claim.role & 31;
    const int np = npull(xcd), nc = ncomm(xcd);
    if (claim.role != asked || (slot >= nc && wid > 0)) {
        if (slot >= nc) for (int k = 0; k < wid; ++k) __builtin_amdgcn_s_sleep(BOOST_GAP);
        peer = P.wsx[link_of(xcd) & 7]; fetch_first(claim.role, ~0u); WAIT_FETCH(nxt);
    }
    if (slot < np) puller(P, peer, lds + wid * WAVE_LDS, (unsigned)(size_t)lds + PACK_LUT, word + 2, link_of(xcd), wid * PULL_WGS + slot, claim.call, nxt);
    else { const int g = first_piece(xcd, slot); packer(P, lds + wid * WAVE_LDS, (unsigned)(size_t)lds + PACK_LUT, g, slot < nc || g >= NCH * 64 ? NCH * 64 : g + 1, claim.call, nxt); }
    if (slot < nc) {
        expander(P, lds + wid * WAVE_LDS, PICK(xcd, comm) + wid * nc + slot, claim.call);
    } else {
        const int g = slot - nc, rt = row_tile(xcd, g >> 2);
        if (xcd != 1 && wid >= 6 && g < 24) unpack_pushed(P, lds + wid * WAVE_LDS, (unsigned)(size_t)lds + PACK_LUT, (g * 2 + wid - 6) * 7 + (xcd ? xcd - 1 : 0), claim.call);
        if (xcd < 4) gemm_tile<SL, false>(P, lds, rt, g & 3, claim.call);
        else gemm_tile<SR, true>(P, lds, rt, g & 3, claim.call);
    }
}
}

extern "C" void kernel_launch(int rank, int nranks, ncclComm_t comm, hipStream_t stream,
                              void* const* d_in, const int* in_sizes, int n_in,
                              void* d_out, int out_size, const int* in_dtypes, int out_dtype,
                              void* const* d_out_all, void* const* d_ws_all, size_t ws_bytes) {
    (void)comm; (void)in_dtypes; (void)out_dtype; (void)d_out_all;
    if (nranks != NR || (unsigned)rank >= (unsigned)NR || n_in != 2 || in_sizes[0] != 1024 * KDIM || in_sizes[1] != KDIM * 1024 || out_size != KDIM * 1024 || ws_bytes < WS_NEED) {
        fprintf(stderr, "kernel_launch: unexpected problem geometry\n");
        return;
    }
    static bool attr_set = false;
    if (!attr_set) { HIP_CHECK(hipFuncSetAttribute((const void*)ag_gemm_kernel, hipFuncAttributeMaxDynamicSharedMemorySize, LDS_BYTES)); attr_set = true; }
    Params P;
    P.x = (const char*)d_in[0]; P.w = (const char*)d_in[1]; P.out = (char*)d_out;
    for (int d = 0; d < NR; ++d) P.wsx[d] = (char*)d_ws_all[renum(renum(rank) ^ d)];
    P.rank = renum(rank); P.pad = 0;
    hipLaunchKernelGGL(ag_gemm_kernel, dim3(GRID), dim3(NTHR), LDS_BYTES, stream, P);
    HIP_CHECK(hipGetLastError());
}
```
